```python
import numpy as np

kernel_name = "gpu_a2a_gemm_m4096_k4096_n16384_w8a8_dqrelu"


ABSMAX_RTOL = 0.001

COMPUTE_DTYPE = "int8"
EPILOGUE = "dequant_relu"
IN_QUANT = "int8"
OUT_QUANT = None
IN_DTYPES = {"x": "int8", "w_mat": "int8",
             "scale_x": "float32", "scale_w": "float32"}
WORLD_SIZE = 8

def setup_inputs():

    x = np.random.randint(-127, 128, size=(4096, 4096)).astype(np.int8)
    w_mat = np.random.randint(-127, 128, size=(4096, 16384)).astype(np.int8)
    scale_x = (0.02 * (1.0 + np.random.rand(1))).astype(np.float32)
    scale_w = (0.02 * (1.0 + np.random.rand(1))).astype(np.float32)
    return {"x": x, "w_mat": w_mat, "scale_x": scale_x, "scale_w": scale_w}

def reference(x, w_mat, scale_x, scale_w):
    acc = x.astype(np.float64) @ w_mat.astype(np.float64)
    y = (acc.astype(np.float32) * (scale_x * scale_w)).astype(np.float32)
    return np.maximum(y, 0.0).astype(np.float32)
```

```cpp
#include <hip/hip_runtime.h>
#include <rccl/rccl.h>
#include <cstdio>

#define HIP_CHECK(expr) do { hipError_t _e = (expr); if (_e != hipSuccess) { \
    fprintf(stderr, "HIP error %s at %s:%d\n", hipGetErrorString(_e), __FILE__, __LINE__); \
    return; } } while (0)

#define GAS __attribute__((address_space(1)))
#define LAS __attribute__((address_space(3)))
typedef int v4i __attribute__((ext_vector_type(4)));

namespace {
constexpr int NR = 8;
constexpr int MLOC = 512, NN = 16384, KK = 4096, KC = 512;
constexpr int NSTEP = 64;
constexpr int NS = 18;
constexpr int SLOT = 8192;
constexpr int CTRL = 19 * SLOT;
constexpr int LDS_BYTES = CTRL + 256 + 1024;
constexpr int NWG = 256;
constexpr int NTHREADS = 64 * 8;
constexpr size_t WS_CTR = 0;
constexpr size_t WS_TMO = 32768;
constexpr size_t WS_CNT = 33024;
constexpr size_t WS_FLAG2 = 40960;
constexpr size_t WS_FLAG = 65536;
constexpr size_t FLAG_MT = 8 * 2048;
constexpr size_t WS_A = 2u << 20;
constexpr size_t A_PAR = 2u << 20;
constexpr size_t PAD = 0x1E0000;
constexpr size_t WS_NEED = WS_A + 2 * A_PAR;
constexpr unsigned long long DEADLINE = 50000000ull;
constexpr unsigned BIG = 100000u;
constexpr int ABIAS = 4;
constexpr int CL = 2;
constexpr int LLEAD = 14, LFOL = 8;
constexpr int LASTN = 53;
constexpr int HSTART = 8, HAHEAD = 4;
constexpr int PBASE = 240, PG = 90, POLL0 = 360;
struct Peers { char* ws[NR]; };
}

#define CONSUMER_ASM \
  "s_mov_b64 exec, -1\n" \
  "v_mbcnt_lo_u32_b32 v12, -1, 0\n" \
  "v_mbcnt_hi_u32_b32 v12, -1, v12\n" \
  "v_lshrrev_b32 v128, 3, v12\n" \
  "v_and_b32 v129, 6, v128\n" \
  "v_and_b32 v112, 7, v12\n" \
  "v_xor_b32 v112, v112, v129\n" \
  "v_lshlrev_b32 v112, 4, v112\n" \
  "v_lshl_add_u32 v112, v128, 9, v112\n" \
  "s_mov_b32 s66, %[sxb]\n" \
  "s_mov_b64 s[40:41], %[sa0]\n" \
  "s_cmp_eq_u32 %[shold], 0\n" \
  "s_cbranch_scc1 LNH1_%=\n" \
  "v_mov_b32 v9, 155728\n" \
  "LHW_%=:\n" \
  "s_sleep 1\n" \
  "ds_read_b32 v10, v9\n" \
  "s_waitcnt lgkmcnt(0)\n" \
  "v_cmp_eq_u32 vcc, 0, v10\n" \
  "s_cbranch_vccnz LHW_%=\n" \
  "LNH1_%=:\n" \
  "s_add_u32 m0, s66, 0\n" \
  "s_add_u32 s68, s40, 0\n" \
  "s_addc_u32 s69, s41, 0\n" \
  "global_load_lds_dwordx4 v112, s[68:69]\n" \
  "s_add_u32 m0, s66, 1024\n" \
  "s_add_u32 s68, s40, 4096\n" \
  "s_addc_u32 s69, s41, 0\n" \
  "global_load_lds_dwordx4 v112, s[68:69]\n" \
  "s_add_u32 m0, s66, 2048\n" \
  "s_add_u32 s68, s40, 8192\n" \
  "s_addc_u32 s69, s41, 0\n" \
  "global_load_lds_dwordx4 v112, s[68:69]\n" \
  "s_add_u32 m0, s66, 3072\n" \
  "s_add_u32 s68, s40, 12288\n" \
  "s_addc_u32 s69, s41, 0\n" \
  "global_load_lds_dwordx4 v112, s[68:69]\n" \
  "s_add_u32 m0, s66, 4096\n" \
  "s_add_u32 s68, s40, 16384\n" \
  "s_addc_u32 s69, s41, 0\n" \
  "global_load_lds_dwordx4 v112, s[68:69]\n" \
  "s_add_u32 m0, s66, 5120\n" \
  "s_add_u32 s68, s40, 20480\n" \
  "s_addc_u32 s69, s41, 0\n" \
  "global_load_lds_dwordx4 v112, s[68:69]\n" \
  "s_add_u32 m0, s66, 6144\n" \
  "s_add_u32 s68, s40, 24576\n" \
  "s_addc_u32 s69, s41, 0\n" \
  "global_load_lds_dwordx4 v112, s[68:69]\n" \
  "s_add_u32 m0, s66, 7168\n" \
  "s_add_u32 s68, s40, 28672\n" \
  "s_addc_u32 s69, s41, 0\n" \
  "global_load_lds_dwordx4 v112, s[68:69]\n" \
  "s_add_u32 m0, s66, 32768\n" \
  "s_add_u32 s68, s40, 128\n" \
  "s_addc_u32 s69, s41, 0\n" \
  "global_load_lds_dwordx4 v112, s[68:69]\n" \
  "s_add_u32 m0, s66, 33792\n" \
  "s_add_u32 s68, s40, 4224\n" \
  "s_addc_u32 s69, s41, 0\n" \
  "global_load_lds_dwordx4 v112, s[68:69]\n" \
  "s_add_u32 m0, s66, 34816\n" \
  "s_add_u32 s68, s40, 8320\n" \
  "s_addc_u32 s69, s41, 0\n" \
  "global_load_lds_dwordx4 v112, s[68:69]\n" \
  "s_add_u32 m0, s66, 35840\n" \
  "s_add_u32 s68, s40, 12416\n" \
  "s_addc_u32 s69, s41, 0\n" \
  "global_load_lds_dwordx4 v112, s[68:69]\n" \
  "s_add_u32 m0, s66, 36864\n" \
  "s_add_u32 s68, s40, 16512\n" \
  "s_addc_u32 s69, s41, 0\n" \
  "global_load_lds_dwordx4 v112, s[68:69]\n" \
  "s_add_u32 m0, s66, 37888\n" \
  "s_add_u32 s68, s40, 20608\n" \
  "s_addc_u32 s69, s41, 0\n" \
  "global_load_lds_dwordx4 v112, s[68:69]\n" \
  "s_add_u32 m0, s66, 38912\n" \
  "s_add_u32 s68, s40, 24704\n" \
  "s_addc_u32 s69, s41, 0\n" \
  "global_load_lds_dwordx4 v112, s[68:69]\n" \
  "s_add_u32 m0, s66, 39936\n" \
  "s_add_u32 s68, s40, 28800\n" \
  "s_addc_u32 s69, s41, 0\n" \
  "global_load_lds_dwordx4 v112, s[68:69]\n" \
  "s_load_dword s50, %[psx], 0\n" \
  "s_load_dword s51, %[psw], 0\n" \
  "v_and_b32 v10, 15, v12\n" \
  "v_lshrrev_b32 v11, 4, v12\n" \
  "v_lshrrev_b32 v8, 1, v10\n" \
  "v_lshl_add_u32 v0, v11, 4, v8\n" \
  "v_lshlrev_b32 v0, 7, v0\n" \
  "v_and_b32 v9, 1, v10\n" \
  "v_lshl_add_u32 v0, v9, 3, v0\n" \
  "v_and_b32 v8, 6, v8\n" \
  "v_and_b32 v9, 1, v11\n" \
  "v_or_b32 v8, v8, v9\n" \
  "v_lshl_add_u32 v0, v8, 4, v0\n" \
  "s_lshr_b32 s47, %[sw7], 2\n" \
  "v_xor_b32 v0, s47, v0\n" \
  "v_xor_b32 v1, 16, v0\n" \
  "v_xor_b32 v2, 32, v0\n" \
  "v_xor_b32 v3, 48, v0\n" \
  "v_xor_b32 v4, 64, v0\n" \
  "v_xor_b32 v5, 80, v0\n" \
  "v_xor_b32 v6, 96, v0\n" \
  "v_xor_b32 v7, 112, v0\n" \
  "v_and_b32 v128, 6, v10\n" \
  "v_xor_b32 v128, v128, v11\n" \
  "v_lshlrev_b32 v128, 4, v128\n" \
  "v_lshl_add_u32 v13, v10, 7, v128\n" \
  "v_add_u32 v13, s66, v13\n" \
  "v_xor_b32 v8, 64, v13\n" \
  "v_and_b32 v9, 3, v12\n" \
  "v_lshlrev_b32 v9, 2, v9\n" \
  "v_add_u32 v9, 155648, v9\n" \
  "v_lshlrev_b32 v12, 2, v12\n" \
  "v_add_u32 v12, %[sfree], v12\n" \
  "v_mov_b32 v11, 2\n" \
  "v_mov_b32 v128, 0\n" \
  "v_mov_b32 v129, 0\n" \
  "v_mov_b32 v130, 0\n" \
  "v_mov_b32 v131, 0\n" \
  "v_mov_b32 v132, 0\n" \
  "v_mov_b32 v133, 0\n" \
  "v_mov_b32 v134, 0\n" \
  "v_mov_b32 v135, 0\n" \
  "v_mov_b32 v136, 0\n" \
  "v_mov_b32 v137, 0\n" \
  "v_mov_b32 v138, 0\n" \
  "v_mov_b32 v139, 0\n" \
  "v_mov_b32 v140, 0\n" \
  "v_mov_b32 v141, 0\n" \
  "v_mov_b32 v142, 0\n" \
  "v_mov_b32 v143, 0\n" \
  "v_mov_b32 v144, 0\n" \
  "v_mov_b32 v145, 0\n" \
  "v_mov_b32 v146, 0\n" \
  "v_mov_b32 v147, 0\n" \
  "v_mov_b32 v148, 0\n" \
  "v_mov_b32 v149, 0\n" \
  "v_mov_b32 v150, 0\n" \
  "v_mov_b32 v151, 0\n" \
  "v_mov_b32 v152, 0\n" \
  "v_mov_b32 v153, 0\n" \
  "v_mov_b32 v154, 0\n" \
  "v_mov_b32 v155, 0\n" \
  "v_mov_b32 v156, 0\n" \
  "v_mov_b32 v157, 0\n" \
  "v_mov_b32 v158, 0\n" \
  "v_mov_b32 v159, 0\n" \
  "v_mov_b32 v160, 0\n" \
  "v_mov_b32 v161, 0\n" \
  "v_mov_b32 v162, 0\n" \
  "v_mov_b32 v163, 0\n" \
  "v_mov_b32 v164, 0\n" \
  "v_mov_b32 v165, 0\n" \
  "v_mov_b32 v166, 0\n" \
  "v_mov_b32 v167, 0\n" \
  "v_mov_b32 v168, 0\n" \
  "v_mov_b32 v169, 0\n" \
  "v_mov_b32 v170, 0\n" \
  "v_mov_b32 v171, 0\n" \
  "v_mov_b32 v172, 0\n" \
  "v_mov_b32 v173, 0\n" \
  "v_mov_b32 v174, 0\n" \
  "v_mov_b32 v175, 0\n" \
  "v_mov_b32 v176, 0\n" \
  "v_mov_b32 v177, 0\n" \
  "v_mov_b32 v178, 0\n" \
  "v_mov_b32 v179, 0\n" \
  "v_mov_b32 v180, 0\n" \
  "v_mov_b32 v181, 0\n" \
  "v_mov_b32 v182, 0\n" \
  "v_mov_b32 v183, 0\n" \
  "v_mov_b32 v184, 0\n" \
  "v_mov_b32 v185, 0\n" \
  "v_mov_b32 v186, 0\n" \
  "v_mov_b32 v187, 0\n" \
  "v_mov_b32 v188, 0\n" \
  "v_mov_b32 v189, 0\n" \
  "v_mov_b32 v190, 0\n" \
  "v_mov_b32 v191, 0\n" \
  "v_mov_b32 v192, 0\n" \
  "v_mov_b32 v193, 0\n" \
  "v_mov_b32 v194, 0\n" \
  "v_mov_b32 v195, 0\n" \
  "v_mov_b32 v196, 0\n" \
  "v_mov_b32 v197, 0\n" \
  "v_mov_b32 v198, 0\n" \
  "v_mov_b32 v199, 0\n" \
  "v_mov_b32 v200, 0\n" \
  "v_mov_b32 v201, 0\n" \
  "v_mov_b32 v202, 0\n" \
  "v_mov_b32 v203, 0\n" \
  "v_mov_b32 v204, 0\n" \
  "v_mov_b32 v205, 0\n" \
  "v_mov_b32 v206, 0\n" \
  "v_mov_b32 v207, 0\n" \
  "v_mov_b32 v208, 0\n" \
  "v_mov_b32 v209, 0\n" \
  "v_mov_b32 v210, 0\n" \
  "v_mov_b32 v211, 0\n" \
  "v_mov_b32 v212, 0\n" \
  "v_mov_b32 v213, 0\n" \
  "v_mov_b32 v214, 0\n" \
  "v_mov_b32 v215, 0\n" \
  "v_mov_b32 v216, 0\n" \
  "v_mov_b32 v217, 0\n" \
  "v_mov_b32 v218, 0\n" \
  "v_mov_b32 v219, 0\n" \
  "v_mov_b32 v220, 0\n" \
  "v_mov_b32 v221, 0\n" \
  "v_mov_b32 v222, 0\n" \
  "v_mov_b32 v223, 0\n" \
  "v_mov_b32 v224, 0\n" \
  "v_mov_b32 v225, 0\n" \
  "v_mov_b32 v226, 0\n" \
  "v_mov_b32 v227, 0\n" \
  "v_mov_b32 v228, 0\n" \
  "v_mov_b32 v229, 0\n" \
  "v_mov_b32 v230, 0\n" \
  "v_mov_b32 v231, 0\n" \
  "v_mov_b32 v232, 0\n" \
  "v_mov_b32 v233, 0\n" \
  "v_mov_b32 v234, 0\n" \
  "v_mov_b32 v235, 0\n" \
  "v_mov_b32 v236, 0\n" \
  "v_mov_b32 v237, 0\n" \
  "v_mov_b32 v238, 0\n" \
  "v_mov_b32 v239, 0\n" \
  "v_mov_b32 v240, 0\n" \
  "v_mov_b32 v241, 0\n" \
  "v_mov_b32 v242, 0\n" \
  "v_mov_b32 v243, 0\n" \
  "v_mov_b32 v244, 0\n" \
  "v_mov_b32 v245, 0\n" \
  "v_mov_b32 v246, 0\n" \
  "v_mov_b32 v247, 0\n" \
  "v_mov_b32 v248, 0\n" \
  "v_mov_b32 v249, 0\n" \
  "v_mov_b32 v250, 0\n" \
  "v_mov_b32 v251, 0\n" \
  "v_mov_b32 v252, 0\n" \
  "v_mov_b32 v253, 0\n" \
  "v_mov_b32 v254, 0\n" \
  "v_mov_b32 v255, 0\n" \
  "s_waitcnt vmcnt(8)\n" \
  "ds_read_b128 v[32:35], v13 offset:0\n" \
  "ds_read_b128 v[36:39], v13 offset:2048\n" \
  "ds_read_b128 v[40:43], v13 offset:4096\n" \
  "ds_read_b128 v[44:47], v13 offset:6144\n" \
  "ds_read_b128 v[48:51], v8 offset:0\n" \
  "ds_read_b128 v[52:55], v8 offset:2048\n" \
  "ds_read_b128 v[56:59], v8 offset:4096\n" \
  "ds_read_b128 v[60:63], v8 offset:6144\n" \
  "LPRO_%=:\n" \
  "ds_read_b32 v10, v9\n" \
  "s_waitcnt lgkmcnt(0)\n" \
  "v_cmp_gt_u32 vcc, 2, v10\n" \
  "s_cbranch_vccz LPRO2_%=\n" \
  "s_sleep 1\n" \
  "s_branch LPRO_%=\n" \
  "LPRO2_%=:\n" \
  "ds_read_b64_tr_b8 v[16:17], v0 offset:0\n" \
  "ds_read_b64_tr_b8 v[18:19], v0 offset:1024\n" \
  "ds_read_b64_tr_b8 v[20:21], v0 offset:8192\n" \
  "ds_read_b64_tr_b8 v[22:23], v0 offset:9216\n" \
  "s_waitcnt lgkmcnt(0)\n" \
  "v_mfma_i32_16x16x64_i8 v[128:131], v[32:35], v[16:19], v[128:131]\n" \
  "ds_read_b64_tr_b8 v[24:25], v1 offset:0\n" \
  "v_mfma_i32_16x16x64_i8 v[128:131], v[48:51], v[20:23], v[128:131]\n" \
  "ds_read_b64_tr_b8 v[26:27], v1 offset:1024\n" \
  "v_mfma_i32_16x16x64_i8 v[160:163], v[52:55], v[20:23], v[160:163]\n" \
  "ds_read_b64_tr_b8 v[28:29], v1 offset:8192\n" \
  "v_mfma_i32_16x16x64_i8 v[160:163], v[36:39], v[16:19], v[160:163]\n" \
  "ds_read_b64_tr_b8 v[30:31], v1 offset:9216\n" \
  "v_mfma_i32_16x16x64_i8 v[192:195], v[40:43], v[16:19], v[192:195]\n" \
  "s_add_u32 m0, s66, 0\n" \
  "s_add_u32 s68, s40, 256\n" \
  "s_addc_u32 s69, s41, 0\n" \
  "global_load_lds_dwordx4 v112, s[68:69]\n" \
  "v_mfma_i32_16x16x64_i8 v[192:195], v[56:59], v[20:23], v[192:195]\n" \
  "s_add_u32 m0, s66, 1024\n" \
  "s_add_u32 s68, s40, 4352\n" \
  "s_addc_u32 s69, s41, 0\n" \
  "global_load_lds_dwordx4 v112, s[68:69]\n" \
  "v_mfma_i32_16x16x64_i8 v[224:227], v[60:63], v[20:23], v[224:227]\n" \
  "v_mfma_i32_16x16x64_i8 v[224:227], v[44:47], v[16:19], v[224:227]\n" \
  "s_waitcnt vmcnt(2) lgkmcnt(0)\n" \
  "v_mfma_i32_16x16x64_i8 v[228:231], v[44:47], v[24:27], v[228:231]\n" \
  "ds_read_b64_tr_b8 v[16:17], v2 offset:0\n" \
  "v_mfma_i32_16x16x64_i8 v[228:231], v[60:63], v[28:31], v[228:231]\n" \
  "ds_read_b64_tr_b8 v[18:19], v2 offset:1024\n" \
  "v_mfma_i32_16x16x64_i8 v[196:199], v[56:59], v[28:31], v[196:199]\n" \
  "ds_read_b64_tr_b8 v[20:21], v2 offset:8192\n" \
  "v_mfma_i32_16x16x64_i8 v[196:199], v[40:43], v[24:27], v[196:199]\n" \
  "ds_read_b64_tr_b8 v[22:23], v2 offset:9216\n" \
  "v_mfma_i32_16x16x64_i8 v[164:167], v[36:39], v[24:27], v[164:167]\n" \
  "s_add_u32 m0, s66, 2048\n" \
  "s_add_u32 s68, s40, 8448\n" \
  "s_addc_u32 s69, s41, 0\n" \
  "global_load_lds_dwordx4 v112, s[68:69]\n" \
  "s_add_u32 m0, s66, 3072\n" \
  "s_add_u32 s68, s40, 12544\n" \
  "s_addc_u32 s69, s41, 0\n" \
  "global_load_lds_dwordx4 v112, s[68:69]\n" \
  "v_mfma_i32_16x16x64_i8 v[164:167], v[52:55], v[28:31], v[164:167]\n" \
  "ds_read_b128 v[64:67], v13 offset:32768\n" \
  "ds_read_b128 v[68:71], v13 offset:34816\n" \
  "v_mfma_i32_16x16x64_i8 v[132:135], v[48:51], v[28:31], v[132:135]\n" \
  "ds_read_b128 v[72:75], v13 offset:36864\n" \
  "ds_read_b128 v[76:79], v13 offset:38912\n" \
  "v_mfma_i32_16x16x64_i8 v[132:135], v[32:35], v[24:27], v[132:135]\n" \
  "s_waitcnt lgkmcnt(4)\n" \
  "v_mfma_i32_16x16x64_i8 v[136:139], v[32:35], v[16:19], v[136:139]\n" \
  "ds_read_b64_tr_b8 v[24:25], v3 offset:0\n" \
  "v_mfma_i32_16x16x64_i8 v[136:139], v[48:51], v[20:23], v[136:139]\n" \
  "ds_read_b64_tr_b8 v[26:27], v3 offset:1024\n" \
  "v_mfma_i32_16x16x64_i8 v[168:171], v[52:55], v[20:23], v[168:171]\n" \
  "ds_read_b64_tr_b8 v[28:29], v3 offset:8192\n" \
  "v_mfma_i32_16x16x64_i8 v[168:171], v[36:39], v[16:19], v[168:171]\n" \
  "ds_read_b64_tr_b8 v[30:31], v3 offset:9216\n" \
  "v_mfma_i32_16x16x64_i8 v[200:203], v[40:43], v[16:19], v[200:203]\n" \
  "s_add_u32 m0, s66, 4096\n" \
  "s_add_u32 s68, s40, 16640\n" \
  "s_addc_u32 s69, s41, 0\n" \
  "global_load_lds_dwordx4 v112, s[68:69]\n" \
  "s_add_u32 m0, s66, 5120\n" \
  "s_add_u32 s68, s40, 20736\n" \
  "s_addc_u32 s69, s41, 0\n" \
  "global_load_lds_dwordx4 v112, s[68:69]\n" \
  "v_mfma_i32_16x16x64_i8 v[200:203], v[56:59], v[20:23], v[200:203]\n" \
  "ds_read_b128 v[80:83], v8 offset:32768\n" \
  "ds_read_b128 v[84:87], v8 offset:34816\n" \
  "v_mfma_i32_16x16x64_i8 v[232:235], v[60:63], v[20:23], v[232:235]\n" \
  "ds_read_b128 v[88:91], v8 offset:36864\n" \
  "ds_read_b128 v[92:95], v8 offset:38912\n" \
  "v_mfma_i32_16x16x64_i8 v[232:235], v[44:47], v[16:19], v[232:235]\n" \
  "s_waitcnt lgkmcnt(4)\n" \
  "v_mfma_i32_16x16x64_i8 v[236:239], v[44:47], v[24:27], v[236:239]\n" \
  "ds_read_b64_tr_b8 v[16:17], v4 offset:0\n" \
  "v_mfma_i32_16x16x64_i8 v[236:239], v[60:63], v[28:31], v[236:239]\n" \
  "ds_read_b64_tr_b8 v[18:19], v4 offset:1024\n" \
  "v_mfma_i32_16x16x64_i8 v[204:207], v[56:59], v[28:31], v[204:207]\n" \
  "ds_read_b64_tr_b8 v[20:21], v4 offset:8192\n" \
  "v_mfma_i32_16x16x64_i8 v[204:207], v[40:43], v[24:27], v[204:207]\n" \
  "ds_read_b64_tr_b8 v[22:23], v4 offset:9216\n" \
  "v_mfma_i32_16x16x64_i8 v[172:175], v[36:39], v[24:27], v[172:175]\n" \
  "s_add_u32 m0, s66, 6144\n" \
  "s_add_u32 s68, s40, 24832\n" \
  "s_addc_u32 s69, s41, 0\n" \
  "global_load_lds_dwordx4 v112, s[68:69]\n" \
  "v_mfma_i32_16x16x64_i8 v[172:175], v[52:55], v[28:31], v[172:175]\n" \
  "s_add_u32 m0, s66, 7168\n" \
  "s_add_u32 s68, s40, 28928\n" \
  "s_addc_u32 s69, s41, 0\n" \
  "global_load_lds_dwordx4 v112, s[68:69]\n" \
  "v_mfma_i32_16x16x64_i8 v[140:143], v[48:51], v[28:31], v[140:143]\n" \
  "v_mfma_i32_16x16x64_i8 v[140:143], v[32:35], v[24:27], v[140:143]\n" \
  "s_waitcnt lgkmcnt(0)\n" \
  "v_mfma_i32_16x16x64_i8 v[144:147], v[32:35], v[16:19], v[144:147]\n" \
  "ds_read_b64_tr_b8 v[24:25], v5 offset:0\n" \
  "v_mfma_i32_16x16x64_i8 v[144:147], v[48:51], v[20:23], v[144:147]\n" \
  "ds_read_b64_tr_b8 v[26:27], v5 offset:1024\n" \
  "v_mfma_i32_16x16x64_i8 v[176:179], v[52:55], v[20:23], v[176:179]\n" \
  "ds_read_b64_tr_b8 v[28:29], v5 offset:8192\n" \
  "v_mfma_i32_16x16x64_i8 v[176:179], v[36:39], v[16:19], v[176:179]\n" \
  "ds_read_b64_tr_b8 v[30:31], v5 offset:9216\n" \
  "v_mfma_i32_16x16x64_i8 v[208:211], v[40:43], v[16:19], v[208:211]\n" \
  "ds_read_b32 v10, v9\n" \
  "s_waitcnt lgkmcnt(0)\n" \
  "v_mfma_i32_16x16x64_i8 v[208:211], v[56:59], v[20:23], v[208:211]\n" \
  "s_add_u32 m0, s66, 32768\n" \
  "s_add_u32 s68, s40, 384\n" \
  "s_addc_u32 s69, s41, 0\n" \
  "global_load_lds_dwordx4 v112, s[68:69]\n" \
  "v_mfma_i32_16x16x64_i8 v[240:243], v[60:63], v[20:23], v[240:243]\n" \
  "s_add_u32 m0, s66, 33792\n" \
  "s_add_u32 s68, s40, 4480\n" \
  "s_addc_u32 s69, s41, 0\n" \
  "global_load_lds_dwordx4 v112, s[68:69]\n" \
  "v_mfma_i32_16x16x64_i8 v[240:243], v[44:47], v[16:19], v[240:243]\n" \
  "s_waitcnt lgkmcnt(1)\n" \
  "v_mfma_i32_16x16x64_i8 v[244:247], v[44:47], v[24:27], v[244:247]\n" \
  "ds_read_b64_tr_b8 v[16:17], v6 offset:0\n" \
  "v_mfma_i32_16x16x64_i8 v[244:247], v[60:63], v[28:31], v[244:247]\n" \
  "ds_read_b64_tr_b8 v[18:19], v6 offset:1024\n" \
  "v_mfma_i32_16x16x64_i8 v[212:215], v[56:59], v[28:31], v[212:215]\n" \
  "ds_read_b64_tr_b8 v[20:21], v6 offset:8192\n" \
  "v_mfma_i32_16x16x64_i8 v[212:215], v[40:43], v[24:27], v[212:215]\n" \
  "ds_read_b64_tr_b8 v[22:23], v6 offset:9216\n" \
  "v_mfma_i32_16x16x64_i8 v[180:183], v[36:39], v[24:27], v[180:183]\n" \
  "s_add_u32 m0, s66, 34816\n" \
  "s_add_u32 s68, s40, 8576\n" \
  "s_addc_u32 s69, s41, 0\n" \
  "global_load_lds_dwordx4 v112, s[68:69]\n" \
  "v_mfma_i32_16x16x64_i8 v[180:183], v[52:55], v[28:31], v[180:183]\n" \
  "s_add_u32 m0, s66, 35840\n" \
  "s_add_u32 s68, s40, 12672\n" \
  "s_addc_u32 s69, s41, 0\n" \
  "global_load_lds_dwordx4 v112, s[68:69]\n" \
  "v_mfma_i32_16x16x64_i8 v[148:151], v[48:51], v[28:31], v[148:151]\n" \
  "v_mfma_i32_16x16x64_i8 v[148:151], v[32:35], v[24:27], v[148:151]\n" \
  "s_waitcnt lgkmcnt(0)\n" \
  "v_mfma_i32_16x16x64_i8 v[152:155], v[32:35], v[16:19], v[152:155]\n" \
  "ds_read_b64_tr_b8 v[24:25], v7 offset:0\n" \
  "v_mfma_i32_16x16x64_i8 v[152:155], v[48:51], v[20:23], v[152:155]\n" \
  "ds_read_b64_tr_b8 v[26:27], v7 offset:1024\n" \
  "v_mfma_i32_16x16x64_i8 v[184:187], v[52:55], v[20:23], v[184:187]\n" \
  "ds_read_b64_tr_b8 v[28:29], v7 offset:8192\n" \
  "v_mfma_i32_16x16x64_i8 v[184:187], v[36:39], v[16:19], v[184:187]\n" \
  "ds_read_b64_tr_b8 v[30:31], v7 offset:9216\n" \
  "v_mfma_i32_16x16x64_i8 v[216:219], v[40:43], v[16:19], v[216:219]\n" \
  "s_add_u32 m0, s66, 36864\n" \
  "s_add_u32 s68, s40, 16768\n" \
  "s_addc_u32 s69, s41, 0\n" \
  "global_load_lds_dwordx4 v112, s[68:69]\n" \
  "v_mfma_i32_16x16x64_i8 v[216:219], v[56:59], v[20:23], v[216:219]\n" \
  "s_add_u32 m0, s66, 37888\n" \
  "s_add_u32 s68, s40, 20864\n" \
  "s_addc_u32 s69, s41, 0\n" \
  "global_load_lds_dwordx4 v112, s[68:69]\n" \
  "v_mfma_i32_16x16x64_i8 v[248:251], v[60:63], v[20:23], v[248:251]\n" \
  "v_cmp_gt_u32 vcc, 4, v10\n" \
  "s_and_b64 vcc, vcc, 13\n" \
  "v_mfma_i32_16x16x64_i8 v[248:251], v[44:47], v[16:19], v[248:251]\n" \
  "s_waitcnt lgkmcnt(0)\n" \
  "v_mfma_i32_16x16x64_i8 v[252:255], v[44:47], v[24:27], v[252:255]\n" \
  "s_cbranch_vccnz LS0_%=\n" \
  "LB0_%=:\n" \
  "ds_read_b64_tr_b8 v[16:17], v0 offset:16384\n" \
  "v_mfma_i32_16x16x64_i8 v[252:255], v[60:63], v[28:31], v[252:255]\n" \
  "ds_read_b64_tr_b8 v[18:19], v0 offset:17408\n" \
  "v_mfma_i32_16x16x64_i8 v[220:223], v[56:59], v[28:31], v[220:223]\n" \
  "ds_read_b64_tr_b8 v[20:21], v0 offset:24576\n" \
  "v_mfma_i32_16x16x64_i8 v[220:223], v[40:43], v[24:27], v[220:223]\n" \
  "ds_read_b64_tr_b8 v[22:23], v0 offset:25600\n" \
  "v_mfma_i32_16x16x64_i8 v[188:191], v[36:39], v[24:27], v[188:191]\n" \
  "s_add_u32 m0, s66, 38912\n" \
  "s_add_u32 s68, s40, 24960\n" \
  "s_addc_u32 s69, s41, 0\n" \
  "global_load_lds_dwordx4 v112, s[68:69]\n" \
  "v_mfma_i32_16x16x64_i8 v[188:191], v[52:55], v[28:31], v[188:191]\n" \
  "s_add_u32 m0, s66, 39936\n" \
  "s_add_u32 s68, s40, 29056\n" \
  "s_addc_u32 s69, s41, 0\n" \
  "global_load_lds_dwordx4 v112, s[68:69]\n" \
  "v_mfma_i32_16x16x64_i8 v[156:159], v[48:51], v[28:31], v[156:159]\n" \
  "ds_add_u32 v12, v11\n" \
  "v_mfma_i32_16x16x64_i8 v[156:159], v[32:35], v[24:27], v[156:159]\n" \
  "s_waitcnt vmcnt(8) lgkmcnt(1)\n" \
  "v_mfma_i32_16x16x64_i8 v[128:131], v[64:67], v[16:19], v[128:131]\n" \
  "ds_read_b64_tr_b8 v[24:25], v1 offset:16384\n" \
  "v_mfma_i32_16x16x64_i8 v[128:131], v[80:83], v[20:23], v[128:131]\n" \
  "ds_read_b64_tr_b8 v[26:27], v1 offset:17408\n" \
  "v_mfma_i32_16x16x64_i8 v[160:163], v[84:87], v[20:23], v[160:163]\n" \
  "ds_read_b64_tr_b8 v[28:29], v1 offset:24576\n" \
  "v_mfma_i32_16x16x64_i8 v[160:163], v[68:71], v[16:19], v[160:163]\n" \
  "ds_read_b64_tr_b8 v[30:31], v1 offset:25600\n" \
  "v_mfma_i32_16x16x64_i8 v[192:195], v[72:75], v[16:19], v[192:195]\n" \
  "ds_read_b128 v[96:99], v13 offset:0\n" \
  "v_mfma_i32_16x16x64_i8 v[192:195], v[88:91], v[20:23], v[192:195]\n" \
  "ds_read_b128 v[100:103], v13 offset:2048\n" \
  "v_mfma_i32_16x16x64_i8 v[224:227], v[92:95], v[20:23], v[224:227]\n" \
  "v_mfma_i32_16x16x64_i8 v[224:227], v[76:79], v[16:19], v[224:227]\n" \
  "s_waitcnt lgkmcnt(2)\n" \
  "v_mfma_i32_16x16x64_i8 v[228:231], v[76:79], v[24:27], v[228:231]\n" \
  "ds_read_b64_tr_b8 v[16:17], v2 offset:16384\n" \
  "v_mfma_i32_16x16x64_i8 v[228:231], v[92:95], v[28:31], v[228:231]\n" \
  "ds_read_b64_tr_b8 v[18:19], v2 offset:17408\n" \
  "v_mfma_i32_16x16x64_i8 v[196:199], v[88:91], v[28:31], v[196:199]\n" \
  "ds_read_b64_tr_b8 v[20:21], v2 offset:24576\n" \
  "v_mfma_i32_16x16x64_i8 v[196:199], v[72:75], v[24:27], v[196:199]\n" \
  "ds_read_b64_tr_b8 v[22:23], v2 offset:25600\n" \
  "v_mfma_i32_16x16x64_i8 v[164:167], v[68:71], v[24:27], v[164:167]\n" \
  "ds_read_b128 v[104:107], v13 offset:4096\n" \
  "v_mfma_i32_16x16x64_i8 v[164:167], v[84:87], v[28:31], v[164:167]\n" \
  "ds_read_b128 v[108:111], v13 offset:6144\n" \
  "v_mfma_i32_16x16x64_i8 v[132:135], v[80:83], v[28:31], v[132:135]\n" \
  "v_mfma_i32_16x16x64_i8 v[132:135], v[64:67], v[24:27], v[132:135]\n" \
  "s_waitcnt lgkmcnt(2)\n" \
  "v_mfma_i32_16x16x64_i8 v[136:139], v[64:67], v[16:19], v[136:139]\n" \
  "ds_read_b64_tr_b8 v[24:25], v3 offset:16384\n" \
  "v_mfma_i32_16x16x64_i8 v[136:139], v[80:83], v[20:23], v[136:139]\n" \
  "ds_read_b64_tr_b8 v[26:27], v3 offset:17408\n" \
  "v_mfma_i32_16x16x64_i8 v[168:171], v[84:87], v[20:23], v[168:171]\n" \
  "ds_read_b64_tr_b8 v[28:29], v3 offset:24576\n" \
  "v_mfma_i32_16x16x64_i8 v[168:171], v[68:71], v[16:19], v[168:171]\n" \
  "ds_read_b64_tr_b8 v[30:31], v3 offset:25600\n" \
  "v_mfma_i32_16x16x64_i8 v[200:203], v[72:75], v[16:19], v[200:203]\n" \
  "ds_read_b128 v[112:115], v8 offset:0\n" \
  "v_mfma_i32_16x16x64_i8 v[200:203], v[88:91], v[20:23], v[200:203]\n" \
  "ds_read_b128 v[116:119], v8 offset:2048\n" \
  "v_mfma_i32_16x16x64_i8 v[232:235], v[92:95], v[20:23], v[232:235]\n" \
  "v_mfma_i32_16x16x64_i8 v[232:235], v[76:79], v[16:19], v[232:235]\n" \
  "s_waitcnt lgkmcnt(2)\n" \
  "v_mfma_i32_16x16x64_i8 v[236:239], v[76:79], v[24:27], v[236:239]\n" \
  "ds_read_b64_tr_b8 v[16:17], v4 offset:16384\n" \
  "v_mfma_i32_16x16x64_i8 v[236:239], v[92:95], v[28:31], v[236:239]\n" \
  "ds_read_b64_tr_b8 v[18:19], v4 offset:17408\n" \
  "v_mfma_i32_16x16x64_i8 v[204:207], v[88:91], v[28:31], v[204:207]\n" \
  "ds_read_b64_tr_b8 v[20:21], v4 offset:24576\n" \
  "v_mfma_i32_16x16x64_i8 v[204:207], v[72:75], v[24:27], v[204:207]\n" \
  "ds_read_b64_tr_b8 v[22:23], v4 offset:25600\n" \
  "v_mfma_i32_16x16x64_i8 v[172:175], v[68:71], v[24:27], v[172:175]\n" \
  "ds_read_b128 v[120:123], v8 offset:4096\n" \
  "v_mfma_i32_16x16x64_i8 v[172:175], v[84:87], v[28:31], v[172:175]\n" \
  "ds_read_b128 v[124:127], v8 offset:6144\n" \
  "v_mfma_i32_16x16x64_i8 v[140:143], v[80:83], v[28:31], v[140:143]\n" \
  "v_mfma_i32_16x16x64_i8 v[140:143], v[64:67], v[24:27], v[140:143]\n" \
  "s_waitcnt vmcnt(0) lgkmcnt(2)\n" \
  "v_mfma_i32_16x16x64_i8 v[144:147], v[64:67], v[16:19], v[144:147]\n" \
  "ds_read_b64_tr_b8 v[24:25], v5 offset:16384\n" \
  "v_mfma_i32_16x16x64_i8 v[144:147], v[80:83], v[20:23], v[144:147]\n" \
  "ds_read_b64_tr_b8 v[26:27], v5 offset:17408\n" \
  "v_mfma_i32_16x16x64_i8 v[176:179], v[84:87], v[20:23], v[176:179]\n" \
  "ds_read_b64_tr_b8 v[28:29], v5 offset:24576\n" \
  "v_mfma_i32_16x16x64_i8 v[176:179], v[68:71], v[16:19], v[176:179]\n" \
  "ds_read_b64_tr_b8 v[30:31], v5 offset:25600\n" \
  "v_mfma_i32_16x16x64_i8 v[208:211], v[72:75], v[16:19], v[208:211]\n" \
  "ds_read_b32 v10, v9\n" \
  "v_mfma_i32_16x16x64_i8 v[208:211], v[88:91], v[20:23], v[208:211]\n" \
  "ds_read_b128 v[32:35], v13 offset:32768\n" \
  "v_mfma_i32_16x16x64_i8 v[240:243], v[92:95], v[20:23], v[240:243]\n" \
  "ds_read_b128 v[36:39], v13 offset:34816\n" \
  "v_mfma_i32_16x16x64_i8 v[240:243], v[76:79], v[16:19], v[240:243]\n" \
  "s_waitcnt lgkmcnt(3)\n" \
  "v_mfma_i32_16x16x64_i8 v[244:247], v[76:79], v[24:27], v[244:247]\n" \
  "ds_read_b64_tr_b8 v[16:17], v6 offset:16384\n" \
  "v_mfma_i32_16x16x64_i8 v[244:247], v[92:95], v[28:31], v[244:247]\n" \
  "ds_read_b64_tr_b8 v[18:19], v6 offset:17408\n" \
  "v_mfma_i32_16x16x64_i8 v[212:215], v[88:91], v[28:31], v[212:215]\n" \
  "ds_read_b64_tr_b8 v[20:21], v6 offset:24576\n" \
  "v_mfma_i32_16x16x64_i8 v[212:215], v[72:75], v[24:27], v[212:215]\n" \
  "ds_read_b64_tr_b8 v[22:23], v6 offset:25600\n" \
  "v_mfma_i32_16x16x64_i8 v[180:183], v[68:71], v[24:27], v[180:183]\n" \
  "ds_read_b128 v[40:43], v13 offset:36864\n" \
  "v_mfma_i32_16x16x64_i8 v[180:183], v[84:87], v[28:31], v[180:183]\n" \
  "ds_read_b128 v[44:47], v13 offset:38912\n" \
  "v_mfma_i32_16x16x64_i8 v[148:151], v[80:83], v[28:31], v[148:151]\n" \
  "v_mfma_i32_16x16x64_i8 v[148:151], v[64:67], v[24:27], v[148:151]\n" \
  "s_waitcnt lgkmcnt(2)\n" \
  "v_mfma_i32_16x16x64_i8 v[152:155], v[64:67], v[16:19], v[152:155]\n" \
  "ds_read_b64_tr_b8 v[24:25], v7 offset:16384\n" \
  "v_mfma_i32_16x16x64_i8 v[152:155], v[80:83], v[20:23], v[152:155]\n" \
  "ds_read_b64_tr_b8 v[26:27], v7 offset:17408\n" \
  "v_mfma_i32_16x16x64_i8 v[184:187], v[84:87], v[20:23], v[184:187]\n" \
  "ds_read_b64_tr_b8 v[28:29], v7 offset:24576\n" \
  "v_mfma_i32_16x16x64_i8 v[184:187], v[68:71], v[16:19], v[184:187]\n" \
  "ds_read_b64_tr_b8 v[30:31], v7 offset:25600\n" \
  "v_mfma_i32_16x16x64_i8 v[216:219], v[72:75], v[16:19], v[216:219]\n" \
  "ds_read_b128 v[48:51], v8 offset:32768\n" \
  "v_mfma_i32_16x16x64_i8 v[216:219], v[88:91], v[20:23], v[216:219]\n" \
  "ds_read_b128 v[52:55], v8 offset:34816\n" \
  "v_mfma_i32_16x16x64_i8 v[248:251], v[92:95], v[20:23], v[248:251]\n" \
  "v_cmp_gt_u32 vcc, 6, v10\n" \
  "s_and_b64 vcc, vcc, 13\n" \
  "v_mfma_i32_16x16x64_i8 v[248:251], v[76:79], v[16:19], v[248:251]\n" \
  "s_waitcnt lgkmcnt(2)\n" \
  "v_mfma_i32_16x16x64_i8 v[252:255], v[76:79], v[24:27], v[252:255]\n" \
  "s_cbranch_vccnz LS1_%=\n" \
  "LB1_%=:\n" \
  "ds_read_b64_tr_b8 v[16:17], v0 offset:32768\n" \
  "v_mfma_i32_16x16x64_i8 v[252:255], v[92:95], v[28:31], v[252:255]\n" \
  "ds_read_b64_tr_b8 v[18:19], v0 offset:33792\n" \
  "v_mfma_i32_16x16x64_i8 v[220:223], v[88:91], v[28:31], v[220:223]\n" \
  "ds_read_b64_tr_b8 v[20:21], v0 offset:40960\n" \
  "v_mfma_i32_16x16x64_i8 v[220:223], v[72:75], v[24:27], v[220:223]\n" \
  "ds_read_b64_tr_b8 v[22:23], v0 offset:41984\n" \
  "v_mfma_i32_16x16x64_i8 v[188:191], v[68:71], v[24:27], v[188:191]\n" \
  "ds_read_b128 v[56:59], v8 offset:36864\n" \
  "ds_read_b128 v[60:63], v8 offset:38912\n" \
  "v_mfma_i32_16x16x64_i8 v[188:191], v[84:87], v[28:31], v[188:191]\n" \
  "s_waitcnt lgkmcnt(0)\n" \
  "v_mfma_i32_16x16x64_i8 v[156:159], v[80:83], v[28:31], v[156:159]\n" \
  "ds_add_u32 v12, v11\n" \
  "v_mfma_i32_16x16x64_i8 v[156:159], v[64:67], v[24:27], v[156:159]\n" \
  "s_waitcnt lgkmcnt(3)\n" \
  "v_mfma_i32_16x16x64_i8 v[128:131], v[96:99], v[16:19], v[128:131]\n" \
  "ds_read_b64_tr_b8 v[24:25], v1 offset:32768\n" \
  "v_mfma_i32_16x16x64_i8 v[128:131], v[112:115], v[20:23], v[128:131]\n" \
  "ds_read_b64_tr_b8 v[26:27], v1 offset:33792\n" \
  "v_mfma_i32_16x16x64_i8 v[160:163], v[116:119], v[20:23], v[160:163]\n" \
  "ds_read_b64_tr_b8 v[28:29], v1 offset:40960\n" \
  "v_mfma_i32_16x16x64_i8 v[160:163], v[100:103], v[16:19], v[160:163]\n" \
  "ds_read_b64_tr_b8 v[30:31], v1 offset:41984\n" \
  "v_mfma_i32_16x16x64_i8 v[192:195], v[104:107], v[16:19], v[192:195]\n" \
  "ds_read_b32 v10, v9\n" \
  "v_mfma_i32_16x16x64_i8 v[192:195], v[120:123], v[20:23], v[192:195]\n" \
  "v_mfma_i32_16x16x64_i8 v[224:227], v[124:127], v[20:23], v[224:227]\n" \
  "v_mfma_i32_16x16x64_i8 v[224:227], v[108:111], v[16:19], v[224:227]\n" \
  "s_waitcnt lgkmcnt(1)\n" \
  "v_mfma_i32_16x16x64_i8 v[228:231], v[108:111], v[24:27], v[228:231]\n" \
  "ds_read_b64_tr_b8 v[16:17], v2 offset:32768\n" \
  "v_mfma_i32_16x16x64_i8 v[228:231], v[124:127], v[28:31], v[228:231]\n" \
  "ds_read_b64_tr_b8 v[18:19], v2 offset:33792\n" \
  "v_mfma_i32_16x16x64_i8 v[196:199], v[120:123], v[28:31], v[196:199]\n" \
  "ds_read_b64_tr_b8 v[20:21], v2 offset:40960\n" \
  "v_mfma_i32_16x16x64_i8 v[196:199], v[104:107], v[24:27], v[196:199]\n" \
  "ds_read_b64_tr_b8 v[22:23], v2 offset:41984\n" \
  "v_mfma_i32_16x16x64_i8 v[164:167], v[100:103], v[24:27], v[164:167]\n" \
  "v_mfma_i32_16x16x64_i8 v[164:167], v[116:119], v[28:31], v[164:167]\n" \
  "v_mfma_i32_16x16x64_i8 v[132:135], v[112:115], v[28:31], v[132:135]\n" \
  "v_mfma_i32_16x16x64_i8 v[132:135], v[96:99], v[24:27], v[132:135]\n" \
  "s_waitcnt lgkmcnt(0)\n" \
  "v_mfma_i32_16x16x64_i8 v[136:139], v[96:99], v[16:19], v[136:139]\n" \
  "ds_read_b64_tr_b8 v[24:25], v3 offset:32768\n" \
  "v_mfma_i32_16x16x64_i8 v[136:139], v[112:115], v[20:23], v[136:139]\n" \
  "ds_read_b64_tr_b8 v[26:27], v3 offset:33792\n" \
  "v_mfma_i32_16x16x64_i8 v[168:171], v[116:119], v[20:23], v[168:171]\n" \
  "ds_read_b64_tr_b8 v[28:29], v3 offset:40960\n" \
  "v_mfma_i32_16x16x64_i8 v[168:171], v[100:103], v[16:19], v[168:171]\n" \
  "ds_read_b64_tr_b8 v[30:31], v3 offset:41984\n" \
  "v_mfma_i32_16x16x64_i8 v[200:203], v[104:107], v[16:19], v[200:203]\n" \
  "v_mfma_i32_16x16x64_i8 v[200:203], v[120:123], v[20:23], v[200:203]\n" \
  "v_mfma_i32_16x16x64_i8 v[232:235], v[124:127], v[20:23], v[232:235]\n" \
  "v_cmp_gt_u32 vcc, 6, v10\n" \
  "s_and_b64 vcc, vcc, 2\n" \
  "v_mfma_i32_16x16x64_i8 v[232:235], v[108:111], v[16:19], v[232:235]\n" \
  "s_waitcnt lgkmcnt(0)\n" \
  "v_mfma_i32_16x16x64_i8 v[236:239], v[108:111], v[24:27], v[236:239]\n" \
  "s_cbranch_vccnz LSX_%=\n" \
  "LBX_%=:\n" \
  "ds_read_b32 v13, v9 offset:64\n" \
  "s_mov_b64 s[40:41], %[sa1]\n" \
  "v_mbcnt_lo_u32_b32 v8, -1, 0\n" \
  "v_mbcnt_hi_u32_b32 v8, -1, v8\n" \
  "v_lshlrev_b32 v8, 4, v8\n" \
  "s_waitcnt lgkmcnt(0)\n" \
  "v_readfirstlane_b32 s47, v13\n" \
  "s_add_u32 s40, s40, s47\n" \
  "s_addc_u32 s41, s41, 0\n" \
  "s_nop 4\n" \
  "ds_read_b64_tr_b8 v[16:17], v4 offset:32768\n" \
  "v_mfma_i32_16x16x64_i8 v[236:239], v[124:127], v[28:31], v[236:239]\n" \
  "ds_read_b64_tr_b8 v[18:19], v4 offset:33792\n" \
  "v_mfma_i32_16x16x64_i8 v[204:207], v[120:123], v[28:31], v[204:207]\n" \
  "ds_read_b64_tr_b8 v[20:21], v4 offset:40960\n" \
  "v_mfma_i32_16x16x64_i8 v[204:207], v[104:107], v[24:27], v[204:207]\n" \
  "ds_read_b64_tr_b8 v[22:23], v4 offset:41984\n" \
  "v_mfma_i32_16x16x64_i8 v[172:175], v[100:103], v[24:27], v[172:175]\n" \
  "global_load_dwordx4 v[64:67], v8, s[40:41] offset:0 sc0 sc1\n" \
  "v_mfma_i32_16x16x64_i8 v[172:175], v[116:119], v[28:31], v[172:175]\n" \
  "global_load_dwordx4 v[68:71], v8, s[40:41] offset:1024 sc0 sc1\n" \
  "v_mfma_i32_16x16x64_i8 v[140:143], v[112:115], v[28:31], v[140:143]\n" \
  "v_mfma_i32_16x16x64_i8 v[140:143], v[96:99], v[24:27], v[140:143]\n" \
  "s_waitcnt lgkmcnt(0)\n" \
  "v_mfma_i32_16x16x64_i8 v[144:147], v[96:99], v[16:19], v[144:147]\n" \
  "ds_read_b64_tr_b8 v[24:25], v5 offset:32768\n" \
  "v_mfma_i32_16x16x64_i8 v[144:147], v[112:115], v[20:23], v[144:147]\n" \
  "ds_read_b64_tr_b8 v[26:27], v5 offset:33792\n" \
  "v_mfma_i32_16x16x64_i8 v[176:179], v[116:119], v[20:23], v[176:179]\n" \
  "ds_read_b64_tr_b8 v[28:29], v5 offset:40960\n" \
  "v_mfma_i32_16x16x64_i8 v[176:179], v[100:103], v[16:19], v[176:179]\n" \
  "ds_read_b64_tr_b8 v[30:31], v5 offset:41984\n" \
  "v_mfma_i32_16x16x64_i8 v[208:211], v[104:107], v[16:19], v[208:211]\n" \
  "ds_read_b32 v10, v9\n" \
  "global_load_dwordx4 v[72:75], v8, s[40:41] offset:2048 sc0 sc1\n" \
  "v_mfma_i32_16x16x64_i8 v[208:211], v[120:123], v[20:23], v[208:211]\n" \
  "global_load_dwordx4 v[76:79], v8, s[40:41] offset:3072 sc0 sc1\n" \
  "v_mfma_i32_16x16x64_i8 v[240:243], v[124:127], v[20:23], v[240:243]\n" \
  "v_add_u32 v8, 0x8000, v8\n" \
  "v_mfma_i32_16x16x64_i8 v[240:243], v[108:111], v[16:19], v[240:243]\n" \
  "s_waitcnt lgkmcnt(1)\n" \
  "v_mfma_i32_16x16x64_i8 v[244:247], v[108:111], v[24:27], v[244:247]\n" \
  "ds_read_b64_tr_b8 v[16:17], v6 offset:32768\n" \
  "v_mfma_i32_16x16x64_i8 v[244:247], v[124:127], v[28:31], v[244:247]\n" \
  "ds_read_b64_tr_b8 v[18:19], v6 offset:33792\n" \
  "v_mfma_i32_16x16x64_i8 v[212:215], v[120:123], v[28:31], v[212:215]\n" \
  "ds_read_b64_tr_b8 v[20:21], v6 offset:40960\n" \
  "v_mfma_i32_16x16x64_i8 v[212:215], v[104:107], v[24:27], v[212:215]\n" \
  "ds_read_b64_tr_b8 v[22:23], v6 offset:41984\n" \
  "v_mfma_i32_16x16x64_i8 v[180:183], v[100:103], v[24:27], v[180:183]\n" \
  "global_load_dwordx4 v[80:83], v8, s[40:41] offset:0 sc0 sc1\n" \
  "v_mfma_i32_16x16x64_i8 v[180:183], v[116:119], v[28:31], v[180:183]\n" \
  "global_load_dwordx4 v[84:87], v8, s[40:41] offset:1024 sc0 sc1\n" \
  "v_mfma_i32_16x16x64_i8 v[148:151], v[112:115], v[28:31], v[148:151]\n" \
  "v_mfma_i32_16x16x64_i8 v[148:151], v[96:99], v[24:27], v[148:151]\n" \
  "s_waitcnt lgkmcnt(0)\n" \
  "v_mfma_i32_16x16x64_i8 v[152:155], v[96:99], v[16:19], v[152:155]\n" \
  "ds_read_b64_tr_b8 v[24:25], v7 offset:32768\n" \
  "v_mfma_i32_16x16x64_i8 v[152:155], v[112:115], v[20:23], v[152:155]\n" \
  "ds_read_b64_tr_b8 v[26:27], v7 offset:33792\n" \
  "v_mfma_i32_16x16x64_i8 v[184:187], v[116:119], v[20:23], v[184:187]\n" \
  "ds_read_b64_tr_b8 v[28:29], v7 offset:40960\n" \
  "v_mfma_i32_16x16x64_i8 v[184:187], v[100:103], v[16:19], v[184:187]\n" \
  "ds_read_b64_tr_b8 v[30:31], v7 offset:41984\n" \
  "v_mfma_i32_16x16x64_i8 v[216:219], v[104:107], v[16:19], v[216:219]\n" \
  "global_load_dwordx4 v[88:91], v8, s[40:41] offset:2048 sc0 sc1\n" \
  "v_mfma_i32_16x16x64_i8 v[216:219], v[120:123], v[20:23], v[216:219]\n" \
  "global_load_dwordx4 v[92:95], v8, s[40:41] offset:3072 sc0 sc1\n" \
  "v_mfma_i32_16x16x64_i8 v[248:251], v[124:127], v[20:23], v[248:251]\n" \
  "v_add_u32 v8, 0x8000, v8\n" \
  "v_cmp_gt_u32 vcc, 8, v10\n" \
  "v_mfma_i32_16x16x64_i8 v[248:251], v[108:111], v[16:19], v[248:251]\n" \
  "s_waitcnt lgkmcnt(0)\n" \
  "v_mfma_i32_16x16x64_i8 v[252:255], v[108:111], v[24:27], v[252:255]\n" \
  "s_cbranch_vccnz LS2_%=\n" \
  "LB2_%=:\n" \
  "v_add_u32 v0, 0xc000, v0\n" \
  "ds_read_b64_tr_b8 v[16:17], v0 offset:0\n" \
  "v_mfma_i32_16x16x64_i8 v[252:255], v[124:127], v[28:31], v[252:255]\n" \
  "ds_read_b64_tr_b8 v[18:19], v0 offset:1024\n" \
  "v_mfma_i32_16x16x64_i8 v[220:223], v[120:123], v[28:31], v[220:223]\n" \
  "ds_read_b64_tr_b8 v[20:21], v0 offset:8192\n" \
  "v_mfma_i32_16x16x64_i8 v[220:223], v[104:107], v[24:27], v[220:223]\n" \
  "ds_read_b64_tr_b8 v[22:23], v0 offset:9216\n" \
  "v_mfma_i32_16x16x64_i8 v[188:191], v[100:103], v[24:27], v[188:191]\n" \
  "ds_add_u32 v12, v11\n" \
  "v_mfma_i32_16x16x64_i8 v[188:191], v[116:119], v[28:31], v[188:191]\n" \
  "v_mfma_i32_16x16x64_i8 v[156:159], v[112:115], v[28:31], v[156:159]\n" \
  "v_mfma_i32_16x16x64_i8 v[156:159], v[96:99], v[24:27], v[156:159]\n" \
  "s_waitcnt lgkmcnt(1)\n" \
  "v_mfma_i32_16x16x64_i8 v[128:131], v[32:35], v[16:19], v[128:131]\n" \
  "v_add_u32 v1, 0xc000, v1\n" \
  "ds_read_b64_tr_b8 v[24:25], v1 offset:0\n" \
  "v_mfma_i32_16x16x64_i8 v[128:131], v[48:51], v[20:23], v[128:131]\n" \
  "ds_read_b64_tr_b8 v[26:27], v1 offset:1024\n" \
  "v_mfma_i32_16x16x64_i8 v[160:163], v[52:55], v[20:23], v[160:163]\n" \
  "ds_read_b64_tr_b8 v[28:29], v1 offset:8192\n" \
  "v_mfma_i32_16x16x64_i8 v[160:163], v[36:39], v[16:19], v[160:163]\n" \
  "ds_read_b64_tr_b8 v[30:31], v1 offset:9216\n" \
  "v_mfma_i32_16x16x64_i8 v[192:195], v[40:43], v[16:19], v[192:195]\n" \
  "global_load_dwordx4 v[96:99], v8, s[40:41] offset:0 sc0 sc1\n" \
  "v_mfma_i32_16x16x64_i8 v[192:195], v[56:59], v[20:23], v[192:195]\n" \
  "global_load_dwordx4 v[100:103], v8, s[40:41] offset:1024 sc0 sc1\n" \
  "v_mfma_i32_16x16x64_i8 v[224:227], v[60:63], v[20:23], v[224:227]\n" \
  "v_mfma_i32_16x16x64_i8 v[224:227], v[44:47], v[16:19], v[224:227]\n" \
  "s_waitcnt lgkmcnt(0)\n" \
  "v_mfma_i32_16x16x64_i8 v[228:231], v[44:47], v[24:27], v[228:231]\n" \
  "v_add_u32 v2, 0xc000, v2\n" \
  "ds_read_b64_tr_b8 v[16:17], v2 offset:0\n" \
  "v_mfma_i32_16x16x64_i8 v[228:231], v[60:63], v[28:31], v[228:231]\n" \
  "ds_read_b64_tr_b8 v[18:19], v2 offset:1024\n" \
  "v_mfma_i32_16x16x64_i8 v[196:199], v[56:59], v[28:31], v[196:199]\n" \
  "ds_read_b64_tr_b8 v[20:21], v2 offset:8192\n" \
  "v_mfma_i32_16x16x64_i8 v[196:199], v[40:43], v[24:27], v[196:199]\n" \
  "ds_read_b64_tr_b8 v[22:23], v2 offset:9216\n" \
  "v_mfma_i32_16x16x64_i8 v[164:167], v[36:39], v[24:27], v[164:167]\n" \
  "global_load_dwordx4 v[104:107], v8, s[40:41] offset:2048 sc0 sc1\n" \
  "v_mfma_i32_16x16x64_i8 v[164:167], v[52:55], v[28:31], v[164:167]\n" \
  "global_load_dwordx4 v[108:111], v8, s[40:41] offset:3072 sc0 sc1\n" \
  "v_mfma_i32_16x16x64_i8 v[132:135], v[48:51], v[28:31], v[132:135]\n" \
  "v_add_u32 v8, 0x8000, v8\n" \
  "v_mfma_i32_16x16x64_i8 v[132:135], v[32:35], v[24:27], v[132:135]\n" \
  "s_waitcnt lgkmcnt(0)\n" \
  "v_mfma_i32_16x16x64_i8 v[136:139], v[32:35], v[16:19], v[136:139]\n" \
  "v_add_u32 v3, 0xc000, v3\n" \
  "ds_read_b64_tr_b8 v[24:25], v3 offset:0\n" \
  "v_mfma_i32_16x16x64_i8 v[136:139], v[48:51], v[20:23], v[136:139]\n" \
  "ds_read_b64_tr_b8 v[26:27], v3 offset:1024\n" \
  "v_mfma_i32_16x16x64_i8 v[168:171], v[52:55], v[20:23], v[168:171]\n" \
  "ds_read_b64_tr_b8 v[28:29], v3 offset:8192\n" \
  "v_mfma_i32_16x16x64_i8 v[168:171], v[36:39], v[16:19], v[168:171]\n" \
  "ds_read_b64_tr_b8 v[30:31], v3 offset:9216\n" \
  "v_mfma_i32_16x16x64_i8 v[200:203], v[40:43], v[16:19], v[200:203]\n" \
  "global_load_dwordx4 v[112:115], v8, s[40:41] offset:0 sc0 sc1\n" \
  "v_mfma_i32_16x16x64_i8 v[200:203], v[56:59], v[20:23], v[200:203]\n" \
  "global_load_dwordx4 v[116:119], v8, s[40:41] offset:1024 sc0 sc1\n" \
  "v_mfma_i32_16x16x64_i8 v[232:235], v[60:63], v[20:23], v[232:235]\n" \
  "v_mfma_i32_16x16x64_i8 v[232:235], v[44:47], v[16:19], v[232:235]\n" \
  "s_waitcnt lgkmcnt(0)\n" \
  "v_mfma_i32_16x16x64_i8 v[236:239], v[44:47], v[24:27], v[236:239]\n" \
  "v_add_u32 v4, 0xc000, v4\n" \
  "ds_read_b64_tr_b8 v[16:17], v4 offset:0\n" \
  "v_mfma_i32_16x16x64_i8 v[236:239], v[60:63], v[28:31], v[236:239]\n" \
  "ds_read_b64_tr_b8 v[18:19], v4 offset:1024\n" \
  "v_mfma_i32_16x16x64_i8 v[204:207], v[56:59], v[28:31], v[204:207]\n" \
  "ds_read_b64_tr_b8 v[20:21], v4 offset:8192\n" \
  "v_mfma_i32_16x16x64_i8 v[204:207], v[40:43], v[24:27], v[204:207]\n" \
  "ds_read_b64_tr_b8 v[22:23], v4 offset:9216\n" \
  "v_mfma_i32_16x16x64_i8 v[172:175], v[36:39], v[24:27], v[172:175]\n" \
  "global_load_dwordx4 v[120:123], v8, s[40:41] offset:2048 sc0 sc1\n" \
  "v_mfma_i32_16x16x64_i8 v[172:175], v[52:55], v[28:31], v[172:175]\n" \
  "global_load_dwordx4 v[124:127], v8, s[40:41] offset:3072 sc0 sc1\n" \
  "v_mfma_i32_16x16x64_i8 v[140:143], v[48:51], v[28:31], v[140:143]\n" \
  "v_add_u32 v8, 0x8000, v8\n" \
  "v_mfma_i32_16x16x64_i8 v[140:143], v[32:35], v[24:27], v[140:143]\n" \
  "s_waitcnt lgkmcnt(0)\n" \
  "v_mfma_i32_16x16x64_i8 v[144:147], v[32:35], v[16:19], v[144:147]\n" \
  "v_add_u32 v5, 0xc000, v5\n" \
  "ds_read_b64_tr_b8 v[24:25], v5 offset:0\n" \
  "v_mfma_i32_16x16x64_i8 v[144:147], v[48:51], v[20:23], v[144:147]\n" \
  "ds_read_b64_tr_b8 v[26:27], v5 offset:1024\n" \
  "v_mfma_i32_16x16x64_i8 v[176:179], v[52:55], v[20:23], v[176:179]\n" \
  "ds_read_b64_tr_b8 v[28:29], v5 offset:8192\n" \
  "v_mfma_i32_16x16x64_i8 v[176:179], v[36:39], v[16:19], v[176:179]\n" \
  "ds_read_b64_tr_b8 v[30:31], v5 offset:9216\n" \
  "v_mfma_i32_16x16x64_i8 v[208:211], v[40:43], v[16:19], v[208:211]\n" \
  "ds_read_b32 v10, v9\n" \
  "v_mfma_i32_16x16x64_i8 v[208:211], v[56:59], v[20:23], v[208:211]\n" \
  "v_mfma_i32_16x16x64_i8 v[240:243], v[60:63], v[20:23], v[240:243]\n" \
  "v_mfma_i32_16x16x64_i8 v[240:243], v[44:47], v[16:19], v[240:243]\n" \
  "s_waitcnt lgkmcnt(1)\n" \
  "v_mfma_i32_16x16x64_i8 v[244:247], v[44:47], v[24:27], v[244:247]\n" \
  "v_add_u32 v6, 0xc000, v6\n" \
  "ds_read_b64_tr_b8 v[16:17], v6 offset:0\n" \
  "v_mfma_i32_16x16x64_i8 v[244:247], v[60:63], v[28:31], v[244:247]\n" \
  "ds_read_b64_tr_b8 v[18:19], v6 offset:1024\n" \
  "v_mfma_i32_16x16x64_i8 v[212:215], v[56:59], v[28:31], v[212:215]\n" \
  "ds_read_b64_tr_b8 v[20:21], v6 offset:8192\n" \
  "v_mfma_i32_16x16x64_i8 v[212:215], v[40:43], v[24:27], v[212:215]\n" \
  "ds_read_b64_tr_b8 v[22:23], v6 offset:9216\n" \
  "v_mfma_i32_16x16x64_i8 v[180:183], v[36:39], v[24:27], v[180:183]\n" \
  "v_mfma_i32_16x16x64_i8 v[180:183], v[52:55], v[28:31], v[180:183]\n" \
  "v_mfma_i32_16x16x64_i8 v[148:151], v[48:51], v[28:31], v[148:151]\n" \
  "v_mfma_i32_16x16x64_i8 v[148:151], v[32:35], v[24:27], v[148:151]\n" \
  "s_waitcnt lgkmcnt(0)\n" \
  "v_mfma_i32_16x16x64_i8 v[152:155], v[32:35], v[16:19], v[152:155]\n" \
  "v_add_u32 v7, 0xc000, v7\n" \
  "ds_read_b64_tr_b8 v[24:25], v7 offset:0\n" \
  "v_mfma_i32_16x16x64_i8 v[152:155], v[48:51], v[20:23], v[152:155]\n" \
  "ds_read_b64_tr_b8 v[26:27], v7 offset:1024\n" \
  "v_mfma_i32_16x16x64_i8 v[184:187], v[52:55], v[20:23], v[184:187]\n" \
  "ds_read_b64_tr_b8 v[28:29], v7 offset:8192\n" \
  "v_mfma_i32_16x16x64_i8 v[184:187], v[36:39], v[16:19], v[184:187]\n" \
  "ds_read_b64_tr_b8 v[30:31], v7 offset:9216\n" \
  "v_mfma_i32_16x16x64_i8 v[216:219], v[40:43], v[16:19], v[216:219]\n" \
  "v_mfma_i32_16x16x64_i8 v[216:219], v[56:59], v[20:23], v[216:219]\n" \
  "v_mfma_i32_16x16x64_i8 v[248:251], v[60:63], v[20:23], v[248:251]\n" \
  "v_cmp_gt_u32 vcc, 10, v10\n" \
  "v_mfma_i32_16x16x64_i8 v[248:251], v[44:47], v[16:19], v[248:251]\n" \
  "s_waitcnt lgkmcnt(0)\n" \
  "v_mfma_i32_16x16x64_i8 v[252:255], v[44:47], v[24:27], v[252:255]\n" \
  "s_cbranch_vccnz LS3_%=\n" \
  "LB3_%=:\n" \
  "ds_read_b64_tr_b8 v[16:17], v0 offset:16384\n" \
  "v_mfma_i32_16x16x64_i8 v[252:255], v[60:63], v[28:31], v[252:255]\n" \
  "ds_read_b64_tr_b8 v[18:19], v0 offset:17408\n" \
  "v_mfma_i32_16x16x64_i8 v[220:223], v[56:59], v[28:31], v[220:223]\n" \
  "ds_read_b64_tr_b8 v[20:21], v0 offset:24576\n" \
  "v_mfma_i32_16x16x64_i8 v[220:223], v[40:43], v[24:27], v[220:223]\n" \
  "ds_read_b64_tr_b8 v[22:23], v0 offset:25600\n" \
  "v_mfma_i32_16x16x64_i8 v[188:191], v[36:39], v[24:27], v[188:191]\n" \
  "ds_add_u32 v12, v11\n" \
  "v_mfma_i32_16x16x64_i8 v[188:191], v[52:55], v[28:31], v[188:191]\n" \
  "v_mfma_i32_16x16x64_i8 v[156:159], v[48:51], v[28:31], v[156:159]\n" \
  "v_mfma_i32_16x16x64_i8 v[156:159], v[32:35], v[24:27], v[156:159]\n" \
  "s_waitcnt vmcnt(8) lgkmcnt(1)\n" \
  "v_mfma_i32_16x16x64_i8 v[128:131], v[64:67], v[16:19], v[128:131]\n" \
  "ds_read_b64_tr_b8 v[24:25], v1 offset:16384\n" \
  "v_mfma_i32_16x16x64_i8 v[128:131], v[80:83], v[20:23], v[128:131]\n" \
  "ds_read_b64_tr_b8 v[26:27], v1 offset:17408\n" \
  "v_mfma_i32_16x16x64_i8 v[160:163], v[84:87], v[20:23], v[160:163]\n" \
  "ds_read_b64_tr_b8 v[28:29], v1 offset:24576\n" \
  "v_mfma_i32_16x16x64_i8 v[160:163], v[68:71], v[16:19], v[160:163]\n" \
  "ds_read_b64_tr_b8 v[30:31], v1 offset:25600\n" \
  "v_mfma_i32_16x16x64_i8 v[192:195], v[72:75], v[16:19], v[192:195]\n" \
  "global_load_dwordx4 v[32:35], v8, s[40:41] offset:0 sc0 sc1\n" \
  "v_mfma_i32_16x16x64_i8 v[192:195], v[88:91], v[20:23], v[192:195]\n" \
  "global_load_dwordx4 v[36:39], v8, s[40:41] offset:1024 sc0 sc1\n" \
  "v_mfma_i32_16x16x64_i8 v[224:227], v[92:95], v[20:23], v[224:227]\n" \
  "v_mfma_i32_16x16x64_i8 v[224:227], v[76:79], v[16:19], v[224:227]\n" \
  "s_waitcnt lgkmcnt(0)\n" \
  "v_mfma_i32_16x16x64_i8 v[228:231], v[76:79], v[24:27], v[228:231]\n" \
  "ds_read_b64_tr_b8 v[16:17], v2 offset:16384\n" \
  "v_mfma_i32_16x16x64_i8 v[228:231], v[92:95], v[28:31], v[228:231]\n" \
  "ds_read_b64_tr_b8 v[18:19], v2 offset:17408\n" \
  "v_mfma_i32_16x16x64_i8 v[196:199], v[88:91], v[28:31], v[196:199]\n" \
  "ds_read_b64_tr_b8 v[20:21], v2 offset:24576\n" \
  "v_mfma_i32_16x16x64_i8 v[196:199], v[72:75], v[24:27], v[196:199]\n" \
  "ds_read_b64_tr_b8 v[22:23], v2 offset:25600\n" \
  "v_mfma_i32_16x16x64_i8 v[164:167], v[68:71], v[24:27], v[164:167]\n" \
  "global_load_dwordx4 v[40:43], v8, s[40:41] offset:2048 sc0 sc1\n" \
  "v_mfma_i32_16x16x64_i8 v[164:167], v[84:87], v[28:31], v[164:167]\n" \
  "global_load_dwordx4 v[44:47], v8, s[40:41] offset:3072 sc0 sc1\n" \
  "v_mfma_i32_16x16x64_i8 v[132:135], v[80:83], v[28:31], v[132:135]\n" \
  "v_add_u32 v8, 0x8000, v8\n" \
  "v_mfma_i32_16x16x64_i8 v[132:135], v[64:67], v[24:27], v[132:135]\n" \
  "s_waitcnt lgkmcnt(0)\n" \
  "v_mfma_i32_16x16x64_i8 v[136:139], v[64:67], v[16:19], v[136:139]\n" \
  "ds_read_b64_tr_b8 v[24:25], v3 offset:16384\n" \
  "v_mfma_i32_16x16x64_i8 v[136:139], v[80:83], v[20:23], v[136:139]\n" \
  "ds_read_b64_tr_b8 v[26:27], v3 offset:17408\n" \
  "v_mfma_i32_16x16x64_i8 v[168:171], v[84:87], v[20:23], v[168:171]\n" \
  "ds_read_b64_tr_b8 v[28:29], v3 offset:24576\n" \
  "v_mfma_i32_16x16x64_i8 v[168:171], v[68:71], v[16:19], v[168:171]\n" \
  "ds_read_b64_tr_b8 v[30:31], v3 offset:25600\n" \
  "v_mfma_i32_16x16x64_i8 v[200:203], v[72:75], v[16:19], v[200:203]\n" \
  "global_load_dwordx4 v[48:51], v8, s[40:41] offset:0 sc0 sc1\n" \
  "v_mfma_i32_16x16x64_i8 v[200:203], v[88:91], v[20:23], v[200:203]\n" \
  "global_load_dwordx4 v[52:55], v8, s[40:41] offset:1024 sc0 sc1\n" \
  "v_mfma_i32_16x16x64_i8 v[232:235], v[92:95], v[20:23], v[232:235]\n" \
  "v_mfma_i32_16x16x64_i8 v[232:235], v[76:79], v[16:19], v[232:235]\n" \
  "s_waitcnt lgkmcnt(0)\n" \
  "v_mfma_i32_16x16x64_i8 v[236:239], v[76:79], v[24:27], v[236:239]\n" \
  "ds_read_b64_tr_b8 v[16:17], v4 offset:16384\n" \
  "v_mfma_i32_16x16x64_i8 v[236:239], v[92:95], v[28:31], v[236:239]\n" \
  "ds_read_b64_tr_b8 v[18:19], v4 offset:17408\n" \
  "v_mfma_i32_16x16x64_i8 v[204:207], v[88:91], v[28:31], v[204:207]\n" \
  "ds_read_b64_tr_b8 v[20:21], v4 offset:24576\n" \
  "v_mfma_i32_16x16x64_i8 v[204:207], v[72:75], v[24:27], v[204:207]\n" \
  "ds_read_b64_tr_b8 v[22:23], v4 offset:25600\n" \
  "v_mfma_i32_16x16x64_i8 v[172:175], v[68:71], v[24:27], v[172:175]\n" \
  "global_load_dwordx4 v[56:59], v8, s[40:41] offset:2048 sc0 sc1\n" \
  "v_mfma_i32_16x16x64_i8 v[172:175], v[84:87], v[28:31], v[172:175]\n" \
  "global_load_dwordx4 v[60:63], v8, s[40:41] offset:3072 sc0 sc1\n" \
  "v_mfma_i32_16x16x64_i8 v[140:143], v[80:83], v[28:31], v[140:143]\n" \
  "v_add_u32 v8, 0x8000, v8\n" \
  "v_mfma_i32_16x16x64_i8 v[140:143], v[64:67], v[24:27], v[140:143]\n" \
  "s_waitcnt lgkmcnt(0)\n" \
  "v_mfma_i32_16x16x64_i8 v[144:147], v[64:67], v[16:19], v[144:147]\n" \
  "ds_read_b64_tr_b8 v[24:25], v5 offset:16384\n" \
  "v_mfma_i32_16x16x64_i8 v[144:147], v[80:83], v[20:23], v[144:147]\n" \
  "ds_read_b64_tr_b8 v[26:27], v5 offset:17408\n" \
  "v_mfma_i32_16x16x64_i8 v[176:179], v[84:87], v[20:23], v[176:179]\n" \
  "ds_read_b64_tr_b8 v[28:29], v5 offset:24576\n" \
  "v_mfma_i32_16x16x64_i8 v[176:179], v[68:71], v[16:19], v[176:179]\n" \
  "ds_read_b64_tr_b8 v[30:31], v5 offset:25600\n" \
  "v_mfma_i32_16x16x64_i8 v[208:211], v[72:75], v[16:19], v[208:211]\n" \
  "ds_read_b32 v10, v9\n" \
  "v_mfma_i32_16x16x64_i8 v[208:211], v[88:91], v[20:23], v[208:211]\n" \
  "v_mfma_i32_16x16x64_i8 v[240:243], v[92:95], v[20:23], v[240:243]\n" \
  "v_mfma_i32_16x16x64_i8 v[240:243], v[76:79], v[16:19], v[240:243]\n" \
  "s_waitcnt lgkmcnt(1)\n" \
  "v_mfma_i32_16x16x64_i8 v[244:247], v[76:79], v[24:27], v[244:247]\n" \
  "ds_read_b64_tr_b8 v[16:17], v6 offset:16384\n" \
  "v_mfma_i32_16x16x64_i8 v[244:247], v[92:95], v[28:31], v[244:247]\n" \
  "ds_read_b64_tr_b8 v[18:19], v6 offset:17408\n" \
  "v_mfma_i32_16x16x64_i8 v[212:215], v[88:91], v[28:31], v[212:215]\n" \
  "ds_read_b64_tr_b8 v[20:21], v6 offset:24576\n" \
  "v_mfma_i32_16x16x64_i8 v[212:215], v[72:75], v[24:27], v[212:215]\n" \
  "ds_read_b64_tr_b8 v[22:23], v6 offset:25600\n" \
  "v_mfma_i32_16x16x64_i8 v[180:183], v[68:71], v[24:27], v[180:183]\n" \
  "v_mfma_i32_16x16x64_i8 v[180:183], v[84:87], v[28:31], v[180:183]\n" \
  "v_mfma_i32_16x16x64_i8 v[148:151], v[80:83], v[28:31], v[148:151]\n" \
  "v_mfma_i32_16x16x64_i8 v[148:151], v[64:67], v[24:27], v[148:151]\n" \
  "s_waitcnt lgkmcnt(0)\n" \
  "v_mfma_i32_16x16x64_i8 v[152:155], v[64:67], v[16:19], v[152:155]\n" \
  "ds_read_b64_tr_b8 v[24:25], v7 offset:16384\n" \
  "v_mfma_i32_16x16x64_i8 v[152:155], v[80:83], v[20:23], v[152:155]\n" \
  "ds_read_b64_tr_b8 v[26:27], v7 offset:17408\n" \
  "v_mfma_i32_16x16x64_i8 v[184:187], v[84:87], v[20:23], v[184:187]\n" \
  "ds_read_b64_tr_b8 v[28:29], v7 offset:24576\n" \
  "v_mfma_i32_16x16x64_i8 v[184:187], v[68:71], v[16:19], v[184:187]\n" \
  "ds_read_b64_tr_b8 v[30:31], v7 offset:25600\n" \
  "v_mfma_i32_16x16x64_i8 v[216:219], v[72:75], v[16:19], v[216:219]\n" \
  "v_mfma_i32_16x16x64_i8 v[216:219], v[88:91], v[20:23], v[216:219]\n" \
  "v_mfma_i32_16x16x64_i8 v[248:251], v[92:95], v[20:23], v[248:251]\n" \
  "v_cmp_gt_u32 vcc, 12, v10\n" \
  "v_mfma_i32_16x16x64_i8 v[248:251], v[76:79], v[16:19], v[248:251]\n" \
  "s_waitcnt lgkmcnt(0)\n" \
  "v_mfma_i32_16x16x64_i8 v[252:255], v[76:79], v[24:27], v[252:255]\n" \
  "s_cbranch_vccnz LS4_%=\n" \
  "LB4_%=:\n" \
  "ds_read_b64_tr_b8 v[16:17], v0 offset:32768\n" \
  "v_mfma_i32_16x16x64_i8 v[252:255], v[92:95], v[28:31], v[252:255]\n" \
  "ds_read_b64_tr_b8 v[18:19], v0 offset:33792\n" \
  "v_mfma_i32_16x16x64_i8 v[220:223], v[88:91], v[28:31], v[220:223]\n" \
  "ds_read_b64_tr_b8 v[20:21], v0 offset:40960\n" \
  "v_mfma_i32_16x16x64_i8 v[220:223], v[72:75], v[24:27], v[220:223]\n" \
  "ds_read_b64_tr_b8 v[22:23], v0 offset:41984\n" \
  "v_mfma_i32_16x16x64_i8 v[188:191], v[68:71], v[24:27], v[188:191]\n" \
  "ds_add_u32 v12, v11\n" \
  "v_mfma_i32_16x16x64_i8 v[188:191], v[84:87], v[28:31], v[188:191]\n" \
  "v_mfma_i32_16x16x64_i8 v[156:159], v[80:83], v[28:31], v[156:159]\n" \
  "v_mfma_i32_16x16x64_i8 v[156:159], v[64:67], v[24:27], v[156:159]\n" \
  "s_waitcnt vmcnt(8) lgkmcnt(1)\n" \
  "v_mfma_i32_16x16x64_i8 v[128:131], v[96:99], v[16:19], v[128:131]\n" \
  "ds_read_b64_tr_b8 v[24:25], v1 offset:32768\n" \
  "v_mfma_i32_16x16x64_i8 v[128:131], v[112:115], v[20:23], v[128:131]\n" \
  "ds_read_b64_tr_b8 v[26:27], v1 offset:33792\n" \
  "v_mfma_i32_16x16x64_i8 v[160:163], v[116:119], v[20:23], v[160:163]\n" \
  "ds_read_b64_tr_b8 v[28:29], v1 offset:40960\n" \
  "v_mfma_i32_16x16x64_i8 v[160:163], v[100:103], v[16:19], v[160:163]\n" \
  "ds_read_b64_tr_b8 v[30:31], v1 offset:41984\n" \
  "v_mfma_i32_16x16x64_i8 v[192:195], v[104:107], v[16:19], v[192:195]\n" \
  "global_load_dwordx4 v[64:67], v8, s[40:41] offset:0 sc0 sc1\n" \
  "v_mfma_i32_16x16x64_i8 v[192:195], v[120:123], v[20:23], v[192:195]\n" \
  "global_load_dwordx4 v[68:71], v8, s[40:41] offset:1024 sc0 sc1\n" \
  "v_mfma_i32_16x16x64_i8 v[224:227], v[124:127], v[20:23], v[224:227]\n" \
  "v_mfma_i32_16x16x64_i8 v[224:227], v[108:111], v[16:19], v[224:227]\n" \
  "s_waitcnt lgkmcnt(0)\n" \
  "v_mfma_i32_16x16x64_i8 v[228:231], v[108:111], v[24:27], v[228:231]\n" \
  "ds_read_b64_tr_b8 v[16:17], v2 offset:32768\n" \
  "v_mfma_i32_16x16x64_i8 v[228:231], v[124:127], v[28:31], v[228:231]\n" \
  "ds_read_b64_tr_b8 v[18:19], v2 offset:33792\n" \
  "v_mfma_i32_16x16x64_i8 v[196:199], v[120:123], v[28:31], v[196:199]\n" \
  "ds_read_b64_tr_b8 v[20:21], v2 offset:40960\n" \
  "v_mfma_i32_16x16x64_i8 v[196:199], v[104:107], v[24:27], v[196:199]\n" \
  "ds_read_b64_tr_b8 v[22:23], v2 offset:41984\n" \
  "v_mfma_i32_16x16x64_i8 v[164:167], v[100:103], v[24:27], v[164:167]\n" \
  "global_load_dwordx4 v[72:75], v8, s[40:41] offset:2048 sc0 sc1\n" \
  "v_mfma_i32_16x16x64_i8 v[164:167], v[116:119], v[28:31], v[164:167]\n" \
  "global_load_dwordx4 v[76:79], v8, s[40:41] offset:3072 sc0 sc1\n" \
  "v_mfma_i32_16x16x64_i8 v[132:135], v[112:115], v[28:31], v[132:135]\n" \
  "v_add_u32 v8, 0x8000, v8\n" \
  "v_mfma_i32_16x16x64_i8 v[132:135], v[96:99], v[24:27], v[132:135]\n" \
  "s_waitcnt lgkmcnt(0)\n" \
  "v_mfma_i32_16x16x64_i8 v[136:139], v[96:99], v[16:19], v[136:139]\n" \
  "ds_read_b64_tr_b8 v[24:25], v3 offset:32768\n" \
  "v_mfma_i32_16x16x64_i8 v[136:139], v[112:115], v[20:23], v[136:139]\n" \
  "ds_read_b64_tr_b8 v[26:27], v3 offset:33792\n" \
  "v_mfma_i32_16x16x64_i8 v[168:171], v[116:119], v[20:23], v[168:171]\n" \
  "ds_read_b64_tr_b8 v[28:29], v3 offset:40960\n" \
  "v_mfma_i32_16x16x64_i8 v[168:171], v[100:103], v[16:19], v[168:171]\n" \
  "ds_read_b64_tr_b8 v[30:31], v3 offset:41984\n" \
  "v_mfma_i32_16x16x64_i8 v[200:203], v[104:107], v[16:19], v[200:203]\n" \
  "global_load_dwordx4 v[80:83], v8, s[40:41] offset:0 sc0 sc1\n" \
  "v_mfma_i32_16x16x64_i8 v[200:203], v[120:123], v[20:23], v[200:203]\n" \
  "global_load_dwordx4 v[84:87], v8, s[40:41] offset:1024 sc0 sc1\n" \
  "v_mfma_i32_16x16x64_i8 v[232:235], v[124:127], v[20:23], v[232:235]\n" \
  "v_mfma_i32_16x16x64_i8 v[232:235], v[108:111], v[16:19], v[232:235]\n" \
  "s_waitcnt lgkmcnt(0)\n" \
  "v_mfma_i32_16x16x64_i8 v[236:239], v[108:111], v[24:27], v[236:239]\n" \
  "ds_read_b64_tr_b8 v[16:17], v4 offset:32768\n" \
  "v_mfma_i32_16x16x64_i8 v[236:239], v[124:127], v[28:31], v[236:239]\n" \
  "ds_read_b64_tr_b8 v[18:19], v4 offset:33792\n" \
  "v_mfma_i32_16x16x64_i8 v[204:207], v[120:123], v[28:31], v[204:207]\n" \
  "ds_read_b64_tr_b8 v[20:21], v4 offset:40960\n" \
  "v_mfma_i32_16x16x64_i8 v[204:207], v[104:107], v[24:27], v[204:207]\n" \
  "ds_read_b64_tr_b8 v[22:23], v4 offset:41984\n" \
  "v_mfma_i32_16x16x64_i8 v[172:175], v[100:103], v[24:27], v[172:175]\n" \
  "global_load_dwordx4 v[88:91], v8, s[40:41] offset:2048 sc0 sc1\n" \
  "v_mfma_i32_16x16x64_i8 v[172:175], v[116:119], v[28:31], v[172:175]\n" \
  "global_load_dwordx4 v[92:95], v8, s[40:41] offset:3072 sc0 sc1\n" \
  "v_mfma_i32_16x16x64_i8 v[140:143], v[112:115], v[28:31], v[140:143]\n" \
  "v_add_u32 v8, 0x8000, v8\n" \
  "v_mfma_i32_16x16x64_i8 v[140:143], v[96:99], v[24:27], v[140:143]\n" \
  "s_waitcnt lgkmcnt(0)\n" \
  "v_mfma_i32_16x16x64_i8 v[144:147], v[96:99], v[16:19], v[144:147]\n" \
  "ds_read_b64_tr_b8 v[24:25], v5 offset:32768\n" \
  "v_mfma_i32_16x16x64_i8 v[144:147], v[112:115], v[20:23], v[144:147]\n" \
  "ds_read_b64_tr_b8 v[26:27], v5 offset:33792\n" \
  "v_mfma_i32_16x16x64_i8 v[176:179], v[116:119], v[20:23], v[176:179]\n" \
  "ds_read_b64_tr_b8 v[28:29], v5 offset:40960\n" \
  "v_mfma_i32_16x16x64_i8 v[176:179], v[100:103], v[16:19], v[176:179]\n" \
  "ds_read_b64_tr_b8 v[30:31], v5 offset:41984\n" \
  "v_mfma_i32_16x16x64_i8 v[208:211], v[104:107], v[16:19], v[208:211]\n" \
  "ds_read_b32 v10, v9\n" \
  "v_mfma_i32_16x16x64_i8 v[208:211], v[120:123], v[20:23], v[208:211]\n" \
  "v_mfma_i32_16x16x64_i8 v[240:243], v[124:127], v[20:23], v[240:243]\n" \
  "v_mfma_i32_16x16x64_i8 v[240:243], v[108:111], v[16:19], v[240:243]\n" \
  "s_waitcnt lgkmcnt(1)\n" \
  "v_mfma_i32_16x16x64_i8 v[244:247], v[108:111], v[24:27], v[244:247]\n" \
  "ds_read_b64_tr_b8 v[16:17], v6 offset:32768\n" \
  "v_mfma_i32_16x16x64_i8 v[244:247], v[124:127], v[28:31], v[244:247]\n" \
  "ds_read_b64_tr_b8 v[18:19], v6 offset:33792\n" \
  "v_mfma_i32_16x16x64_i8 v[212:215], v[120:123], v[28:31], v[212:215]\n" \
  "ds_read_b64_tr_b8 v[20:21], v6 offset:40960\n" \
  "v_mfma_i32_16x16x64_i8 v[212:215], v[104:107], v[24:27], v[212:215]\n" \
  "ds_read_b64_tr_b8 v[22:23], v6 offset:41984\n" \
  "v_mfma_i32_16x16x64_i8 v[180:183], v[100:103], v[24:27], v[180:183]\n" \
  "v_mfma_i32_16x16x64_i8 v[180:183], v[116:119], v[28:31], v[180:183]\n" \
  "v_mfma_i32_16x16x64_i8 v[148:151], v[112:115], v[28:31], v[148:151]\n" \
  "v_mfma_i32_16x16x64_i8 v[148:151], v[96:99], v[24:27], v[148:151]\n" \
  "s_waitcnt lgkmcnt(0)\n" \
  "v_mfma_i32_16x16x64_i8 v[152:155], v[96:99], v[16:19], v[152:155]\n" \
  "ds_read_b64_tr_b8 v[24:25], v7 offset:32768\n" \
  "v_mfma_i32_16x16x64_i8 v[152:155], v[112:115], v[20:23], v[152:155]\n" \
  "ds_read_b64_tr_b8 v[26:27], v7 offset:33792\n" \
  "v_mfma_i32_16x16x64_i8 v[184:187], v[116:119], v[20:23], v[184:187]\n" \
  "ds_read_b64_tr_b8 v[28:29], v7 offset:40960\n" \
  "v_mfma_i32_16x16x64_i8 v[184:187], v[100:103], v[16:19], v[184:187]\n" \
  "ds_read_b64_tr_b8 v[30:31], v7 offset:41984\n" \
  "v_mfma_i32_16x16x64_i8 v[216:219], v[104:107], v[16:19], v[216:219]\n" \
  "v_mfma_i32_16x16x64_i8 v[216:219], v[120:123], v[20:23], v[216:219]\n" \
  "v_mfma_i32_16x16x64_i8 v[248:251], v[124:127], v[20:23], v[248:251]\n" \
  "v_cmp_gt_u32 vcc, 14, v10\n" \
  "v_mfma_i32_16x16x64_i8 v[248:251], v[108:111], v[16:19], v[248:251]\n" \
  "s_waitcnt lgkmcnt(0)\n" \
  "v_mfma_i32_16x16x64_i8 v[252:255], v[108:111], v[24:27], v[252:255]\n" \
  "s_cbranch_vccnz LS5_%=\n" \
  "LB5_%=:\n" \
  "v_add_u32 v0, 0xc000, v0\n" \
  "ds_read_b64_tr_b8 v[16:17], v0 offset:0\n" \
  "v_mfma_i32_16x16x64_i8 v[252:255], v[124:127], v[28:31], v[252:255]\n" \
  "ds_read_b64_tr_b8 v[18:19], v0 offset:1024\n" \
  "v_mfma_i32_16x16x64_i8 v[220:223], v[120:123], v[28:31], v[220:223]\n" \
  "ds_read_b64_tr_b8 v[20:21], v0 offset:8192\n" \
  "v_mfma_i32_16x16x64_i8 v[220:223], v[104:107], v[24:27], v[220:223]\n" \
  "ds_read_b64_tr_b8 v[22:23], v0 offset:9216\n" \
  "v_mfma_i32_16x16x64_i8 v[188:191], v[100:103], v[24:27], v[188:191]\n" \
  "ds_add_u32 v12, v11\n" \
  "v_mfma_i32_16x16x64_i8 v[188:191], v[116:119], v[28:31], v[188:191]\n" \
  "v_mfma_i32_16x16x64_i8 v[156:159], v[112:115], v[28:31], v[156:159]\n" \
  "v_mfma_i32_16x16x64_i8 v[156:159], v[96:99], v[24:27], v[156:159]\n" \
  "s_waitcnt vmcnt(8) lgkmcnt(1)\n" \
  "v_mfma_i32_16x16x64_i8 v[128:131], v[32:35], v[16:19], v[128:131]\n" \
  "v_add_u32 v1, 0xc000, v1\n" \
  "ds_read_b64_tr_b8 v[24:25], v1 offset:0\n" \
  "v_mfma_i32_16x16x64_i8 v[128:131], v[48:51], v[20:23], v[128:131]\n" \
  "ds_read_b64_tr_b8 v[26:27], v1 offset:1024\n" \
  "v_mfma_i32_16x16x64_i8 v[160:163], v[52:55], v[20:23], v[160:163]\n" \
  "ds_read_b64_tr_b8 v[28:29], v1 offset:8192\n" \
  "v_mfma_i32_16x16x64_i8 v[160:163], v[36:39], v[16:19], v[160:163]\n" \
  "ds_read_b64_tr_b8 v[30:31], v1 offset:9216\n" \
  "v_mfma_i32_16x16x64_i8 v[192:195], v[40:43], v[16:19], v[192:195]\n" \
  "global_load_dwordx4 v[96:99], v8, s[40:41] offset:0 sc0 sc1\n" \
  "v_mfma_i32_16x16x64_i8 v[192:195], v[56:59], v[20:23], v[192:195]\n" \
  "global_load_dwordx4 v[100:103], v8, s[40:41] offset:1024 sc0 sc1\n" \
  "v_mfma_i32_16x16x64_i8 v[224:227], v[60:63], v[20:23], v[224:227]\n" \
  "v_mfma_i32_16x16x64_i8 v[224:227], v[44:47], v[16:19], v[224:227]\n" \
  "s_waitcnt lgkmcnt(0)\n" \
  "v_mfma_i32_16x16x64_i8 v[228:231], v[44:47], v[24:27], v[228:231]\n" \
  "v_add_u32 v2, 0xc000, v2\n" \
  "ds_read_b64_tr_b8 v[16:17], v2 offset:0\n" \
  "v_mfma_i32_16x16x64_i8 v[228:231], v[60:63], v[28:31], v[228:231]\n" \
  "ds_read_b64_tr_b8 v[18:19], v2 offset:1024\n" \
  "v_mfma_i32_16x16x64_i8 v[196:199], v[56:59], v[28:31], v[196:199]\n" \
  "ds_read_b64_tr_b8 v[20:21], v2 offset:8192\n" \
  "v_mfma_i32_16x16x64_i8 v[196:199], v[40:43], v[24:27], v[196:199]\n" \
  "ds_read_b64_tr_b8 v[22:23], v2 offset:9216\n" \
  "v_mfma_i32_16x16x64_i8 v[164:167], v[36:39], v[24:27], v[164:167]\n" \
  "global_load_dwordx4 v[104:107], v8, s[40:41] offset:2048 sc0 sc1\n" \
  "v_mfma_i32_16x16x64_i8 v[164:167], v[52:55], v[28:31], v[164:167]\n" \
  "global_load_dwordx4 v[108:111], v8, s[40:41] offset:3072 sc0 sc1\n" \
  "v_mfma_i32_16x16x64_i8 v[132:135], v[48:51], v[28:31], v[132:135]\n" \
  "v_add_u32 v8, 0x8000, v8\n" \
  "v_mfma_i32_16x16x64_i8 v[132:135], v[32:35], v[24:27], v[132:135]\n" \
  "s_waitcnt lgkmcnt(0)\n" \
  "v_mfma_i32_16x16x64_i8 v[136:139], v[32:35], v[16:19], v[136:139]\n" \
  "v_add_u32 v3, 0xc000, v3\n" \
  "ds_read_b64_tr_b8 v[24:25], v3 offset:0\n" \
  "v_mfma_i32_16x16x64_i8 v[136:139], v[48:51], v[20:23], v[136:139]\n" \
  "ds_read_b64_tr_b8 v[26:27], v3 offset:1024\n" \
  "v_mfma_i32_16x16x64_i8 v[168:171], v[52:55], v[20:23], v[168:171]\n" \
  "ds_read_b64_tr_b8 v[28:29], v3 offset:8192\n" \
  "v_mfma_i32_16x16x64_i8 v[168:171], v[36:39], v[16:19], v[168:171]\n" \
  "ds_read_b64_tr_b8 v[30:31], v3 offset:9216\n" \
  "v_mfma_i32_16x16x64_i8 v[200:203], v[40:43], v[16:19], v[200:203]\n" \
  "global_load_dwordx4 v[112:115], v8, s[40:41] offset:0 sc0 sc1\n" \
  "v_mfma_i32_16x16x64_i8 v[200:203], v[56:59], v[20:23], v[200:203]\n" \
  "global_load_dwordx4 v[116:119], v8, s[40:41] offset:1024 sc0 sc1\n" \
  "v_mfma_i32_16x16x64_i8 v[232:235], v[60:63], v[20:23], v[232:235]\n" \
  "v_mfma_i32_16x16x64_i8 v[232:235], v[44:47], v[16:19], v[232:235]\n" \
  "s_waitcnt lgkmcnt(0)\n" \
  "v_mfma_i32_16x16x64_i8 v[236:239], v[44:47], v[24:27], v[236:239]\n" \
  "v_add_u32 v4, 0xc000, v4\n" \
  "ds_read_b64_tr_b8 v[16:17], v4 offset:0\n" \
  "v_mfma_i32_16x16x64_i8 v[236:239], v[60:63], v[28:31], v[236:239]\n" \
  "ds_read_b64_tr_b8 v[18:19], v4 offset:1024\n" \
  "v_mfma_i32_16x16x64_i8 v[204:207], v[56:59], v[28:31], v[204:207]\n" \
  "ds_read_b64_tr_b8 v[20:21], v4 offset:8192\n" \
  "v_mfma_i32_16x16x64_i8 v[204:207], v[40:43], v[24:27], v[204:207]\n" \
  "ds_read_b64_tr_b8 v[22:23], v4 offset:9216\n" \
  "v_mfma_i32_16x16x64_i8 v[172:175], v[36:39], v[24:27], v[172:175]\n" \
  "global_load_dwordx4 v[120:123], v8, s[40:41] offset:2048 sc0 sc1\n" \
  "v_mfma_i32_16x16x64_i8 v[172:175], v[52:55], v[28:31], v[172:175]\n" \
  "global_load_dwordx4 v[124:127], v8, s[40:41] offset:3072 sc0 sc1\n" \
  "v_mfma_i32_16x16x64_i8 v[140:143], v[48:51], v[28:31], v[140:143]\n" \
  "v_add_u32 v8, 0x8000, v8\n" \
  "v_mfma_i32_16x16x64_i8 v[140:143], v[32:35], v[24:27], v[140:143]\n" \
  "s_waitcnt lgkmcnt(0)\n" \
  "v_mfma_i32_16x16x64_i8 v[144:147], v[32:35], v[16:19], v[144:147]\n" \
  "v_add_u32 v5, 0xc000, v5\n" \
  "ds_read_b64_tr_b8 v[24:25], v5 offset:0\n" \
  "v_mfma_i32_16x16x64_i8 v[144:147], v[48:51], v[20:23], v[144:147]\n" \
  "ds_read_b64_tr_b8 v[26:27], v5 offset:1024\n" \
  "v_mfma_i32_16x16x64_i8 v[176:179], v[52:55], v[20:23], v[176:179]\n" \
  "ds_read_b64_tr_b8 v[28:29], v5 offset:8192\n" \
  "v_mfma_i32_16x16x64_i8 v[176:179], v[36:39], v[16:19], v[176:179]\n" \
  "ds_read_b64_tr_b8 v[30:31], v5 offset:9216\n" \
  "v_mfma_i32_16x16x64_i8 v[208:211], v[40:43], v[16:19], v[208:211]\n" \
  "ds_read_b32 v10, v9\n" \
  "v_mfma_i32_16x16x64_i8 v[208:211], v[56:59], v[20:23], v[208:211]\n" \
  "v_mfma_i32_16x16x64_i8 v[240:243], v[60:63], v[20:23], v[240:243]\n" \
  "v_mfma_i32_16x16x64_i8 v[240:243], v[44:47], v[16:19], v[240:243]\n" \
  "s_waitcnt lgkmcnt(1)\n" \
  "v_mfma_i32_16x16x64_i8 v[244:247], v[44:47], v[24:27], v[244:247]\n" \
  "v_add_u32 v6, 0xc000, v6\n" \
  "ds_read_b64_tr_b8 v[16:17], v6 offset:0\n" \
  "v_mfma_i32_16x16x64_i8 v[244:247], v[60:63], v[28:31], v[244:247]\n" \
  "ds_read_b64_tr_b8 v[18:19], v6 offset:1024\n" \
  "v_mfma_i32_16x16x64_i8 v[212:215], v[56:59], v[28:31], v[212:215]\n" \
  "ds_read_b64_tr_b8 v[20:21], v6 offset:8192\n" \
  "v_mfma_i32_16x16x64_i8 v[212:215], v[40:43], v[24:27], v[212:215]\n" \
  "ds_read_b64_tr_b8 v[22:23], v6 offset:9216\n" \
  "v_mfma_i32_16x16x64_i8 v[180:183], v[36:39], v[24:27], v[180:183]\n" \
  "v_mfma_i32_16x16x64_i8 v[180:183], v[52:55], v[28:31], v[180:183]\n" \
  "v_mfma_i32_16x16x64_i8 v[148:151], v[48:51], v[28:31], v[148:151]\n" \
  "v_mfma_i32_16x16x64_i8 v[148:151], v[32:35], v[24:27], v[148:151]\n" \
  "s_waitcnt lgkmcnt(0)\n" \
  "v_mfma_i32_16x16x64_i8 v[152:155], v[32:35], v[16:19], v[152:155]\n" \
  "v_add_u32 v7, 0xc000, v7\n" \
  "ds_read_b64_tr_b8 v[24:25], v7 offset:0\n" \
  "v_mfma_i32_16x16x64_i8 v[152:155], v[48:51], v[20:23], v[152:155]\n" \
  "ds_read_b64_tr_b8 v[26:27], v7 offset:1024\n" \
  "v_mfma_i32_16x16x64_i8 v[184:187], v[52:55], v[20:23], v[184:187]\n" \
  "ds_read_b64_tr_b8 v[28:29], v7 offset:8192\n" \
  "v_mfma_i32_16x16x64_i8 v[184:187], v[36:39], v[16:19], v[184:187]\n" \
  "ds_read_b64_tr_b8 v[30:31], v7 offset:9216\n" \
  "v_mfma_i32_16x16x64_i8 v[216:219], v[40:43], v[16:19], v[216:219]\n" \
  "v_mfma_i32_16x16x64_i8 v[216:219], v[56:59], v[20:23], v[216:219]\n" \
  "v_mfma_i32_16x16x64_i8 v[248:251], v[60:63], v[20:23], v[248:251]\n" \
  "v_cmp_gt_u32 vcc, 16, v10\n" \
  "v_mfma_i32_16x16x64_i8 v[248:251], v[44:47], v[16:19], v[248:251]\n" \
  "s_waitcnt lgkmcnt(0)\n" \
  "v_mfma_i32_16x16x64_i8 v[252:255], v[44:47], v[24:27], v[252:255]\n" \
  "s_cbranch_vccnz LS6_%=\n" \
  "LB6_%=:\n" \
  "ds_read_b64_tr_b8 v[16:17], v0 offset:16384\n" \
  "v_mfma_i32_16x16x64_i8 v[252:255], v[60:63], v[28:31], v[252:255]\n" \
  "ds_read_b64_tr_b8 v[18:19], v0 offset:17408\n" \
  "v_mfma_i32_16x16x64_i8 v[220:223], v[56:59], v[28:31], v[220:223]\n" \
  "ds_read_b64_tr_b8 v[20:21], v0 offset:24576\n" \
  "v_mfma_i32_16x16x64_i8 v[220:223], v[40:43], v[24:27], v[220:223]\n" \
  "ds_read_b64_tr_b8 v[22:23], v0 offset:25600\n" \
  "v_mfma_i32_16x16x64_i8 v[188:191], v[36:39], v[24:27], v[188:191]\n" \
  "ds_add_u32 v12, v11\n" \
  "v_mfma_i32_16x16x64_i8 v[188:191], v[52:55], v[28:31], v[188:191]\n" \
  "v_mfma_i32_16x16x64_i8 v[156:159], v[48:51], v[28:31], v[156:159]\n" \
  "v_mfma_i32_16x16x64_i8 v[156:159], v[32:35], v[24:27], v[156:159]\n" \
  "s_waitcnt vmcnt(8) lgkmcnt(1)\n" \
  "v_mfma_i32_16x16x64_i8 v[128:131], v[64:67], v[16:19], v[128:131]\n" \
  "ds_read_b64_tr_b8 v[24:25], v1 offset:16384\n" \
  "v_mfma_i32_16x16x64_i8 v[128:131], v[80:83], v[20:23], v[128:131]\n" \
  "ds_read_b64_tr_b8 v[26:27], v1 offset:17408\n" \
  "v_mfma_i32_16x16x64_i8 v[160:163], v[84:87], v[20:23], v[160:163]\n" \
  "ds_read_b64_tr_b8 v[28:29], v1 offset:24576\n" \
  "v_mfma_i32_16x16x64_i8 v[160:163], v[68:71], v[16:19], v[160:163]\n" \
  "ds_read_b64_tr_b8 v[30:31], v1 offset:25600\n" \
  "v_mfma_i32_16x16x64_i8 v[192:195], v[72:75], v[16:19], v[192:195]\n" \
  "global_load_dwordx4 v[32:35], v8, s[40:41] offset:0 sc0 sc1\n" \
  "v_mfma_i32_16x16x64_i8 v[192:195], v[88:91], v[20:23], v[192:195]\n" \
  "global_load_dwordx4 v[36:39], v8, s[40:41] offset:1024 sc0 sc1\n" \
  "v_mfma_i32_16x16x64_i8 v[224:227], v[92:95], v[20:23], v[224:227]\n" \
  "v_mfma_i32_16x16x64_i8 v[224:227], v[76:79], v[16:19], v[224:227]\n" \
  "s_waitcnt lgkmcnt(0)\n" \
  "v_mfma_i32_16x16x64_i8 v[228:231], v[76:79], v[24:27], v[228:231]\n" \
  "ds_read_b64_tr_b8 v[16:17], v2 offset:16384\n" \
  "v_mfma_i32_16x16x64_i8 v[228:231], v[92:95], v[28:31], v[228:231]\n" \
  "ds_read_b64_tr_b8 v[18:19], v2 offset:17408\n" \
  "v_mfma_i32_16x16x64_i8 v[196:199], v[88:91], v[28:31], v[196:199]\n" \
  "ds_read_b64_tr_b8 v[20:21], v2 offset:24576\n" \
  "v_mfma_i32_16x16x64_i8 v[196:199], v[72:75], v[24:27], v[196:199]\n" \
  "ds_read_b64_tr_b8 v[22:23], v2 offset:25600\n" \
  "v_mfma_i32_16x16x64_i8 v[164:167], v[68:71], v[24:27], v[164:167]\n" \
  "global_load_dwordx4 v[40:43], v8, s[40:41] offset:2048 sc0 sc1\n" \
  "v_mfma_i32_16x16x64_i8 v[164:167], v[84:87], v[28:31], v[164:167]\n" \
  "global_load_dwordx4 v[44:47], v8, s[40:41] offset:3072 sc0 sc1\n" \
  "v_mfma_i32_16x16x64_i8 v[132:135], v[80:83], v[28:31], v[132:135]\n" \
  "v_add_u32 v8, 0x8000, v8\n" \
  "v_mfma_i32_16x16x64_i8 v[132:135], v[64:67], v[24:27], v[132:135]\n" \
  "s_waitcnt lgkmcnt(0)\n" \
  "v_mfma_i32_16x16x64_i8 v[136:139], v[64:67], v[16:19], v[136:139]\n" \
  "ds_read_b64_tr_b8 v[24:25], v3 offset:16384\n" \
  "v_mfma_i32_16x16x64_i8 v[136:139], v[80:83], v[20:23], v[136:139]\n" \
  "ds_read_b64_tr_b8 v[26:27], v3 offset:17408\n" \
  "v_mfma_i32_16x16x64_i8 v[168:171], v[84:87], v[20:23], v[168:171]\n" \
  "ds_read_b64_tr_b8 v[28:29], v3 offset:24576\n" \
  "v_mfma_i32_16x16x64_i8 v[168:171], v[68:71], v[16:19], v[168:171]\n" \
  "ds_read_b64_tr_b8 v[30:31], v3 offset:25600\n" \
  "v_mfma_i32_16x16x64_i8 v[200:203], v[72:75], v[16:19], v[200:203]\n" \
  "global_load_dwordx4 v[48:51], v8, s[40:41] offset:0 sc0 sc1\n" \
  "v_mfma_i32_16x16x64_i8 v[200:203], v[88:91], v[20:23], v[200:203]\n" \
  "global_load_dwordx4 v[52:55], v8, s[40:41] offset:1024 sc0 sc1\n" \
  "v_mfma_i32_16x16x64_i8 v[232:235], v[92:95], v[20:23], v[232:235]\n" \
  "v_mfma_i32_16x16x64_i8 v[232:235], v[76:79], v[16:19], v[232:235]\n" \
  "s_waitcnt lgkmcnt(0)\n" \
  "v_mfma_i32_16x16x64_i8 v[236:239], v[76:79], v[24:27], v[236:239]\n" \
  "ds_read_b64_tr_b8 v[16:17], v4 offset:16384\n" \
  "v_mfma_i32_16x16x64_i8 v[236:239], v[92:95], v[28:31], v[236:239]\n" \
  "ds_read_b64_tr_b8 v[18:19], v4 offset:17408\n" \
  "v_mfma_i32_16x16x64_i8 v[204:207], v[88:91], v[28:31], v[204:207]\n" \
  "ds_read_b64_tr_b8 v[20:21], v4 offset:24576\n" \
  "v_mfma_i32_16x16x64_i8 v[204:207], v[72:75], v[24:27], v[204:207]\n" \
  "ds_read_b64_tr_b8 v[22:23], v4 offset:25600\n" \
  "v_mfma_i32_16x16x64_i8 v[172:175], v[68:71], v[24:27], v[172:175]\n" \
  "global_load_dwordx4 v[56:59], v8, s[40:41] offset:2048 sc0 sc1\n" \
  "v_mfma_i32_16x16x64_i8 v[172:175], v[84:87], v[28:31], v[172:175]\n" \
  "global_load_dwordx4 v[60:63], v8, s[40:41] offset:3072 sc0 sc1\n" \
  "v_mfma_i32_16x16x64_i8 v[140:143], v[80:83], v[28:31], v[140:143]\n" \
  "v_add_u32 v8, 0x8000, v8\n" \
  "v_mfma_i32_16x16x64_i8 v[140:143], v[64:67], v[24:27], v[140:143]\n" \
  "s_waitcnt lgkmcnt(0)\n" \
  "v_mfma_i32_16x16x64_i8 v[144:147], v[64:67], v[16:19], v[144:147]\n" \
  "ds_read_b64_tr_b8 v[24:25], v5 offset:16384\n" \
  "v_mfma_i32_16x16x64_i8 v[144:147], v[80:83], v[20:23], v[144:147]\n" \
  "ds_read_b64_tr_b8 v[26:27], v5 offset:17408\n" \
  "v_mfma_i32_16x16x64_i8 v[176:179], v[84:87], v[20:23], v[176:179]\n" \
  "ds_read_b64_tr_b8 v[28:29], v5 offset:24576\n" \
  "v_mfma_i32_16x16x64_i8 v[176:179], v[68:71], v[16:19], v[176:179]\n" \
  "ds_read_b64_tr_b8 v[30:31], v5 offset:25600\n" \
  "v_mfma_i32_16x16x64_i8 v[208:211], v[72:75], v[16:19], v[208:211]\n" \
  "ds_read_b32 v10, v9\n" \
  "v_mfma_i32_16x16x64_i8 v[208:211], v[88:91], v[20:23], v[208:211]\n" \
  "v_mfma_i32_16x16x64_i8 v[240:243], v[92:95], v[20:23], v[240:243]\n" \
  "v_mfma_i32_16x16x64_i8 v[240:243], v[76:79], v[16:19], v[240:243]\n" \
  "s_waitcnt lgkmcnt(1)\n" \
  "v_mfma_i32_16x16x64_i8 v[244:247], v[76:79], v[24:27], v[244:247]\n" \
  "ds_read_b64_tr_b8 v[16:17], v6 offset:16384\n" \
  "v_mfma_i32_16x16x64_i8 v[244:247], v[92:95], v[28:31], v[244:247]\n" \
  "ds_read_b64_tr_b8 v[18:19], v6 offset:17408\n" \
  "v_mfma_i32_16x16x64_i8 v[212:215], v[88:91], v[28:31], v[212:215]\n" \
  "ds_read_b64_tr_b8 v[20:21], v6 offset:24576\n" \
  "v_mfma_i32_16x16x64_i8 v[212:215], v[72:75], v[24:27], v[212:215]\n" \
  "ds_read_b64_tr_b8 v[22:23], v6 offset:25600\n" \
  "v_mfma_i32_16x16x64_i8 v[180:183], v[68:71], v[24:27], v[180:183]\n" \
  "v_mfma_i32_16x16x64_i8 v[180:183], v[84:87], v[28:31], v[180:183]\n" \
  "v_mfma_i32_16x16x64_i8 v[148:151], v[80:83], v[28:31], v[148:151]\n" \
  "v_mfma_i32_16x16x64_i8 v[148:151], v[64:67], v[24:27], v[148:151]\n" \
  "s_waitcnt lgkmcnt(0)\n" \
  "v_mfma_i32_16x16x64_i8 v[152:155], v[64:67], v[16:19], v[152:155]\n" \
  "ds_read_b64_tr_b8 v[24:25], v7 offset:16384\n" \
  "v_mfma_i32_16x16x64_i8 v[152:155], v[80:83], v[20:23], v[152:155]\n" \
  "ds_read_b64_tr_b8 v[26:27], v7 offset:17408\n" \
  "v_mfma_i32_16x16x64_i8 v[184:187], v[84:87], v[20:23], v[184:187]\n" \
  "ds_read_b64_tr_b8 v[28:29], v7 offset:24576\n" \
  "v_mfma_i32_16x16x64_i8 v[184:187], v[68:71], v[16:19], v[184:187]\n" \
  "ds_read_b64_tr_b8 v[30:31], v7 offset:25600\n" \
  "v_mfma_i32_16x16x64_i8 v[216:219], v[72:75], v[16:19], v[216:219]\n" \
  "v_mfma_i32_16x16x64_i8 v[216:219], v[88:91], v[20:23], v[216:219]\n" \
  "v_mfma_i32_16x16x64_i8 v[248:251], v[92:95], v[20:23], v[248:251]\n" \
  "v_cmp_gt_u32 vcc, 18, v10\n" \
  "v_mfma_i32_16x16x64_i8 v[248:251], v[76:79], v[16:19], v[248:251]\n" \
  "s_waitcnt lgkmcnt(0)\n" \
  "v_mfma_i32_16x16x64_i8 v[252:255], v[76:79], v[24:27], v[252:255]\n" \
  "s_cbranch_vccnz LS7_%=\n" \
  "LB7_%=:\n" \
  "ds_read_b64_tr_b8 v[16:17], v0 offset:32768\n" \
  "v_mfma_i32_16x16x64_i8 v[252:255], v[92:95], v[28:31], v[252:255]\n" \
  "ds_read_b64_tr_b8 v[18:19], v0 offset:33792\n" \
  "v_mfma_i32_16x16x64_i8 v[220:223], v[88:91], v[28:31], v[220:223]\n" \
  "ds_read_b64_tr_b8 v[20:21], v0 offset:40960\n" \
  "v_mfma_i32_16x16x64_i8 v[220:223], v[72:75], v[24:27], v[220:223]\n" \
  "ds_read_b64_tr_b8 v[22:23], v0 offset:41984\n" \
  "v_mfma_i32_16x16x64_i8 v[188:191], v[68:71], v[24:27], v[188:191]\n" \
  "ds_add_u32 v12, v11\n" \
  "v_mfma_i32_16x16x64_i8 v[188:191], v[84:87], v[28:31], v[188:191]\n" \
  "v_mfma_i32_16x16x64_i8 v[156:159], v[80:83], v[28:31], v[156:159]\n" \
  "v_mfma_i32_16x16x64_i8 v[156:159], v[64:67], v[24:27], v[156:159]\n" \
  "s_waitcnt vmcnt(8) lgkmcnt(1)\n" \
  "v_mfma_i32_16x16x64_i8 v[128:131], v[96:99], v[16:19], v[128:131]\n" \
  "ds_read_b64_tr_b8 v[24:25], v1 offset:32768\n" \
  "v_mfma_i32_16x16x64_i8 v[128:131], v[112:115], v[20:23], v[128:131]\n" \
  "ds_read_b64_tr_b8 v[26:27], v1 offset:33792\n" \
  "v_mfma_i32_16x16x64_i8 v[160:163], v[116:119], v[20:23], v[160:163]\n" \
  "ds_read_b64_tr_b8 v[28:29], v1 offset:40960\n" \
  "v_mfma_i32_16x16x64_i8 v[160:163], v[100:103], v[16:19], v[160:163]\n" \
  "ds_read_b64_tr_b8 v[30:31], v1 offset:41984\n" \
  "v_mfma_i32_16x16x64_i8 v[192:195], v[104:107], v[16:19], v[192:195]\n" \
  "global_load_dwordx4 v[64:67], v8, s[40:41] offset:0 sc0 sc1\n" \
  "v_mfma_i32_16x16x64_i8 v[192:195], v[120:123], v[20:23], v[192:195]\n" \
  "global_load_dwordx4 v[68:71], v8, s[40:41] offset:1024 sc0 sc1\n" \
  "v_mfma_i32_16x16x64_i8 v[224:227], v[124:127], v[20:23], v[224:227]\n" \
  "v_mfma_i32_16x16x64_i8 v[224:227], v[108:111], v[16:19], v[224:227]\n" \
  "s_waitcnt lgkmcnt(0)\n" \
  "v_mfma_i32_16x16x64_i8 v[228:231], v[108:111], v[24:27], v[228:231]\n" \
  "ds_read_b64_tr_b8 v[16:17], v2 offset:32768\n" \
  "v_mfma_i32_16x16x64_i8 v[228:231], v[124:127], v[28:31], v[228:231]\n" \
  "ds_read_b64_tr_b8 v[18:19], v2 offset:33792\n" \
  "v_mfma_i32_16x16x64_i8 v[196:199], v[120:123], v[28:31], v[196:199]\n" \
  "ds_read_b64_tr_b8 v[20:21], v2 offset:40960\n" \
  "v_mfma_i32_16x16x64_i8 v[196:199], v[104:107], v[24:27], v[196:199]\n" \
  "ds_read_b64_tr_b8 v[22:23], v2 offset:41984\n" \
  "v_mfma_i32_16x16x64_i8 v[164:167], v[100:103], v[24:27], v[164:167]\n" \
  "global_load_dwordx4 v[72:75], v8, s[40:41] offset:2048 sc0 sc1\n" \
  "v_mfma_i32_16x16x64_i8 v[164:167], v[116:119], v[28:31], v[164:167]\n" \
  "global_load_dwordx4 v[76:79], v8, s[40:41] offset:3072 sc0 sc1\n" \
  "v_mfma_i32_16x16x64_i8 v[132:135], v[112:115], v[28:31], v[132:135]\n" \
  "v_add_u32 v8, 0x8000, v8\n" \
  "v_mfma_i32_16x16x64_i8 v[132:135], v[96:99], v[24:27], v[132:135]\n" \
  "s_waitcnt lgkmcnt(0)\n" \
  "v_mfma_i32_16x16x64_i8 v[136:139], v[96:99], v[16:19], v[136:139]\n" \
  "ds_read_b64_tr_b8 v[24:25], v3 offset:32768\n" \
  "v_mfma_i32_16x16x64_i8 v[136:139], v[112:115], v[20:23], v[136:139]\n" \
  "ds_read_b64_tr_b8 v[26:27], v3 offset:33792\n" \
  "v_mfma_i32_16x16x64_i8 v[168:171], v[116:119], v[20:23], v[168:171]\n" \
  "ds_read_b64_tr_b8 v[28:29], v3 offset:40960\n" \
  "v_mfma_i32_16x16x64_i8 v[168:171], v[100:103], v[16:19], v[168:171]\n" \
  "ds_read_b64_tr_b8 v[30:31], v3 offset:41984\n" \
  "v_mfma_i32_16x16x64_i8 v[200:203], v[104:107], v[16:19], v[200:203]\n" \
  "global_load_dwordx4 v[80:83], v8, s[40:41] offset:0 sc0 sc1\n" \
  "v_mfma_i32_16x16x64_i8 v[200:203], v[120:123], v[20:23], v[200:203]\n" \
  "global_load_dwordx4 v[84:87], v8, s[40:41] offset:1024 sc0 sc1\n" \
  "v_mfma_i32_16x16x64_i8 v[232:235], v[124:127], v[20:23], v[232:235]\n" \
  "v_mfma_i32_16x16x64_i8 v[232:235], v[108:111], v[16:19], v[232:235]\n" \
  "s_waitcnt lgkmcnt(0)\n" \
  "v_mfma_i32_16x16x64_i8 v[236:239], v[108:111], v[24:27], v[236:239]\n" \
  "ds_read_b64_tr_b8 v[16:17], v4 offset:32768\n" \
  "v_mfma_i32_16x16x64_i8 v[236:239], v[124:127], v[28:31], v[236:239]\n" \
  "ds_read_b64_tr_b8 v[18:19], v4 offset:33792\n" \
  "v_mfma_i32_16x16x64_i8 v[204:207], v[120:123], v[28:31], v[204:207]\n" \
  "ds_read_b64_tr_b8 v[20:21], v4 offset:40960\n" \
  "v_mfma_i32_16x16x64_i8 v[204:207], v[104:107], v[24:27], v[204:207]\n" \
  "ds_read_b64_tr_b8 v[22:23], v4 offset:41984\n" \
  "v_mfma_i32_16x16x64_i8 v[172:175], v[100:103], v[24:27], v[172:175]\n" \
  "global_load_dwordx4 v[88:91], v8, s[40:41] offset:2048 sc0 sc1\n" \
  "v_mfma_i32_16x16x64_i8 v[172:175], v[116:119], v[28:31], v[172:175]\n" \
  "global_load_dwordx4 v[92:95], v8, s[40:41] offset:3072 sc0 sc1\n" \
  "v_mfma_i32_16x16x64_i8 v[140:143], v[112:115], v[28:31], v[140:143]\n" \
  "v_add_u32 v8, 0x8000, v8\n" \
  "v_mfma_i32_16x16x64_i8 v[140:143], v[96:99], v[24:27], v[140:143]\n" \
  "s_waitcnt lgkmcnt(0)\n" \
  "v_mfma_i32_16x16x64_i8 v[144:147], v[96:99], v[16:19], v[144:147]\n" \
  "ds_read_b64_tr_b8 v[24:25], v5 offset:32768\n" \
  "v_mfma_i32_16x16x64_i8 v[144:147], v[112:115], v[20:23], v[144:147]\n" \
  "ds_read_b64_tr_b8 v[26:27], v5 offset:33792\n" \
  "v_mfma_i32_16x16x64_i8 v[176:179], v[116:119], v[20:23], v[176:179]\n" \
  "ds_read_b64_tr_b8 v[28:29], v5 offset:40960\n" \
  "v_mfma_i32_16x16x64_i8 v[176:179], v[100:103], v[16:19], v[176:179]\n" \
  "ds_read_b64_tr_b8 v[30:31], v5 offset:41984\n" \
  "v_mfma_i32_16x16x64_i8 v[208:211], v[104:107], v[16:19], v[208:211]\n" \
  "ds_read_b32 v10, v9\n" \
  "v_mfma_i32_16x16x64_i8 v[208:211], v[120:123], v[20:23], v[208:211]\n" \
  "v_mfma_i32_16x16x64_i8 v[240:243], v[124:127], v[20:23], v[240:243]\n" \
  "v_mfma_i32_16x16x64_i8 v[240:243], v[108:111], v[16:19], v[240:243]\n" \
  "s_waitcnt lgkmcnt(1)\n" \
  "v_mfma_i32_16x16x64_i8 v[244:247], v[108:111], v[24:27], v[244:247]\n" \
  "ds_read_b64_tr_b8 v[16:17], v6 offset:32768\n" \
  "v_mfma_i32_16x16x64_i8 v[244:247], v[124:127], v[28:31], v[244:247]\n" \
  "ds_read_b64_tr_b8 v[18:19], v6 offset:33792\n" \
  "v_mfma_i32_16x16x64_i8 v[212:215], v[120:123], v[28:31], v[212:215]\n" \
  "ds_read_b64_tr_b8 v[20:21], v6 offset:40960\n" \
  "v_mfma_i32_16x16x64_i8 v[212:215], v[104:107], v[24:27], v[212:215]\n" \
  "ds_read_b64_tr_b8 v[22:23], v6 offset:41984\n" \
  "v_mfma_i32_16x16x64_i8 v[180:183], v[100:103], v[24:27], v[180:183]\n" \
  "v_mfma_i32_16x16x64_i8 v[180:183], v[116:119], v[28:31], v[180:183]\n" \
  "v_mfma_i32_16x16x64_i8 v[148:151], v[112:115], v[28:31], v[148:151]\n" \
  "v_mfma_i32_16x16x64_i8 v[148:151], v[96:99], v[24:27], v[148:151]\n" \
  "s_waitcnt lgkmcnt(0)\n" \
  "v_mfma_i32_16x16x64_i8 v[152:155], v[96:99], v[16:19], v[152:155]\n" \
  "ds_read_b64_tr_b8 v[24:25], v7 offset:32768\n" \
  "v_mfma_i32_16x16x64_i8 v[152:155], v[112:115], v[20:23], v[152:155]\n" \
  "ds_read_b64_tr_b8 v[26:27], v7 offset:33792\n" \
  "v_mfma_i32_16x16x64_i8 v[184:187], v[116:119], v[20:23], v[184:187]\n" \
  "ds_read_b64_tr_b8 v[28:29], v7 offset:40960\n" \
  "v_mfma_i32_16x16x64_i8 v[184:187], v[100:103], v[16:19], v[184:187]\n" \
  "ds_read_b64_tr_b8 v[30:31], v7 offset:41984\n" \
  "v_mfma_i32_16x16x64_i8 v[216:219], v[104:107], v[16:19], v[216:219]\n" \
  "v_mfma_i32_16x16x64_i8 v[216:219], v[120:123], v[20:23], v[216:219]\n" \
  "v_mfma_i32_16x16x64_i8 v[248:251], v[124:127], v[20:23], v[248:251]\n" \
  "v_cmp_gt_u32 vcc, 20, v10\n" \
  "v_mfma_i32_16x16x64_i8 v[248:251], v[108:111], v[16:19], v[248:251]\n" \
  "s_waitcnt lgkmcnt(0)\n" \
  "v_mfma_i32_16x16x64_i8 v[252:255], v[108:111], v[24:27], v[252:255]\n" \
  "s_cbranch_vccnz LS8_%=\n" \
  "LB8_%=:\n" \
  "v_add_u32 v0, 0xfffe8000, v0\n" \
  "ds_read_b64_tr_b8 v[16:17], v0 offset:0\n" \
  "v_mfma_i32_16x16x64_i8 v[252:255], v[124:127], v[28:31], v[252:255]\n" \
  "ds_read_b64_tr_b8 v[18:19], v0 offset:1024\n" \
  "v_mfma_i32_16x16x64_i8 v[220:223], v[120:123], v[28:31], v[220:223]\n" \
  "ds_read_b64_tr_b8 v[20:21], v0 offset:8192\n" \
  "v_mfma_i32_16x16x64_i8 v[220:223], v[104:107], v[24:27], v[220:223]\n" \
  "ds_read_b64_tr_b8 v[22:23], v0 offset:9216\n" \
  "v_mfma_i32_16x16x64_i8 v[188:191], v[100:103], v[24:27], v[188:191]\n" \
  "ds_add_u32 v12, v11\n" \
  "v_mfma_i32_16x16x64_i8 v[188:191], v[116:119], v[28:31], v[188:191]\n" \
  "v_mfma_i32_16x16x64_i8 v[156:159], v[112:115], v[28:31], v[156:159]\n" \
  "v_mfma_i32_16x16x64_i8 v[156:159], v[96:99], v[24:27], v[156:159]\n" \
  "s_waitcnt vmcnt(8) lgkmcnt(1)\n" \
  "v_mfma_i32_16x16x64_i8 v[128:131], v[32:35], v[16:19], v[128:131]\n" \
  "v_add_u32 v1, 0xfffe8000, v1\n" \
  "ds_read_b64_tr_b8 v[24:25], v1 offset:0\n" \
  "v_mfma_i32_16x16x64_i8 v[128:131], v[48:51], v[20:23], v[128:131]\n" \
  "ds_read_b64_tr_b8 v[26:27], v1 offset:1024\n" \
  "v_mfma_i32_16x16x64_i8 v[160:163], v[52:55], v[20:23], v[160:163]\n" \
  "ds_read_b64_tr_b8 v[28:29], v1 offset:8192\n" \
  "v_mfma_i32_16x16x64_i8 v[160:163], v[36:39], v[16:19], v[160:163]\n" \
  "ds_read_b64_tr_b8 v[30:31], v1 offset:9216\n" \
  "v_mfma_i32_16x16x64_i8 v[192:195], v[40:43], v[16:19], v[192:195]\n" \
  "global_load_dwordx4 v[96:99], v8, s[40:41] offset:0 sc0 sc1\n" \
  "v_mfma_i32_16x16x64_i8 v[192:195], v[56:59], v[20:23], v[192:195]\n" \
  "global_load_dwordx4 v[100:103], v8, s[40:41] offset:1024 sc0 sc1\n" \
  "v_mfma_i32_16x16x64_i8 v[224:227], v[60:63], v[20:23], v[224:227]\n" \
  "v_mfma_i32_16x16x64_i8 v[224:227], v[44:47], v[16:19], v[224:227]\n" \
  "s_waitcnt lgkmcnt(0)\n" \
  "v_mfma_i32_16x16x64_i8 v[228:231], v[44:47], v[24:27], v[228:231]\n" \
  "v_add_u32 v2, 0xfffe8000, v2\n" \
  "ds_read_b64_tr_b8 v[16:17], v2 offset:0\n" \
  "v_mfma_i32_16x16x64_i8 v[228:231], v[60:63], v[28:31], v[228:231]\n" \
  "ds_read_b64_tr_b8 v[18:19], v2 offset:1024\n" \
  "v_mfma_i32_16x16x64_i8 v[196:199], v[56:59], v[28:31], v[196:199]\n" \
  "ds_read_b64_tr_b8 v[20:21], v2 offset:8192\n" \
  "v_mfma_i32_16x16x64_i8 v[196:199], v[40:43], v[24:27], v[196:199]\n" \
  "ds_read_b64_tr_b8 v[22:23], v2 offset:9216\n" \
  "v_mfma_i32_16x16x64_i8 v[164:167], v[36:39], v[24:27], v[164:167]\n" \
  "global_load_dwordx4 v[104:107], v8, s[40:41] offset:2048 sc0 sc1\n" \
  "v_mfma_i32_16x16x64_i8 v[164:167], v[52:55], v[28:31], v[164:167]\n" \
  "global_load_dwordx4 v[108:111], v8, s[40:41] offset:3072 sc0 sc1\n" \
  "v_mfma_i32_16x16x64_i8 v[132:135], v[48:51], v[28:31], v[132:135]\n" \
  "v_add_u32 v8, 0x8000, v8\n" \
  "v_mfma_i32_16x16x64_i8 v[132:135], v[32:35], v[24:27], v[132:135]\n" \
  "s_waitcnt lgkmcnt(0)\n" \
  "v_mfma_i32_16x16x64_i8 v[136:139], v[32:35], v[16:19], v[136:139]\n" \
  "v_add_u32 v3, 0xfffe8000, v3\n" \
  "ds_read_b64_tr_b8 v[24:25], v3 offset:0\n" \
  "v_mfma_i32_16x16x64_i8 v[136:139], v[48:51], v[20:23], v[136:139]\n" \
  "ds_read_b64_tr_b8 v[26:27], v3 offset:1024\n" \
  "v_mfma_i32_16x16x64_i8 v[168:171], v[52:55], v[20:23], v[168:171]\n" \
  "ds_read_b64_tr_b8 v[28:29], v3 offset:8192\n" \
  "v_mfma_i32_16x16x64_i8 v[168:171], v[36:39], v[16:19], v[168:171]\n" \
  "ds_read_b64_tr_b8 v[30:31], v3 offset:9216\n" \
  "v_mfma_i32_16x16x64_i8 v[200:203], v[40:43], v[16:19], v[200:203]\n" \
  "global_load_dwordx4 v[112:115], v8, s[40:41] offset:0 sc0 sc1\n" \
  "v_mfma_i32_16x16x64_i8 v[200:203], v[56:59], v[20:23], v[200:203]\n" \
  "global_load_dwordx4 v[116:119], v8, s[40:41] offset:1024 sc0 sc1\n" \
  "v_mfma_i32_16x16x64_i8 v[232:235], v[60:63], v[20:23], v[232:235]\n" \
  "v_mfma_i32_16x16x64_i8 v[232:235], v[44:47], v[16:19], v[232:235]\n" \
  "s_waitcnt lgkmcnt(0)\n" \
  "v_mfma_i32_16x16x64_i8 v[236:239], v[44:47], v[24:27], v[236:239]\n" \
  "v_add_u32 v4, 0xfffe8000, v4\n" \
  "ds_read_b64_tr_b8 v[16:17], v4 offset:0\n" \
  "v_mfma_i32_16x16x64_i8 v[236:239], v[60:63], v[28:31], v[236:239]\n" \
  "ds_read_b64_tr_b8 v[18:19], v4 offset:1024\n" \
  "v_mfma_i32_16x16x64_i8 v[204:207], v[56:59], v[28:31], v[204:207]\n" \
  "ds_read_b64_tr_b8 v[20:21], v4 offset:8192\n" \
  "v_mfma_i32_16x16x64_i8 v[204:207], v[40:43], v[24:27], v[204:207]\n" \
  "ds_read_b64_tr_b8 v[22:23], v4 offset:9216\n" \
  "v_mfma_i32_16x16x64_i8 v[172:175], v[36:39], v[24:27], v[172:175]\n" \
  "global_load_dwordx4 v[120:123], v8, s[40:41] offset:2048 sc0 sc1\n" \
  "v_mfma_i32_16x16x64_i8 v[172:175], v[52:55], v[28:31], v[172:175]\n" \
  "global_load_dwordx4 v[124:127], v8, s[40:41] offset:3072 sc0 sc1\n" \
  "v_mfma_i32_16x16x64_i8 v[140:143], v[48:51], v[28:31], v[140:143]\n" \
  "v_add_u32 v8, 0x8000, v8\n" \
  "v_mfma_i32_16x16x64_i8 v[140:143], v[32:35], v[24:27], v[140:143]\n" \
  "s_waitcnt lgkmcnt(0)\n" \
  "v_mfma_i32_16x16x64_i8 v[144:147], v[32:35], v[16:19], v[144:147]\n" \
  "v_add_u32 v5, 0xfffe8000, v5\n" \
  "ds_read_b64_tr_b8 v[24:25], v5 offset:0\n" \
  "v_mfma_i32_16x16x64_i8 v[144:147], v[48:51], v[20:23], v[144:147]\n" \
  "ds_read_b64_tr_b8 v[26:27], v5 offset:1024\n" \
  "v_mfma_i32_16x16x64_i8 v[176:179], v[52:55], v[20:23], v[176:179]\n" \
  "ds_read_b64_tr_b8 v[28:29], v5 offset:8192\n" \
  "v_mfma_i32_16x16x64_i8 v[176:179], v[36:39], v[16:19], v[176:179]\n" \
  "ds_read_b64_tr_b8 v[30:31], v5 offset:9216\n" \
  "v_mfma_i32_16x16x64_i8 v[208:211], v[40:43], v[16:19], v[208:211]\n" \
  "ds_read_b32 v10, v9\n" \
  "v_mfma_i32_16x16x64_i8 v[208:211], v[56:59], v[20:23], v[208:211]\n" \
  "v_mfma_i32_16x16x64_i8 v[240:243], v[60:63], v[20:23], v[240:243]\n" \
  "v_mfma_i32_16x16x64_i8 v[240:243], v[44:47], v[16:19], v[240:243]\n" \
  "s_waitcnt lgkmcnt(1)\n" \
  "v_mfma_i32_16x16x64_i8 v[244:247], v[44:47], v[24:27], v[244:247]\n" \
  "v_add_u32 v6, 0xfffe8000, v6\n" \
  "ds_read_b64_tr_b8 v[16:17], v6 offset:0\n" \
  "v_mfma_i32_16x16x64_i8 v[244:247], v[60:63], v[28:31], v[244:247]\n" \
  "ds_read_b64_tr_b8 v[18:19], v6 offset:1024\n" \
  "v_mfma_i32_16x16x64_i8 v[212:215], v[56:59], v[28:31], v[212:215]\n" \
  "ds_read_b64_tr_b8 v[20:21], v6 offset:8192\n" \
  "v_mfma_i32_16x16x64_i8 v[212:215], v[40:43], v[24:27], v[212:215]\n" \
  "ds_read_b64_tr_b8 v[22:23], v6 offset:9216\n" \
  "v_mfma_i32_16x16x64_i8 v[180:183], v[36:39], v[24:27], v[180:183]\n" \
  "v_mfma_i32_16x16x64_i8 v[180:183], v[52:55], v[28:31], v[180:183]\n" \
  "v_mfma_i32_16x16x64_i8 v[148:151], v[48:51], v[28:31], v[148:151]\n" \
  "v_mfma_i32_16x16x64_i8 v[148:151], v[32:35], v[24:27], v[148:151]\n" \
  "s_waitcnt lgkmcnt(0)\n" \
  "v_mfma_i32_16x16x64_i8 v[152:155], v[32:35], v[16:19], v[152:155]\n" \
  "v_add_u32 v7, 0xfffe8000, v7\n" \
  "ds_read_b64_tr_b8 v[24:25], v7 offset:0\n" \
  "v_mfma_i32_16x16x64_i8 v[152:155], v[48:51], v[20:23], v[152:155]\n" \
  "ds_read_b64_tr_b8 v[26:27], v7 offset:1024\n" \
  "v_mfma_i32_16x16x64_i8 v[184:187], v[52:55], v[20:23], v[184:187]\n" \
  "ds_read_b64_tr_b8 v[28:29], v7 offset:8192\n" \
  "v_mfma_i32_16x16x64_i8 v[184:187], v[36:39], v[16:19], v[184:187]\n" \
  "ds_read_b64_tr_b8 v[30:31], v7 offset:9216\n" \
  "v_mfma_i32_16x16x64_i8 v[216:219], v[40:43], v[16:19], v[216:219]\n" \
  "v_mfma_i32_16x16x64_i8 v[216:219], v[56:59], v[20:23], v[216:219]\n" \
  "v_mfma_i32_16x16x64_i8 v[248:251], v[60:63], v[20:23], v[248:251]\n" \
  "v_cmp_gt_u32 vcc, 22, v10\n" \
  "v_mfma_i32_16x16x64_i8 v[248:251], v[44:47], v[16:19], v[248:251]\n" \
  "s_waitcnt lgkmcnt(0)\n" \
  "v_mfma_i32_16x16x64_i8 v[252:255], v[44:47], v[24:27], v[252:255]\n" \
  "s_cbranch_vccnz LS9_%=\n" \
  "LB9_%=:\n" \
  "ds_read_b64_tr_b8 v[16:17], v0 offset:16384\n" \
  "v_mfma_i32_16x16x64_i8 v[252:255], v[60:63], v[28:31], v[252:255]\n" \
  "ds_read_b64_tr_b8 v[18:19], v0 offset:17408\n" \
  "v_mfma_i32_16x16x64_i8 v[220:223], v[56:59], v[28:31], v[220:223]\n" \
  "ds_read_b64_tr_b8 v[20:21], v0 offset:24576\n" \
  "v_mfma_i32_16x16x64_i8 v[220:223], v[40:43], v[24:27], v[220:223]\n" \
  "ds_read_b64_tr_b8 v[22:23], v0 offset:25600\n" \
  "v_mfma_i32_16x16x64_i8 v[188:191], v[36:39], v[24:27], v[188:191]\n" \
  "ds_add_u32 v12, v11\n" \
  "v_mfma_i32_16x16x64_i8 v[188:191], v[52:55], v[28:31], v[188:191]\n" \
  "v_mfma_i32_16x16x64_i8 v[156:159], v[48:51], v[28:31], v[156:159]\n" \
  "v_mfma_i32_16x16x64_i8 v[156:159], v[32:35], v[24:27], v[156:159]\n" \
  "s_waitcnt vmcnt(8) lgkmcnt(1)\n" \
  "v_mfma_i32_16x16x64_i8 v[128:131], v[64:67], v[16:19], v[128:131]\n" \
  "ds_read_b64_tr_b8 v[24:25], v1 offset:16384\n" \
  "v_mfma_i32_16x16x64_i8 v[128:131], v[80:83], v[20:23], v[128:131]\n" \
  "ds_read_b64_tr_b8 v[26:27], v1 offset:17408\n" \
  "v_mfma_i32_16x16x64_i8 v[160:163], v[84:87], v[20:23], v[160:163]\n" \
  "ds_read_b64_tr_b8 v[28:29], v1 offset:24576\n" \
  "v_mfma_i32_16x16x64_i8 v[160:163], v[68:71], v[16:19], v[160:163]\n" \
  "ds_read_b64_tr_b8 v[30:31], v1 offset:25600\n" \
  "v_mfma_i32_16x16x64_i8 v[192:195], v[72:75], v[16:19], v[192:195]\n" \
  "global_load_dwordx4 v[32:35], v8, s[40:41] offset:0 sc0 sc1\n" \
  "v_mfma_i32_16x16x64_i8 v[192:195], v[88:91], v[20:23], v[192:195]\n" \
  "global_load_dwordx4 v[36:39], v8, s[40:41] offset:1024 sc0 sc1\n" \
  "v_mfma_i32_16x16x64_i8 v[224:227], v[92:95], v[20:23], v[224:227]\n" \
  "v_mfma_i32_16x16x64_i8 v[224:227], v[76:79], v[16:19], v[224:227]\n" \
  "s_waitcnt lgkmcnt(0)\n" \
  "v_mfma_i32_16x16x64_i8 v[228:231], v[76:79], v[24:27], v[228:231]\n" \
  "ds_read_b64_tr_b8 v[16:17], v2 offset:16384\n" \
  "v_mfma_i32_16x16x64_i8 v[228:231], v[92:95], v[28:31], v[228:231]\n" \
  "ds_read_b64_tr_b8 v[18:19], v2 offset:17408\n" \
  "v_mfma_i32_16x16x64_i8 v[196:199], v[88:91], v[28:31], v[196:199]\n" \
  "ds_read_b64_tr_b8 v[20:21], v2 offset:24576\n" \
  "v_mfma_i32_16x16x64_i8 v[196:199], v[72:75], v[24:27], v[196:199]\n" \
  "ds_read_b64_tr_b8 v[22:23], v2 offset:25600\n" \
  "v_mfma_i32_16x16x64_i8 v[164:167], v[68:71], v[24:27], v[164:167]\n" \
  "global_load_dwordx4 v[40:43], v8, s[40:41] offset:2048 sc0 sc1\n" \
  "v_mfma_i32_16x16x64_i8 v[164:167], v[84:87], v[28:31], v[164:167]\n" \
  "global_load_dwordx4 v[44:47], v8, s[40:41] offset:3072 sc0 sc1\n" \
  "v_mfma_i32_16x16x64_i8 v[132:135], v[80:83], v[28:31], v[132:135]\n" \
  "v_add_u32 v8, 0x8000, v8\n" \
  "v_mfma_i32_16x16x64_i8 v[132:135], v[64:67], v[24:27], v[132:135]\n" \
  "s_waitcnt lgkmcnt(0)\n" \
  "v_mfma_i32_16x16x64_i8 v[136:139], v[64:67], v[16:19], v[136:139]\n" \
  "ds_read_b64_tr_b8 v[24:25], v3 offset:16384\n" \
  "v_mfma_i32_16x16x64_i8 v[136:139], v[80:83], v[20:23], v[136:139]\n" \
  "ds_read_b64_tr_b8 v[26:27], v3 offset:17408\n" \
  "v_mfma_i32_16x16x64_i8 v[168:171], v[84:87], v[20:23], v[168:171]\n" \
  "ds_read_b64_tr_b8 v[28:29], v3 offset:24576\n" \
  "v_mfma_i32_16x16x64_i8 v[168:171], v[68:71], v[16:19], v[168:171]\n" \
  "ds_read_b64_tr_b8 v[30:31], v3 offset:25600\n" \
  "v_mfma_i32_16x16x64_i8 v[200:203], v[72:75], v[16:19], v[200:203]\n" \
  "global_load_dwordx4 v[48:51], v8, s[40:41] offset:0 sc0 sc1\n" \
  "v_mfma_i32_16x16x64_i8 v[200:203], v[88:91], v[20:23], v[200:203]\n" \
  "global_load_dwordx4 v[52:55], v8, s[40:41] offset:1024 sc0 sc1\n" \
  "v_mfma_i32_16x16x64_i8 v[232:235], v[92:95], v[20:23], v[232:235]\n" \
  "v_mfma_i32_16x16x64_i8 v[232:235], v[76:79], v[16:19], v[232:235]\n" \
  "s_waitcnt lgkmcnt(0)\n" \
  "v_mfma_i32_16x16x64_i8 v[236:239], v[76:79], v[24:27], v[236:239]\n" \
  "ds_read_b64_tr_b8 v[16:17], v4 offset:16384\n" \
  "v_mfma_i32_16x16x64_i8 v[236:239], v[92:95], v[28:31], v[236:239]\n" \
  "ds_read_b64_tr_b8 v[18:19], v4 offset:17408\n" \
  "v_mfma_i32_16x16x64_i8 v[204:207], v[88:91], v[28:31], v[204:207]\n" \
  "ds_read_b64_tr_b8 v[20:21], v4 offset:24576\n" \
  "v_mfma_i32_16x16x64_i8 v[204:207], v[72:75], v[24:27], v[204:207]\n" \
  "ds_read_b64_tr_b8 v[22:23], v4 offset:25600\n" \
  "v_mfma_i32_16x16x64_i8 v[172:175], v[68:71], v[24:27], v[172:175]\n" \
  "global_load_dwordx4 v[56:59], v8, s[40:41] offset:2048 sc0 sc1\n" \
  "v_mfma_i32_16x16x64_i8 v[172:175], v[84:87], v[28:31], v[172:175]\n" \
  "global_load_dwordx4 v[60:63], v8, s[40:41] offset:3072 sc0 sc1\n" \
  "v_mfma_i32_16x16x64_i8 v[140:143], v[80:83], v[28:31], v[140:143]\n" \
  "v_add_u32 v8, 0x8000, v8\n" \
  "v_mfma_i32_16x16x64_i8 v[140:143], v[64:67], v[24:27], v[140:143]\n" \
  "s_waitcnt lgkmcnt(0)\n" \
  "v_mfma_i32_16x16x64_i8 v[144:147], v[64:67], v[16:19], v[144:147]\n" \
  "ds_read_b64_tr_b8 v[24:25], v5 offset:16384\n" \
  "v_mfma_i32_16x16x64_i8 v[144:147], v[80:83], v[20:23], v[144:147]\n" \
  "ds_read_b64_tr_b8 v[26:27], v5 offset:17408\n" \
  "v_mfma_i32_16x16x64_i8 v[176:179], v[84:87], v[20:23], v[176:179]\n" \
  "ds_read_b64_tr_b8 v[28:29], v5 offset:24576\n" \
  "v_mfma_i32_16x16x64_i8 v[176:179], v[68:71], v[16:19], v[176:179]\n" \
  "ds_read_b64_tr_b8 v[30:31], v5 offset:25600\n" \
  "v_mfma_i32_16x16x64_i8 v[208:211], v[72:75], v[16:19], v[208:211]\n" \
  "ds_read_b32 v10, v9\n" \
  "v_mfma_i32_16x16x64_i8 v[208:211], v[88:91], v[20:23], v[208:211]\n" \
  "v_mfma_i32_16x16x64_i8 v[240:243], v[92:95], v[20:23], v[240:243]\n" \
  "v_mfma_i32_16x16x64_i8 v[240:243], v[76:79], v[16:19], v[240:243]\n" \
  "s_waitcnt lgkmcnt(1)\n" \
  "v_mfma_i32_16x16x64_i8 v[244:247], v[76:79], v[24:27], v[244:247]\n" \
  "ds_read_b64_tr_b8 v[16:17], v6 offset:16384\n" \
  "v_mfma_i32_16x16x64_i8 v[244:247], v[92:95], v[28:31], v[244:247]\n" \
  "ds_read_b64_tr_b8 v[18:19], v6 offset:17408\n" \
  "v_mfma_i32_16x16x64_i8 v[212:215], v[88:91], v[28:31], v[212:215]\n" \
  "ds_read_b64_tr_b8 v[20:21], v6 offset:24576\n" \
  "v_mfma_i32_16x16x64_i8 v[212:215], v[72:75], v[24:27], v[212:215]\n" \
  "ds_read_b64_tr_b8 v[22:23], v6 offset:25600\n" \
  "v_mfma_i32_16x16x64_i8 v[180:183], v[68:71], v[24:27], v[180:183]\n" \
  "v_mfma_i32_16x16x64_i8 v[180:183], v[84:87], v[28:31], v[180:183]\n" \
  "v_mfma_i32_16x16x64_i8 v[148:151], v[80:83], v[28:31], v[148:151]\n" \
  "v_mfma_i32_16x16x64_i8 v[148:151], v[64:67], v[24:27], v[148:151]\n" \
  "s_waitcnt lgkmcnt(0)\n" \
  "v_mfma_i32_16x16x64_i8 v[152:155], v[64:67], v[16:19], v[152:155]\n" \
  "ds_read_b64_tr_b8 v[24:25], v7 offset:16384\n" \
  "v_mfma_i32_16x16x64_i8 v[152:155], v[80:83], v[20:23], v[152:155]\n" \
  "ds_read_b64_tr_b8 v[26:27], v7 offset:17408\n" \
  "v_mfma_i32_16x16x64_i8 v[184:187], v[84:87], v[20:23], v[184:187]\n" \
  "ds_read_b64_tr_b8 v[28:29], v7 offset:24576\n" \
  "v_mfma_i32_16x16x64_i8 v[184:187], v[68:71], v[16:19], v[184:187]\n" \
  "ds_read_b64_tr_b8 v[30:31], v7 offset:25600\n" \
  "v_mfma_i32_16x16x64_i8 v[216:219], v[72:75], v[16:19], v[216:219]\n" \
  "v_mfma_i32_16x16x64_i8 v[216:219], v[88:91], v[20:23], v[216:219]\n" \
  "v_mfma_i32_16x16x64_i8 v[248:251], v[92:95], v[20:23], v[248:251]\n" \
  "v_cmp_gt_u32 vcc, 24, v10\n" \
  "v_mfma_i32_16x16x64_i8 v[248:251], v[76:79], v[16:19], v[248:251]\n" \
  "s_waitcnt lgkmcnt(0)\n" \
  "v_mfma_i32_16x16x64_i8 v[252:255], v[76:79], v[24:27], v[252:255]\n" \
  "s_cbranch_vccnz LS10_%=\n" \
  "LB10_%=:\n" \
  "ds_read_b64_tr_b8 v[16:17], v0 offset:32768\n" \
  "v_mfma_i32_16x16x64_i8 v[252:255], v[92:95], v[28:31], v[252:255]\n" \
  "ds_read_b64_tr_b8 v[18:19], v0 offset:33792\n" \
  "v_mfma_i32_16x16x64_i8 v[220:223], v[88:91], v[28:31], v[220:223]\n" \
  "ds_read_b64_tr_b8 v[20:21], v0 offset:40960\n" \
  "v_mfma_i32_16x16x64_i8 v[220:223], v[72:75], v[24:27], v[220:223]\n" \
  "ds_read_b64_tr_b8 v[22:23], v0 offset:41984\n" \
  "v_mfma_i32_16x16x64_i8 v[188:191], v[68:71], v[24:27], v[188:191]\n" \
  "ds_add_u32 v12, v11\n" \
  "v_mfma_i32_16x16x64_i8 v[188:191], v[84:87], v[28:31], v[188:191]\n" \
  "v_mfma_i32_16x16x64_i8 v[156:159], v[80:83], v[28:31], v[156:159]\n" \
  "v_mfma_i32_16x16x64_i8 v[156:159], v[64:67], v[24:27], v[156:159]\n" \
  "s_waitcnt vmcnt(8) lgkmcnt(1)\n" \
  "v_mfma_i32_16x16x64_i8 v[128:131], v[96:99], v[16:19], v[128:131]\n" \
  "ds_read_b64_tr_b8 v[24:25], v1 offset:32768\n" \
  "v_mfma_i32_16x16x64_i8 v[128:131], v[112:115], v[20:23], v[128:131]\n" \
  "ds_read_b64_tr_b8 v[26:27], v1 offset:33792\n" \
  "v_mfma_i32_16x16x64_i8 v[160:163], v[116:119], v[20:23], v[160:163]\n" \
  "ds_read_b64_tr_b8 v[28:29], v1 offset:40960\n" \
  "v_mfma_i32_16x16x64_i8 v[160:163], v[100:103], v[16:19], v[160:163]\n" \
  "ds_read_b64_tr_b8 v[30:31], v1 offset:41984\n" \
  "v_mfma_i32_16x16x64_i8 v[192:195], v[104:107], v[16:19], v[192:195]\n" \
  "global_load_dwordx4 v[64:67], v8, s[40:41] offset:0 sc0 sc1\n" \
  "v_mfma_i32_16x16x64_i8 v[192:195], v[120:123], v[20:23], v[192:195]\n" \
  "global_load_dwordx4 v[68:71], v8, s[40:41] offset:1024 sc0 sc1\n" \
  "v_mfma_i32_16x16x64_i8 v[224:227], v[124:127], v[20:23], v[224:227]\n" \
  "v_mfma_i32_16x16x64_i8 v[224:227], v[108:111], v[16:19], v[224:227]\n" \
  "s_waitcnt lgkmcnt(0)\n" \
  "v_mfma_i32_16x16x64_i8 v[228:231], v[108:111], v[24:27], v[228:231]\n" \
  "ds_read_b64_tr_b8 v[16:17], v2 offset:32768\n" \
  "v_mfma_i32_16x16x64_i8 v[228:231], v[124:127], v[28:31], v[228:231]\n" \
  "ds_read_b64_tr_b8 v[18:19], v2 offset:33792\n" \
  "v_mfma_i32_16x16x64_i8 v[196:199], v[120:123], v[28:31], v[196:199]\n" \
  "ds_read_b64_tr_b8 v[20:21], v2 offset:40960\n" \
  "v_mfma_i32_16x16x64_i8 v[196:199], v[104:107], v[24:27], v[196:199]\n" \
  "ds_read_b64_tr_b8 v[22:23], v2 offset:41984\n" \
  "v_mfma_i32_16x16x64_i8 v[164:167], v[100:103], v[24:27], v[164:167]\n" \
  "global_load_dwordx4 v[72:75], v8, s[40:41] offset:2048 sc0 sc1\n" \
  "v_mfma_i32_16x16x64_i8 v[164:167], v[116:119], v[28:31], v[164:167]\n" \
  "global_load_dwordx4 v[76:79], v8, s[40:41] offset:3072 sc0 sc1\n" \
  "v_mfma_i32_16x16x64_i8 v[132:135], v[112:115], v[28:31], v[132:135]\n" \
  "v_add_u32 v8, 0x8000, v8\n" \
  "v_mfma_i32_16x16x64_i8 v[132:135], v[96:99], v[24:27], v[132:135]\n" \
  "s_waitcnt lgkmcnt(0)\n" \
  "v_mfma_i32_16x16x64_i8 v[136:139], v[96:99], v[16:19], v[136:139]\n" \
  "ds_read_b64_tr_b8 v[24:25], v3 offset:32768\n" \
  "v_mfma_i32_16x16x64_i8 v[136:139], v[112:115], v[20:23], v[136:139]\n" \
  "ds_read_b64_tr_b8 v[26:27], v3 offset:33792\n" \
  "v_mfma_i32_16x16x64_i8 v[168:171], v[116:119], v[20:23], v[168:171]\n" \
  "ds_read_b64_tr_b8 v[28:29], v3 offset:40960\n" \
  "v_mfma_i32_16x16x64_i8 v[168:171], v[100:103], v[16:19], v[168:171]\n" \
  "ds_read_b64_tr_b8 v[30:31], v3 offset:41984\n" \
  "v_mfma_i32_16x16x64_i8 v[200:203], v[104:107], v[16:19], v[200:203]\n" \
  "global_load_dwordx4 v[80:83], v8, s[40:41] offset:0 sc0 sc1\n" \
  "v_mfma_i32_16x16x64_i8 v[200:203], v[120:123], v[20:23], v[200:203]\n" \
  "global_load_dwordx4 v[84:87], v8, s[40:41] offset:1024 sc0 sc1\n" \
  "v_mfma_i32_16x16x64_i8 v[232:235], v[124:127], v[20:23], v[232:235]\n" \
  "v_mfma_i32_16x16x64_i8 v[232:235], v[108:111], v[16:19], v[232:235]\n" \
  "s_waitcnt lgkmcnt(0)\n" \
  "v_mfma_i32_16x16x64_i8 v[236:239], v[108:111], v[24:27], v[236:239]\n" \
  "ds_read_b64_tr_b8 v[16:17], v4 offset:32768\n" \
  "v_mfma_i32_16x16x64_i8 v[236:239], v[124:127], v[28:31], v[236:239]\n" \
  "ds_read_b64_tr_b8 v[18:19], v4 offset:33792\n" \
  "v_mfma_i32_16x16x64_i8 v[204:207], v[120:123], v[28:31], v[204:207]\n" \
  "ds_read_b64_tr_b8 v[20:21], v4 offset:40960\n" \
  "v_mfma_i32_16x16x64_i8 v[204:207], v[104:107], v[24:27], v[204:207]\n" \
  "ds_read_b64_tr_b8 v[22:23], v4 offset:41984\n" \
  "v_mfma_i32_16x16x64_i8 v[172:175], v[100:103], v[24:27], v[172:175]\n" \
  "global_load_dwordx4 v[88:91], v8, s[40:41] offset:2048 sc0 sc1\n" \
  "v_mfma_i32_16x16x64_i8 v[172:175], v[116:119], v[28:31], v[172:175]\n" \
  "global_load_dwordx4 v[92:95], v8, s[40:41] offset:3072 sc0 sc1\n" \
  "v_mfma_i32_16x16x64_i8 v[140:143], v[112:115], v[28:31], v[140:143]\n" \
  "v_add_u32 v8, 0x8000, v8\n" \
  "v_mfma_i32_16x16x64_i8 v[140:143], v[96:99], v[24:27], v[140:143]\n" \
  "s_waitcnt lgkmcnt(0)\n" \
  "v_mfma_i32_16x16x64_i8 v[144:147], v[96:99], v[16:19], v[144:147]\n" \
  "ds_read_b64_tr_b8 v[24:25], v5 offset:32768\n" \
  "v_mfma_i32_16x16x64_i8 v[144:147], v[112:115], v[20:23], v[144:147]\n" \
  "ds_read_b64_tr_b8 v[26:27], v5 offset:33792\n" \
  "v_mfma_i32_16x16x64_i8 v[176:179], v[116:119], v[20:23], v[176:179]\n" \
  "ds_read_b64_tr_b8 v[28:29], v5 offset:40960\n" \
  "v_mfma_i32_16x16x64_i8 v[176:179], v[100:103], v[16:19], v[176:179]\n" \
  "ds_read_b64_tr_b8 v[30:31], v5 offset:41984\n" \
  "v_mfma_i32_16x16x64_i8 v[208:211], v[104:107], v[16:19], v[208:211]\n" \
  "ds_read_b32 v10, v9\n" \
  "v_mfma_i32_16x16x64_i8 v[208:211], v[120:123], v[20:23], v[208:211]\n" \
  "v_mfma_i32_16x16x64_i8 v[240:243], v[124:127], v[20:23], v[240:243]\n" \
  "v_mfma_i32_16x16x64_i8 v[240:243], v[108:111], v[16:19], v[240:243]\n" \
  "s_waitcnt lgkmcnt(1)\n" \
  "v_mfma_i32_16x16x64_i8 v[244:247], v[108:111], v[24:27], v[244:247]\n" \
  "ds_read_b64_tr_b8 v[16:17], v6 offset:32768\n" \
  "v_mfma_i32_16x16x64_i8 v[244:247], v[124:127], v[28:31], v[244:247]\n" \
  "ds_read_b64_tr_b8 v[18:19], v6 offset:33792\n" \
  "v_mfma_i32_16x16x64_i8 v[212:215], v[120:123], v[28:31], v[212:215]\n" \
  "ds_read_b64_tr_b8 v[20:21], v6 offset:40960\n" \
  "v_mfma_i32_16x16x64_i8 v[212:215], v[104:107], v[24:27], v[212:215]\n" \
  "ds_read_b64_tr_b8 v[22:23], v6 offset:41984\n" \
  "v_mfma_i32_16x16x64_i8 v[180:183], v[100:103], v[24:27], v[180:183]\n" \
  "v_mfma_i32_16x16x64_i8 v[180:183], v[116:119], v[28:31], v[180:183]\n" \
  "v_mfma_i32_16x16x64_i8 v[148:151], v[112:115], v[28:31], v[148:151]\n" \
  "v_mfma_i32_16x16x64_i8 v[148:151], v[96:99], v[24:27], v[148:151]\n" \
  "s_waitcnt lgkmcnt(0)\n" \
  "v_mfma_i32_16x16x64_i8 v[152:155], v[96:99], v[16:19], v[152:155]\n" \
  "ds_read_b64_tr_b8 v[24:25], v7 offset:32768\n" \
  "v_mfma_i32_16x16x64_i8 v[152:155], v[112:115], v[20:23], v[152:155]\n" \
  "ds_read_b64_tr_b8 v[26:27], v7 offset:33792\n" \
  "v_mfma_i32_16x16x64_i8 v[184:187], v[116:119], v[20:23], v[184:187]\n" \
  "ds_read_b64_tr_b8 v[28:29], v7 offset:40960\n" \
  "v_mfma_i32_16x16x64_i8 v[184:187], v[100:103], v[16:19], v[184:187]\n" \
  "ds_read_b64_tr_b8 v[30:31], v7 offset:41984\n" \
  "v_mfma_i32_16x16x64_i8 v[216:219], v[104:107], v[16:19], v[216:219]\n" \
  "v_mfma_i32_16x16x64_i8 v[216:219], v[120:123], v[20:23], v[216:219]\n" \
  "v_mfma_i32_16x16x64_i8 v[248:251], v[124:127], v[20:23], v[248:251]\n" \
  "v_cmp_gt_u32 vcc, 26, v10\n" \
  "v_mfma_i32_16x16x64_i8 v[248:251], v[108:111], v[16:19], v[248:251]\n" \
  "s_waitcnt lgkmcnt(0)\n" \
  "v_mfma_i32_16x16x64_i8 v[252:255], v[108:111], v[24:27], v[252:255]\n" \
  "s_cbranch_vccnz LS11_%=\n" \
  "LB11_%=:\n" \
  "v_add_u32 v0, 0xc000, v0\n" \
  "ds_read_b64_tr_b8 v[16:17], v0 offset:0\n" \
  "v_mfma_i32_16x16x64_i8 v[252:255], v[124:127], v[28:31], v[252:255]\n" \
  "ds_read_b64_tr_b8 v[18:19], v0 offset:1024\n" \
  "v_mfma_i32_16x16x64_i8 v[220:223], v[120:123], v[28:31], v[220:223]\n" \
  "ds_read_b64_tr_b8 v[20:21], v0 offset:8192\n" \
  "v_mfma_i32_16x16x64_i8 v[220:223], v[104:107], v[24:27], v[220:223]\n" \
  "ds_read_b64_tr_b8 v[22:23], v0 offset:9216\n" \
  "v_mfma_i32_16x16x64_i8 v[188:191], v[100:103], v[24:27], v[188:191]\n" \
  "ds_add_u32 v12, v11\n" \
  "v_mfma_i32_16x16x64_i8 v[188:191], v[116:119], v[28:31], v[188:191]\n" \
  "v_mfma_i32_16x16x64_i8 v[156:159], v[112:115], v[28:31], v[156:159]\n" \
  "v_mfma_i32_16x16x64_i8 v[156:159], v[96:99], v[24:27], v[156:159]\n" \
  "s_waitcnt vmcnt(8) lgkmcnt(1)\n" \
  "v_mfma_i32_16x16x64_i8 v[128:131], v[32:35], v[16:19], v[128:131]\n" \
  "v_add_u32 v1, 0xc000, v1\n" \
  "ds_read_b64_tr_b8 v[24:25], v1 offset:0\n" \
  "v_mfma_i32_16x16x64_i8 v[128:131], v[48:51], v[20:23], v[128:131]\n" \
  "ds_read_b64_tr_b8 v[26:27], v1 offset:1024\n" \
  "v_mfma_i32_16x16x64_i8 v[160:163], v[52:55], v[20:23], v[160:163]\n" \
  "ds_read_b64_tr_b8 v[28:29], v1 offset:8192\n" \
  "v_mfma_i32_16x16x64_i8 v[160:163], v[36:39], v[16:19], v[160:163]\n" \
  "ds_read_b64_tr_b8 v[30:31], v1 offset:9216\n" \
  "v_mfma_i32_16x16x64_i8 v[192:195], v[40:43], v[16:19], v[192:195]\n" \
  "global_load_dwordx4 v[96:99], v8, s[40:41] offset:0 sc0 sc1\n" \
  "v_mfma_i32_16x16x64_i8 v[192:195], v[56:59], v[20:23], v[192:195]\n" \
  "global_load_dwordx4 v[100:103], v8, s[40:41] offset:1024 sc0 sc1\n" \
  "v_mfma_i32_16x16x64_i8 v[224:227], v[60:63], v[20:23], v[224:227]\n" \
  "v_mfma_i32_16x16x64_i8 v[224:227], v[44:47], v[16:19], v[224:227]\n" \
  "s_waitcnt lgkmcnt(0)\n" \
  "v_mfma_i32_16x16x64_i8 v[228:231], v[44:47], v[24:27], v[228:231]\n" \
  "v_add_u32 v2, 0xc000, v2\n" \
  "ds_read_b64_tr_b8 v[16:17], v2 offset:0\n" \
  "v_mfma_i32_16x16x64_i8 v[228:231], v[60:63], v[28:31], v[228:231]\n" \
  "ds_read_b64_tr_b8 v[18:19], v2 offset:1024\n" \
  "v_mfma_i32_16x16x64_i8 v[196:199], v[56:59], v[28:31], v[196:199]\n" \
  "ds_read_b64_tr_b8 v[20:21], v2 offset:8192\n" \
  "v_mfma_i32_16x16x64_i8 v[196:199], v[40:43], v[24:27], v[196:199]\n" \
  "ds_read_b64_tr_b8 v[22:23], v2 offset:9216\n" \
  "v_mfma_i32_16x16x64_i8 v[164:167], v[36:39], v[24:27], v[164:167]\n" \
  "global_load_dwordx4 v[104:107], v8, s[40:41] offset:2048 sc0 sc1\n" \
  "v_mfma_i32_16x16x64_i8 v[164:167], v[52:55], v[28:31], v[164:167]\n" \
  "global_load_dwordx4 v[108:111], v8, s[40:41] offset:3072 sc0 sc1\n" \
  "v_mfma_i32_16x16x64_i8 v[132:135], v[48:51], v[28:31], v[132:135]\n" \
  "v_add_u32 v8, 0x8000, v8\n" \
  "v_mfma_i32_16x16x64_i8 v[132:135], v[32:35], v[24:27], v[132:135]\n" \
  "s_waitcnt lgkmcnt(0)\n" \
  "v_mfma_i32_16x16x64_i8 v[136:139], v[32:35], v[16:19], v[136:139]\n" \
  "v_add_u32 v3, 0xc000, v3\n" \
  "ds_read_b64_tr_b8 v[24:25], v3 offset:0\n" \
  "v_mfma_i32_16x16x64_i8 v[136:139], v[48:51], v[20:23], v[136:139]\n" \
  "ds_read_b64_tr_b8 v[26:27], v3 offset:1024\n" \
  "v_mfma_i32_16x16x64_i8 v[168:171], v[52:55], v[20:23], v[168:171]\n" \
  "ds_read_b64_tr_b8 v[28:29], v3 offset:8192\n" \
  "v_mfma_i32_16x16x64_i8 v[168:171], v[36:39], v[16:19], v[168:171]\n" \
  "ds_read_b64_tr_b8 v[30:31], v3 offset:9216\n" \
  "v_mfma_i32_16x16x64_i8 v[200:203], v[40:43], v[16:19], v[200:203]\n" \
  "global_load_dwordx4 v[112:115], v8, s[40:41] offset:0 sc0 sc1\n" \
  "v_mfma_i32_16x16x64_i8 v[200:203], v[56:59], v[20:23], v[200:203]\n" \
  "global_load_dwordx4 v[116:119], v8, s[40:41] offset:1024 sc0 sc1\n" \
  "v_mfma_i32_16x16x64_i8 v[232:235], v[60:63], v[20:23], v[232:235]\n" \
  "v_mfma_i32_16x16x64_i8 v[232:235], v[44:47], v[16:19], v[232:235]\n" \
  "s_waitcnt lgkmcnt(0)\n" \
  "v_mfma_i32_16x16x64_i8 v[236:239], v[44:47], v[24:27], v[236:239]\n" \
  "v_add_u32 v4, 0xc000, v4\n" \
  "ds_read_b64_tr_b8 v[16:17], v4 offset:0\n" \
  "v_mfma_i32_16x16x64_i8 v[236:239], v[60:63], v[28:31], v[236:239]\n" \
  "ds_read_b64_tr_b8 v[18:19], v4 offset:1024\n" \
  "v_mfma_i32_16x16x64_i8 v[204:207], v[56:59], v[28:31], v[204:207]\n" \
  "ds_read_b64_tr_b8 v[20:21], v4 offset:8192\n" \
  "v_mfma_i32_16x16x64_i8 v[204:207], v[40:43], v[24:27], v[204:207]\n" \
  "ds_read_b64_tr_b8 v[22:23], v4 offset:9216\n" \
  "v_mfma_i32_16x16x64_i8 v[172:175], v[36:39], v[24:27], v[172:175]\n" \
  "global_load_dwordx4 v[120:123], v8, s[40:41] offset:2048 sc0 sc1\n" \
  "v_mfma_i32_16x16x64_i8 v[172:175], v[52:55], v[28:31], v[172:175]\n" \
  "global_load_dwordx4 v[124:127], v8, s[40:41] offset:3072 sc0 sc1\n" \
  "v_mfma_i32_16x16x64_i8 v[140:143], v[48:51], v[28:31], v[140:143]\n" \
  "v_add_u32 v8, 0x8000, v8\n" \
  "v_mfma_i32_16x16x64_i8 v[140:143], v[32:35], v[24:27], v[140:143]\n" \
  "s_waitcnt lgkmcnt(0)\n" \
  "v_mfma_i32_16x16x64_i8 v[144:147], v[32:35], v[16:19], v[144:147]\n" \
  "v_add_u32 v5, 0xc000, v5\n" \
  "ds_read_b64_tr_b8 v[24:25], v5 offset:0\n" \
  "v_mfma_i32_16x16x64_i8 v[144:147], v[48:51], v[20:23], v[144:147]\n" \
  "ds_read_b64_tr_b8 v[26:27], v5 offset:1024\n" \
  "v_mfma_i32_16x16x64_i8 v[176:179], v[52:55], v[20:23], v[176:179]\n" \
  "ds_read_b64_tr_b8 v[28:29], v5 offset:8192\n" \
  "v_mfma_i32_16x16x64_i8 v[176:179], v[36:39], v[16:19], v[176:179]\n" \
  "ds_read_b64_tr_b8 v[30:31], v5 offset:9216\n" \
  "v_mfma_i32_16x16x64_i8 v[208:211], v[40:43], v[16:19], v[208:211]\n" \
  "ds_read_b32 v10, v9\n" \
  "v_mfma_i32_16x16x64_i8 v[208:211], v[56:59], v[20:23], v[208:211]\n" \
  "v_mfma_i32_16x16x64_i8 v[240:243], v[60:63], v[20:23], v[240:243]\n" \
  "v_mfma_i32_16x16x64_i8 v[240:243], v[44:47], v[16:19], v[240:243]\n" \
  "s_waitcnt lgkmcnt(1)\n" \
  "v_mfma_i32_16x16x64_i8 v[244:247], v[44:47], v[24:27], v[244:247]\n" \
  "v_add_u32 v6, 0xc000, v6\n" \
  "ds_read_b64_tr_b8 v[16:17], v6 offset:0\n" \
  "v_mfma_i32_16x16x64_i8 v[244:247], v[60:63], v[28:31], v[244:247]\n" \
  "ds_read_b64_tr_b8 v[18:19], v6 offset:1024\n" \
  "v_mfma_i32_16x16x64_i8 v[212:215], v[56:59], v[28:31], v[212:215]\n" \
  "ds_read_b64_tr_b8 v[20:21], v6 offset:8192\n" \
  "v_mfma_i32_16x16x64_i8 v[212:215], v[40:43], v[24:27], v[212:215]\n" \
  "ds_read_b64_tr_b8 v[22:23], v6 offset:9216\n" \
  "v_mfma_i32_16x16x64_i8 v[180:183], v[36:39], v[24:27], v[180:183]\n" \
  "v_mfma_i32_16x16x64_i8 v[180:183], v[52:55], v[28:31], v[180:183]\n" \
  "v_mfma_i32_16x16x64_i8 v[148:151], v[48:51], v[28:31], v[148:151]\n" \
  "v_mfma_i32_16x16x64_i8 v[148:151], v[32:35], v[24:27], v[148:151]\n" \
  "s_waitcnt lgkmcnt(0)\n" \
  "v_mfma_i32_16x16x64_i8 v[152:155], v[32:35], v[16:19], v[152:155]\n" \
  "v_add_u32 v7, 0xc000, v7\n" \
  "ds_read_b64_tr_b8 v[24:25], v7 offset:0\n" \
  "v_mfma_i32_16x16x64_i8 v[152:155], v[48:51], v[20:23], v[152:155]\n" \
  "ds_read_b64_tr_b8 v[26:27], v7 offset:1024\n" \
  "v_mfma_i32_16x16x64_i8 v[184:187], v[52:55], v[20:23], v[184:187]\n" \
  "ds_read_b64_tr_b8 v[28:29], v7 offset:8192\n" \
  "v_mfma_i32_16x16x64_i8 v[184:187], v[36:39], v[16:19], v[184:187]\n" \
  "ds_read_b64_tr_b8 v[30:31], v7 offset:9216\n" \
  "v_mfma_i32_16x16x64_i8 v[216:219], v[40:43], v[16:19], v[216:219]\n" \
  "v_mfma_i32_16x16x64_i8 v[216:219], v[56:59], v[20:23], v[216:219]\n" \
  "v_mfma_i32_16x16x64_i8 v[248:251], v[60:63], v[20:23], v[248:251]\n" \
  "v_cmp_gt_u32 vcc, 28, v10\n" \
  "v_mfma_i32_16x16x64_i8 v[248:251], v[44:47], v[16:19], v[248:251]\n" \
  "s_waitcnt lgkmcnt(0)\n" \
  "v_mfma_i32_16x16x64_i8 v[252:255], v[44:47], v[24:27], v[252:255]\n" \
  "s_cbranch_vccnz LS12_%=\n" \
  "LB12_%=:\n" \
  "ds_read_b64_tr_b8 v[16:17], v0 offset:16384\n" \
  "v_mfma_i32_16x16x64_i8 v[252:255], v[60:63], v[28:31], v[252:255]\n" \
  "ds_read_b64_tr_b8 v[18:19], v0 offset:17408\n" \
  "v_mfma_i32_16x16x64_i8 v[220:223], v[56:59], v[28:31], v[220:223]\n" \
  "ds_read_b64_tr_b8 v[20:21], v0 offset:24576\n" \
  "v_mfma_i32_16x16x64_i8 v[220:223], v[40:43], v[24:27], v[220:223]\n" \
  "ds_read_b64_tr_b8 v[22:23], v0 offset:25600\n" \
  "v_mfma_i32_16x16x64_i8 v[188:191], v[36:39], v[24:27], v[188:191]\n" \
  "ds_add_u32 v12, v11\n" \
  "v_mfma_i32_16x16x64_i8 v[188:191], v[52:55], v[28:31], v[188:191]\n" \
  "v_mfma_i32_16x16x64_i8 v[156:159], v[48:51], v[28:31], v[156:159]\n" \
  "v_mfma_i32_16x16x64_i8 v[156:159], v[32:35], v[24:27], v[156:159]\n" \
  "s_waitcnt vmcnt(8) lgkmcnt(1)\n" \
  "v_mfma_i32_16x16x64_i8 v[128:131], v[64:67], v[16:19], v[128:131]\n" \
  "ds_read_b64_tr_b8 v[24:25], v1 offset:16384\n" \
  "v_mfma_i32_16x16x64_i8 v[128:131], v[80:83], v[20:23], v[128:131]\n" \
  "ds_read_b64_tr_b8 v[26:27], v1 offset:17408\n" \
  "v_mfma_i32_16x16x64_i8 v[160:163], v[84:87], v[20:23], v[160:163]\n" \
  "ds_read_b64_tr_b8 v[28:29], v1 offset:24576\n" \
  "v_mfma_i32_16x16x64_i8 v[160:163], v[68:71], v[16:19], v[160:163]\n" \
  "ds_read_b64_tr_b8 v[30:31], v1 offset:25600\n" \
  "v_mfma_i32_16x16x64_i8 v[192:195], v[72:75], v[16:19], v[192:195]\n" \
  "global_load_dwordx4 v[32:35], v8, s[40:41] offset:0 sc0 sc1\n" \
  "v_mfma_i32_16x16x64_i8 v[192:195], v[88:91], v[20:23], v[192:195]\n" \
  "global_load_dwordx4 v[36:39], v8, s[40:41] offset:1024 sc0 sc1\n" \
  "v_mfma_i32_16x16x64_i8 v[224:227], v[92:95], v[20:23], v[224:227]\n" \
  "v_mfma_i32_16x16x64_i8 v[224:227], v[76:79], v[16:19], v[224:227]\n" \
  "s_waitcnt lgkmcnt(0)\n" \
  "v_mfma_i32_16x16x64_i8 v[228:231], v[76:79], v[24:27], v[228:231]\n" \
  "ds_read_b64_tr_b8 v[16:17], v2 offset:16384\n" \
  "v_mfma_i32_16x16x64_i8 v[228:231], v[92:95], v[28:31], v[228:231]\n" \
  "ds_read_b64_tr_b8 v[18:19], v2 offset:17408\n" \
  "v_mfma_i32_16x16x64_i8 v[196:199], v[88:91], v[28:31], v[196:199]\n" \
  "ds_read_b64_tr_b8 v[20:21], v2 offset:24576\n" \
  "v_mfma_i32_16x16x64_i8 v[196:199], v[72:75], v[24:27], v[196:199]\n" \
  "ds_read_b64_tr_b8 v[22:23], v2 offset:25600\n" \
  "v_mfma_i32_16x16x64_i8 v[164:167], v[68:71], v[24:27], v[164:167]\n" \
  "global_load_dwordx4 v[40:43], v8, s[40:41] offset:2048 sc0 sc1\n" \
  "v_mfma_i32_16x16x64_i8 v[164:167], v[84:87], v[28:31], v[164:167]\n" \
  "global_load_dwordx4 v[44:47], v8, s[40:41] offset:3072 sc0 sc1\n" \
  "v_mfma_i32_16x16x64_i8 v[132:135], v[80:83], v[28:31], v[132:135]\n" \
  "v_add_u32 v8, 0x8000, v8\n" \
  "v_mfma_i32_16x16x64_i8 v[132:135], v[64:67], v[24:27], v[132:135]\n" \
  "s_waitcnt lgkmcnt(0)\n" \
  "v_mfma_i32_16x16x64_i8 v[136:139], v[64:67], v[16:19], v[136:139]\n" \
  "ds_read_b64_tr_b8 v[24:25], v3 offset:16384\n" \
  "v_mfma_i32_16x16x64_i8 v[136:139], v[80:83], v[20:23], v[136:139]\n" \
  "ds_read_b64_tr_b8 v[26:27], v3 offset:17408\n" \
  "v_mfma_i32_16x16x64_i8 v[168:171], v[84:87], v[20:23], v[168:171]\n" \
  "ds_read_b64_tr_b8 v[28:29], v3 offset:24576\n" \
  "v_mfma_i32_16x16x64_i8 v[168:171], v[68:71], v[16:19], v[168:171]\n" \
  "ds_read_b64_tr_b8 v[30:31], v3 offset:25600\n" \
  "v_mfma_i32_16x16x64_i8 v[200:203], v[72:75], v[16:19], v[200:203]\n" \
  "global_load_dwordx4 v[48:51], v8, s[40:41] offset:0 sc0 sc1\n" \
  "v_mfma_i32_16x16x64_i8 v[200:203], v[88:91], v[20:23], v[200:203]\n" \
  "global_load_dwordx4 v[52:55], v8, s[40:41] offset:1024 sc0 sc1\n" \
  "v_mfma_i32_16x16x64_i8 v[232:235], v[92:95], v[20:23], v[232:235]\n" \
  "v_mfma_i32_16x16x64_i8 v[232:235], v[76:79], v[16:19], v[232:235]\n" \
  "s_waitcnt lgkmcnt(0)\n" \
  "v_mfma_i32_16x16x64_i8 v[236:239], v[76:79], v[24:27], v[236:239]\n" \
  "ds_read_b64_tr_b8 v[16:17], v4 offset:16384\n" \
  "v_mfma_i32_16x16x64_i8 v[236:239], v[92:95], v[28:31], v[236:239]\n" \
  "ds_read_b64_tr_b8 v[18:19], v4 offset:17408\n" \
  "v_mfma_i32_16x16x64_i8 v[204:207], v[88:91], v[28:31], v[204:207]\n" \
  "ds_read_b64_tr_b8 v[20:21], v4 offset:24576\n" \
  "v_mfma_i32_16x16x64_i8 v[204:207], v[72:75], v[24:27], v[204:207]\n" \
  "ds_read_b64_tr_b8 v[22:23], v4 offset:25600\n" \
  "v_mfma_i32_16x16x64_i8 v[172:175], v[68:71], v[24:27], v[172:175]\n" \
  "global_load_dwordx4 v[56:59], v8, s[40:41] offset:2048 sc0 sc1\n" \
  "v_mfma_i32_16x16x64_i8 v[172:175], v[84:87], v[28:31], v[172:175]\n" \
  "global_load_dwordx4 v[60:63], v8, s[40:41] offset:3072 sc0 sc1\n" \
  "v_mfma_i32_16x16x64_i8 v[140:143], v[80:83], v[28:31], v[140:143]\n" \
  "v_add_u32 v8, 0x8000, v8\n" \
  "v_mfma_i32_16x16x64_i8 v[140:143], v[64:67], v[24:27], v[140:143]\n" \
  "s_waitcnt lgkmcnt(0)\n" \
  "v_mfma_i32_16x16x64_i8 v[144:147], v[64:67], v[16:19], v[144:147]\n" \
  "ds_read_b64_tr_b8 v[24:25], v5 offset:16384\n" \
  "v_mfma_i32_16x16x64_i8 v[144:147], v[80:83], v[20:23], v[144:147]\n" \
  "ds_read_b64_tr_b8 v[26:27], v5 offset:17408\n" \
  "v_mfma_i32_16x16x64_i8 v[176:179], v[84:87], v[20:23], v[176:179]\n" \
  "ds_read_b64_tr_b8 v[28:29], v5 offset:24576\n" \
  "v_mfma_i32_16x16x64_i8 v[176:179], v[68:71], v[16:19], v[176:179]\n" \
  "ds_read_b64_tr_b8 v[30:31], v5 offset:25600\n" \
  "v_mfma_i32_16x16x64_i8 v[208:211], v[72:75], v[16:19], v[208:211]\n" \
  "ds_read_b32 v10, v9\n" \
  "v_mfma_i32_16x16x64_i8 v[208:211], v[88:91], v[20:23], v[208:211]\n" \
  "v_mfma_i32_16x16x64_i8 v[240:243], v[92:95], v[20:23], v[240:243]\n" \
  "v_mfma_i32_16x16x64_i8 v[240:243], v[76:79], v[16:19], v[240:243]\n" \
  "s_waitcnt lgkmcnt(1)\n" \
  "v_mfma_i32_16x16x64_i8 v[244:247], v[76:79], v[24:27], v[244:247]\n" \
  "ds_read_b64_tr_b8 v[16:17], v6 offset:16384\n" \
  "v_mfma_i32_16x16x64_i8 v[244:247], v[92:95], v[28:31], v[244:247]\n" \
  "ds_read_b64_tr_b8 v[18:19], v6 offset:17408\n" \
  "v_mfma_i32_16x16x64_i8 v[212:215], v[88:91], v[28:31], v[212:215]\n" \
  "ds_read_b64_tr_b8 v[20:21], v6 offset:24576\n" \
  "v_mfma_i32_16x16x64_i8 v[212:215], v[72:75], v[24:27], v[212:215]\n" \
  "ds_read_b64_tr_b8 v[22:23], v6 offset:25600\n" \
  "v_mfma_i32_16x16x64_i8 v[180:183], v[68:71], v[24:27], v[180:183]\n" \
  "v_mfma_i32_16x16x64_i8 v[180:183], v[84:87], v[28:31], v[180:183]\n" \
  "v_mfma_i32_16x16x64_i8 v[148:151], v[80:83], v[28:31], v[148:151]\n" \
  "v_mfma_i32_16x16x64_i8 v[148:151], v[64:67], v[24:27], v[148:151]\n" \
  "s_waitcnt lgkmcnt(0)\n" \
  "v_mfma_i32_16x16x64_i8 v[152:155], v[64:67], v[16:19], v[152:155]\n" \
  "ds_read_b64_tr_b8 v[24:25], v7 offset:16384\n" \
  "v_mfma_i32_16x16x64_i8 v[152:155], v[80:83], v[20:23], v[152:155]\n" \
  "ds_read_b64_tr_b8 v[26:27], v7 offset:17408\n" \
  "v_mfma_i32_16x16x64_i8 v[184:187], v[84:87], v[20:23], v[184:187]\n" \
  "ds_read_b64_tr_b8 v[28:29], v7 offset:24576\n" \
  "v_mfma_i32_16x16x64_i8 v[184:187], v[68:71], v[16:19], v[184:187]\n" \
  "ds_read_b64_tr_b8 v[30:31], v7 offset:25600\n" \
  "v_mfma_i32_16x16x64_i8 v[216:219], v[72:75], v[16:19], v[216:219]\n" \
  "v_mfma_i32_16x16x64_i8 v[216:219], v[88:91], v[20:23], v[216:219]\n" \
  "v_mfma_i32_16x16x64_i8 v[248:251], v[92:95], v[20:23], v[248:251]\n" \
  "v_cmp_gt_u32 vcc, 30, v10\n" \
  "v_mfma_i32_16x16x64_i8 v[248:251], v[76:79], v[16:19], v[248:251]\n" \
  "s_waitcnt lgkmcnt(0)\n" \
  "v_mfma_i32_16x16x64_i8 v[252:255], v[76:79], v[24:27], v[252:255]\n" \
  "s_cbranch_vccnz LS13_%=\n" \
  "LB13_%=:\n" \
  "ds_read_b64_tr_b8 v[16:17], v0 offset:32768\n" \
  "v_mfma_i32_16x16x64_i8 v[252:255], v[92:95], v[28:31], v[252:255]\n" \
  "ds_read_b64_tr_b8 v[18:19], v0 offset:33792\n" \
  "v_mfma_i32_16x16x64_i8 v[220:223], v[88:91], v[28:31], v[220:223]\n" \
  "ds_read_b64_tr_b8 v[20:21], v0 offset:40960\n" \
  "v_mfma_i32_16x16x64_i8 v[220:223], v[72:75], v[24:27], v[220:223]\n" \
  "ds_read_b64_tr_b8 v[22:23], v0 offset:41984\n" \
  "v_mfma_i32_16x16x64_i8 v[188:191], v[68:71], v[24:27], v[188:191]\n" \
  "ds_add_u32 v12, v11\n" \
  "v_mfma_i32_16x16x64_i8 v[188:191], v[84:87], v[28:31], v[188:191]\n" \
  "v_mfma_i32_16x16x64_i8 v[156:159], v[80:83], v[28:31], v[156:159]\n" \
  "v_mfma_i32_16x16x64_i8 v[156:159], v[64:67], v[24:27], v[156:159]\n" \
  "s_waitcnt vmcnt(8) lgkmcnt(1)\n" \
  "v_mfma_i32_16x16x64_i8 v[128:131], v[96:99], v[16:19], v[128:131]\n" \
  "ds_read_b64_tr_b8 v[24:25], v1 offset:32768\n" \
  "v_mfma_i32_16x16x64_i8 v[128:131], v[112:115], v[20:23], v[128:131]\n" \
  "ds_read_b64_tr_b8 v[26:27], v1 offset:33792\n" \
  "v_mfma_i32_16x16x64_i8 v[160:163], v[116:119], v[20:23], v[160:163]\n" \
  "ds_read_b64_tr_b8 v[28:29], v1 offset:40960\n" \
  "v_mfma_i32_16x16x64_i8 v[160:163], v[100:103], v[16:19], v[160:163]\n" \
  "ds_read_b64_tr_b8 v[30:31], v1 offset:41984\n" \
  "v_mfma_i32_16x16x64_i8 v[192:195], v[104:107], v[16:19], v[192:195]\n" \
  "global_load_dwordx4 v[64:67], v8, s[40:41] offset:0 sc0 sc1\n" \
  "v_mfma_i32_16x16x64_i8 v[192:195], v[120:123], v[20:23], v[192:195]\n" \
  "global_load_dwordx4 v[68:71], v8, s[40:41] offset:1024 sc0 sc1\n" \
  "v_mfma_i32_16x16x64_i8 v[224:227], v[124:127], v[20:23], v[224:227]\n" \
  "v_mfma_i32_16x16x64_i8 v[224:227], v[108:111], v[16:19], v[224:227]\n" \
  "s_waitcnt lgkmcnt(0)\n" \
  "v_mfma_i32_16x16x64_i8 v[228:231], v[108:111], v[24:27], v[228:231]\n" \
  "ds_read_b64_tr_b8 v[16:17], v2 offset:32768\n" \
  "v_mfma_i32_16x16x64_i8 v[228:231], v[124:127], v[28:31], v[228:231]\n" \
  "ds_read_b64_tr_b8 v[18:19], v2 offset:33792\n" \
  "v_mfma_i32_16x16x64_i8 v[196:199], v[120:123], v[28:31], v[196:199]\n" \
  "ds_read_b64_tr_b8 v[20:21], v2 offset:40960\n" \
  "v_mfma_i32_16x16x64_i8 v[196:199], v[104:107], v[24:27], v[196:199]\n" \
  "ds_read_b64_tr_b8 v[22:23], v2 offset:41984\n" \
  "v_mfma_i32_16x16x64_i8 v[164:167], v[100:103], v[24:27], v[164:167]\n" \
  "global_load_dwordx4 v[72:75], v8, s[40:41] offset:2048 sc0 sc1\n" \
  "v_mfma_i32_16x16x64_i8 v[164:167], v[116:119], v[28:31], v[164:167]\n" \
  "global_load_dwordx4 v[76:79], v8, s[40:41] offset:3072 sc0 sc1\n" \
  "v_mfma_i32_16x16x64_i8 v[132:135], v[112:115], v[28:31], v[132:135]\n" \
  "v_add_u32 v8, 0x8000, v8\n" \
  "v_mfma_i32_16x16x64_i8 v[132:135], v[96:99], v[24:27], v[132:135]\n" \
  "s_waitcnt lgkmcnt(0)\n" \
  "v_mfma_i32_16x16x64_i8 v[136:139], v[96:99], v[16:19], v[136:139]\n" \
  "ds_read_b64_tr_b8 v[24:25], v3 offset:32768\n" \
  "v_mfma_i32_16x16x64_i8 v[136:139], v[112:115], v[20:23], v[136:139]\n" \
  "ds_read_b64_tr_b8 v[26:27], v3 offset:33792\n" \
  "v_mfma_i32_16x16x64_i8 v[168:171], v[116:119], v[20:23], v[168:171]\n" \
  "ds_read_b64_tr_b8 v[28:29], v3 offset:40960\n" \
  "v_mfma_i32_16x16x64_i8 v[168:171], v[100:103], v[16:19], v[168:171]\n" \
  "ds_read_b64_tr_b8 v[30:31], v3 offset:41984\n" \
  "v_mfma_i32_16x16x64_i8 v[200:203], v[104:107], v[16:19], v[200:203]\n" \
  "global_load_dwordx4 v[80:83], v8, s[40:41] offset:0 sc0 sc1\n" \
  "v_mfma_i32_16x16x64_i8 v[200:203], v[120:123], v[20:23], v[200:203]\n" \
  "global_load_dwordx4 v[84:87], v8, s[40:41] offset:1024 sc0 sc1\n" \
  "v_mfma_i32_16x16x64_i8 v[232:235], v[124:127], v[20:23], v[232:235]\n" \
  "v_mfma_i32_16x16x64_i8 v[232:235], v[108:111], v[16:19], v[232:235]\n" \
  "s_waitcnt lgkmcnt(0)\n" \
  "v_mfma_i32_16x16x64_i8 v[236:239], v[108:111], v[24:27], v[236:239]\n" \
  "ds_read_b64_tr_b8 v[16:17], v4 offset:32768\n" \
  "v_mfma_i32_16x16x64_i8 v[236:239], v[124:127], v[28:31], v[236:239]\n" \
  "ds_read_b64_tr_b8 v[18:19], v4 offset:33792\n" \
  "v_mfma_i32_16x16x64_i8 v[204:207], v[120:123], v[28:31], v[204:207]\n" \
  "ds_read_b64_tr_b8 v[20:21], v4 offset:40960\n" \
  "v_mfma_i32_16x16x64_i8 v[204:207], v[104:107], v[24:27], v[204:207]\n" \
  "ds_read_b64_tr_b8 v[22:23], v4 offset:41984\n" \
  "v_mfma_i32_16x16x64_i8 v[172:175], v[100:103], v[24:27], v[172:175]\n" \
  "global_load_dwordx4 v[88:91], v8, s[40:41] offset:2048 sc0 sc1\n" \
  "v_mfma_i32_16x16x64_i8 v[172:175], v[116:119], v[28:31], v[172:175]\n" \
  "global_load_dwordx4 v[92:95], v8, s[40:41] offset:3072 sc0 sc1\n" \
  "v_mfma_i32_16x16x64_i8 v[140:143], v[112:115], v[28:31], v[140:143]\n" \
  "v_add_u32 v8, 0x8000, v8\n" \
  "v_mfma_i32_16x16x64_i8 v[140:143], v[96:99], v[24:27], v[140:143]\n" \
  "s_waitcnt lgkmcnt(0)\n" \
  "v_mfma_i32_16x16x64_i8 v[144:147], v[96:99], v[16:19], v[144:147]\n" \
  "ds_read_b64_tr_b8 v[24:25], v5 offset:32768\n" \
  "v_mfma_i32_16x16x64_i8 v[144:147], v[112:115], v[20:23], v[144:147]\n" \
  "ds_read_b64_tr_b8 v[26:27], v5 offset:33792\n" \
  "v_mfma_i32_16x16x64_i8 v[176:179], v[116:119], v[20:23], v[176:179]\n" \
  "ds_read_b64_tr_b8 v[28:29], v5 offset:40960\n" \
  "v_mfma_i32_16x16x64_i8 v[176:179], v[100:103], v[16:19], v[176:179]\n" \
  "ds_read_b64_tr_b8 v[30:31], v5 offset:41984\n" \
  "v_mfma_i32_16x16x64_i8 v[208:211], v[104:107], v[16:19], v[208:211]\n" \
  "ds_read_b32 v10, v9\n" \
  "v_mfma_i32_16x16x64_i8 v[208:211], v[120:123], v[20:23], v[208:211]\n" \
  "v_mfma_i32_16x16x64_i8 v[240:243], v[124:127], v[20:23], v[240:243]\n" \
  "v_mfma_i32_16x16x64_i8 v[240:243], v[108:111], v[16:19], v[240:243]\n" \
  "s_waitcnt lgkmcnt(1)\n" \
  "v_mfma_i32_16x16x64_i8 v[244:247], v[108:111], v[24:27], v[244:247]\n" \
  "ds_read_b64_tr_b8 v[16:17], v6 offset:32768\n" \
  "v_mfma_i32_16x16x64_i8 v[244:247], v[124:127], v[28:31], v[244:247]\n" \
  "ds_read_b64_tr_b8 v[18:19], v6 offset:33792\n" \
  "v_mfma_i32_16x16x64_i8 v[212:215], v[120:123], v[28:31], v[212:215]\n" \
  "ds_read_b64_tr_b8 v[20:21], v6 offset:40960\n" \
  "v_mfma_i32_16x16x64_i8 v[212:215], v[104:107], v[24:27], v[212:215]\n" \
  "ds_read_b64_tr_b8 v[22:23], v6 offset:41984\n" \
  "v_mfma_i32_16x16x64_i8 v[180:183], v[100:103], v[24:27], v[180:183]\n" \
  "v_mfma_i32_16x16x64_i8 v[180:183], v[116:119], v[28:31], v[180:183]\n" \
  "v_mfma_i32_16x16x64_i8 v[148:151], v[112:115], v[28:31], v[148:151]\n" \
  "v_mfma_i32_16x16x64_i8 v[148:151], v[96:99], v[24:27], v[148:151]\n" \
  "s_waitcnt lgkmcnt(0)\n" \
  "v_mfma_i32_16x16x64_i8 v[152:155], v[96:99], v[16:19], v[152:155]\n" \
  "ds_read_b64_tr_b8 v[24:25], v7 offset:32768\n" \
  "v_mfma_i32_16x16x64_i8 v[152:155], v[112:115], v[20:23], v[152:155]\n" \
  "ds_read_b64_tr_b8 v[26:27], v7 offset:33792\n" \
  "v_mfma_i32_16x16x64_i8 v[184:187], v[116:119], v[20:23], v[184:187]\n" \
  "ds_read_b64_tr_b8 v[28:29], v7 offset:40960\n" \
  "v_mfma_i32_16x16x64_i8 v[184:187], v[100:103], v[16:19], v[184:187]\n" \
  "ds_read_b64_tr_b8 v[30:31], v7 offset:41984\n" \
  "v_mfma_i32_16x16x64_i8 v[216:219], v[104:107], v[16:19], v[216:219]\n" \
  "v_mfma_i32_16x16x64_i8 v[216:219], v[120:123], v[20:23], v[216:219]\n" \
  "v_mfma_i32_16x16x64_i8 v[248:251], v[124:127], v[20:23], v[248:251]\n" \
  "v_cmp_gt_u32 vcc, 32, v10\n" \
  "v_mfma_i32_16x16x64_i8 v[248:251], v[108:111], v[16:19], v[248:251]\n" \
  "s_waitcnt lgkmcnt(0)\n" \
  "v_mfma_i32_16x16x64_i8 v[252:255], v[108:111], v[24:27], v[252:255]\n" \
  "s_cbranch_vccnz LS14_%=\n" \
  "LB14_%=:\n" \
  "v_add_u32 v0, 0xc000, v0\n" \
  "ds_read_b64_tr_b8 v[16:17], v0 offset:0\n" \
  "v_mfma_i32_16x16x64_i8 v[252:255], v[124:127], v[28:31], v[252:255]\n" \
  "ds_read_b64_tr_b8 v[18:19], v0 offset:1024\n" \
  "v_mfma_i32_16x16x64_i8 v[220:223], v[120:123], v[28:31], v[220:223]\n" \
  "ds_read_b64_tr_b8 v[20:21], v0 offset:8192\n" \
  "v_mfma_i32_16x16x64_i8 v[220:223], v[104:107], v[24:27], v[220:223]\n" \
  "ds_read_b64_tr_b8 v[22:23], v0 offset:9216\n" \
  "v_mfma_i32_16x16x64_i8 v[188:191], v[100:103], v[24:27], v[188:191]\n" \
  "ds_add_u32 v12, v11\n" \
  "v_mfma_i32_16x16x64_i8 v[188:191], v[116:119], v[28:31], v[188:191]\n" \
  "v_mfma_i32_16x16x64_i8 v[156:159], v[112:115], v[28:31], v[156:159]\n" \
  "v_mfma_i32_16x16x64_i8 v[156:159], v[96:99], v[24:27], v[156:159]\n" \
  "s_waitcnt vmcnt(8) lgkmcnt(1)\n" \
  "v_mfma_i32_16x16x64_i8 v[128:131], v[32:35], v[16:19], v[128:131]\n" \
  "v_add_u32 v1, 0xc000, v1\n" \
  "ds_read_b64_tr_b8 v[24:25], v1 offset:0\n" \
  "v_mfma_i32_16x16x64_i8 v[128:131], v[48:51], v[20:23], v[128:131]\n" \
  "ds_read_b64_tr_b8 v[26:27], v1 offset:1024\n" \
  "v_mfma_i32_16x16x64_i8 v[160:163], v[52:55], v[20:23], v[160:163]\n" \
  "ds_read_b64_tr_b8 v[28:29], v1 offset:8192\n" \
  "v_mfma_i32_16x16x64_i8 v[160:163], v[36:39], v[16:19], v[160:163]\n" \
  "ds_read_b64_tr_b8 v[30:31], v1 offset:9216\n" \
  "v_mfma_i32_16x16x64_i8 v[192:195], v[40:43], v[16:19], v[192:195]\n" \
  "global_load_dwordx4 v[96:99], v8, s[40:41] offset:0 sc0 sc1\n" \
  "v_mfma_i32_16x16x64_i8 v[192:195], v[56:59], v[20:23], v[192:195]\n" \
  "global_load_dwordx4 v[100:103], v8, s[40:41] offset:1024 sc0 sc1\n" \
  "v_mfma_i32_16x16x64_i8 v[224:227], v[60:63], v[20:23], v[224:227]\n" \
  "v_mfma_i32_16x16x64_i8 v[224:227], v[44:47], v[16:19], v[224:227]\n" \
  "s_waitcnt lgkmcnt(0)\n" \
  "v_mfma_i32_16x16x64_i8 v[228:231], v[44:47], v[24:27], v[228:231]\n" \
  "v_add_u32 v2, 0xc000, v2\n" \
  "ds_read_b64_tr_b8 v[16:17], v2 offset:0\n" \
  "v_mfma_i32_16x16x64_i8 v[228:231], v[60:63], v[28:31], v[228:231]\n" \
  "ds_read_b64_tr_b8 v[18:19], v2 offset:1024\n" \
  "v_mfma_i32_16x16x64_i8 v[196:199], v[56:59], v[28:31], v[196:199]\n" \
  "ds_read_b64_tr_b8 v[20:21], v2 offset:8192\n" \
  "v_mfma_i32_16x16x64_i8 v[196:199], v[40:43], v[24:27], v[196:199]\n" \
  "ds_read_b64_tr_b8 v[22:23], v2 offset:9216\n" \
  "v_mfma_i32_16x16x64_i8 v[164:167], v[36:39], v[24:27], v[164:167]\n" \
  "global_load_dwordx4 v[104:107], v8, s[40:41] offset:2048 sc0 sc1\n" \
  "v_mfma_i32_16x16x64_i8 v[164:167], v[52:55], v[28:31], v[164:167]\n" \
  "global_load_dwordx4 v[108:111], v8, s[40:41] offset:3072 sc0 sc1\n" \
  "v_mfma_i32_16x16x64_i8 v[132:135], v[48:51], v[28:31], v[132:135]\n" \
  "v_add_u32 v8, 0x8000, v8\n" \
  "v_mfma_i32_16x16x64_i8 v[132:135], v[32:35], v[24:27], v[132:135]\n" \
  "s_waitcnt lgkmcnt(0)\n" \
  "v_mfma_i32_16x16x64_i8 v[136:139], v[32:35], v[16:19], v[136:139]\n" \
  "v_add_u32 v3, 0xc000, v3\n" \
  "ds_read_b64_tr_b8 v[24:25], v3 offset:0\n" \
  "v_mfma_i32_16x16x64_i8 v[136:139], v[48:51], v[20:23], v[136:139]\n" \
  "ds_read_b64_tr_b8 v[26:27], v3 offset:1024\n" \
  "v_mfma_i32_16x16x64_i8 v[168:171], v[52:55], v[20:23], v[168:171]\n" \
  "ds_read_b64_tr_b8 v[28:29], v3 offset:8192\n" \
  "v_mfma_i32_16x16x64_i8 v[168:171], v[36:39], v[16:19], v[168:171]\n" \
  "ds_read_b64_tr_b8 v[30:31], v3 offset:9216\n" \
  "v_mfma_i32_16x16x64_i8 v[200:203], v[40:43], v[16:19], v[200:203]\n" \
  "global_load_dwordx4 v[112:115], v8, s[40:41] offset:0 sc0 sc1\n" \
  "v_mfma_i32_16x16x64_i8 v[200:203], v[56:59], v[20:23], v[200:203]\n" \
  "global_load_dwordx4 v[116:119], v8, s[40:41] offset:1024 sc0 sc1\n" \
  "v_mfma_i32_16x16x64_i8 v[232:235], v[60:63], v[20:23], v[232:235]\n" \
  "v_mfma_i32_16x16x64_i8 v[232:235], v[44:47], v[16:19], v[232:235]\n" \
  "s_waitcnt lgkmcnt(0)\n" \
  "v_mfma_i32_16x16x64_i8 v[236:239], v[44:47], v[24:27], v[236:239]\n" \
  "v_add_u32 v4, 0xc000, v4\n" \
  "ds_read_b64_tr_b8 v[16:17], v4 offset:0\n" \
  "v_mfma_i32_16x16x64_i8 v[236:239], v[60:63], v[28:31], v[236:239]\n" \
  "ds_read_b64_tr_b8 v[18:19], v4 offset:1024\n" \
  "v_mfma_i32_16x16x64_i8 v[204:207], v[56:59], v[28:31], v[204:207]\n" \
  "ds_read_b64_tr_b8 v[20:21], v4 offset:8192\n" \
  "v_mfma_i32_16x16x64_i8 v[204:207], v[40:43], v[24:27], v[204:207]\n" \
  "ds_read_b64_tr_b8 v[22:23], v4 offset:9216\n" \
  "v_mfma_i32_16x16x64_i8 v[172:175], v[36:39], v[24:27], v[172:175]\n" \
  "global_load_dwordx4 v[120:123], v8, s[40:41] offset:2048 sc0 sc1\n" \
  "v_mfma_i32_16x16x64_i8 v[172:175], v[52:55], v[28:31], v[172:175]\n" \
  "global_load_dwordx4 v[124:127], v8, s[40:41] offset:3072 sc0 sc1\n" \
  "v_mfma_i32_16x16x64_i8 v[140:143], v[48:51], v[28:31], v[140:143]\n" \
  "v_add_u32 v8, 0x8000, v8\n" \
  "v_mfma_i32_16x16x64_i8 v[140:143], v[32:35], v[24:27], v[140:143]\n" \
  "s_waitcnt lgkmcnt(0)\n" \
  "v_mfma_i32_16x16x64_i8 v[144:147], v[32:35], v[16:19], v[144:147]\n" \
  "v_add_u32 v5, 0xc000, v5\n" \
  "ds_read_b64_tr_b8 v[24:25], v5 offset:0\n" \
  "v_mfma_i32_16x16x64_i8 v[144:147], v[48:51], v[20:23], v[144:147]\n" \
  "ds_read_b64_tr_b8 v[26:27], v5 offset:1024\n" \
  "v_mfma_i32_16x16x64_i8 v[176:179], v[52:55], v[20:23], v[176:179]\n" \
  "ds_read_b64_tr_b8 v[28:29], v5 offset:8192\n" \
  "v_mfma_i32_16x16x64_i8 v[176:179], v[36:39], v[16:19], v[176:179]\n" \
  "ds_read_b64_tr_b8 v[30:31], v5 offset:9216\n" \
  "v_mfma_i32_16x16x64_i8 v[208:211], v[40:43], v[16:19], v[208:211]\n" \
  "ds_read_b32 v10, v9\n" \
  "v_mfma_i32_16x16x64_i8 v[208:211], v[56:59], v[20:23], v[208:211]\n" \
  "v_mfma_i32_16x16x64_i8 v[240:243], v[60:63], v[20:23], v[240:243]\n" \
  "v_mfma_i32_16x16x64_i8 v[240:243], v[44:47], v[16:19], v[240:243]\n" \
  "s_waitcnt lgkmcnt(1)\n" \
  "v_mfma_i32_16x16x64_i8 v[244:247], v[44:47], v[24:27], v[244:247]\n" \
  "v_add_u32 v6, 0xc000, v6\n" \
  "ds_read_b64_tr_b8 v[16:17], v6 offset:0\n" \
  "v_mfma_i32_16x16x64_i8 v[244:247], v[60:63], v[28:31], v[244:247]\n" \
  "ds_read_b64_tr_b8 v[18:19], v6 offset:1024\n" \
  "v_mfma_i32_16x16x64_i8 v[212:215], v[56:59], v[28:31], v[212:215]\n" \
  "ds_read_b64_tr_b8 v[20:21], v6 offset:8192\n" \
  "v_mfma_i32_16x16x64_i8 v[212:215], v[40:43], v[24:27], v[212:215]\n" \
  "ds_read_b64_tr_b8 v[22:23], v6 offset:9216\n" \
  "v_mfma_i32_16x16x64_i8 v[180:183], v[36:39], v[24:27], v[180:183]\n" \
  "v_mfma_i32_16x16x64_i8 v[180:183], v[52:55], v[28:31], v[180:183]\n" \
  "v_mfma_i32_16x16x64_i8 v[148:151], v[48:51], v[28:31], v[148:151]\n" \
  "v_mfma_i32_16x16x64_i8 v[148:151], v[32:35], v[24:27], v[148:151]\n" \
  "s_waitcnt lgkmcnt(0)\n" \
  "v_mfma_i32_16x16x64_i8 v[152:155], v[32:35], v[16:19], v[152:155]\n" \
  "v_add_u32 v7, 0xc000, v7\n" \
  "ds_read_b64_tr_b8 v[24:25], v7 offset:0\n" \
  "v_mfma_i32_16x16x64_i8 v[152:155], v[48:51], v[20:23], v[152:155]\n" \
  "ds_read_b64_tr_b8 v[26:27], v7 offset:1024\n" \
  "v_mfma_i32_16x16x64_i8 v[184:187], v[52:55], v[20:23], v[184:187]\n" \
  "ds_read_b64_tr_b8 v[28:29], v7 offset:8192\n" \
  "v_mfma_i32_16x16x64_i8 v[184:187], v[36:39], v[16:19], v[184:187]\n" \
  "ds_read_b64_tr_b8 v[30:31], v7 offset:9216\n" \
  "v_mfma_i32_16x16x64_i8 v[216:219], v[40:43], v[16:19], v[216:219]\n" \
  "v_mfma_i32_16x16x64_i8 v[216:219], v[56:59], v[20:23], v[216:219]\n" \
  "v_mfma_i32_16x16x64_i8 v[248:251], v[60:63], v[20:23], v[248:251]\n" \
  "v_cmp_gt_u32 vcc, 34, v10\n" \
  "v_mfma_i32_16x16x64_i8 v[248:251], v[44:47], v[16:19], v[248:251]\n" \
  "s_waitcnt lgkmcnt(0)\n" \
  "v_mfma_i32_16x16x64_i8 v[252:255], v[44:47], v[24:27], v[252:255]\n" \
  "s_cbranch_vccnz LS15_%=\n" \
  "LB15_%=:\n" \
  "ds_read_b64_tr_b8 v[16:17], v0 offset:16384\n" \
  "v_mfma_i32_16x16x64_i8 v[252:255], v[60:63], v[28:31], v[252:255]\n" \
  "ds_read_b64_tr_b8 v[18:19], v0 offset:17408\n" \
  "v_mfma_i32_16x16x64_i8 v[220:223], v[56:59], v[28:31], v[220:223]\n" \
  "ds_read_b64_tr_b8 v[20:21], v0 offset:24576\n" \
  "v_mfma_i32_16x16x64_i8 v[220:223], v[40:43], v[24:27], v[220:223]\n" \
  "ds_read_b64_tr_b8 v[22:23], v0 offset:25600\n" \
  "v_mfma_i32_16x16x64_i8 v[188:191], v[36:39], v[24:27], v[188:191]\n" \
  "ds_add_u32 v12, v11\n" \
  "v_mfma_i32_16x16x64_i8 v[188:191], v[52:55], v[28:31], v[188:191]\n" \
  "v_mfma_i32_16x16x64_i8 v[156:159], v[48:51], v[28:31], v[156:159]\n" \
  "v_mfma_i32_16x16x64_i8 v[156:159], v[32:35], v[24:27], v[156:159]\n" \
  "s_waitcnt vmcnt(8) lgkmcnt(1)\n" \
  "v_mfma_i32_16x16x64_i8 v[128:131], v[64:67], v[16:19], v[128:131]\n" \
  "ds_read_b64_tr_b8 v[24:25], v1 offset:16384\n" \
  "v_mfma_i32_16x16x64_i8 v[128:131], v[80:83], v[20:23], v[128:131]\n" \
  "ds_read_b64_tr_b8 v[26:27], v1 offset:17408\n" \
  "v_mfma_i32_16x16x64_i8 v[160:163], v[84:87], v[20:23], v[160:163]\n" \
  "ds_read_b64_tr_b8 v[28:29], v1 offset:24576\n" \
  "v_mfma_i32_16x16x64_i8 v[160:163], v[68:71], v[16:19], v[160:163]\n" \
  "ds_read_b64_tr_b8 v[30:31], v1 offset:25600\n" \
  "v_mfma_i32_16x16x64_i8 v[192:195], v[72:75], v[16:19], v[192:195]\n" \
  "global_load_dwordx4 v[32:35], v8, s[40:41] offset:0 sc0 sc1\n" \
  "v_mfma_i32_16x16x64_i8 v[192:195], v[88:91], v[20:23], v[192:195]\n" \
  "global_load_dwordx4 v[36:39], v8, s[40:41] offset:1024 sc0 sc1\n" \
  "v_mfma_i32_16x16x64_i8 v[224:227], v[92:95], v[20:23], v[224:227]\n" \
  "v_mfma_i32_16x16x64_i8 v[224:227], v[76:79], v[16:19], v[224:227]\n" \
  "s_waitcnt lgkmcnt(0)\n" \
  "v_mfma_i32_16x16x64_i8 v[228:231], v[76:79], v[24:27], v[228:231]\n" \
  "ds_read_b64_tr_b8 v[16:17], v2 offset:16384\n" \
  "v_mfma_i32_16x16x64_i8 v[228:231], v[92:95], v[28:31], v[228:231]\n" \
  "ds_read_b64_tr_b8 v[18:19], v2 offset:17408\n" \
  "v_mfma_i32_16x16x64_i8 v[196:199], v[88:91], v[28:31], v[196:199]\n" \
  "ds_read_b64_tr_b8 v[20:21], v2 offset:24576\n" \
  "v_mfma_i32_16x16x64_i8 v[196:199], v[72:75], v[24:27], v[196:199]\n" \
  "ds_read_b64_tr_b8 v[22:23], v2 offset:25600\n" \
  "v_mfma_i32_16x16x64_i8 v[164:167], v[68:71], v[24:27], v[164:167]\n" \
  "global_load_dwordx4 v[40:43], v8, s[40:41] offset:2048 sc0 sc1\n" \
  "v_mfma_i32_16x16x64_i8 v[164:167], v[84:87], v[28:31], v[164:167]\n" \
  "global_load_dwordx4 v[44:47], v8, s[40:41] offset:3072 sc0 sc1\n" \
  "v_mfma_i32_16x16x64_i8 v[132:135], v[80:83], v[28:31], v[132:135]\n" \
  "v_add_u32 v8, 0x8000, v8\n" \
  "v_mfma_i32_16x16x64_i8 v[132:135], v[64:67], v[24:27], v[132:135]\n" \
  "s_waitcnt lgkmcnt(0)\n" \
  "v_mfma_i32_16x16x64_i8 v[136:139], v[64:67], v[16:19], v[136:139]\n" \
  "ds_read_b64_tr_b8 v[24:25], v3 offset:16384\n" \
  "v_mfma_i32_16x16x64_i8 v[136:139], v[80:83], v[20:23], v[136:139]\n" \
  "ds_read_b64_tr_b8 v[26:27], v3 offset:17408\n" \
  "v_mfma_i32_16x16x64_i8 v[168:171], v[84:87], v[20:23], v[168:171]\n" \
  "ds_read_b64_tr_b8 v[28:29], v3 offset:24576\n" \
  "v_mfma_i32_16x16x64_i8 v[168:171], v[68:71], v[16:19], v[168:171]\n" \
  "ds_read_b64_tr_b8 v[30:31], v3 offset:25600\n" \
  "v_mfma_i32_16x16x64_i8 v[200:203], v[72:75], v[16:19], v[200:203]\n" \
  "global_load_dwordx4 v[48:51], v8, s[40:41] offset:0 sc0 sc1\n" \
  "v_mfma_i32_16x16x64_i8 v[200:203], v[88:91], v[20:23], v[200:203]\n" \
  "global_load_dwordx4 v[52:55], v8, s[40:41] offset:1024 sc0 sc1\n" \
  "v_mfma_i32_16x16x64_i8 v[232:235], v[92:95], v[20:23], v[232:235]\n" \
  "v_mfma_i32_16x16x64_i8 v[232:235], v[76:79], v[16:19], v[232:235]\n" \
  "s_waitcnt lgkmcnt(0)\n" \
  "v_mfma_i32_16x16x64_i8 v[236:239], v[76:79], v[24:27], v[236:239]\n" \
  "ds_read_b64_tr_b8 v[16:17], v4 offset:16384\n" \
  "v_mfma_i32_16x16x64_i8 v[236:239], v[92:95], v[28:31], v[236:239]\n" \
  "ds_read_b64_tr_b8 v[18:19], v4 offset:17408\n" \
  "v_mfma_i32_16x16x64_i8 v[204:207], v[88:91], v[28:31], v[204:207]\n" \
  "ds_read_b64_tr_b8 v[20:21], v4 offset:24576\n" \
  "v_mfma_i32_16x16x64_i8 v[204:207], v[72:75], v[24:27], v[204:207]\n" \
  "ds_read_b64_tr_b8 v[22:23], v4 offset:25600\n" \
  "v_mfma_i32_16x16x64_i8 v[172:175], v[68:71], v[24:27], v[172:175]\n" \
  "global_load_dwordx4 v[56:59], v8, s[40:41] offset:2048 sc0 sc1\n" \
  "v_mfma_i32_16x16x64_i8 v[172:175], v[84:87], v[28:31], v[172:175]\n" \
  "global_load_dwordx4 v[60:63], v8, s[40:41] offset:3072 sc0 sc1\n" \
  "v_mfma_i32_16x16x64_i8 v[140:143], v[80:83], v[28:31], v[140:143]\n" \
  "v_add_u32 v8, 0x8000, v8\n" \
  "v_mfma_i32_16x16x64_i8 v[140:143], v[64:67], v[24:27], v[140:143]\n" \
  "s_waitcnt lgkmcnt(0)\n" \
  "v_mfma_i32_16x16x64_i8 v[144:147], v[64:67], v[16:19], v[144:147]\n" \
  "ds_read_b64_tr_b8 v[24:25], v5 offset:16384\n" \
  "v_mfma_i32_16x16x64_i8 v[144:147], v[80:83], v[20:23], v[144:147]\n" \
  "ds_read_b64_tr_b8 v[26:27], v5 offset:17408\n" \
  "v_mfma_i32_16x16x64_i8 v[176:179], v[84:87], v[20:23], v[176:179]\n" \
  "ds_read_b64_tr_b8 v[28:29], v5 offset:24576\n" \
  "v_mfma_i32_16x16x64_i8 v[176:179], v[68:71], v[16:19], v[176:179]\n" \
  "ds_read_b64_tr_b8 v[30:31], v5 offset:25600\n" \
  "v_mfma_i32_16x16x64_i8 v[208:211], v[72:75], v[16:19], v[208:211]\n" \
  "ds_read_b32 v10, v9\n" \
  "v_mfma_i32_16x16x64_i8 v[208:211], v[88:91], v[20:23], v[208:211]\n" \
  "v_mfma_i32_16x16x64_i8 v[240:243], v[92:95], v[20:23], v[240:243]\n" \
  "v_mfma_i32_16x16x64_i8 v[240:243], v[76:79], v[16:19], v[240:243]\n" \
  "s_waitcnt lgkmcnt(1)\n" \
  "v_mfma_i32_16x16x64_i8 v[244:247], v[76:79], v[24:27], v[244:247]\n" \
  "ds_read_b64_tr_b8 v[16:17], v6 offset:16384\n" \
  "v_mfma_i32_16x16x64_i8 v[244:247], v[92:95], v[28:31], v[244:247]\n" \
  "ds_read_b64_tr_b8 v[18:19], v6 offset:17408\n" \
  "v_mfma_i32_16x16x64_i8 v[212:215], v[88:91], v[28:31], v[212:215]\n" \
  "ds_read_b64_tr_b8 v[20:21], v6 offset:24576\n" \
  "v_mfma_i32_16x16x64_i8 v[212:215], v[72:75], v[24:27], v[212:215]\n" \
  "ds_read_b64_tr_b8 v[22:23], v6 offset:25600\n" \
  "v_mfma_i32_16x16x64_i8 v[180:183], v[68:71], v[24:27], v[180:183]\n" \
  "v_mfma_i32_16x16x64_i8 v[180:183], v[84:87], v[28:31], v[180:183]\n" \
  "v_mfma_i32_16x16x64_i8 v[148:151], v[80:83], v[28:31], v[148:151]\n" \
  "v_mfma_i32_16x16x64_i8 v[148:151], v[64:67], v[24:27], v[148:151]\n" \
  "s_waitcnt lgkmcnt(0)\n" \
  "v_mfma_i32_16x16x64_i8 v[152:155], v[64:67], v[16:19], v[152:155]\n" \
  "ds_read_b64_tr_b8 v[24:25], v7 offset:16384\n" \
  "v_mfma_i32_16x16x64_i8 v[152:155], v[80:83], v[20:23], v[152:155]\n" \
  "ds_read_b64_tr_b8 v[26:27], v7 offset:17408\n" \
  "v_mfma_i32_16x16x64_i8 v[184:187], v[84:87], v[20:23], v[184:187]\n" \
  "ds_read_b64_tr_b8 v[28:29], v7 offset:24576\n" \
  "v_mfma_i32_16x16x64_i8 v[184:187], v[68:71], v[16:19], v[184:187]\n" \
  "ds_read_b64_tr_b8 v[30:31], v7 offset:25600\n" \
  "v_mfma_i32_16x16x64_i8 v[216:219], v[72:75], v[16:19], v[216:219]\n" \
  "v_mfma_i32_16x16x64_i8 v[216:219], v[88:91], v[20:23], v[216:219]\n" \
  "v_mfma_i32_16x16x64_i8 v[248:251], v[92:95], v[20:23], v[248:251]\n" \
  "v_cmp_gt_u32 vcc, 36, v10\n" \
  "v_mfma_i32_16x16x64_i8 v[248:251], v[76:79], v[16:19], v[248:251]\n" \
  "s_waitcnt lgkmcnt(0)\n" \
  "v_mfma_i32_16x16x64_i8 v[252:255], v[76:79], v[24:27], v[252:255]\n" \
  "s_cbranch_vccnz LS16_%=\n" \
  "LB16_%=:\n" \
  "ds_read_b64_tr_b8 v[16:17], v0 offset:32768\n" \
  "v_mfma_i32_16x16x64_i8 v[252:255], v[92:95], v[28:31], v[252:255]\n" \
  "ds_read_b64_tr_b8 v[18:19], v0 offset:33792\n" \
  "v_mfma_i32_16x16x64_i8 v[220:223], v[88:91], v[28:31], v[220:223]\n" \
  "ds_read_b64_tr_b8 v[20:21], v0 offset:40960\n" \
  "v_mfma_i32_16x16x64_i8 v[220:223], v[72:75], v[24:27], v[220:223]\n" \
  "ds_read_b64_tr_b8 v[22:23], v0 offset:41984\n" \
  "v_mfma_i32_16x16x64_i8 v[188:191], v[68:71], v[24:27], v[188:191]\n" \
  "ds_add_u32 v12, v11\n" \
  "v_mfma_i32_16x16x64_i8 v[188:191], v[84:87], v[28:31], v[188:191]\n" \
  "v_mfma_i32_16x16x64_i8 v[156:159], v[80:83], v[28:31], v[156:159]\n" \
  "v_mfma_i32_16x16x64_i8 v[156:159], v[64:67], v[24:27], v[156:159]\n" \
  "s_waitcnt vmcnt(8) lgkmcnt(1)\n" \
  "v_mfma_i32_16x16x64_i8 v[128:131], v[96:99], v[16:19], v[128:131]\n" \
  "ds_read_b64_tr_b8 v[24:25], v1 offset:32768\n" \
  "v_mfma_i32_16x16x64_i8 v[128:131], v[112:115], v[20:23], v[128:131]\n" \
  "ds_read_b64_tr_b8 v[26:27], v1 offset:33792\n" \
  "v_mfma_i32_16x16x64_i8 v[160:163], v[116:119], v[20:23], v[160:163]\n" \
  "ds_read_b64_tr_b8 v[28:29], v1 offset:40960\n" \
  "v_mfma_i32_16x16x64_i8 v[160:163], v[100:103], v[16:19], v[160:163]\n" \
  "ds_read_b64_tr_b8 v[30:31], v1 offset:41984\n" \
  "v_mfma_i32_16x16x64_i8 v[192:195], v[104:107], v[16:19], v[192:195]\n" \
  "global_load_dwordx4 v[64:67], v8, s[40:41] offset:0 sc0 sc1\n" \
  "v_mfma_i32_16x16x64_i8 v[192:195], v[120:123], v[20:23], v[192:195]\n" \
  "global_load_dwordx4 v[68:71], v8, s[40:41] offset:1024 sc0 sc1\n" \
  "v_mfma_i32_16x16x64_i8 v[224:227], v[124:127], v[20:23], v[224:227]\n" \
  "v_mfma_i32_16x16x64_i8 v[224:227], v[108:111], v[16:19], v[224:227]\n" \
  "s_waitcnt lgkmcnt(0)\n" \
  "v_mfma_i32_16x16x64_i8 v[228:231], v[108:111], v[24:27], v[228:231]\n" \
  "ds_read_b64_tr_b8 v[16:17], v2 offset:32768\n" \
  "v_mfma_i32_16x16x64_i8 v[228:231], v[124:127], v[28:31], v[228:231]\n" \
  "ds_read_b64_tr_b8 v[18:19], v2 offset:33792\n" \
  "v_mfma_i32_16x16x64_i8 v[196:199], v[120:123], v[28:31], v[196:199]\n" \
  "ds_read_b64_tr_b8 v[20:21], v2 offset:40960\n" \
  "v_mfma_i32_16x16x64_i8 v[196:199], v[104:107], v[24:27], v[196:199]\n" \
  "ds_read_b64_tr_b8 v[22:23], v2 offset:41984\n" \
  "v_mfma_i32_16x16x64_i8 v[164:167], v[100:103], v[24:27], v[164:167]\n" \
  "global_load_dwordx4 v[72:75], v8, s[40:41] offset:2048 sc0 sc1\n" \
  "v_mfma_i32_16x16x64_i8 v[164:167], v[116:119], v[28:31], v[164:167]\n" \
  "global_load_dwordx4 v[76:79], v8, s[40:41] offset:3072 sc0 sc1\n" \
  "v_mfma_i32_16x16x64_i8 v[132:135], v[112:115], v[28:31], v[132:135]\n" \
  "v_add_u32 v8, 0x8000, v8\n" \
  "v_mfma_i32_16x16x64_i8 v[132:135], v[96:99], v[24:27], v[132:135]\n" \
  "s_waitcnt lgkmcnt(0)\n" \
  "v_mfma_i32_16x16x64_i8 v[136:139], v[96:99], v[16:19], v[136:139]\n" \
  "ds_read_b64_tr_b8 v[24:25], v3 offset:32768\n" \
  "v_mfma_i32_16x16x64_i8 v[136:139], v[112:115], v[20:23], v[136:139]\n" \
  "ds_read_b64_tr_b8 v[26:27], v3 offset:33792\n" \
  "v_mfma_i32_16x16x64_i8 v[168:171], v[116:119], v[20:23], v[168:171]\n" \
  "ds_read_b64_tr_b8 v[28:29], v3 offset:40960\n" \
  "v_mfma_i32_16x16x64_i8 v[168:171], v[100:103], v[16:19], v[168:171]\n" \
  "ds_read_b64_tr_b8 v[30:31], v3 offset:41984\n" \
  "v_mfma_i32_16x16x64_i8 v[200:203], v[104:107], v[16:19], v[200:203]\n" \
  "global_load_dwordx4 v[80:83], v8, s[40:41] offset:0 sc0 sc1\n" \
  "v_mfma_i32_16x16x64_i8 v[200:203], v[120:123], v[20:23], v[200:203]\n" \
  "global_load_dwordx4 v[84:87], v8, s[40:41] offset:1024 sc0 sc1\n" \
  "v_mfma_i32_16x16x64_i8 v[232:235], v[124:127], v[20:23], v[232:235]\n" \
  "v_mfma_i32_16x16x64_i8 v[232:235], v[108:111], v[16:19], v[232:235]\n" \
  "s_waitcnt lgkmcnt(0)\n" \
  "v_mfma_i32_16x16x64_i8 v[236:239], v[108:111], v[24:27], v[236:239]\n" \
  "ds_read_b64_tr_b8 v[16:17], v4 offset:32768\n" \
  "v_mfma_i32_16x16x64_i8 v[236:239], v[124:127], v[28:31], v[236:239]\n" \
  "ds_read_b64_tr_b8 v[18:19], v4 offset:33792\n" \
  "v_mfma_i32_16x16x64_i8 v[204:207], v[120:123], v[28:31], v[204:207]\n" \
  "ds_read_b64_tr_b8 v[20:21], v4 offset:40960\n" \
  "v_mfma_i32_16x16x64_i8 v[204:207], v[104:107], v[24:27], v[204:207]\n" \
  "ds_read_b64_tr_b8 v[22:23], v4 offset:41984\n" \
  "v_mfma_i32_16x16x64_i8 v[172:175], v[100:103], v[24:27], v[172:175]\n" \
  "global_load_dwordx4 v[88:91], v8, s[40:41] offset:2048 sc0 sc1\n" \
  "v_mfma_i32_16x16x64_i8 v[172:175], v[116:119], v[28:31], v[172:175]\n" \
  "global_load_dwordx4 v[92:95], v8, s[40:41] offset:3072 sc0 sc1\n" \
  "v_mfma_i32_16x16x64_i8 v[140:143], v[112:115], v[28:31], v[140:143]\n" \
  "v_add_u32 v8, 0x8000, v8\n" \
  "v_mfma_i32_16x16x64_i8 v[140:143], v[96:99], v[24:27], v[140:143]\n" \
  "s_waitcnt lgkmcnt(0)\n" \
  "v_mfma_i32_16x16x64_i8 v[144:147], v[96:99], v[16:19], v[144:147]\n" \
  "ds_read_b64_tr_b8 v[24:25], v5 offset:32768\n" \
  "v_mfma_i32_16x16x64_i8 v[144:147], v[112:115], v[20:23], v[144:147]\n" \
  "ds_read_b64_tr_b8 v[26:27], v5 offset:33792\n" \
  "v_mfma_i32_16x16x64_i8 v[176:179], v[116:119], v[20:23], v[176:179]\n" \
  "ds_read_b64_tr_b8 v[28:29], v5 offset:40960\n" \
  "v_mfma_i32_16x16x64_i8 v[176:179], v[100:103], v[16:19], v[176:179]\n" \
  "ds_read_b64_tr_b8 v[30:31], v5 offset:41984\n" \
  "v_mfma_i32_16x16x64_i8 v[208:211], v[104:107], v[16:19], v[208:211]\n" \
  "ds_read_b32 v10, v9\n" \
  "v_mfma_i32_16x16x64_i8 v[208:211], v[120:123], v[20:23], v[208:211]\n" \
  "v_mfma_i32_16x16x64_i8 v[240:243], v[124:127], v[20:23], v[240:243]\n" \
  "v_mfma_i32_16x16x64_i8 v[240:243], v[108:111], v[16:19], v[240:243]\n" \
  "s_waitcnt lgkmcnt(1)\n" \
  "v_mfma_i32_16x16x64_i8 v[244:247], v[108:111], v[24:27], v[244:247]\n" \
  "ds_read_b64_tr_b8 v[16:17], v6 offset:32768\n" \
  "v_mfma_i32_16x16x64_i8 v[244:247], v[124:127], v[28:31], v[244:247]\n" \
  "ds_read_b64_tr_b8 v[18:19], v6 offset:33792\n" \
  "v_mfma_i32_16x16x64_i8 v[212:215], v[120:123], v[28:31], v[212:215]\n" \
  "ds_read_b64_tr_b8 v[20:21], v6 offset:40960\n" \
  "v_mfma_i32_16x16x64_i8 v[212:215], v[104:107], v[24:27], v[212:215]\n" \
  "ds_read_b64_tr_b8 v[22:23], v6 offset:41984\n" \
  "v_mfma_i32_16x16x64_i8 v[180:183], v[100:103], v[24:27], v[180:183]\n" \
  "v_mfma_i32_16x16x64_i8 v[180:183], v[116:119], v[28:31], v[180:183]\n" \
  "v_mfma_i32_16x16x64_i8 v[148:151], v[112:115], v[28:31], v[148:151]\n" \
  "v_mfma_i32_16x16x64_i8 v[148:151], v[96:99], v[24:27], v[148:151]\n" \
  "s_waitcnt lgkmcnt(0)\n" \
  "v_mfma_i32_16x16x64_i8 v[152:155], v[96:99], v[16:19], v[152:155]\n" \
  "ds_read_b64_tr_b8 v[24:25], v7 offset:32768\n" \
  "v_mfma_i32_16x16x64_i8 v[152:155], v[112:115], v[20:23], v[152:155]\n" \
  "ds_read_b64_tr_b8 v[26:27], v7 offset:33792\n" \
  "v_mfma_i32_16x16x64_i8 v[184:187], v[116:119], v[20:23], v[184:187]\n" \
  "ds_read_b64_tr_b8 v[28:29], v7 offset:40960\n" \
  "v_mfma_i32_16x16x64_i8 v[184:187], v[100:103], v[16:19], v[184:187]\n" \
  "ds_read_b64_tr_b8 v[30:31], v7 offset:41984\n" \
  "v_mfma_i32_16x16x64_i8 v[216:219], v[104:107], v[16:19], v[216:219]\n" \
  "v_mfma_i32_16x16x64_i8 v[216:219], v[120:123], v[20:23], v[216:219]\n" \
  "v_mfma_i32_16x16x64_i8 v[248:251], v[124:127], v[20:23], v[248:251]\n" \
  "v_cmp_gt_u32 vcc, 38, v10\n" \
  "v_mfma_i32_16x16x64_i8 v[248:251], v[108:111], v[16:19], v[248:251]\n" \
  "s_waitcnt lgkmcnt(0)\n" \
  "v_mfma_i32_16x16x64_i8 v[252:255], v[108:111], v[24:27], v[252:255]\n" \
  "s_cbranch_vccnz LS17_%=\n" \
  "LB17_%=:\n" \
  "v_add_u32 v0, 0xfffe8000, v0\n" \
  "ds_read_b64_tr_b8 v[16:17], v0 offset:0\n" \
  "v_mfma_i32_16x16x64_i8 v[252:255], v[124:127], v[28:31], v[252:255]\n" \
  "ds_read_b64_tr_b8 v[18:19], v0 offset:1024\n" \
  "v_mfma_i32_16x16x64_i8 v[220:223], v[120:123], v[28:31], v[220:223]\n" \
  "ds_read_b64_tr_b8 v[20:21], v0 offset:8192\n" \
  "v_mfma_i32_16x16x64_i8 v[220:223], v[104:107], v[24:27], v[220:223]\n" \
  "ds_read_b64_tr_b8 v[22:23], v0 offset:9216\n" \
  "v_mfma_i32_16x16x64_i8 v[188:191], v[100:103], v[24:27], v[188:191]\n" \
  "ds_add_u32 v12, v11\n" \
  "v_mfma_i32_16x16x64_i8 v[188:191], v[116:119], v[28:31], v[188:191]\n" \
  "v_mfma_i32_16x16x64_i8 v[156:159], v[112:115], v[28:31], v[156:159]\n" \
  "v_mfma_i32_16x16x64_i8 v[156:159], v[96:99], v[24:27], v[156:159]\n" \
  "s_waitcnt vmcnt(8) lgkmcnt(1)\n" \
  "v_mfma_i32_16x16x64_i8 v[128:131], v[32:35], v[16:19], v[128:131]\n" \
  "v_add_u32 v1, 0xfffe8000, v1\n" \
  "ds_read_b64_tr_b8 v[24:25], v1 offset:0\n" \
  "v_mfma_i32_16x16x64_i8 v[128:131], v[48:51], v[20:23], v[128:131]\n" \
  "ds_read_b64_tr_b8 v[26:27], v1 offset:1024\n" \
  "v_mfma_i32_16x16x64_i8 v[160:163], v[52:55], v[20:23], v[160:163]\n" \
  "ds_read_b64_tr_b8 v[28:29], v1 offset:8192\n" \
  "v_mfma_i32_16x16x64_i8 v[160:163], v[36:39], v[16:19], v[160:163]\n" \
  "ds_read_b64_tr_b8 v[30:31], v1 offset:9216\n" \
  "v_mfma_i32_16x16x64_i8 v[192:195], v[40:43], v[16:19], v[192:195]\n" \
  "global_load_dwordx4 v[96:99], v8, s[40:41] offset:0 sc0 sc1\n" \
  "v_mfma_i32_16x16x64_i8 v[192:195], v[56:59], v[20:23], v[192:195]\n" \
  "global_load_dwordx4 v[100:103], v8, s[40:41] offset:1024 sc0 sc1\n" \
  "v_mfma_i32_16x16x64_i8 v[224:227], v[60:63], v[20:23], v[224:227]\n" \
  "v_mfma_i32_16x16x64_i8 v[224:227], v[44:47], v[16:19], v[224:227]\n" \
  "s_waitcnt lgkmcnt(0)\n" \
  "v_mfma_i32_16x16x64_i8 v[228:231], v[44:47], v[24:27], v[228:231]\n" \
  "v_add_u32 v2, 0xfffe8000, v2\n" \
  "ds_read_b64_tr_b8 v[16:17], v2 offset:0\n" \
  "v_mfma_i32_16x16x64_i8 v[228:231], v[60:63], v[28:31], v[228:231]\n" \
  "ds_read_b64_tr_b8 v[18:19], v2 offset:1024\n" \
  "v_mfma_i32_16x16x64_i8 v[196:199], v[56:59], v[28:31], v[196:199]\n" \
  "ds_read_b64_tr_b8 v[20:21], v2 offset:8192\n" \
  "v_mfma_i32_16x16x64_i8 v[196:199], v[40:43], v[24:27], v[196:199]\n" \
  "ds_read_b64_tr_b8 v[22:23], v2 offset:9216\n" \
  "v_mfma_i32_16x16x64_i8 v[164:167], v[36:39], v[24:27], v[164:167]\n" \
  "global_load_dwordx4 v[104:107], v8, s[40:41] offset:2048 sc0 sc1\n" \
  "v_mfma_i32_16x16x64_i8 v[164:167], v[52:55], v[28:31], v[164:167]\n" \
  "global_load_dwordx4 v[108:111], v8, s[40:41] offset:3072 sc0 sc1\n" \
  "v_mfma_i32_16x16x64_i8 v[132:135], v[48:51], v[28:31], v[132:135]\n" \
  "v_add_u32 v8, 0x8000, v8\n" \
  "v_mfma_i32_16x16x64_i8 v[132:135], v[32:35], v[24:27], v[132:135]\n" \
  "s_waitcnt lgkmcnt(0)\n" \
  "v_mfma_i32_16x16x64_i8 v[136:139], v[32:35], v[16:19], v[136:139]\n" \
  "v_add_u32 v3, 0xfffe8000, v3\n" \
  "ds_read_b64_tr_b8 v[24:25], v3 offset:0\n" \
  "v_mfma_i32_16x16x64_i8 v[136:139], v[48:51], v[20:23], v[136:139]\n" \
  "ds_read_b64_tr_b8 v[26:27], v3 offset:1024\n" \
  "v_mfma_i32_16x16x64_i8 v[168:171], v[52:55], v[20:23], v[168:171]\n" \
  "ds_read_b64_tr_b8 v[28:29], v3 offset:8192\n" \
  "v_mfma_i32_16x16x64_i8 v[168:171], v[36:39], v[16:19], v[168:171]\n" \
  "ds_read_b64_tr_b8 v[30:31], v3 offset:9216\n" \
  "v_mfma_i32_16x16x64_i8 v[200:203], v[40:43], v[16:19], v[200:203]\n" \
  "global_load_dwordx4 v[112:115], v8, s[40:41] offset:0 sc0 sc1\n" \
  "v_mfma_i32_16x16x64_i8 v[200:203], v[56:59], v[20:23], v[200:203]\n" \
  "global_load_dwordx4 v[116:119], v8, s[40:41] offset:1024 sc0 sc1\n" \
  "v_mfma_i32_16x16x64_i8 v[232:235], v[60:63], v[20:23], v[232:235]\n" \
  "v_mfma_i32_16x16x64_i8 v[232:235], v[44:47], v[16:19], v[232:235]\n" \
  "s_waitcnt lgkmcnt(0)\n" \
  "v_mfma_i32_16x16x64_i8 v[236:239], v[44:47], v[24:27], v[236:239]\n" \
  "v_add_u32 v4, 0xfffe8000, v4\n" \
  "ds_read_b64_tr_b8 v[16:17], v4 offset:0\n" \
  "v_mfma_i32_16x16x64_i8 v[236:239], v[60:63], v[28:31], v[236:239]\n" \
  "ds_read_b64_tr_b8 v[18:19], v4 offset:1024\n" \
  "v_mfma_i32_16x16x64_i8 v[204:207], v[56:59], v[28:31], v[204:207]\n" \
  "ds_read_b64_tr_b8 v[20:21], v4 offset:8192\n" \
  "v_mfma_i32_16x16x64_i8 v[204:207], v[40:43], v[24:27], v[204:207]\n" \
  "ds_read_b64_tr_b8 v[22:23], v4 offset:9216\n" \
  "v_mfma_i32_16x16x64_i8 v[172:175], v[36:39], v[24:27], v[172:175]\n" \
  "global_load_dwordx4 v[120:123], v8, s[40:41] offset:2048 sc0 sc1\n" \
  "v_mfma_i32_16x16x64_i8 v[172:175], v[52:55], v[28:31], v[172:175]\n" \
  "global_load_dwordx4 v[124:127], v8, s[40:41] offset:3072 sc0 sc1\n" \
  "v_mfma_i32_16x16x64_i8 v[140:143], v[48:51], v[28:31], v[140:143]\n" \
  "v_add_u32 v8, 0x8000, v8\n" \
  "v_mfma_i32_16x16x64_i8 v[140:143], v[32:35], v[24:27], v[140:143]\n" \
  "s_waitcnt lgkmcnt(0)\n" \
  "v_mfma_i32_16x16x64_i8 v[144:147], v[32:35], v[16:19], v[144:147]\n" \
  "v_add_u32 v5, 0xfffe8000, v5\n" \
  "ds_read_b64_tr_b8 v[24:25], v5 offset:0\n" \
  "v_mfma_i32_16x16x64_i8 v[144:147], v[48:51], v[20:23], v[144:147]\n" \
  "ds_read_b64_tr_b8 v[26:27], v5 offset:1024\n" \
  "v_mfma_i32_16x16x64_i8 v[176:179], v[52:55], v[20:23], v[176:179]\n" \
  "ds_read_b64_tr_b8 v[28:29], v5 offset:8192\n" \
  "v_mfma_i32_16x16x64_i8 v[176:179], v[36:39], v[16:19], v[176:179]\n" \
  "ds_read_b64_tr_b8 v[30:31], v5 offset:9216\n" \
  "v_mfma_i32_16x16x64_i8 v[208:211], v[40:43], v[16:19], v[208:211]\n" \
  "ds_read_b32 v10, v9\n" \
  "v_mfma_i32_16x16x64_i8 v[208:211], v[56:59], v[20:23], v[208:211]\n" \
  "v_mfma_i32_16x16x64_i8 v[240:243], v[60:63], v[20:23], v[240:243]\n" \
  "v_mfma_i32_16x16x64_i8 v[240:243], v[44:47], v[16:19], v[240:243]\n" \
  "s_waitcnt lgkmcnt(1)\n" \
  "v_mfma_i32_16x16x64_i8 v[244:247], v[44:47], v[24:27], v[244:247]\n" \
  "v_add_u32 v6, 0xfffe8000, v6\n" \
  "ds_read_b64_tr_b8 v[16:17], v6 offset:0\n" \
  "v_mfma_i32_16x16x64_i8 v[244:247], v[60:63], v[28:31], v[244:247]\n" \
  "ds_read_b64_tr_b8 v[18:19], v6 offset:1024\n" \
  "v_mfma_i32_16x16x64_i8 v[212:215], v[56:59], v[28:31], v[212:215]\n" \
  "ds_read_b64_tr_b8 v[20:21], v6 offset:8192\n" \
  "v_mfma_i32_16x16x64_i8 v[212:215], v[40:43], v[24:27], v[212:215]\n" \
  "ds_read_b64_tr_b8 v[22:23], v6 offset:9216\n" \
  "v_mfma_i32_16x16x64_i8 v[180:183], v[36:39], v[24:27], v[180:183]\n" \
  "v_mfma_i32_16x16x64_i8 v[180:183], v[52:55], v[28:31], v[180:183]\n" \
  "v_mfma_i32_16x16x64_i8 v[148:151], v[48:51], v[28:31], v[148:151]\n" \
  "v_mfma_i32_16x16x64_i8 v[148:151], v[32:35], v[24:27], v[148:151]\n" \
  "s_waitcnt lgkmcnt(0)\n" \
  "v_mfma_i32_16x16x64_i8 v[152:155], v[32:35], v[16:19], v[152:155]\n" \
  "v_add_u32 v7, 0xfffe8000, v7\n" \
  "ds_read_b64_tr_b8 v[24:25], v7 offset:0\n" \
  "v_mfma_i32_16x16x64_i8 v[152:155], v[48:51], v[20:23], v[152:155]\n" \
  "ds_read_b64_tr_b8 v[26:27], v7 offset:1024\n" \
  "v_mfma_i32_16x16x64_i8 v[184:187], v[52:55], v[20:23], v[184:187]\n" \
  "ds_read_b64_tr_b8 v[28:29], v7 offset:8192\n" \
  "v_mfma_i32_16x16x64_i8 v[184:187], v[36:39], v[16:19], v[184:187]\n" \
  "ds_read_b64_tr_b8 v[30:31], v7 offset:9216\n" \
  "v_mfma_i32_16x16x64_i8 v[216:219], v[40:43], v[16:19], v[216:219]\n" \
  "v_mfma_i32_16x16x64_i8 v[216:219], v[56:59], v[20:23], v[216:219]\n" \
  "v_mfma_i32_16x16x64_i8 v[248:251], v[60:63], v[20:23], v[248:251]\n" \
  "v_cmp_gt_u32 vcc, 40, v10\n" \
  "v_mfma_i32_16x16x64_i8 v[248:251], v[44:47], v[16:19], v[248:251]\n" \
  "s_waitcnt lgkmcnt(0)\n" \
  "v_mfma_i32_16x16x64_i8 v[252:255], v[44:47], v[24:27], v[252:255]\n" \
  "s_cbranch_vccnz LS18_%=\n" \
  "LB18_%=:\n" \
  "ds_read_b64_tr_b8 v[16:17], v0 offset:16384\n" \
  "v_mfma_i32_16x16x64_i8 v[252:255], v[60:63], v[28:31], v[252:255]\n" \
  "ds_read_b64_tr_b8 v[18:19], v0 offset:17408\n" \
  "v_mfma_i32_16x16x64_i8 v[220:223], v[56:59], v[28:31], v[220:223]\n" \
  "ds_read_b64_tr_b8 v[20:21], v0 offset:24576\n" \
  "v_mfma_i32_16x16x64_i8 v[220:223], v[40:43], v[24:27], v[220:223]\n" \
  "ds_read_b64_tr_b8 v[22:23], v0 offset:25600\n" \
  "v_mfma_i32_16x16x64_i8 v[188:191], v[36:39], v[24:27], v[188:191]\n" \
  "ds_add_u32 v12, v11\n" \
  "v_mfma_i32_16x16x64_i8 v[188:191], v[52:55], v[28:31], v[188:191]\n" \
  "v_mfma_i32_16x16x64_i8 v[156:159], v[48:51], v[28:31], v[156:159]\n" \
  "v_mfma_i32_16x16x64_i8 v[156:159], v[32:35], v[24:27], v[156:159]\n" \
  "s_waitcnt vmcnt(8) lgkmcnt(1)\n" \
  "v_mfma_i32_16x16x64_i8 v[128:131], v[64:67], v[16:19], v[128:131]\n" \
  "ds_read_b64_tr_b8 v[24:25], v1 offset:16384\n" \
  "v_mfma_i32_16x16x64_i8 v[128:131], v[80:83], v[20:23], v[128:131]\n" \
  "ds_read_b64_tr_b8 v[26:27], v1 offset:17408\n" \
  "v_mfma_i32_16x16x64_i8 v[160:163], v[84:87], v[20:23], v[160:163]\n" \
  "ds_read_b64_tr_b8 v[28:29], v1 offset:24576\n" \
  "v_mfma_i32_16x16x64_i8 v[160:163], v[68:71], v[16:19], v[160:163]\n" \
  "ds_read_b64_tr_b8 v[30:31], v1 offset:25600\n" \
  "v_mfma_i32_16x16x64_i8 v[192:195], v[72:75], v[16:19], v[192:195]\n" \
  "global_load_dwordx4 v[32:35], v8, s[40:41] offset:0 sc0 sc1\n" \
  "v_mfma_i32_16x16x64_i8 v[192:195], v[88:91], v[20:23], v[192:195]\n" \
  "global_load_dwordx4 v[36:39], v8, s[40:41] offset:1024 sc0 sc1\n" \
  "v_mfma_i32_16x16x64_i8 v[224:227], v[92:95], v[20:23], v[224:227]\n" \
  "v_mfma_i32_16x16x64_i8 v[224:227], v[76:79], v[16:19], v[224:227]\n" \
  "s_waitcnt lgkmcnt(0)\n" \
  "v_mfma_i32_16x16x64_i8 v[228:231], v[76:79], v[24:27], v[228:231]\n" \
  "ds_read_b64_tr_b8 v[16:17], v2 offset:16384\n" \
  "v_mfma_i32_16x16x64_i8 v[228:231], v[92:95], v[28:31], v[228:231]\n" \
  "ds_read_b64_tr_b8 v[18:19], v2 offset:17408\n" \
  "v_mfma_i32_16x16x64_i8 v[196:199], v[88:91], v[28:31], v[196:199]\n" \
  "ds_read_b64_tr_b8 v[20:21], v2 offset:24576\n" \
  "v_mfma_i32_16x16x64_i8 v[196:199], v[72:75], v[24:27], v[196:199]\n" \
  "ds_read_b64_tr_b8 v[22:23], v2 offset:25600\n" \
  "v_mfma_i32_16x16x64_i8 v[164:167], v[68:71], v[24:27], v[164:167]\n" \
  "global_load_dwordx4 v[40:43], v8, s[40:41] offset:2048 sc0 sc1\n" \
  "v_mfma_i32_16x16x64_i8 v[164:167], v[84:87], v[28:31], v[164:167]\n" \
  "global_load_dwordx4 v[44:47], v8, s[40:41] offset:3072 sc0 sc1\n" \
  "v_mfma_i32_16x16x64_i8 v[132:135], v[80:83], v[28:31], v[132:135]\n" \
  "v_add_u32 v8, 0x8000, v8\n" \
  "v_mfma_i32_16x16x64_i8 v[132:135], v[64:67], v[24:27], v[132:135]\n" \
  "s_waitcnt lgkmcnt(0)\n" \
  "v_mfma_i32_16x16x64_i8 v[136:139], v[64:67], v[16:19], v[136:139]\n" \
  "ds_read_b64_tr_b8 v[24:25], v3 offset:16384\n" \
  "v_mfma_i32_16x16x64_i8 v[136:139], v[80:83], v[20:23], v[136:139]\n" \
  "ds_read_b64_tr_b8 v[26:27], v3 offset:17408\n" \
  "v_mfma_i32_16x16x64_i8 v[168:171], v[84:87], v[20:23], v[168:171]\n" \
  "ds_read_b64_tr_b8 v[28:29], v3 offset:24576\n" \
  "v_mfma_i32_16x16x64_i8 v[168:171], v[68:71], v[16:19], v[168:171]\n" \
  "ds_read_b64_tr_b8 v[30:31], v3 offset:25600\n" \
  "v_mfma_i32_16x16x64_i8 v[200:203], v[72:75], v[16:19], v[200:203]\n" \
  "global_load_dwordx4 v[48:51], v8, s[40:41] offset:0 sc0 sc1\n" \
  "v_mfma_i32_16x16x64_i8 v[200:203], v[88:91], v[20:23], v[200:203]\n" \
  "global_load_dwordx4 v[52:55], v8, s[40:41] offset:1024 sc0 sc1\n" \
  "v_mfma_i32_16x16x64_i8 v[232:235], v[92:95], v[20:23], v[232:235]\n" \
  "v_mfma_i32_16x16x64_i8 v[232:235], v[76:79], v[16:19], v[232:235]\n" \
  "s_waitcnt lgkmcnt(0)\n" \
  "v_mfma_i32_16x16x64_i8 v[236:239], v[76:79], v[24:27], v[236:239]\n" \
  "ds_read_b64_tr_b8 v[16:17], v4 offset:16384\n" \
  "v_mfma_i32_16x16x64_i8 v[236:239], v[92:95], v[28:31], v[236:239]\n" \
  "ds_read_b64_tr_b8 v[18:19], v4 offset:17408\n" \
  "v_mfma_i32_16x16x64_i8 v[204:207], v[88:91], v[28:31], v[204:207]\n" \
  "ds_read_b64_tr_b8 v[20:21], v4 offset:24576\n" \
  "v_mfma_i32_16x16x64_i8 v[204:207], v[72:75], v[24:27], v[204:207]\n" \
  "ds_read_b64_tr_b8 v[22:23], v4 offset:25600\n" \
  "v_mfma_i32_16x16x64_i8 v[172:175], v[68:71], v[24:27], v[172:175]\n" \
  "global_load_dwordx4 v[56:59], v8, s[40:41] offset:2048 sc0 sc1\n" \
  "v_mfma_i32_16x16x64_i8 v[172:175], v[84:87], v[28:31], v[172:175]\n" \
  "global_load_dwordx4 v[60:63], v8, s[40:41] offset:3072 sc0 sc1\n" \
  "v_mfma_i32_16x16x64_i8 v[140:143], v[80:83], v[28:31], v[140:143]\n" \
  "v_add_u32 v8, 0x8000, v8\n" \
  "v_mfma_i32_16x16x64_i8 v[140:143], v[64:67], v[24:27], v[140:143]\n" \
  "s_waitcnt lgkmcnt(0)\n" \
  "v_mfma_i32_16x16x64_i8 v[144:147], v[64:67], v[16:19], v[144:147]\n" \
  "ds_read_b64_tr_b8 v[24:25], v5 offset:16384\n" \
  "v_mfma_i32_16x16x64_i8 v[144:147], v[80:83], v[20:23], v[144:147]\n" \
  "ds_read_b64_tr_b8 v[26:27], v5 offset:17408\n" \
  "v_mfma_i32_16x16x64_i8 v[176:179], v[84:87], v[20:23], v[176:179]\n" \
  "ds_read_b64_tr_b8 v[28:29], v5 offset:24576\n" \
  "v_mfma_i32_16x16x64_i8 v[176:179], v[68:71], v[16:19], v[176:179]\n" \
  "ds_read_b64_tr_b8 v[30:31], v5 offset:25600\n" \
  "v_mfma_i32_16x16x64_i8 v[208:211], v[72:75], v[16:19], v[208:211]\n" \
  "ds_read_b32 v10, v9\n" \
  "v_mfma_i32_16x16x64_i8 v[208:211], v[88:91], v[20:23], v[208:211]\n" \
  "v_mfma_i32_16x16x64_i8 v[240:243], v[92:95], v[20:23], v[240:243]\n" \
  "v_mfma_i32_16x16x64_i8 v[240:243], v[76:79], v[16:19], v[240:243]\n" \
  "s_waitcnt lgkmcnt(1)\n" \
  "v_mfma_i32_16x16x64_i8 v[244:247], v[76:79], v[24:27], v[244:247]\n" \
  "ds_read_b64_tr_b8 v[16:17], v6 offset:16384\n" \
  "v_mfma_i32_16x16x64_i8 v[244:247], v[92:95], v[28:31], v[244:247]\n" \
  "ds_read_b64_tr_b8 v[18:19], v6 offset:17408\n" \
  "v_mfma_i32_16x16x64_i8 v[212:215], v[88:91], v[28:31], v[212:215]\n" \
  "ds_read_b64_tr_b8 v[20:21], v6 offset:24576\n" \
  "v_mfma_i32_16x16x64_i8 v[212:215], v[72:75], v[24:27], v[212:215]\n" \
  "ds_read_b64_tr_b8 v[22:23], v6 offset:25600\n" \
  "v_mfma_i32_16x16x64_i8 v[180:183], v[68:71], v[24:27], v[180:183]\n" \
  "v_mfma_i32_16x16x64_i8 v[180:183], v[84:87], v[28:31], v[180:183]\n" \
  "v_mfma_i32_16x16x64_i8 v[148:151], v[80:83], v[28:31], v[148:151]\n" \
  "v_mfma_i32_16x16x64_i8 v[148:151], v[64:67], v[24:27], v[148:151]\n" \
  "s_waitcnt lgkmcnt(0)\n" \
  "v_mfma_i32_16x16x64_i8 v[152:155], v[64:67], v[16:19], v[152:155]\n" \
  "ds_read_b64_tr_b8 v[24:25], v7 offset:16384\n" \
  "v_mfma_i32_16x16x64_i8 v[152:155], v[80:83], v[20:23], v[152:155]\n" \
  "ds_read_b64_tr_b8 v[26:27], v7 offset:17408\n" \
  "v_mfma_i32_16x16x64_i8 v[184:187], v[84:87], v[20:23], v[184:187]\n" \
  "ds_read_b64_tr_b8 v[28:29], v7 offset:24576\n" \
  "v_mfma_i32_16x16x64_i8 v[184:187], v[68:71], v[16:19], v[184:187]\n" \
  "ds_read_b64_tr_b8 v[30:31], v7 offset:25600\n" \
  "v_mfma_i32_16x16x64_i8 v[216:219], v[72:75], v[16:19], v[216:219]\n" \
  "v_mfma_i32_16x16x64_i8 v[216:219], v[88:91], v[20:23], v[216:219]\n" \
  "v_mfma_i32_16x16x64_i8 v[248:251], v[92:95], v[20:23], v[248:251]\n" \
  "v_cmp_gt_u32 vcc, 42, v10\n" \
  "v_mfma_i32_16x16x64_i8 v[248:251], v[76:79], v[16:19], v[248:251]\n" \
  "s_waitcnt lgkmcnt(0)\n" \
  "v_mfma_i32_16x16x64_i8 v[252:255], v[76:79], v[24:27], v[252:255]\n" \
  "s_cbranch_vccnz LS19_%=\n" \
  "LB19_%=:\n" \
  "ds_read_b64_tr_b8 v[16:17], v0 offset:32768\n" \
  "v_mfma_i32_16x16x64_i8 v[252:255], v[92:95], v[28:31], v[252:255]\n" \
  "ds_read_b64_tr_b8 v[18:19], v0 offset:33792\n" \
  "v_mfma_i32_16x16x64_i8 v[220:223], v[88:91], v[28:31], v[220:223]\n" \
  "ds_read_b64_tr_b8 v[20:21], v0 offset:40960\n" \
  "v_mfma_i32_16x16x64_i8 v[220:223], v[72:75], v[24:27], v[220:223]\n" \
  "ds_read_b64_tr_b8 v[22:23], v0 offset:41984\n" \
  "v_mfma_i32_16x16x64_i8 v[188:191], v[68:71], v[24:27], v[188:191]\n" \
  "ds_add_u32 v12, v11\n" \
  "v_mfma_i32_16x16x64_i8 v[188:191], v[84:87], v[28:31], v[188:191]\n" \
  "v_mfma_i32_16x16x64_i8 v[156:159], v[80:83], v[28:31], v[156:159]\n" \
  "v_mfma_i32_16x16x64_i8 v[156:159], v[64:67], v[24:27], v[156:159]\n" \
  "s_waitcnt vmcnt(8) lgkmcnt(1)\n" \
  "v_mfma_i32_16x16x64_i8 v[128:131], v[96:99], v[16:19], v[128:131]\n" \
  "ds_read_b64_tr_b8 v[24:25], v1 offset:32768\n" \
  "v_mfma_i32_16x16x64_i8 v[128:131], v[112:115], v[20:23], v[128:131]\n" \
  "ds_read_b64_tr_b8 v[26:27], v1 offset:33792\n" \
  "v_mfma_i32_16x16x64_i8 v[160:163], v[116:119], v[20:23], v[160:163]\n" \
  "ds_read_b64_tr_b8 v[28:29], v1 offset:40960\n" \
  "v_mfma_i32_16x16x64_i8 v[160:163], v[100:103], v[16:19], v[160:163]\n" \
  "ds_read_b64_tr_b8 v[30:31], v1 offset:41984\n" \
  "v_mfma_i32_16x16x64_i8 v[192:195], v[104:107], v[16:19], v[192:195]\n" \
  "global_load_dwordx4 v[64:67], v8, s[40:41] offset:0 sc0 sc1\n" \
  "v_mfma_i32_16x16x64_i8 v[192:195], v[120:123], v[20:23], v[192:195]\n" \
  "global_load_dwordx4 v[68:71], v8, s[40:41] offset:1024 sc0 sc1\n" \
  "v_mfma_i32_16x16x64_i8 v[224:227], v[124:127], v[20:23], v[224:227]\n" \
  "v_mfma_i32_16x16x64_i8 v[224:227], v[108:111], v[16:19], v[224:227]\n" \
  "s_waitcnt lgkmcnt(0)\n" \
  "v_mfma_i32_16x16x64_i8 v[228:231], v[108:111], v[24:27], v[228:231]\n" \
  "ds_read_b64_tr_b8 v[16:17], v2 offset:32768\n" \
  "v_mfma_i32_16x16x64_i8 v[228:231], v[124:127], v[28:31], v[228:231]\n" \
  "ds_read_b64_tr_b8 v[18:19], v2 offset:33792\n" \
  "v_mfma_i32_16x16x64_i8 v[196:199], v[120:123], v[28:31], v[196:199]\n" \
  "ds_read_b64_tr_b8 v[20:21], v2 offset:40960\n" \
  "v_mfma_i32_16x16x64_i8 v[196:199], v[104:107], v[24:27], v[196:199]\n" \
  "ds_read_b64_tr_b8 v[22:23], v2 offset:41984\n" \
  "v_mfma_i32_16x16x64_i8 v[164:167], v[100:103], v[24:27], v[164:167]\n" \
  "global_load_dwordx4 v[72:75], v8, s[40:41] offset:2048 sc0 sc1\n" \
  "v_mfma_i32_16x16x64_i8 v[164:167], v[116:119], v[28:31], v[164:167]\n" \
  "global_load_dwordx4 v[76:79], v8, s[40:41] offset:3072 sc0 sc1\n" \
  "v_mfma_i32_16x16x64_i8 v[132:135], v[112:115], v[28:31], v[132:135]\n" \
  "v_add_u32 v8, 0x8000, v8\n" \
  "v_mfma_i32_16x16x64_i8 v[132:135], v[96:99], v[24:27], v[132:135]\n" \
  "s_waitcnt lgkmcnt(0)\n" \
  "v_mfma_i32_16x16x64_i8 v[136:139], v[96:99], v[16:19], v[136:139]\n" \
  "ds_read_b64_tr_b8 v[24:25], v3 offset:32768\n" \
  "v_mfma_i32_16x16x64_i8 v[136:139], v[112:115], v[20:23], v[136:139]\n" \
  "ds_read_b64_tr_b8 v[26:27], v3 offset:33792\n" \
  "v_mfma_i32_16x16x64_i8 v[168:171], v[116:119], v[20:23], v[168:171]\n" \
  "ds_read_b64_tr_b8 v[28:29], v3 offset:40960\n" \
  "v_mfma_i32_16x16x64_i8 v[168:171], v[100:103], v[16:19], v[168:171]\n" \
  "ds_read_b64_tr_b8 v[30:31], v3 offset:41984\n" \
  "v_mfma_i32_16x16x64_i8 v[200:203], v[104:107], v[16:19], v[200:203]\n" \
  "global_load_dwordx4 v[80:83], v8, s[40:41] offset:0 sc0 sc1\n" \
  "v_mfma_i32_16x16x64_i8 v[200:203], v[120:123], v[20:23], v[200:203]\n" \
  "global_load_dwordx4 v[84:87], v8, s[40:41] offset:1024 sc0 sc1\n" \
  "v_mfma_i32_16x16x64_i8 v[232:235], v[124:127], v[20:23], v[232:235]\n" \
  "v_mfma_i32_16x16x64_i8 v[232:235], v[108:111], v[16:19], v[232:235]\n" \
  "s_waitcnt lgkmcnt(0)\n" \
  "v_mfma_i32_16x16x64_i8 v[236:239], v[108:111], v[24:27], v[236:239]\n" \
  "ds_read_b64_tr_b8 v[16:17], v4 offset:32768\n" \
  "v_mfma_i32_16x16x64_i8 v[236:239], v[124:127], v[28:31], v[236:239]\n" \
  "ds_read_b64_tr_b8 v[18:19], v4 offset:33792\n" \
  "v_mfma_i32_16x16x64_i8 v[204:207], v[120:123], v[28:31], v[204:207]\n" \
  "ds_read_b64_tr_b8 v[20:21], v4 offset:40960\n" \
  "v_mfma_i32_16x16x64_i8 v[204:207], v[104:107], v[24:27], v[204:207]\n" \
  "ds_read_b64_tr_b8 v[22:23], v4 offset:41984\n" \
  "v_mfma_i32_16x16x64_i8 v[172:175], v[100:103], v[24:27], v[172:175]\n" \
  "global_load_dwordx4 v[88:91], v8, s[40:41] offset:2048 sc0 sc1\n" \
  "v_mfma_i32_16x16x64_i8 v[172:175], v[116:119], v[28:31], v[172:175]\n" \
  "global_load_dwordx4 v[92:95], v8, s[40:41] offset:3072 sc0 sc1\n" \
  "v_mfma_i32_16x16x64_i8 v[140:143], v[112:115], v[28:31], v[140:143]\n" \
  "v_add_u32 v8, 0x8000, v8\n" \
  "v_mfma_i32_16x16x64_i8 v[140:143], v[96:99], v[24:27], v[140:143]\n" \
  "s_waitcnt lgkmcnt(0)\n" \
  "v_mfma_i32_16x16x64_i8 v[144:147], v[96:99], v[16:19], v[144:147]\n" \
  "ds_read_b64_tr_b8 v[24:25], v5 offset:32768\n" \
  "v_mfma_i32_16x16x64_i8 v[144:147], v[112:115], v[20:23], v[144:147]\n" \
  "ds_read_b64_tr_b8 v[26:27], v5 offset:33792\n" \
  "v_mfma_i32_16x16x64_i8 v[176:179], v[116:119], v[20:23], v[176:179]\n" \
  "ds_read_b64_tr_b8 v[28:29], v5 offset:40960\n" \
  "v_mfma_i32_16x16x64_i8 v[176:179], v[100:103], v[16:19], v[176:179]\n" \
  "ds_read_b64_tr_b8 v[30:31], v5 offset:41984\n" \
  "v_mfma_i32_16x16x64_i8 v[208:211], v[104:107], v[16:19], v[208:211]\n" \
  "ds_read_b32 v10, v9\n" \
  "v_mfma_i32_16x16x64_i8 v[208:211], v[120:123], v[20:23], v[208:211]\n" \
  "v_mfma_i32_16x16x64_i8 v[240:243], v[124:127], v[20:23], v[240:243]\n" \
  "v_mfma_i32_16x16x64_i8 v[240:243], v[108:111], v[16:19], v[240:243]\n" \
  "s_waitcnt lgkmcnt(1)\n" \
  "v_mfma_i32_16x16x64_i8 v[244:247], v[108:111], v[24:27], v[244:247]\n" \
  "ds_read_b64_tr_b8 v[16:17], v6 offset:32768\n" \
  "v_mfma_i32_16x16x64_i8 v[244:247], v[124:127], v[28:31], v[244:247]\n" \
  "ds_read_b64_tr_b8 v[18:19], v6 offset:33792\n" \
  "v_mfma_i32_16x16x64_i8 v[212:215], v[120:123], v[28:31], v[212:215]\n" \
  "ds_read_b64_tr_b8 v[20:21], v6 offset:40960\n" \
  "v_mfma_i32_16x16x64_i8 v[212:215], v[104:107], v[24:27], v[212:215]\n" \
  "ds_read_b64_tr_b8 v[22:23], v6 offset:41984\n" \
  "v_mfma_i32_16x16x64_i8 v[180:183], v[100:103], v[24:27], v[180:183]\n" \
  "v_mfma_i32_16x16x64_i8 v[180:183], v[116:119], v[28:31], v[180:183]\n" \
  "v_mfma_i32_16x16x64_i8 v[148:151], v[112:115], v[28:31], v[148:151]\n" \
  "v_mfma_i32_16x16x64_i8 v[148:151], v[96:99], v[24:27], v[148:151]\n" \
  "s_waitcnt lgkmcnt(0)\n" \
  "v_mfma_i32_16x16x64_i8 v[152:155], v[96:99], v[16:19], v[152:155]\n" \
  "ds_read_b64_tr_b8 v[24:25], v7 offset:32768\n" \
  "v_mfma_i32_16x16x64_i8 v[152:155], v[112:115], v[20:23], v[152:155]\n" \
  "ds_read_b64_tr_b8 v[26:27], v7 offset:33792\n" \
  "v_mfma_i32_16x16x64_i8 v[184:187], v[116:119], v[20:23], v[184:187]\n" \
  "ds_read_b64_tr_b8 v[28:29], v7 offset:40960\n" \
  "v_mfma_i32_16x16x64_i8 v[184:187], v[100:103], v[16:19], v[184:187]\n" \
  "ds_read_b64_tr_b8 v[30:31], v7 offset:41984\n" \
  "v_mfma_i32_16x16x64_i8 v[216:219], v[104:107], v[16:19], v[216:219]\n" \
  "v_mfma_i32_16x16x64_i8 v[216:219], v[120:123], v[20:23], v[216:219]\n" \
  "v_mfma_i32_16x16x64_i8 v[248:251], v[124:127], v[20:23], v[248:251]\n" \
  "v_cmp_gt_u32 vcc, 44, v10\n" \
  "v_mfma_i32_16x16x64_i8 v[248:251], v[108:111], v[16:19], v[248:251]\n" \
  "s_waitcnt lgkmcnt(0)\n" \
  "v_mfma_i32_16x16x64_i8 v[252:255], v[108:111], v[24:27], v[252:255]\n" \
  "s_cbranch_vccnz LS20_%=\n" \
  "LB20_%=:\n" \
  "v_add_u32 v0, 0xc000, v0\n" \
  "ds_read_b64_tr_b8 v[16:17], v0 offset:0\n" \
  "v_mfma_i32_16x16x64_i8 v[252:255], v[124:127], v[28:31], v[252:255]\n" \
  "ds_read_b64_tr_b8 v[18:19], v0 offset:1024\n" \
  "v_mfma_i32_16x16x64_i8 v[220:223], v[120:123], v[28:31], v[220:223]\n" \
  "ds_read_b64_tr_b8 v[20:21], v0 offset:8192\n" \
  "v_mfma_i32_16x16x64_i8 v[220:223], v[104:107], v[24:27], v[220:223]\n" \
  "ds_read_b64_tr_b8 v[22:23], v0 offset:9216\n" \
  "v_mfma_i32_16x16x64_i8 v[188:191], v[100:103], v[24:27], v[188:191]\n" \
  "ds_add_u32 v12, v11\n" \
  "v_mfma_i32_16x16x64_i8 v[188:191], v[116:119], v[28:31], v[188:191]\n" \
  "v_mfma_i32_16x16x64_i8 v[156:159], v[112:115], v[28:31], v[156:159]\n" \
  "v_mfma_i32_16x16x64_i8 v[156:159], v[96:99], v[24:27], v[156:159]\n" \
  "s_waitcnt vmcnt(8) lgkmcnt(1)\n" \
  "v_mfma_i32_16x16x64_i8 v[128:131], v[32:35], v[16:19], v[128:131]\n" \
  "v_add_u32 v1, 0xc000, v1\n" \
  "ds_read_b64_tr_b8 v[24:25], v1 offset:0\n" \
  "v_mfma_i32_16x16x64_i8 v[128:131], v[48:51], v[20:23], v[128:131]\n" \
  "ds_read_b64_tr_b8 v[26:27], v1 offset:1024\n" \
  "v_mfma_i32_16x16x64_i8 v[160:163], v[52:55], v[20:23], v[160:163]\n" \
  "ds_read_b64_tr_b8 v[28:29], v1 offset:8192\n" \
  "v_mfma_i32_16x16x64_i8 v[160:163], v[36:39], v[16:19], v[160:163]\n" \
  "ds_read_b64_tr_b8 v[30:31], v1 offset:9216\n" \
  "v_mfma_i32_16x16x64_i8 v[192:195], v[40:43], v[16:19], v[192:195]\n" \
  "global_load_dwordx4 v[96:99], v8, s[40:41] offset:0 sc0 sc1\n" \
  "v_mfma_i32_16x16x64_i8 v[192:195], v[56:59], v[20:23], v[192:195]\n" \
  "global_load_dwordx4 v[100:103], v8, s[40:41] offset:1024 sc0 sc1\n" \
  "v_mfma_i32_16x16x64_i8 v[224:227], v[60:63], v[20:23], v[224:227]\n" \
  "v_mfma_i32_16x16x64_i8 v[224:227], v[44:47], v[16:19], v[224:227]\n" \
  "s_waitcnt lgkmcnt(0)\n" \
  "v_mfma_i32_16x16x64_i8 v[228:231], v[44:47], v[24:27], v[228:231]\n" \
  "v_add_u32 v2, 0xc000, v2\n" \
  "ds_read_b64_tr_b8 v[16:17], v2 offset:0\n" \
  "v_mfma_i32_16x16x64_i8 v[228:231], v[60:63], v[28:31], v[228:231]\n" \
  "ds_read_b64_tr_b8 v[18:19], v2 offset:1024\n" \
  "v_mfma_i32_16x16x64_i8 v[196:199], v[56:59], v[28:31], v[196:199]\n" \
  "ds_read_b64_tr_b8 v[20:21], v2 offset:8192\n" \
  "v_mfma_i32_16x16x64_i8 v[196:199], v[40:43], v[24:27], v[196:199]\n" \
  "ds_read_b64_tr_b8 v[22:23], v2 offset:9216\n" \
  "v_mfma_i32_16x16x64_i8 v[164:167], v[36:39], v[24:27], v[164:167]\n" \
  "global_load_dwordx4 v[104:107], v8, s[40:41] offset:2048 sc0 sc1\n" \
  "v_mfma_i32_16x16x64_i8 v[164:167], v[52:55], v[28:31], v[164:167]\n" \
  "global_load_dwordx4 v[108:111], v8, s[40:41] offset:3072 sc0 sc1\n" \
  "v_mfma_i32_16x16x64_i8 v[132:135], v[48:51], v[28:31], v[132:135]\n" \
  "v_add_u32 v8, 0x8000, v8\n" \
  "v_mfma_i32_16x16x64_i8 v[132:135], v[32:35], v[24:27], v[132:135]\n" \
  "s_waitcnt lgkmcnt(0)\n" \
  "v_mfma_i32_16x16x64_i8 v[136:139], v[32:35], v[16:19], v[136:139]\n" \
  "v_add_u32 v3, 0xc000, v3\n" \
  "ds_read_b64_tr_b8 v[24:25], v3 offset:0\n" \
  "v_mfma_i32_16x16x64_i8 v[136:139], v[48:51], v[20:23], v[136:139]\n" \
  "ds_read_b64_tr_b8 v[26:27], v3 offset:1024\n" \
  "v_mfma_i32_16x16x64_i8 v[168:171], v[52:55], v[20:23], v[168:171]\n" \
  "ds_read_b64_tr_b8 v[28:29], v3 offset:8192\n" \
  "v_mfma_i32_16x16x64_i8 v[168:171], v[36:39], v[16:19], v[168:171]\n" \
  "ds_read_b64_tr_b8 v[30:31], v3 offset:9216\n" \
  "v_mfma_i32_16x16x64_i8 v[200:203], v[40:43], v[16:19], v[200:203]\n" \
  "global_load_dwordx4 v[112:115], v8, s[40:41] offset:0 sc0 sc1\n" \
  "v_mfma_i32_16x16x64_i8 v[200:203], v[56:59], v[20:23], v[200:203]\n" \
  "global_load_dwordx4 v[116:119], v8, s[40:41] offset:1024 sc0 sc1\n" \
  "v_mfma_i32_16x16x64_i8 v[232:235], v[60:63], v[20:23], v[232:235]\n" \
  "v_mfma_i32_16x16x64_i8 v[232:235], v[44:47], v[16:19], v[232:235]\n" \
  "s_waitcnt lgkmcnt(0)\n" \
  "v_mfma_i32_16x16x64_i8 v[236:239], v[44:47], v[24:27], v[236:239]\n" \
  "v_add_u32 v4, 0xc000, v4\n" \
  "ds_read_b64_tr_b8 v[16:17], v4 offset:0\n" \
  "v_mfma_i32_16x16x64_i8 v[236:239], v[60:63], v[28:31], v[236:239]\n" \
  "ds_read_b64_tr_b8 v[18:19], v4 offset:1024\n" \
  "v_mfma_i32_16x16x64_i8 v[204:207], v[56:59], v[28:31], v[204:207]\n" \
  "ds_read_b64_tr_b8 v[20:21], v4 offset:8192\n" \
  "v_mfma_i32_16x16x64_i8 v[204:207], v[40:43], v[24:27], v[204:207]\n" \
  "ds_read_b64_tr_b8 v[22:23], v4 offset:9216\n" \
  "v_mfma_i32_16x16x64_i8 v[172:175], v[36:39], v[24:27], v[172:175]\n" \
  "global_load_dwordx4 v[120:123], v8, s[40:41] offset:2048 sc0 sc1\n" \
  "v_mfma_i32_16x16x64_i8 v[172:175], v[52:55], v[28:31], v[172:175]\n" \
  "global_load_dwordx4 v[124:127], v8, s[40:41] offset:3072 sc0 sc1\n" \
  "v_mfma_i32_16x16x64_i8 v[140:143], v[48:51], v[28:31], v[140:143]\n" \
  "v_add_u32 v8, 0x8000, v8\n" \
  "v_mfma_i32_16x16x64_i8 v[140:143], v[32:35], v[24:27], v[140:143]\n" \
  "s_waitcnt lgkmcnt(0)\n" \
  "v_mfma_i32_16x16x64_i8 v[144:147], v[32:35], v[16:19], v[144:147]\n" \
  "v_add_u32 v5, 0xc000, v5\n" \
  "ds_read_b64_tr_b8 v[24:25], v5 offset:0\n" \
  "v_mfma_i32_16x16x64_i8 v[144:147], v[48:51], v[20:23], v[144:147]\n" \
  "ds_read_b64_tr_b8 v[26:27], v5 offset:1024\n" \
  "v_mfma_i32_16x16x64_i8 v[176:179], v[52:55], v[20:23], v[176:179]\n" \
  "ds_read_b64_tr_b8 v[28:29], v5 offset:8192\n" \
  "v_mfma_i32_16x16x64_i8 v[176:179], v[36:39], v[16:19], v[176:179]\n" \
  "ds_read_b64_tr_b8 v[30:31], v5 offset:9216\n" \
  "v_mfma_i32_16x16x64_i8 v[208:211], v[40:43], v[16:19], v[208:211]\n" \
  "ds_read_b32 v10, v9\n" \
  "v_mfma_i32_16x16x64_i8 v[208:211], v[56:59], v[20:23], v[208:211]\n" \
  "v_mfma_i32_16x16x64_i8 v[240:243], v[60:63], v[20:23], v[240:243]\n" \
  "v_mfma_i32_16x16x64_i8 v[240:243], v[44:47], v[16:19], v[240:243]\n" \
  "s_waitcnt lgkmcnt(1)\n" \
  "v_mfma_i32_16x16x64_i8 v[244:247], v[44:47], v[24:27], v[244:247]\n" \
  "v_add_u32 v6, 0xc000, v6\n" \
  "ds_read_b64_tr_b8 v[16:17], v6 offset:0\n" \
  "v_mfma_i32_16x16x64_i8 v[244:247], v[60:63], v[28:31], v[244:247]\n" \
  "ds_read_b64_tr_b8 v[18:19], v6 offset:1024\n" \
  "v_mfma_i32_16x16x64_i8 v[212:215], v[56:59], v[28:31], v[212:215]\n" \
  "ds_read_b64_tr_b8 v[20:21], v6 offset:8192\n" \
  "v_mfma_i32_16x16x64_i8 v[212:215], v[40:43], v[24:27], v[212:215]\n" \
  "ds_read_b64_tr_b8 v[22:23], v6 offset:9216\n" \
  "v_mfma_i32_16x16x64_i8 v[180:183], v[36:39], v[24:27], v[180:183]\n" \
  "v_mfma_i32_16x16x64_i8 v[180:183], v[52:55], v[28:31], v[180:183]\n" \
  "v_mfma_i32_16x16x64_i8 v[148:151], v[48:51], v[28:31], v[148:151]\n" \
  "v_mfma_i32_16x16x64_i8 v[148:151], v[32:35], v[24:27], v[148:151]\n" \
  "s_waitcnt lgkmcnt(0)\n" \
  "v_mfma_i32_16x16x64_i8 v[152:155], v[32:35], v[16:19], v[152:155]\n" \
  "v_add_u32 v7, 0xc000, v7\n" \
  "ds_read_b64_tr_b8 v[24:25], v7 offset:0\n" \
  "v_mfma_i32_16x16x64_i8 v[152:155], v[48:51], v[20:23], v[152:155]\n" \
  "ds_read_b64_tr_b8 v[26:27], v7 offset:1024\n" \
  "v_mfma_i32_16x16x64_i8 v[184:187], v[52:55], v[20:23], v[184:187]\n" \
  "ds_read_b64_tr_b8 v[28:29], v7 offset:8192\n" \
  "v_mfma_i32_16x16x64_i8 v[184:187], v[36:39], v[16:19], v[184:187]\n" \
  "ds_read_b64_tr_b8 v[30:31], v7 offset:9216\n" \
  "v_mfma_i32_16x16x64_i8 v[216:219], v[40:43], v[16:19], v[216:219]\n" \
  "v_mfma_i32_16x16x64_i8 v[216:219], v[56:59], v[20:23], v[216:219]\n" \
  "v_mfma_i32_16x16x64_i8 v[248:251], v[60:63], v[20:23], v[248:251]\n" \
  "v_cmp_gt_u32 vcc, 46, v10\n" \
  "v_mfma_i32_16x16x64_i8 v[248:251], v[44:47], v[16:19], v[248:251]\n" \
  "s_waitcnt lgkmcnt(0)\n" \
  "v_mfma_i32_16x16x64_i8 v[252:255], v[44:47], v[24:27], v[252:255]\n" \
  "s_cbranch_vccnz LS21_%=\n" \
  "LB21_%=:\n" \
  "ds_read_b64_tr_b8 v[16:17], v0 offset:16384\n" \
  "v_mfma_i32_16x16x64_i8 v[252:255], v[60:63], v[28:31], v[252:255]\n" \
  "ds_read_b64_tr_b8 v[18:19], v0 offset:17408\n" \
  "v_mfma_i32_16x16x64_i8 v[220:223], v[56:59], v[28:31], v[220:223]\n" \
  "ds_read_b64_tr_b8 v[20:21], v0 offset:24576\n" \
  "v_mfma_i32_16x16x64_i8 v[220:223], v[40:43], v[24:27], v[220:223]\n" \
  "ds_read_b64_tr_b8 v[22:23], v0 offset:25600\n" \
  "v_mfma_i32_16x16x64_i8 v[188:191], v[36:39], v[24:27], v[188:191]\n" \
  "ds_add_u32 v12, v11\n" \
  "v_mfma_i32_16x16x64_i8 v[188:191], v[52:55], v[28:31], v[188:191]\n" \
  "v_mfma_i32_16x16x64_i8 v[156:159], v[48:51], v[28:31], v[156:159]\n" \
  "v_mfma_i32_16x16x64_i8 v[156:159], v[32:35], v[24:27], v[156:159]\n" \
  "s_waitcnt vmcnt(8) lgkmcnt(1)\n" \
  "v_mfma_i32_16x16x64_i8 v[128:131], v[64:67], v[16:19], v[128:131]\n" \
  "ds_read_b64_tr_b8 v[24:25], v1 offset:16384\n" \
  "v_mfma_i32_16x16x64_i8 v[128:131], v[80:83], v[20:23], v[128:131]\n" \
  "ds_read_b64_tr_b8 v[26:27], v1 offset:17408\n" \
  "v_mfma_i32_16x16x64_i8 v[160:163], v[84:87], v[20:23], v[160:163]\n" \
  "ds_read_b64_tr_b8 v[28:29], v1 offset:24576\n" \
  "v_mfma_i32_16x16x64_i8 v[160:163], v[68:71], v[16:19], v[160:163]\n" \
  "ds_read_b64_tr_b8 v[30:31], v1 offset:25600\n" \
  "v_mfma_i32_16x16x64_i8 v[192:195], v[72:75], v[16:19], v[192:195]\n" \
  "global_load_dwordx4 v[32:35], v8, s[40:41] offset:0 sc0 sc1\n" \
  "v_mfma_i32_16x16x64_i8 v[192:195], v[88:91], v[20:23], v[192:195]\n" \
  "global_load_dwordx4 v[36:39], v8, s[40:41] offset:1024 sc0 sc1\n" \
  "v_mfma_i32_16x16x64_i8 v[224:227], v[92:95], v[20:23], v[224:227]\n" \
  "v_mfma_i32_16x16x64_i8 v[224:227], v[76:79], v[16:19], v[224:227]\n" \
  "s_waitcnt lgkmcnt(0)\n" \
  "v_mfma_i32_16x16x64_i8 v[228:231], v[76:79], v[24:27], v[228:231]\n" \
  "ds_read_b64_tr_b8 v[16:17], v2 offset:16384\n" \
  "v_mfma_i32_16x16x64_i8 v[228:231], v[92:95], v[28:31], v[228:231]\n" \
  "ds_read_b64_tr_b8 v[18:19], v2 offset:17408\n" \
  "v_mfma_i32_16x16x64_i8 v[196:199], v[88:91], v[28:31], v[196:199]\n" \
  "ds_read_b64_tr_b8 v[20:21], v2 offset:24576\n" \
  "v_mfma_i32_16x16x64_i8 v[196:199], v[72:75], v[24:27], v[196:199]\n" \
  "ds_read_b64_tr_b8 v[22:23], v2 offset:25600\n" \
  "v_mfma_i32_16x16x64_i8 v[164:167], v[68:71], v[24:27], v[164:167]\n" \
  "global_load_dwordx4 v[40:43], v8, s[40:41] offset:2048 sc0 sc1\n" \
  "v_mfma_i32_16x16x64_i8 v[164:167], v[84:87], v[28:31], v[164:167]\n" \
  "global_load_dwordx4 v[44:47], v8, s[40:41] offset:3072 sc0 sc1\n" \
  "v_mfma_i32_16x16x64_i8 v[132:135], v[80:83], v[28:31], v[132:135]\n" \
  "v_add_u32 v8, 0x8000, v8\n" \
  "v_mfma_i32_16x16x64_i8 v[132:135], v[64:67], v[24:27], v[132:135]\n" \
  "s_waitcnt lgkmcnt(0)\n" \
  "v_mfma_i32_16x16x64_i8 v[136:139], v[64:67], v[16:19], v[136:139]\n" \
  "ds_read_b64_tr_b8 v[24:25], v3 offset:16384\n" \
  "v_mfma_i32_16x16x64_i8 v[136:139], v[80:83], v[20:23], v[136:139]\n" \
  "ds_read_b64_tr_b8 v[26:27], v3 offset:17408\n" \
  "v_mfma_i32_16x16x64_i8 v[168:171], v[84:87], v[20:23], v[168:171]\n" \
  "ds_read_b64_tr_b8 v[28:29], v3 offset:24576\n" \
  "v_mfma_i32_16x16x64_i8 v[168:171], v[68:71], v[16:19], v[168:171]\n" \
  "ds_read_b64_tr_b8 v[30:31], v3 offset:25600\n" \
  "v_mfma_i32_16x16x64_i8 v[200:203], v[72:75], v[16:19], v[200:203]\n" \
  "global_load_dwordx4 v[48:51], v8, s[40:41] offset:0 sc0 sc1\n" \
  "v_mfma_i32_16x16x64_i8 v[200:203], v[88:91], v[20:23], v[200:203]\n" \
  "global_load_dwordx4 v[52:55], v8, s[40:41] offset:1024 sc0 sc1\n" \
  "v_mfma_i32_16x16x64_i8 v[232:235], v[92:95], v[20:23], v[232:235]\n" \
  "v_mfma_i32_16x16x64_i8 v[232:235], v[76:79], v[16:19], v[232:235]\n" \
  "s_waitcnt lgkmcnt(0)\n" \
  "v_mfma_i32_16x16x64_i8 v[236:239], v[76:79], v[24:27], v[236:239]\n" \
  "ds_read_b64_tr_b8 v[16:17], v4 offset:16384\n" \
  "v_mfma_i32_16x16x64_i8 v[236:239], v[92:95], v[28:31], v[236:239]\n" \
  "ds_read_b64_tr_b8 v[18:19], v4 offset:17408\n" \
  "v_mfma_i32_16x16x64_i8 v[204:207], v[88:91], v[28:31], v[204:207]\n" \
  "ds_read_b64_tr_b8 v[20:21], v4 offset:24576\n" \
  "v_mfma_i32_16x16x64_i8 v[204:207], v[72:75], v[24:27], v[204:207]\n" \
  "ds_read_b64_tr_b8 v[22:23], v4 offset:25600\n" \
  "v_mfma_i32_16x16x64_i8 v[172:175], v[68:71], v[24:27], v[172:175]\n" \
  "global_load_dwordx4 v[56:59], v8, s[40:41] offset:2048 sc0 sc1\n" \
  "v_mfma_i32_16x16x64_i8 v[172:175], v[84:87], v[28:31], v[172:175]\n" \
  "global_load_dwordx4 v[60:63], v8, s[40:41] offset:3072 sc0 sc1\n" \
  "v_mfma_i32_16x16x64_i8 v[140:143], v[80:83], v[28:31], v[140:143]\n" \
  "v_add_u32 v8, 0x8000, v8\n" \
  "v_mfma_i32_16x16x64_i8 v[140:143], v[64:67], v[24:27], v[140:143]\n" \
  "s_waitcnt lgkmcnt(0)\n" \
  "v_mfma_i32_16x16x64_i8 v[144:147], v[64:67], v[16:19], v[144:147]\n" \
  "ds_read_b64_tr_b8 v[24:25], v5 offset:16384\n" \
  "v_mfma_i32_16x16x64_i8 v[144:147], v[80:83], v[20:23], v[144:147]\n" \
  "ds_read_b64_tr_b8 v[26:27], v5 offset:17408\n" \
  "v_mfma_i32_16x16x64_i8 v[176:179], v[84:87], v[20:23], v[176:179]\n" \
  "ds_read_b64_tr_b8 v[28:29], v5 offset:24576\n" \
  "v_mfma_i32_16x16x64_i8 v[176:179], v[68:71], v[16:19], v[176:179]\n" \
  "ds_read_b64_tr_b8 v[30:31], v5 offset:25600\n" \
  "v_mfma_i32_16x16x64_i8 v[208:211], v[72:75], v[16:19], v[208:211]\n" \
  "ds_read_b32 v10, v9\n" \
  "v_mfma_i32_16x16x64_i8 v[208:211], v[88:91], v[20:23], v[208:211]\n" \
  "v_mfma_i32_16x16x64_i8 v[240:243], v[92:95], v[20:23], v[240:243]\n" \
  "v_mfma_i32_16x16x64_i8 v[240:243], v[76:79], v[16:19], v[240:243]\n" \
  "s_waitcnt lgkmcnt(1)\n" \
  "v_mfma_i32_16x16x64_i8 v[244:247], v[76:79], v[24:27], v[244:247]\n" \
  "ds_read_b64_tr_b8 v[16:17], v6 offset:16384\n" \
  "v_mfma_i32_16x16x64_i8 v[244:247], v[92:95], v[28:31], v[244:247]\n" \
  "ds_read_b64_tr_b8 v[18:19], v6 offset:17408\n" \
  "v_mfma_i32_16x16x64_i8 v[212:215], v[88:91], v[28:31], v[212:215]\n" \
  "ds_read_b64_tr_b8 v[20:21], v6 offset:24576\n" \
  "v_mfma_i32_16x16x64_i8 v[212:215], v[72:75], v[24:27], v[212:215]\n" \
  "ds_read_b64_tr_b8 v[22:23], v6 offset:25600\n" \
  "v_mfma_i32_16x16x64_i8 v[180:183], v[68:71], v[24:27], v[180:183]\n" \
  "v_mfma_i32_16x16x64_i8 v[180:183], v[84:87], v[28:31], v[180:183]\n" \
  "v_mfma_i32_16x16x64_i8 v[148:151], v[80:83], v[28:31], v[148:151]\n" \
  "v_mfma_i32_16x16x64_i8 v[148:151], v[64:67], v[24:27], v[148:151]\n" \
  "s_waitcnt lgkmcnt(0)\n" \
  "v_mfma_i32_16x16x64_i8 v[152:155], v[64:67], v[16:19], v[152:155]\n" \
  "ds_read_b64_tr_b8 v[24:25], v7 offset:16384\n" \
  "v_mfma_i32_16x16x64_i8 v[152:155], v[80:83], v[20:23], v[152:155]\n" \
  "ds_read_b64_tr_b8 v[26:27], v7 offset:17408\n" \
  "v_mfma_i32_16x16x64_i8 v[184:187], v[84:87], v[20:23], v[184:187]\n" \
  "ds_read_b64_tr_b8 v[28:29], v7 offset:24576\n" \
  "v_mfma_i32_16x16x64_i8 v[184:187], v[68:71], v[16:19], v[184:187]\n" \
  "ds_read_b64_tr_b8 v[30:31], v7 offset:25600\n" \
  "v_mfma_i32_16x16x64_i8 v[216:219], v[72:75], v[16:19], v[216:219]\n" \
  "v_mfma_i32_16x16x64_i8 v[216:219], v[88:91], v[20:23], v[216:219]\n" \
  "v_mfma_i32_16x16x64_i8 v[248:251], v[92:95], v[20:23], v[248:251]\n" \
  "v_cmp_gt_u32 vcc, 48, v10\n" \
  "v_mfma_i32_16x16x64_i8 v[248:251], v[76:79], v[16:19], v[248:251]\n" \
  "s_waitcnt lgkmcnt(0)\n" \
  "v_mfma_i32_16x16x64_i8 v[252:255], v[76:79], v[24:27], v[252:255]\n" \
  "s_cbranch_vccnz LS22_%=\n" \
  "LB22_%=:\n" \
  "ds_read_b64_tr_b8 v[16:17], v0 offset:32768\n" \
  "v_mfma_i32_16x16x64_i8 v[252:255], v[92:95], v[28:31], v[252:255]\n" \
  "ds_read_b64_tr_b8 v[18:19], v0 offset:33792\n" \
  "v_mfma_i32_16x16x64_i8 v[220:223], v[88:91], v[28:31], v[220:223]\n" \
  "ds_read_b64_tr_b8 v[20:21], v0 offset:40960\n" \
  "v_mfma_i32_16x16x64_i8 v[220:223], v[72:75], v[24:27], v[220:223]\n" \
  "ds_read_b64_tr_b8 v[22:23], v0 offset:41984\n" \
  "v_mfma_i32_16x16x64_i8 v[188:191], v[68:71], v[24:27], v[188:191]\n" \
  "ds_add_u32 v12, v11\n" \
  "v_mfma_i32_16x16x64_i8 v[188:191], v[84:87], v[28:31], v[188:191]\n" \
  "v_mfma_i32_16x16x64_i8 v[156:159], v[80:83], v[28:31], v[156:159]\n" \
  "v_mfma_i32_16x16x64_i8 v[156:159], v[64:67], v[24:27], v[156:159]\n" \
  "s_waitcnt vmcnt(8) lgkmcnt(1)\n" \
  "v_mfma_i32_16x16x64_i8 v[128:131], v[96:99], v[16:19], v[128:131]\n" \
  "ds_read_b64_tr_b8 v[24:25], v1 offset:32768\n" \
  "v_mfma_i32_16x16x64_i8 v[128:131], v[112:115], v[20:23], v[128:131]\n" \
  "ds_read_b64_tr_b8 v[26:27], v1 offset:33792\n" \
  "v_mfma_i32_16x16x64_i8 v[160:163], v[116:119], v[20:23], v[160:163]\n" \
  "ds_read_b64_tr_b8 v[28:29], v1 offset:40960\n" \
  "v_mfma_i32_16x16x64_i8 v[160:163], v[100:103], v[16:19], v[160:163]\n" \
  "ds_read_b64_tr_b8 v[30:31], v1 offset:41984\n" \
  "v_mfma_i32_16x16x64_i8 v[192:195], v[104:107], v[16:19], v[192:195]\n" \
  "global_load_dwordx4 v[64:67], v8, s[40:41] offset:0 sc0 sc1\n" \
  "v_mfma_i32_16x16x64_i8 v[192:195], v[120:123], v[20:23], v[192:195]\n" \
  "global_load_dwordx4 v[68:71], v8, s[40:41] offset:1024 sc0 sc1\n" \
  "v_mfma_i32_16x16x64_i8 v[224:227], v[124:127], v[20:23], v[224:227]\n" \
  "v_mfma_i32_16x16x64_i8 v[224:227], v[108:111], v[16:19], v[224:227]\n" \
  "s_waitcnt lgkmcnt(0)\n" \
  "v_mfma_i32_16x16x64_i8 v[228:231], v[108:111], v[24:27], v[228:231]\n" \
  "ds_read_b64_tr_b8 v[16:17], v2 offset:32768\n" \
  "v_mfma_i32_16x16x64_i8 v[228:231], v[124:127], v[28:31], v[228:231]\n" \
  "ds_read_b64_tr_b8 v[18:19], v2 offset:33792\n" \
  "v_mfma_i32_16x16x64_i8 v[196:199], v[120:123], v[28:31], v[196:199]\n" \
  "ds_read_b64_tr_b8 v[20:21], v2 offset:40960\n" \
  "v_mfma_i32_16x16x64_i8 v[196:199], v[104:107], v[24:27], v[196:199]\n" \
  "ds_read_b64_tr_b8 v[22:23], v2 offset:41984\n" \
  "v_mfma_i32_16x16x64_i8 v[164:167], v[100:103], v[24:27], v[164:167]\n" \
  "global_load_dwordx4 v[72:75], v8, s[40:41] offset:2048 sc0 sc1\n" \
  "v_mfma_i32_16x16x64_i8 v[164:167], v[116:119], v[28:31], v[164:167]\n" \
  "global_load_dwordx4 v[76:79], v8, s[40:41] offset:3072 sc0 sc1\n" \
  "v_mfma_i32_16x16x64_i8 v[132:135], v[112:115], v[28:31], v[132:135]\n" \
  "v_add_u32 v8, 0x8000, v8\n" \
  "v_mfma_i32_16x16x64_i8 v[132:135], v[96:99], v[24:27], v[132:135]\n" \
  "s_waitcnt lgkmcnt(0)\n" \
  "v_mfma_i32_16x16x64_i8 v[136:139], v[96:99], v[16:19], v[136:139]\n" \
  "ds_read_b64_tr_b8 v[24:25], v3 offset:32768\n" \
  "v_mfma_i32_16x16x64_i8 v[136:139], v[112:115], v[20:23], v[136:139]\n" \
  "ds_read_b64_tr_b8 v[26:27], v3 offset:33792\n" \
  "v_mfma_i32_16x16x64_i8 v[168:171], v[116:119], v[20:23], v[168:171]\n" \
  "ds_read_b64_tr_b8 v[28:29], v3 offset:40960\n" \
  "v_mfma_i32_16x16x64_i8 v[168:171], v[100:103], v[16:19], v[168:171]\n" \
  "ds_read_b64_tr_b8 v[30:31], v3 offset:41984\n" \
  "v_mfma_i32_16x16x64_i8 v[200:203], v[104:107], v[16:19], v[200:203]\n" \
  "global_load_dwordx4 v[80:83], v8, s[40:41] offset:0 sc0 sc1\n" \
  "v_mfma_i32_16x16x64_i8 v[200:203], v[120:123], v[20:23], v[200:203]\n" \
  "global_load_dwordx4 v[84:87], v8, s[40:41] offset:1024 sc0 sc1\n" \
  "v_mfma_i32_16x16x64_i8 v[232:235], v[124:127], v[20:23], v[232:235]\n" \
  "v_mfma_i32_16x16x64_i8 v[232:235], v[108:111], v[16:19], v[232:235]\n" \
  "s_waitcnt lgkmcnt(0)\n" \
  "v_mfma_i32_16x16x64_i8 v[236:239], v[108:111], v[24:27], v[236:239]\n" \
  "ds_read_b64_tr_b8 v[16:17], v4 offset:32768\n" \
  "v_mfma_i32_16x16x64_i8 v[236:239], v[124:127], v[28:31], v[236:239]\n" \
  "ds_read_b64_tr_b8 v[18:19], v4 offset:33792\n" \
  "v_mfma_i32_16x16x64_i8 v[204:207], v[120:123], v[28:31], v[204:207]\n" \
  "ds_read_b64_tr_b8 v[20:21], v4 offset:40960\n" \
  "v_mfma_i32_16x16x64_i8 v[204:207], v[104:107], v[24:27], v[204:207]\n" \
  "ds_read_b64_tr_b8 v[22:23], v4 offset:41984\n" \
  "v_mfma_i32_16x16x64_i8 v[172:175], v[100:103], v[24:27], v[172:175]\n" \
  "global_load_dwordx4 v[88:91], v8, s[40:41] offset:2048 sc0 sc1\n" \
  "v_mfma_i32_16x16x64_i8 v[172:175], v[116:119], v[28:31], v[172:175]\n" \
  "global_load_dwordx4 v[92:95], v8, s[40:41] offset:3072 sc0 sc1\n" \
  "v_mfma_i32_16x16x64_i8 v[140:143], v[112:115], v[28:31], v[140:143]\n" \
  "v_add_u32 v8, 0x8000, v8\n" \
  "v_mfma_i32_16x16x64_i8 v[140:143], v[96:99], v[24:27], v[140:143]\n" \
  "s_waitcnt lgkmcnt(0)\n" \
  "v_mfma_i32_16x16x64_i8 v[144:147], v[96:99], v[16:19], v[144:147]\n" \
  "ds_read_b64_tr_b8 v[24:25], v5 offset:32768\n" \
  "v_mfma_i32_16x16x64_i8 v[144:147], v[112:115], v[20:23], v[144:147]\n" \
  "ds_read_b64_tr_b8 v[26:27], v5 offset:33792\n" \
  "v_mfma_i32_16x16x64_i8 v[176:179], v[116:119], v[20:23], v[176:179]\n" \
  "ds_read_b64_tr_b8 v[28:29], v5 offset:40960\n" \
  "v_mfma_i32_16x16x64_i8 v[176:179], v[100:103], v[16:19], v[176:179]\n" \
  "ds_read_b64_tr_b8 v[30:31], v5 offset:41984\n" \
  "v_mfma_i32_16x16x64_i8 v[208:211], v[104:107], v[16:19], v[208:211]\n" \
  "ds_read_b32 v10, v9\n" \
  "v_mfma_i32_16x16x64_i8 v[208:211], v[120:123], v[20:23], v[208:211]\n" \
  "v_mfma_i32_16x16x64_i8 v[240:243], v[124:127], v[20:23], v[240:243]\n" \
  "v_mfma_i32_16x16x64_i8 v[240:243], v[108:111], v[16:19], v[240:243]\n" \
  "s_waitcnt lgkmcnt(1)\n" \
  "v_mfma_i32_16x16x64_i8 v[244:247], v[108:111], v[24:27], v[244:247]\n" \
  "ds_read_b64_tr_b8 v[16:17], v6 offset:32768\n" \
  "v_mfma_i32_16x16x64_i8 v[244:247], v[124:127], v[28:31], v[244:247]\n" \
  "ds_read_b64_tr_b8 v[18:19], v6 offset:33792\n" \
  "v_mfma_i32_16x16x64_i8 v[212:215], v[120:123], v[28:31], v[212:215]\n" \
  "ds_read_b64_tr_b8 v[20:21], v6 offset:40960\n" \
  "v_mfma_i32_16x16x64_i8 v[212:215], v[104:107], v[24:27], v[212:215]\n" \
  "ds_read_b64_tr_b8 v[22:23], v6 offset:41984\n" \
  "v_mfma_i32_16x16x64_i8 v[180:183], v[100:103], v[24:27], v[180:183]\n" \
  "v_mfma_i32_16x16x64_i8 v[180:183], v[116:119], v[28:31], v[180:183]\n" \
  "v_mfma_i32_16x16x64_i8 v[148:151], v[112:115], v[28:31], v[148:151]\n" \
  "v_mfma_i32_16x16x64_i8 v[148:151], v[96:99], v[24:27], v[148:151]\n" \
  "s_waitcnt lgkmcnt(0)\n" \
  "v_mfma_i32_16x16x64_i8 v[152:155], v[96:99], v[16:19], v[152:155]\n" \
  "ds_read_b64_tr_b8 v[24:25], v7 offset:32768\n" \
  "v_mfma_i32_16x16x64_i8 v[152:155], v[112:115], v[20:23], v[152:155]\n" \
  "ds_read_b64_tr_b8 v[26:27], v7 offset:33792\n" \
  "v_mfma_i32_16x16x64_i8 v[184:187], v[116:119], v[20:23], v[184:187]\n" \
  "ds_read_b64_tr_b8 v[28:29], v7 offset:40960\n" \
  "v_mfma_i32_16x16x64_i8 v[184:187], v[100:103], v[16:19], v[184:187]\n" \
  "ds_read_b64_tr_b8 v[30:31], v7 offset:41984\n" \
  "v_mfma_i32_16x16x64_i8 v[216:219], v[104:107], v[16:19], v[216:219]\n" \
  "v_mfma_i32_16x16x64_i8 v[216:219], v[120:123], v[20:23], v[216:219]\n" \
  "v_mfma_i32_16x16x64_i8 v[248:251], v[124:127], v[20:23], v[248:251]\n" \
  "v_cmp_gt_u32 vcc, 50, v10\n" \
  "v_mfma_i32_16x16x64_i8 v[248:251], v[108:111], v[16:19], v[248:251]\n" \
  "s_waitcnt lgkmcnt(0)\n" \
  "v_mfma_i32_16x16x64_i8 v[252:255], v[108:111], v[24:27], v[252:255]\n" \
  "s_cbranch_vccnz LS23_%=\n" \
  "LB23_%=:\n" \
  "v_add_u32 v0, 0xc000, v0\n" \
  "ds_read_b64_tr_b8 v[16:17], v0 offset:0\n" \
  "v_mfma_i32_16x16x64_i8 v[252:255], v[124:127], v[28:31], v[252:255]\n" \
  "ds_read_b64_tr_b8 v[18:19], v0 offset:1024\n" \
  "v_mfma_i32_16x16x64_i8 v[220:223], v[120:123], v[28:31], v[220:223]\n" \
  "ds_read_b64_tr_b8 v[20:21], v0 offset:8192\n" \
  "v_mfma_i32_16x16x64_i8 v[220:223], v[104:107], v[24:27], v[220:223]\n" \
  "ds_read_b64_tr_b8 v[22:23], v0 offset:9216\n" \
  "v_mfma_i32_16x16x64_i8 v[188:191], v[100:103], v[24:27], v[188:191]\n" \
  "ds_add_u32 v12, v11\n" \
  "v_mfma_i32_16x16x64_i8 v[188:191], v[116:119], v[28:31], v[188:191]\n" \
  "v_mfma_i32_16x16x64_i8 v[156:159], v[112:115], v[28:31], v[156:159]\n" \
  "v_mfma_i32_16x16x64_i8 v[156:159], v[96:99], v[24:27], v[156:159]\n" \
  "s_waitcnt vmcnt(8) lgkmcnt(1)\n" \
  "v_mfma_i32_16x16x64_i8 v[128:131], v[32:35], v[16:19], v[128:131]\n" \
  "v_add_u32 v1, 0xc000, v1\n" \
  "ds_read_b64_tr_b8 v[24:25], v1 offset:0\n" \
  "v_mfma_i32_16x16x64_i8 v[128:131], v[48:51], v[20:23], v[128:131]\n" \
  "ds_read_b64_tr_b8 v[26:27], v1 offset:1024\n" \
  "v_mfma_i32_16x16x64_i8 v[160:163], v[52:55], v[20:23], v[160:163]\n" \
  "ds_read_b64_tr_b8 v[28:29], v1 offset:8192\n" \
  "v_mfma_i32_16x16x64_i8 v[160:163], v[36:39], v[16:19], v[160:163]\n" \
  "ds_read_b64_tr_b8 v[30:31], v1 offset:9216\n" \
  "v_mfma_i32_16x16x64_i8 v[192:195], v[40:43], v[16:19], v[192:195]\n" \
  "global_load_dwordx4 v[96:99], v8, s[40:41] offset:0 sc0 sc1\n" \
  "v_mfma_i32_16x16x64_i8 v[192:195], v[56:59], v[20:23], v[192:195]\n" \
  "global_load_dwordx4 v[100:103], v8, s[40:41] offset:1024 sc0 sc1\n" \
  "v_mfma_i32_16x16x64_i8 v[224:227], v[60:63], v[20:23], v[224:227]\n" \
  "v_mfma_i32_16x16x64_i8 v[224:227], v[44:47], v[16:19], v[224:227]\n" \
  "s_waitcnt lgkmcnt(0)\n" \
  "v_mfma_i32_16x16x64_i8 v[228:231], v[44:47], v[24:27], v[228:231]\n" \
  "v_add_u32 v2, 0xc000, v2\n" \
  "ds_read_b64_tr_b8 v[16:17], v2 offset:0\n" \
  "v_mfma_i32_16x16x64_i8 v[228:231], v[60:63], v[28:31], v[228:231]\n" \
  "ds_read_b64_tr_b8 v[18:19], v2 offset:1024\n" \
  "v_mfma_i32_16x16x64_i8 v[196:199], v[56:59], v[28:31], v[196:199]\n" \
  "ds_read_b64_tr_b8 v[20:21], v2 offset:8192\n" \
  "v_mfma_i32_16x16x64_i8 v[196:199], v[40:43], v[24:27], v[196:199]\n" \
  "ds_read_b64_tr_b8 v[22:23], v2 offset:9216\n" \
  "v_mfma_i32_16x16x64_i8 v[164:167], v[36:39], v[24:27], v[164:167]\n" \
  "global_load_dwordx4 v[104:107], v8, s[40:41] offset:2048 sc0 sc1\n" \
  "v_mfma_i32_16x16x64_i8 v[164:167], v[52:55], v[28:31], v[164:167]\n" \
  "global_load_dwordx4 v[108:111], v8, s[40:41] offset:3072 sc0 sc1\n" \
  "v_mfma_i32_16x16x64_i8 v[132:135], v[48:51], v[28:31], v[132:135]\n" \
  "v_add_u32 v8, 0x8000, v8\n" \
  "v_mfma_i32_16x16x64_i8 v[132:135], v[32:35], v[24:27], v[132:135]\n" \
  "s_waitcnt lgkmcnt(0)\n" \
  "v_mfma_i32_16x16x64_i8 v[136:139], v[32:35], v[16:19], v[136:139]\n" \
  "v_add_u32 v3, 0xc000, v3\n" \
  "ds_read_b64_tr_b8 v[24:25], v3 offset:0\n" \
  "v_mfma_i32_16x16x64_i8 v[136:139], v[48:51], v[20:23], v[136:139]\n" \
  "ds_read_b64_tr_b8 v[26:27], v3 offset:1024\n" \
  "v_mfma_i32_16x16x64_i8 v[168:171], v[52:55], v[20:23], v[168:171]\n" \
  "ds_read_b64_tr_b8 v[28:29], v3 offset:8192\n" \
  "v_mfma_i32_16x16x64_i8 v[168:171], v[36:39], v[16:19], v[168:171]\n" \
  "ds_read_b64_tr_b8 v[30:31], v3 offset:9216\n" \
  "v_mfma_i32_16x16x64_i8 v[200:203], v[40:43], v[16:19], v[200:203]\n" \
  "global_load_dwordx4 v[112:115], v8, s[40:41] offset:0 sc0 sc1\n" \
  "v_mfma_i32_16x16x64_i8 v[200:203], v[56:59], v[20:23], v[200:203]\n" \
  "global_load_dwordx4 v[116:119], v8, s[40:41] offset:1024 sc0 sc1\n" \
  "v_mfma_i32_16x16x64_i8 v[232:235], v[60:63], v[20:23], v[232:235]\n" \
  "v_mfma_i32_16x16x64_i8 v[232:235], v[44:47], v[16:19], v[232:235]\n" \
  "s_waitcnt lgkmcnt(0)\n" \
  "v_mfma_i32_16x16x64_i8 v[236:239], v[44:47], v[24:27], v[236:239]\n" \
  "v_add_u32 v4, 0xc000, v4\n" \
  "ds_read_b64_tr_b8 v[16:17], v4 offset:0\n" \
  "v_mfma_i32_16x16x64_i8 v[236:239], v[60:63], v[28:31], v[236:239]\n" \
  "ds_read_b64_tr_b8 v[18:19], v4 offset:1024\n" \
  "v_mfma_i32_16x16x64_i8 v[204:207], v[56:59], v[28:31], v[204:207]\n" \
  "ds_read_b64_tr_b8 v[20:21], v4 offset:8192\n" \
  "v_mfma_i32_16x16x64_i8 v[204:207], v[40:43], v[24:27], v[204:207]\n" \
  "ds_read_b64_tr_b8 v[22:23], v4 offset:9216\n" \
  "v_mfma_i32_16x16x64_i8 v[172:175], v[36:39], v[24:27], v[172:175]\n" \
  "global_load_dwordx4 v[120:123], v8, s[40:41] offset:2048 sc0 sc1\n" \
  "v_mfma_i32_16x16x64_i8 v[172:175], v[52:55], v[28:31], v[172:175]\n" \
  "global_load_dwordx4 v[124:127], v8, s[40:41] offset:3072 sc0 sc1\n" \
  "v_mfma_i32_16x16x64_i8 v[140:143], v[48:51], v[28:31], v[140:143]\n" \
  "v_add_u32 v8, 0x8000, v8\n" \
  "v_mfma_i32_16x16x64_i8 v[140:143], v[32:35], v[24:27], v[140:143]\n" \
  "s_waitcnt lgkmcnt(0)\n" \
  "v_mfma_i32_16x16x64_i8 v[144:147], v[32:35], v[16:19], v[144:147]\n" \
  "v_add_u32 v5, 0xc000, v5\n" \
  "ds_read_b64_tr_b8 v[24:25], v5 offset:0\n" \
  "v_mfma_i32_16x16x64_i8 v[144:147], v[48:51], v[20:23], v[144:147]\n" \
  "ds_read_b64_tr_b8 v[26:27], v5 offset:1024\n" \
  "v_mfma_i32_16x16x64_i8 v[176:179], v[52:55], v[20:23], v[176:179]\n" \
  "ds_read_b64_tr_b8 v[28:29], v5 offset:8192\n" \
  "v_mfma_i32_16x16x64_i8 v[176:179], v[36:39], v[16:19], v[176:179]\n" \
  "ds_read_b64_tr_b8 v[30:31], v5 offset:9216\n" \
  "v_mfma_i32_16x16x64_i8 v[208:211], v[40:43], v[16:19], v[208:211]\n" \
  "ds_read_b32 v10, v9\n" \
  "v_mfma_i32_16x16x64_i8 v[208:211], v[56:59], v[20:23], v[208:211]\n" \
  "v_mfma_i32_16x16x64_i8 v[240:243], v[60:63], v[20:23], v[240:243]\n" \
  "v_mfma_i32_16x16x64_i8 v[240:243], v[44:47], v[16:19], v[240:243]\n" \
  "s_waitcnt lgkmcnt(1)\n" \
  "v_mfma_i32_16x16x64_i8 v[244:247], v[44:47], v[24:27], v[244:247]\n" \
  "v_add_u32 v6, 0xc000, v6\n" \
  "ds_read_b64_tr_b8 v[16:17], v6 offset:0\n" \
  "v_mfma_i32_16x16x64_i8 v[244:247], v[60:63], v[28:31], v[244:247]\n" \
  "ds_read_b64_tr_b8 v[18:19], v6 offset:1024\n" \
  "v_mfma_i32_16x16x64_i8 v[212:215], v[56:59], v[28:31], v[212:215]\n" \
  "ds_read_b64_tr_b8 v[20:21], v6 offset:8192\n" \
  "v_mfma_i32_16x16x64_i8 v[212:215], v[40:43], v[24:27], v[212:215]\n" \
  "ds_read_b64_tr_b8 v[22:23], v6 offset:9216\n" \
  "v_mfma_i32_16x16x64_i8 v[180:183], v[36:39], v[24:27], v[180:183]\n" \
  "v_mfma_i32_16x16x64_i8 v[180:183], v[52:55], v[28:31], v[180:183]\n" \
  "v_mfma_i32_16x16x64_i8 v[148:151], v[48:51], v[28:31], v[148:151]\n" \
  "v_mfma_i32_16x16x64_i8 v[148:151], v[32:35], v[24:27], v[148:151]\n" \
  "s_waitcnt lgkmcnt(0)\n" \
  "v_mfma_i32_16x16x64_i8 v[152:155], v[32:35], v[16:19], v[152:155]\n" \
  "v_add_u32 v7, 0xc000, v7\n" \
  "ds_read_b64_tr_b8 v[24:25], v7 offset:0\n" \
  "v_mfma_i32_16x16x64_i8 v[152:155], v[48:51], v[20:23], v[152:155]\n" \
  "ds_read_b64_tr_b8 v[26:27], v7 offset:1024\n" \
  "v_mfma_i32_16x16x64_i8 v[184:187], v[52:55], v[20:23], v[184:187]\n" \
  "ds_read_b64_tr_b8 v[28:29], v7 offset:8192\n" \
  "v_mfma_i32_16x16x64_i8 v[184:187], v[36:39], v[16:19], v[184:187]\n" \
  "ds_read_b64_tr_b8 v[30:31], v7 offset:9216\n" \
  "v_mfma_i32_16x16x64_i8 v[216:219], v[40:43], v[16:19], v[216:219]\n" \
  "v_mfma_i32_16x16x64_i8 v[216:219], v[56:59], v[20:23], v[216:219]\n" \
  "v_mfma_i32_16x16x64_i8 v[248:251], v[60:63], v[20:23], v[248:251]\n" \
  "v_cmp_gt_u32 vcc, 52, v10\n" \
  "v_mfma_i32_16x16x64_i8 v[248:251], v[44:47], v[16:19], v[248:251]\n" \
  "s_waitcnt lgkmcnt(0)\n" \
  "v_mfma_i32_16x16x64_i8 v[252:255], v[44:47], v[24:27], v[252:255]\n" \
  "s_cbranch_vccnz LS24_%=\n" \
  "LB24_%=:\n" \
  "ds_read_b64_tr_b8 v[16:17], v0 offset:16384\n" \
  "v_mfma_i32_16x16x64_i8 v[252:255], v[60:63], v[28:31], v[252:255]\n" \
  "ds_read_b64_tr_b8 v[18:19], v0 offset:17408\n" \
  "v_mfma_i32_16x16x64_i8 v[220:223], v[56:59], v[28:31], v[220:223]\n" \
  "ds_read_b64_tr_b8 v[20:21], v0 offset:24576\n" \
  "v_mfma_i32_16x16x64_i8 v[220:223], v[40:43], v[24:27], v[220:223]\n" \
  "ds_read_b64_tr_b8 v[22:23], v0 offset:25600\n" \
  "v_mfma_i32_16x16x64_i8 v[188:191], v[36:39], v[24:27], v[188:191]\n" \
  "ds_add_u32 v12, v11\n" \
  "v_mfma_i32_16x16x64_i8 v[188:191], v[52:55], v[28:31], v[188:191]\n" \
  "v_mfma_i32_16x16x64_i8 v[156:159], v[48:51], v[28:31], v[156:159]\n" \
  "v_mfma_i32_16x16x64_i8 v[156:159], v[32:35], v[24:27], v[156:159]\n" \
  "s_waitcnt vmcnt(8) lgkmcnt(1)\n" \
  "v_mfma_i32_16x16x64_i8 v[128:131], v[64:67], v[16:19], v[128:131]\n" \
  "ds_read_b64_tr_b8 v[24:25], v1 offset:16384\n" \
  "v_mfma_i32_16x16x64_i8 v[128:131], v[80:83], v[20:23], v[128:131]\n" \
  "ds_read_b64_tr_b8 v[26:27], v1 offset:17408\n" \
  "v_mfma_i32_16x16x64_i8 v[160:163], v[84:87], v[20:23], v[160:163]\n" \
  "ds_read_b64_tr_b8 v[28:29], v1 offset:24576\n" \
  "v_mfma_i32_16x16x64_i8 v[160:163], v[68:71], v[16:19], v[160:163]\n" \
  "ds_read_b64_tr_b8 v[30:31], v1 offset:25600\n" \
  "v_mfma_i32_16x16x64_i8 v[192:195], v[72:75], v[16:19], v[192:195]\n" \
  "global_load_dwordx4 v[32:35], v8, s[40:41] offset:0 sc0 sc1\n" \
  "v_mfma_i32_16x16x64_i8 v[192:195], v[88:91], v[20:23], v[192:195]\n" \
  "global_load_dwordx4 v[36:39], v8, s[40:41] offset:1024 sc0 sc1\n" \
  "v_mfma_i32_16x16x64_i8 v[224:227], v[92:95], v[20:23], v[224:227]\n" \
  "v_mfma_i32_16x16x64_i8 v[224:227], v[76:79], v[16:19], v[224:227]\n" \
  "s_waitcnt lgkmcnt(0)\n" \
  "v_mfma_i32_16x16x64_i8 v[228:231], v[76:79], v[24:27], v[228:231]\n" \
  "ds_read_b64_tr_b8 v[16:17], v2 offset:16384\n" \
  "v_mfma_i32_16x16x64_i8 v[228:231], v[92:95], v[28:31], v[228:231]\n" \
  "ds_read_b64_tr_b8 v[18:19], v2 offset:17408\n" \
  "v_mfma_i32_16x16x64_i8 v[196:199], v[88:91], v[28:31], v[196:199]\n" \
  "ds_read_b64_tr_b8 v[20:21], v2 offset:24576\n" \
  "v_mfma_i32_16x16x64_i8 v[196:199], v[72:75], v[24:27], v[196:199]\n" \
  "ds_read_b64_tr_b8 v[22:23], v2 offset:25600\n" \
  "v_mfma_i32_16x16x64_i8 v[164:167], v[68:71], v[24:27], v[164:167]\n" \
  "global_load_dwordx4 v[40:43], v8, s[40:41] offset:2048 sc0 sc1\n" \
  "v_mfma_i32_16x16x64_i8 v[164:167], v[84:87], v[28:31], v[164:167]\n" \
  "global_load_dwordx4 v[44:47], v8, s[40:41] offset:3072 sc0 sc1\n" \
  "v_mfma_i32_16x16x64_i8 v[132:135], v[80:83], v[28:31], v[132:135]\n" \
  "v_add_u32 v8, 0x8000, v8\n" \
  "v_mfma_i32_16x16x64_i8 v[132:135], v[64:67], v[24:27], v[132:135]\n" \
  "s_waitcnt lgkmcnt(0)\n" \
  "v_mfma_i32_16x16x64_i8 v[136:139], v[64:67], v[16:19], v[136:139]\n" \
  "ds_read_b64_tr_b8 v[24:25], v3 offset:16384\n" \
  "v_mfma_i32_16x16x64_i8 v[136:139], v[80:83], v[20:23], v[136:139]\n" \
  "ds_read_b64_tr_b8 v[26:27], v3 offset:17408\n" \
  "v_mfma_i32_16x16x64_i8 v[168:171], v[84:87], v[20:23], v[168:171]\n" \
  "ds_read_b64_tr_b8 v[28:29], v3 offset:24576\n" \
  "v_mfma_i32_16x16x64_i8 v[168:171], v[68:71], v[16:19], v[168:171]\n" \
  "ds_read_b64_tr_b8 v[30:31], v3 offset:25600\n" \
  "v_mfma_i32_16x16x64_i8 v[200:203], v[72:75], v[16:19], v[200:203]\n" \
  "global_load_dwordx4 v[48:51], v8, s[40:41] offset:0 sc0 sc1\n" \
  "v_mfma_i32_16x16x64_i8 v[200:203], v[88:91], v[20:23], v[200:203]\n" \
  "global_load_dwordx4 v[52:55], v8, s[40:41] offset:1024 sc0 sc1\n" \
  "v_mfma_i32_16x16x64_i8 v[232:235], v[92:95], v[20:23], v[232:235]\n" \
  "v_mfma_i32_16x16x64_i8 v[232:235], v[76:79], v[16:19], v[232:235]\n" \
  "s_waitcnt lgkmcnt(0)\n" \
  "v_mfma_i32_16x16x64_i8 v[236:239], v[76:79], v[24:27], v[236:239]\n" \
  "ds_read_b64_tr_b8 v[16:17], v4 offset:16384\n" \
  "v_mfma_i32_16x16x64_i8 v[236:239], v[92:95], v[28:31], v[236:239]\n" \
  "ds_read_b64_tr_b8 v[18:19], v4 offset:17408\n" \
  "v_mfma_i32_16x16x64_i8 v[204:207], v[88:91], v[28:31], v[204:207]\n" \
  "ds_read_b64_tr_b8 v[20:21], v4 offset:24576\n" \
  "v_mfma_i32_16x16x64_i8 v[204:207], v[72:75], v[24:27], v[204:207]\n" \
  "ds_read_b64_tr_b8 v[22:23], v4 offset:25600\n" \
  "v_mfma_i32_16x16x64_i8 v[172:175], v[68:71], v[24:27], v[172:175]\n" \
  "global_load_dwordx4 v[56:59], v8, s[40:41] offset:2048 sc0 sc1\n" \
  "v_mfma_i32_16x16x64_i8 v[172:175], v[84:87], v[28:31], v[172:175]\n" \
  "global_load_dwordx4 v[60:63], v8, s[40:41] offset:3072 sc0 sc1\n" \
  "v_mfma_i32_16x16x64_i8 v[140:143], v[80:83], v[28:31], v[140:143]\n" \
  "v_add_u32 v8, 0x8000, v8\n" \
  "v_mfma_i32_16x16x64_i8 v[140:143], v[64:67], v[24:27], v[140:143]\n" \
  "s_waitcnt lgkmcnt(0)\n" \
  "v_mfma_i32_16x16x64_i8 v[144:147], v[64:67], v[16:19], v[144:147]\n" \
  "ds_read_b64_tr_b8 v[24:25], v5 offset:16384\n" \
  "v_mfma_i32_16x16x64_i8 v[144:147], v[80:83], v[20:23], v[144:147]\n" \
  "ds_read_b64_tr_b8 v[26:27], v5 offset:17408\n" \
  "v_mfma_i32_16x16x64_i8 v[176:179], v[84:87], v[20:23], v[176:179]\n" \
  "ds_read_b64_tr_b8 v[28:29], v5 offset:24576\n" \
  "v_mfma_i32_16x16x64_i8 v[176:179], v[68:71], v[16:19], v[176:179]\n" \
  "ds_read_b64_tr_b8 v[30:31], v5 offset:25600\n" \
  "v_mfma_i32_16x16x64_i8 v[208:211], v[72:75], v[16:19], v[208:211]\n" \
  "ds_read_b32 v10, v9\n" \
  "v_mfma_i32_16x16x64_i8 v[208:211], v[88:91], v[20:23], v[208:211]\n" \
  "v_mfma_i32_16x16x64_i8 v[240:243], v[92:95], v[20:23], v[240:243]\n" \
  "v_mfma_i32_16x16x64_i8 v[240:243], v[76:79], v[16:19], v[240:243]\n" \
  "s_waitcnt lgkmcnt(1)\n" \
  "v_mfma_i32_16x16x64_i8 v[244:247], v[76:79], v[24:27], v[244:247]\n" \
  "ds_read_b64_tr_b8 v[16:17], v6 offset:16384\n" \
  "v_mfma_i32_16x16x64_i8 v[244:247], v[92:95], v[28:31], v[244:247]\n" \
  "ds_read_b64_tr_b8 v[18:19], v6 offset:17408\n" \
  "v_mfma_i32_16x16x64_i8 v[212:215], v[88:91], v[28:31], v[212:215]\n" \
  "ds_read_b64_tr_b8 v[20:21], v6 offset:24576\n" \
  "v_mfma_i32_16x16x64_i8 v[212:215], v[72:75], v[24:27], v[212:215]\n" \
  "ds_read_b64_tr_b8 v[22:23], v6 offset:25600\n" \
  "v_mfma_i32_16x16x64_i8 v[180:183], v[68:71], v[24:27], v[180:183]\n" \
  "v_mfma_i32_16x16x64_i8 v[180:183], v[84:87], v[28:31], v[180:183]\n" \
  "v_mfma_i32_16x16x64_i8 v[148:151], v[80:83], v[28:31], v[148:151]\n" \
  "v_mfma_i32_16x16x64_i8 v[148:151], v[64:67], v[24:27], v[148:151]\n" \
  "s_waitcnt lgkmcnt(0)\n" \
  "v_mfma_i32_16x16x64_i8 v[152:155], v[64:67], v[16:19], v[152:155]\n" \
  "ds_read_b64_tr_b8 v[24:25], v7 offset:16384\n" \
  "v_mfma_i32_16x16x64_i8 v[152:155], v[80:83], v[20:23], v[152:155]\n" \
  "ds_read_b64_tr_b8 v[26:27], v7 offset:17408\n" \
  "v_mfma_i32_16x16x64_i8 v[184:187], v[84:87], v[20:23], v[184:187]\n" \
  "ds_read_b64_tr_b8 v[28:29], v7 offset:24576\n" \
  "v_mfma_i32_16x16x64_i8 v[184:187], v[68:71], v[16:19], v[184:187]\n" \
  "ds_read_b64_tr_b8 v[30:31], v7 offset:25600\n" \
  "v_mfma_i32_16x16x64_i8 v[216:219], v[72:75], v[16:19], v[216:219]\n" \
  "v_mfma_i32_16x16x64_i8 v[216:219], v[88:91], v[20:23], v[216:219]\n" \
  "v_mfma_i32_16x16x64_i8 v[248:251], v[92:95], v[20:23], v[248:251]\n" \
  "v_cmp_gt_u32 vcc, 54, v10\n" \
  "v_mfma_i32_16x16x64_i8 v[248:251], v[76:79], v[16:19], v[248:251]\n" \
  "s_waitcnt lgkmcnt(0)\n" \
  "v_mfma_i32_16x16x64_i8 v[252:255], v[76:79], v[24:27], v[252:255]\n" \
  "s_cbranch_vccnz LS25_%=\n" \
  "LB25_%=:\n" \
  "ds_read_b64_tr_b8 v[16:17], v0 offset:32768\n" \
  "v_mfma_i32_16x16x64_i8 v[252:255], v[92:95], v[28:31], v[252:255]\n" \
  "ds_read_b64_tr_b8 v[18:19], v0 offset:33792\n" \
  "v_mfma_i32_16x16x64_i8 v[220:223], v[88:91], v[28:31], v[220:223]\n" \
  "ds_read_b64_tr_b8 v[20:21], v0 offset:40960\n" \
  "v_mfma_i32_16x16x64_i8 v[220:223], v[72:75], v[24:27], v[220:223]\n" \
  "ds_read_b64_tr_b8 v[22:23], v0 offset:41984\n" \
  "v_mfma_i32_16x16x64_i8 v[188:191], v[68:71], v[24:27], v[188:191]\n" \
  "ds_add_u32 v12, v11\n" \
  "v_mfma_i32_16x16x64_i8 v[188:191], v[84:87], v[28:31], v[188:191]\n" \
  "v_mfma_i32_16x16x64_i8 v[156:159], v[80:83], v[28:31], v[156:159]\n" \
  "v_mfma_i32_16x16x64_i8 v[156:159], v[64:67], v[24:27], v[156:159]\n" \
  "s_waitcnt vmcnt(8) lgkmcnt(1)\n" \
  "v_mfma_i32_16x16x64_i8 v[128:131], v[96:99], v[16:19], v[128:131]\n" \
  "ds_read_b64_tr_b8 v[24:25], v1 offset:32768\n" \
  "v_mfma_i32_16x16x64_i8 v[128:131], v[112:115], v[20:23], v[128:131]\n" \
  "ds_read_b64_tr_b8 v[26:27], v1 offset:33792\n" \
  "v_mfma_i32_16x16x64_i8 v[160:163], v[116:119], v[20:23], v[160:163]\n" \
  "ds_read_b64_tr_b8 v[28:29], v1 offset:40960\n" \
  "v_mfma_i32_16x16x64_i8 v[160:163], v[100:103], v[16:19], v[160:163]\n" \
  "ds_read_b64_tr_b8 v[30:31], v1 offset:41984\n" \
  "v_mfma_i32_16x16x64_i8 v[192:195], v[104:107], v[16:19], v[192:195]\n" \
  "global_load_dwordx4 v[64:67], v8, s[40:41] offset:0 sc0 sc1\n" \
  "v_mfma_i32_16x16x64_i8 v[192:195], v[120:123], v[20:23], v[192:195]\n" \
  "global_load_dwordx4 v[68:71], v8, s[40:41] offset:1024 sc0 sc1\n" \
  "v_mfma_i32_16x16x64_i8 v[224:227], v[124:127], v[20:23], v[224:227]\n" \
  "v_mfma_i32_16x16x64_i8 v[224:227], v[108:111], v[16:19], v[224:227]\n" \
  "s_waitcnt lgkmcnt(0)\n" \
  "v_mfma_i32_16x16x64_i8 v[228:231], v[108:111], v[24:27], v[228:231]\n" \
  "ds_read_b64_tr_b8 v[16:17], v2 offset:32768\n" \
  "v_mfma_i32_16x16x64_i8 v[228:231], v[124:127], v[28:31], v[228:231]\n" \
  "ds_read_b64_tr_b8 v[18:19], v2 offset:33792\n" \
  "v_mfma_i32_16x16x64_i8 v[196:199], v[120:123], v[28:31], v[196:199]\n" \
  "ds_read_b64_tr_b8 v[20:21], v2 offset:40960\n" \
  "v_mfma_i32_16x16x64_i8 v[196:199], v[104:107], v[24:27], v[196:199]\n" \
  "ds_read_b64_tr_b8 v[22:23], v2 offset:41984\n" \
  "v_mfma_i32_16x16x64_i8 v[164:167], v[100:103], v[24:27], v[164:167]\n" \
  "global_load_dwordx4 v[72:75], v8, s[40:41] offset:2048 sc0 sc1\n" \
  "v_mfma_i32_16x16x64_i8 v[164:167], v[116:119], v[28:31], v[164:167]\n" \
  "global_load_dwordx4 v[76:79], v8, s[40:41] offset:3072 sc0 sc1\n" \
  "v_mfma_i32_16x16x64_i8 v[132:135], v[112:115], v[28:31], v[132:135]\n" \
  "v_add_u32 v8, 0x8000, v8\n" \
  "v_mfma_i32_16x16x64_i8 v[132:135], v[96:99], v[24:27], v[132:135]\n" \
  "s_waitcnt lgkmcnt(0)\n" \
  "v_mfma_i32_16x16x64_i8 v[136:139], v[96:99], v[16:19], v[136:139]\n" \
  "ds_read_b64_tr_b8 v[24:25], v3 offset:32768\n" \
  "v_mfma_i32_16x16x64_i8 v[136:139], v[112:115], v[20:23], v[136:139]\n" \
  "ds_read_b64_tr_b8 v[26:27], v3 offset:33792\n" \
  "v_mfma_i32_16x16x64_i8 v[168:171], v[116:119], v[20:23], v[168:171]\n" \
  "ds_read_b64_tr_b8 v[28:29], v3 offset:40960\n" \
  "v_mfma_i32_16x16x64_i8 v[168:171], v[100:103], v[16:19], v[168:171]\n" \
  "ds_read_b64_tr_b8 v[30:31], v3 offset:41984\n" \
  "v_mfma_i32_16x16x64_i8 v[200:203], v[104:107], v[16:19], v[200:203]\n" \
  "global_load_dwordx4 v[80:83], v8, s[40:41] offset:0 sc0 sc1\n" \
  "v_mfma_i32_16x16x64_i8 v[200:203], v[120:123], v[20:23], v[200:203]\n" \
  "global_load_dwordx4 v[84:87], v8, s[40:41] offset:1024 sc0 sc1\n" \
  "v_mfma_i32_16x16x64_i8 v[232:235], v[124:127], v[20:23], v[232:235]\n" \
  "v_mfma_i32_16x16x64_i8 v[232:235], v[108:111], v[16:19], v[232:235]\n" \
  "s_waitcnt lgkmcnt(0)\n" \
  "v_mfma_i32_16x16x64_i8 v[236:239], v[108:111], v[24:27], v[236:239]\n" \
  "ds_read_b64_tr_b8 v[16:17], v4 offset:32768\n" \
  "v_mfma_i32_16x16x64_i8 v[236:239], v[124:127], v[28:31], v[236:239]\n" \
  "ds_read_b64_tr_b8 v[18:19], v4 offset:33792\n" \
  "v_mfma_i32_16x16x64_i8 v[204:207], v[120:123], v[28:31], v[204:207]\n" \
  "ds_read_b64_tr_b8 v[20:21], v4 offset:40960\n" \
  "v_mfma_i32_16x16x64_i8 v[204:207], v[104:107], v[24:27], v[204:207]\n" \
  "ds_read_b64_tr_b8 v[22:23], v4 offset:41984\n" \
  "v_mfma_i32_16x16x64_i8 v[172:175], v[100:103], v[24:27], v[172:175]\n" \
  "global_load_dwordx4 v[88:91], v8, s[40:41] offset:2048 sc0 sc1\n" \
  "v_mfma_i32_16x16x64_i8 v[172:175], v[116:119], v[28:31], v[172:175]\n" \
  "global_load_dwordx4 v[92:95], v8, s[40:41] offset:3072 sc0 sc1\n" \
  "v_mfma_i32_16x16x64_i8 v[140:143], v[112:115], v[28:31], v[140:143]\n" \
  "v_add_u32 v8, 0x8000, v8\n" \
  "v_mfma_i32_16x16x64_i8 v[140:143], v[96:99], v[24:27], v[140:143]\n" \
  "s_waitcnt lgkmcnt(0)\n" \
  "v_mfma_i32_16x16x64_i8 v[144:147], v[96:99], v[16:19], v[144:147]\n" \
  "ds_read_b64_tr_b8 v[24:25], v5 offset:32768\n" \
  "v_mfma_i32_16x16x64_i8 v[144:147], v[112:115], v[20:23], v[144:147]\n" \
  "ds_read_b64_tr_b8 v[26:27], v5 offset:33792\n" \
  "v_mfma_i32_16x16x64_i8 v[176:179], v[116:119], v[20:23], v[176:179]\n" \
  "ds_read_b64_tr_b8 v[28:29], v5 offset:40960\n" \
  "v_mfma_i32_16x16x64_i8 v[176:179], v[100:103], v[16:19], v[176:179]\n" \
  "ds_read_b64_tr_b8 v[30:31], v5 offset:41984\n" \
  "v_mfma_i32_16x16x64_i8 v[208:211], v[104:107], v[16:19], v[208:211]\n" \
  "ds_read_b32 v10, v9\n" \
  "v_mfma_i32_16x16x64_i8 v[208:211], v[120:123], v[20:23], v[208:211]\n" \
  "v_mfma_i32_16x16x64_i8 v[240:243], v[124:127], v[20:23], v[240:243]\n" \
  "v_mfma_i32_16x16x64_i8 v[240:243], v[108:111], v[16:19], v[240:243]\n" \
  "s_waitcnt lgkmcnt(1)\n" \
  "v_mfma_i32_16x16x64_i8 v[244:247], v[108:111], v[24:27], v[244:247]\n" \
  "ds_read_b64_tr_b8 v[16:17], v6 offset:32768\n" \
  "v_mfma_i32_16x16x64_i8 v[244:247], v[124:127], v[28:31], v[244:247]\n" \
  "ds_read_b64_tr_b8 v[18:19], v6 offset:33792\n" \
  "v_mfma_i32_16x16x64_i8 v[212:215], v[120:123], v[28:31], v[212:215]\n" \
  "ds_read_b64_tr_b8 v[20:21], v6 offset:40960\n" \
  "v_mfma_i32_16x16x64_i8 v[212:215], v[104:107], v[24:27], v[212:215]\n" \
  "ds_read_b64_tr_b8 v[22:23], v6 offset:41984\n" \
  "v_mfma_i32_16x16x64_i8 v[180:183], v[100:103], v[24:27], v[180:183]\n" \
  "v_mfma_i32_16x16x64_i8 v[180:183], v[116:119], v[28:31], v[180:183]\n" \
  "v_mfma_i32_16x16x64_i8 v[148:151], v[112:115], v[28:31], v[148:151]\n" \
  "v_mfma_i32_16x16x64_i8 v[148:151], v[96:99], v[24:27], v[148:151]\n" \
  "s_waitcnt lgkmcnt(0)\n" \
  "v_mfma_i32_16x16x64_i8 v[152:155], v[96:99], v[16:19], v[152:155]\n" \
  "ds_read_b64_tr_b8 v[24:25], v7 offset:32768\n" \
  "v_mfma_i32_16x16x64_i8 v[152:155], v[112:115], v[20:23], v[152:155]\n" \
  "ds_read_b64_tr_b8 v[26:27], v7 offset:33792\n" \
  "v_mfma_i32_16x16x64_i8 v[184:187], v[116:119], v[20:23], v[184:187]\n" \
  "ds_read_b64_tr_b8 v[28:29], v7 offset:40960\n" \
  "v_mfma_i32_16x16x64_i8 v[184:187], v[100:103], v[16:19], v[184:187]\n" \
  "ds_read_b64_tr_b8 v[30:31], v7 offset:41984\n" \
  "v_mfma_i32_16x16x64_i8 v[216:219], v[104:107], v[16:19], v[216:219]\n" \
  "v_mfma_i32_16x16x64_i8 v[216:219], v[120:123], v[20:23], v[216:219]\n" \
  "v_mfma_i32_16x16x64_i8 v[248:251], v[124:127], v[20:23], v[248:251]\n" \
  "v_cmp_gt_u32 vcc, 56, v10\n" \
  "v_mfma_i32_16x16x64_i8 v[248:251], v[108:111], v[16:19], v[248:251]\n" \
  "s_waitcnt lgkmcnt(0)\n" \
  "v_mfma_i32_16x16x64_i8 v[252:255], v[108:111], v[24:27], v[252:255]\n" \
  "s_cbranch_vccnz LS26_%=\n" \
  "LB26_%=:\n" \
  "v_add_u32 v0, 0xfffe8000, v0\n" \
  "ds_read_b64_tr_b8 v[16:17], v0 offset:0\n" \
  "v_mfma_i32_16x16x64_i8 v[252:255], v[124:127], v[28:31], v[252:255]\n" \
  "ds_read_b64_tr_b8 v[18:19], v0 offset:1024\n" \
  "v_mfma_i32_16x16x64_i8 v[220:223], v[120:123], v[28:31], v[220:223]\n" \
  "ds_read_b64_tr_b8 v[20:21], v0 offset:8192\n" \
  "v_mfma_i32_16x16x64_i8 v[220:223], v[104:107], v[24:27], v[220:223]\n" \
  "ds_read_b64_tr_b8 v[22:23], v0 offset:9216\n" \
  "v_mfma_i32_16x16x64_i8 v[188:191], v[100:103], v[24:27], v[188:191]\n" \
  "ds_add_u32 v12, v11\n" \
  "v_mfma_i32_16x16x64_i8 v[188:191], v[116:119], v[28:31], v[188:191]\n" \
  "v_mfma_i32_16x16x64_i8 v[156:159], v[112:115], v[28:31], v[156:159]\n" \
  "v_mfma_i32_16x16x64_i8 v[156:159], v[96:99], v[24:27], v[156:159]\n" \
  "s_waitcnt vmcnt(8) lgkmcnt(1)\n" \
  "v_mfma_i32_16x16x64_i8 v[128:131], v[32:35], v[16:19], v[128:131]\n" \
  "v_add_u32 v1, 0xfffe8000, v1\n" \
  "ds_read_b64_tr_b8 v[24:25], v1 offset:0\n" \
  "v_mfma_i32_16x16x64_i8 v[128:131], v[48:51], v[20:23], v[128:131]\n" \
  "ds_read_b64_tr_b8 v[26:27], v1 offset:1024\n" \
  "v_mfma_i32_16x16x64_i8 v[160:163], v[52:55], v[20:23], v[160:163]\n" \
  "ds_read_b64_tr_b8 v[28:29], v1 offset:8192\n" \
  "v_mfma_i32_16x16x64_i8 v[160:163], v[36:39], v[16:19], v[160:163]\n" \
  "ds_read_b64_tr_b8 v[30:31], v1 offset:9216\n" \
  "v_mfma_i32_16x16x64_i8 v[192:195], v[40:43], v[16:19], v[192:195]\n" \
  "global_load_dwordx4 v[96:99], v8, s[40:41] offset:0 sc0 sc1\n" \
  "v_mfma_i32_16x16x64_i8 v[192:195], v[56:59], v[20:23], v[192:195]\n" \
  "global_load_dwordx4 v[100:103], v8, s[40:41] offset:1024 sc0 sc1\n" \
  "v_mfma_i32_16x16x64_i8 v[224:227], v[60:63], v[20:23], v[224:227]\n" \
  "v_mfma_i32_16x16x64_i8 v[224:227], v[44:47], v[16:19], v[224:227]\n" \
  "s_waitcnt lgkmcnt(0)\n" \
  "v_mfma_i32_16x16x64_i8 v[228:231], v[44:47], v[24:27], v[228:231]\n" \
  "v_add_u32 v2, 0xfffe8000, v2\n" \
  "ds_read_b64_tr_b8 v[16:17], v2 offset:0\n" \
  "v_mfma_i32_16x16x64_i8 v[228:231], v[60:63], v[28:31], v[228:231]\n" \
  "ds_read_b64_tr_b8 v[18:19], v2 offset:1024\n" \
  "v_mfma_i32_16x16x64_i8 v[196:199], v[56:59], v[28:31], v[196:199]\n" \
  "ds_read_b64_tr_b8 v[20:21], v2 offset:8192\n" \
  "v_mfma_i32_16x16x64_i8 v[196:199], v[40:43], v[24:27], v[196:199]\n" \
  "ds_read_b64_tr_b8 v[22:23], v2 offset:9216\n" \
  "v_mfma_i32_16x16x64_i8 v[164:167], v[36:39], v[24:27], v[164:167]\n" \
  "global_load_dwordx4 v[104:107], v8, s[40:41] offset:2048 sc0 sc1\n" \
  "v_mfma_i32_16x16x64_i8 v[164:167], v[52:55], v[28:31], v[164:167]\n" \
  "global_load_dwordx4 v[108:111], v8, s[40:41] offset:3072 sc0 sc1\n" \
  "v_mfma_i32_16x16x64_i8 v[132:135], v[48:51], v[28:31], v[132:135]\n" \
  "v_add_u32 v8, 0x8000, v8\n" \
  "v_mfma_i32_16x16x64_i8 v[132:135], v[32:35], v[24:27], v[132:135]\n" \
  "s_waitcnt lgkmcnt(0)\n" \
  "v_mfma_i32_16x16x64_i8 v[136:139], v[32:35], v[16:19], v[136:139]\n" \
  "v_add_u32 v3, 0xfffe8000, v3\n" \
  "ds_read_b64_tr_b8 v[24:25], v3 offset:0\n" \
  "v_mfma_i32_16x16x64_i8 v[136:139], v[48:51], v[20:23], v[136:139]\n" \
  "ds_read_b64_tr_b8 v[26:27], v3 offset:1024\n" \
  "v_mfma_i32_16x16x64_i8 v[168:171], v[52:55], v[20:23], v[168:171]\n" \
  "ds_read_b64_tr_b8 v[28:29], v3 offset:8192\n" \
  "v_mfma_i32_16x16x64_i8 v[168:171], v[36:39], v[16:19], v[168:171]\n" \
  "ds_read_b64_tr_b8 v[30:31], v3 offset:9216\n" \
  "v_mfma_i32_16x16x64_i8 v[200:203], v[40:43], v[16:19], v[200:203]\n" \
  "global_load_dwordx4 v[112:115], v8, s[40:41] offset:0 sc0 sc1\n" \
  "v_mfma_i32_16x16x64_i8 v[200:203], v[56:59], v[20:23], v[200:203]\n" \
  "global_load_dwordx4 v[116:119], v8, s[40:41] offset:1024 sc0 sc1\n" \
  "v_mfma_i32_16x16x64_i8 v[232:235], v[60:63], v[20:23], v[232:235]\n" \
  "v_mfma_i32_16x16x64_i8 v[232:235], v[44:47], v[16:19], v[232:235]\n" \
  "s_waitcnt lgkmcnt(0)\n" \
  "v_mfma_i32_16x16x64_i8 v[236:239], v[44:47], v[24:27], v[236:239]\n" \
  "v_add_u32 v4, 0xfffe8000, v4\n" \
  "ds_read_b64_tr_b8 v[16:17], v4 offset:0\n" \
  "v_mfma_i32_16x16x64_i8 v[236:239], v[60:63], v[28:31], v[236:239]\n" \
  "ds_read_b64_tr_b8 v[18:19], v4 offset:1024\n" \
  "v_mfma_i32_16x16x64_i8 v[204:207], v[56:59], v[28:31], v[204:207]\n" \
  "ds_read_b64_tr_b8 v[20:21], v4 offset:8192\n" \
  "v_mfma_i32_16x16x64_i8 v[204:207], v[40:43], v[24:27], v[204:207]\n" \
  "ds_read_b64_tr_b8 v[22:23], v4 offset:9216\n" \
  "v_mfma_i32_16x16x64_i8 v[172:175], v[36:39], v[24:27], v[172:175]\n" \
  "global_load_dwordx4 v[120:123], v8, s[40:41] offset:2048 sc0 sc1\n" \
  "v_mfma_i32_16x16x64_i8 v[172:175], v[52:55], v[28:31], v[172:175]\n" \
  "global_load_dwordx4 v[124:127], v8, s[40:41] offset:3072 sc0 sc1\n" \
  "v_mfma_i32_16x16x64_i8 v[140:143], v[48:51], v[28:31], v[140:143]\n" \
  "v_add_u32 v8, 0x8000, v8\n" \
  "v_mfma_i32_16x16x64_i8 v[140:143], v[32:35], v[24:27], v[140:143]\n" \
  "s_waitcnt lgkmcnt(0)\n" \
  "v_mfma_i32_16x16x64_i8 v[144:147], v[32:35], v[16:19], v[144:147]\n" \
  "v_add_u32 v5, 0xfffe8000, v5\n" \
  "ds_read_b64_tr_b8 v[24:25], v5 offset:0\n" \
  "v_mfma_i32_16x16x64_i8 v[144:147], v[48:51], v[20:23], v[144:147]\n" \
  "ds_read_b64_tr_b8 v[26:27], v5 offset:1024\n" \
  "v_mfma_i32_16x16x64_i8 v[176:179], v[52:55], v[20:23], v[176:179]\n" \
  "ds_read_b64_tr_b8 v[28:29], v5 offset:8192\n" \
  "v_mfma_i32_16x16x64_i8 v[176:179], v[36:39], v[16:19], v[176:179]\n" \
  "ds_read_b64_tr_b8 v[30:31], v5 offset:9216\n" \
  "v_mfma_i32_16x16x64_i8 v[208:211], v[40:43], v[16:19], v[208:211]\n" \
  "ds_read_b32 v10, v9\n" \
  "v_mfma_i32_16x16x64_i8 v[208:211], v[56:59], v[20:23], v[208:211]\n" \
  "v_mfma_i32_16x16x64_i8 v[240:243], v[60:63], v[20:23], v[240:243]\n" \
  "v_mfma_i32_16x16x64_i8 v[240:243], v[44:47], v[16:19], v[240:243]\n" \
  "s_waitcnt lgkmcnt(1)\n" \
  "v_mfma_i32_16x16x64_i8 v[244:247], v[44:47], v[24:27], v[244:247]\n" \
  "v_add_u32 v6, 0xfffe8000, v6\n" \
  "ds_read_b64_tr_b8 v[16:17], v6 offset:0\n" \
  "v_mfma_i32_16x16x64_i8 v[244:247], v[60:63], v[28:31], v[244:247]\n" \
  "ds_read_b64_tr_b8 v[18:19], v6 offset:1024\n" \
  "v_mfma_i32_16x16x64_i8 v[212:215], v[56:59], v[28:31], v[212:215]\n" \
  "ds_read_b64_tr_b8 v[20:21], v6 offset:8192\n" \
  "v_mfma_i32_16x16x64_i8 v[212:215], v[40:43], v[24:27], v[212:215]\n" \
  "ds_read_b64_tr_b8 v[22:23], v6 offset:9216\n" \
  "v_mfma_i32_16x16x64_i8 v[180:183], v[36:39], v[24:27], v[180:183]\n" \
  "v_mfma_i32_16x16x64_i8 v[180:183], v[52:55], v[28:31], v[180:183]\n" \
  "v_mfma_i32_16x16x64_i8 v[148:151], v[48:51], v[28:31], v[148:151]\n" \
  "v_mfma_i32_16x16x64_i8 v[148:151], v[32:35], v[24:27], v[148:151]\n" \
  "s_waitcnt lgkmcnt(0)\n" \
  "v_mfma_i32_16x16x64_i8 v[152:155], v[32:35], v[16:19], v[152:155]\n" \
  "v_add_u32 v7, 0xfffe8000, v7\n" \
  "ds_read_b64_tr_b8 v[24:25], v7 offset:0\n" \
  "v_mfma_i32_16x16x64_i8 v[152:155], v[48:51], v[20:23], v[152:155]\n" \
  "ds_read_b64_tr_b8 v[26:27], v7 offset:1024\n" \
  "v_mfma_i32_16x16x64_i8 v[184:187], v[52:55], v[20:23], v[184:187]\n" \
  "ds_read_b64_tr_b8 v[28:29], v7 offset:8192\n" \
  "v_mfma_i32_16x16x64_i8 v[184:187], v[36:39], v[16:19], v[184:187]\n" \
  "ds_read_b64_tr_b8 v[30:31], v7 offset:9216\n" \
  "v_mfma_i32_16x16x64_i8 v[216:219], v[40:43], v[16:19], v[216:219]\n" \
  "v_mfma_i32_16x16x64_i8 v[216:219], v[56:59], v[20:23], v[216:219]\n" \
  "v_mfma_i32_16x16x64_i8 v[248:251], v[60:63], v[20:23], v[248:251]\n" \
  "v_cmp_gt_u32 vcc, 58, v10\n" \
  "v_mfma_i32_16x16x64_i8 v[248:251], v[44:47], v[16:19], v[248:251]\n" \
  "s_waitcnt lgkmcnt(0)\n" \
  "v_mfma_i32_16x16x64_i8 v[252:255], v[44:47], v[24:27], v[252:255]\n" \
  "s_cbranch_vccnz LS27_%=\n" \
  "LB27_%=:\n" \
  "ds_read_b64_tr_b8 v[16:17], v0 offset:16384\n" \
  "v_mfma_i32_16x16x64_i8 v[252:255], v[60:63], v[28:31], v[252:255]\n" \
  "ds_read_b64_tr_b8 v[18:19], v0 offset:17408\n" \
  "v_mfma_i32_16x16x64_i8 v[220:223], v[56:59], v[28:31], v[220:223]\n" \
  "ds_read_b64_tr_b8 v[20:21], v0 offset:24576\n" \
  "v_mfma_i32_16x16x64_i8 v[220:223], v[40:43], v[24:27], v[220:223]\n" \
  "ds_read_b64_tr_b8 v[22:23], v0 offset:25600\n" \
  "v_mfma_i32_16x16x64_i8 v[188:191], v[36:39], v[24:27], v[188:191]\n" \
  "ds_add_u32 v12, v11\n" \
  "v_mfma_i32_16x16x64_i8 v[188:191], v[52:55], v[28:31], v[188:191]\n" \
  "v_mfma_i32_16x16x64_i8 v[156:159], v[48:51], v[28:31], v[156:159]\n" \
  "v_mfma_i32_16x16x64_i8 v[156:159], v[32:35], v[24:27], v[156:159]\n" \
  "s_waitcnt vmcnt(8) lgkmcnt(1)\n" \
  "v_mfma_i32_16x16x64_i8 v[128:131], v[64:67], v[16:19], v[128:131]\n" \
  "ds_read_b64_tr_b8 v[24:25], v1 offset:16384\n" \
  "v_mfma_i32_16x16x64_i8 v[128:131], v[80:83], v[20:23], v[128:131]\n" \
  "ds_read_b64_tr_b8 v[26:27], v1 offset:17408\n" \
  "v_mfma_i32_16x16x64_i8 v[160:163], v[84:87], v[20:23], v[160:163]\n" \
  "ds_read_b64_tr_b8 v[28:29], v1 offset:24576\n" \
  "v_mfma_i32_16x16x64_i8 v[160:163], v[68:71], v[16:19], v[160:163]\n" \
  "ds_read_b64_tr_b8 v[30:31], v1 offset:25600\n" \
  "v_mfma_i32_16x16x64_i8 v[192:195], v[72:75], v[16:19], v[192:195]\n" \
  "global_load_dwordx4 v[32:35], v8, s[40:41] offset:0 sc0 sc1\n" \
  "v_mfma_i32_16x16x64_i8 v[192:195], v[88:91], v[20:23], v[192:195]\n" \
  "global_load_dwordx4 v[36:39], v8, s[40:41] offset:1024 sc0 sc1\n" \
  "v_mfma_i32_16x16x64_i8 v[224:227], v[92:95], v[20:23], v[224:227]\n" \
  "v_mfma_i32_16x16x64_i8 v[224:227], v[76:79], v[16:19], v[224:227]\n" \
  "s_waitcnt lgkmcnt(0)\n" \
  "v_mfma_i32_16x16x64_i8 v[228:231], v[76:79], v[24:27], v[228:231]\n" \
  "ds_read_b64_tr_b8 v[16:17], v2 offset:16384\n" \
  "v_mfma_i32_16x16x64_i8 v[228:231], v[92:95], v[28:31], v[228:231]\n" \
  "ds_read_b64_tr_b8 v[18:19], v2 offset:17408\n" \
  "v_mfma_i32_16x16x64_i8 v[196:199], v[88:91], v[28:31], v[196:199]\n" \
  "ds_read_b64_tr_b8 v[20:21], v2 offset:24576\n" \
  "v_mfma_i32_16x16x64_i8 v[196:199], v[72:75], v[24:27], v[196:199]\n" \
  "ds_read_b64_tr_b8 v[22:23], v2 offset:25600\n" \
  "v_mfma_i32_16x16x64_i8 v[164:167], v[68:71], v[24:27], v[164:167]\n" \
  "global_load_dwordx4 v[40:43], v8, s[40:41] offset:2048 sc0 sc1\n" \
  "v_mfma_i32_16x16x64_i8 v[164:167], v[84:87], v[28:31], v[164:167]\n" \
  "global_load_dwordx4 v[44:47], v8, s[40:41] offset:3072 sc0 sc1\n" \
  "v_mfma_i32_16x16x64_i8 v[132:135], v[80:83], v[28:31], v[132:135]\n" \
  "v_add_u32 v8, 0x8000, v8\n" \
  "v_mfma_i32_16x16x64_i8 v[132:135], v[64:67], v[24:27], v[132:135]\n" \
  "s_waitcnt lgkmcnt(0)\n" \
  "v_mfma_i32_16x16x64_i8 v[136:139], v[64:67], v[16:19], v[136:139]\n" \
  "ds_read_b64_tr_b8 v[24:25], v3 offset:16384\n" \
  "v_mfma_i32_16x16x64_i8 v[136:139], v[80:83], v[20:23], v[136:139]\n" \
  "ds_read_b64_tr_b8 v[26:27], v3 offset:17408\n" \
  "v_mfma_i32_16x16x64_i8 v[168:171], v[84:87], v[20:23], v[168:171]\n" \
  "ds_read_b64_tr_b8 v[28:29], v3 offset:24576\n" \
  "v_mfma_i32_16x16x64_i8 v[168:171], v[68:71], v[16:19], v[168:171]\n" \
  "ds_read_b64_tr_b8 v[30:31], v3 offset:25600\n" \
  "v_mfma_i32_16x16x64_i8 v[200:203], v[72:75], v[16:19], v[200:203]\n" \
  "global_load_dwordx4 v[48:51], v8, s[40:41] offset:0 sc0 sc1\n" \
  "v_mfma_i32_16x16x64_i8 v[200:203], v[88:91], v[20:23], v[200:203]\n" \
  "global_load_dwordx4 v[52:55], v8, s[40:41] offset:1024 sc0 sc1\n" \
  "v_mfma_i32_16x16x64_i8 v[232:235], v[92:95], v[20:23], v[232:235]\n" \
  "v_mfma_i32_16x16x64_i8 v[232:235], v[76:79], v[16:19], v[232:235]\n" \
  "s_waitcnt lgkmcnt(0)\n" \
  "v_mfma_i32_16x16x64_i8 v[236:239], v[76:79], v[24:27], v[236:239]\n" \
  "ds_read_b64_tr_b8 v[16:17], v4 offset:16384\n" \
  "v_mfma_i32_16x16x64_i8 v[236:239], v[92:95], v[28:31], v[236:239]\n" \
  "ds_read_b64_tr_b8 v[18:19], v4 offset:17408\n" \
  "v_mfma_i32_16x16x64_i8 v[204:207], v[88:91], v[28:31], v[204:207]\n" \
  "ds_read_b64_tr_b8 v[20:21], v4 offset:24576\n" \
  "v_mfma_i32_16x16x64_i8 v[204:207], v[72:75], v[24:27], v[204:207]\n" \
  "ds_read_b64_tr_b8 v[22:23], v4 offset:25600\n" \
  "v_mfma_i32_16x16x64_i8 v[172:175], v[68:71], v[24:27], v[172:175]\n" \
  "global_load_dwordx4 v[56:59], v8, s[40:41] offset:2048 sc0 sc1\n" \
  "v_mfma_i32_16x16x64_i8 v[172:175], v[84:87], v[28:31], v[172:175]\n" \
  "global_load_dwordx4 v[60:63], v8, s[40:41] offset:3072 sc0 sc1\n" \
  "v_mfma_i32_16x16x64_i8 v[140:143], v[80:83], v[28:31], v[140:143]\n" \
  "v_add_u32 v8, 0x8000, v8\n" \
  "v_mfma_i32_16x16x64_i8 v[140:143], v[64:67], v[24:27], v[140:143]\n" \
  "s_waitcnt lgkmcnt(0)\n" \
  "v_mfma_i32_16x16x64_i8 v[144:147], v[64:67], v[16:19], v[144:147]\n" \
  "ds_read_b64_tr_b8 v[24:25], v5 offset:16384\n" \
  "v_mfma_i32_16x16x64_i8 v[144:147], v[80:83], v[20:23], v[144:147]\n" \
  "ds_read_b64_tr_b8 v[26:27], v5 offset:17408\n" \
  "v_mfma_i32_16x16x64_i8 v[176:179], v[84:87], v[20:23], v[176:179]\n" \
  "ds_read_b64_tr_b8 v[28:29], v5 offset:24576\n" \
  "v_mfma_i32_16x16x64_i8 v[176:179], v[68:71], v[16:19], v[176:179]\n" \
  "ds_read_b64_tr_b8 v[30:31], v5 offset:25600\n" \
  "v_mfma_i32_16x16x64_i8 v[208:211], v[72:75], v[16:19], v[208:211]\n" \
  "ds_read_b32 v10, v9\n" \
  "v_mfma_i32_16x16x64_i8 v[208:211], v[88:91], v[20:23], v[208:211]\n" \
  "v_mfma_i32_16x16x64_i8 v[240:243], v[92:95], v[20:23], v[240:243]\n" \
  "v_mfma_i32_16x16x64_i8 v[240:243], v[76:79], v[16:19], v[240:243]\n" \
  "s_waitcnt lgkmcnt(1)\n" \
  "v_mfma_i32_16x16x64_i8 v[244:247], v[76:79], v[24:27], v[244:247]\n" \
  "ds_read_b64_tr_b8 v[16:17], v6 offset:16384\n" \
  "v_mfma_i32_16x16x64_i8 v[244:247], v[92:95], v[28:31], v[244:247]\n" \
  "ds_read_b64_tr_b8 v[18:19], v6 offset:17408\n" \
  "v_mfma_i32_16x16x64_i8 v[212:215], v[88:91], v[28:31], v[212:215]\n" \
  "ds_read_b64_tr_b8 v[20:21], v6 offset:24576\n" \
  "v_mfma_i32_16x16x64_i8 v[212:215], v[72:75], v[24:27], v[212:215]\n" \
  "ds_read_b64_tr_b8 v[22:23], v6 offset:25600\n" \
  "v_mfma_i32_16x16x64_i8 v[180:183], v[68:71], v[24:27], v[180:183]\n" \
  "v_mfma_i32_16x16x64_i8 v[180:183], v[84:87], v[28:31], v[180:183]\n" \
  "v_mfma_i32_16x16x64_i8 v[148:151], v[80:83], v[28:31], v[148:151]\n" \
  "v_mfma_i32_16x16x64_i8 v[148:151], v[64:67], v[24:27], v[148:151]\n" \
  "s_waitcnt lgkmcnt(0)\n" \
  "v_mfma_i32_16x16x64_i8 v[152:155], v[64:67], v[16:19], v[152:155]\n" \
  "ds_read_b64_tr_b8 v[24:25], v7 offset:16384\n" \
  "v_mfma_i32_16x16x64_i8 v[152:155], v[80:83], v[20:23], v[152:155]\n" \
  "ds_read_b64_tr_b8 v[26:27], v7 offset:17408\n" \
  "v_mfma_i32_16x16x64_i8 v[184:187], v[84:87], v[20:23], v[184:187]\n" \
  "ds_read_b64_tr_b8 v[28:29], v7 offset:24576\n" \
  "v_mfma_i32_16x16x64_i8 v[184:187], v[68:71], v[16:19], v[184:187]\n" \
  "ds_read_b64_tr_b8 v[30:31], v7 offset:25600\n" \
  "v_mfma_i32_16x16x64_i8 v[216:219], v[72:75], v[16:19], v[216:219]\n" \
  "v_mfma_i32_16x16x64_i8 v[216:219], v[88:91], v[20:23], v[216:219]\n" \
  "v_mfma_i32_16x16x64_i8 v[248:251], v[92:95], v[20:23], v[248:251]\n" \
  "v_cmp_gt_u32 vcc, 60, v10\n" \
  "v_mfma_i32_16x16x64_i8 v[248:251], v[76:79], v[16:19], v[248:251]\n" \
  "s_waitcnt lgkmcnt(0)\n" \
  "v_mfma_i32_16x16x64_i8 v[252:255], v[76:79], v[24:27], v[252:255]\n" \
  "s_cbranch_vccnz LS28_%=\n" \
  "LB28_%=:\n" \
  "ds_read_b64_tr_b8 v[16:17], v0 offset:32768\n" \
  "v_mfma_i32_16x16x64_i8 v[252:255], v[92:95], v[28:31], v[252:255]\n" \
  "ds_read_b64_tr_b8 v[18:19], v0 offset:33792\n" \
  "v_mfma_i32_16x16x64_i8 v[220:223], v[88:91], v[28:31], v[220:223]\n" \
  "ds_read_b64_tr_b8 v[20:21], v0 offset:40960\n" \
  "v_mfma_i32_16x16x64_i8 v[220:223], v[72:75], v[24:27], v[220:223]\n" \
  "ds_read_b64_tr_b8 v[22:23], v0 offset:41984\n" \
  "v_mfma_i32_16x16x64_i8 v[188:191], v[68:71], v[24:27], v[188:191]\n" \
  "ds_add_u32 v12, v11\n" \
  "v_mfma_i32_16x16x64_i8 v[188:191], v[84:87], v[28:31], v[188:191]\n" \
  "v_mfma_i32_16x16x64_i8 v[156:159], v[80:83], v[28:31], v[156:159]\n" \
  "v_mfma_i32_16x16x64_i8 v[156:159], v[64:67], v[24:27], v[156:159]\n" \
  "s_waitcnt vmcnt(8) lgkmcnt(1)\n" \
  "v_mfma_i32_16x16x64_i8 v[128:131], v[96:99], v[16:19], v[128:131]\n" \
  "ds_read_b64_tr_b8 v[24:25], v1 offset:32768\n" \
  "v_mfma_i32_16x16x64_i8 v[128:131], v[112:115], v[20:23], v[128:131]\n" \
  "ds_read_b64_tr_b8 v[26:27], v1 offset:33792\n" \
  "v_mfma_i32_16x16x64_i8 v[160:163], v[116:119], v[20:23], v[160:163]\n" \
  "ds_read_b64_tr_b8 v[28:29], v1 offset:40960\n" \
  "v_mfma_i32_16x16x64_i8 v[160:163], v[100:103], v[16:19], v[160:163]\n" \
  "ds_read_b64_tr_b8 v[30:31], v1 offset:41984\n" \
  "v_mfma_i32_16x16x64_i8 v[192:195], v[104:107], v[16:19], v[192:195]\n" \
  "global_load_dwordx4 v[64:67], v8, s[40:41] offset:0 sc0 sc1\n" \
  "v_mfma_i32_16x16x64_i8 v[192:195], v[120:123], v[20:23], v[192:195]\n" \
  "global_load_dwordx4 v[68:71], v8, s[40:41] offset:1024 sc0 sc1\n" \
  "v_mfma_i32_16x16x64_i8 v[224:227], v[124:127], v[20:23], v[224:227]\n" \
  "v_mfma_i32_16x16x64_i8 v[224:227], v[108:111], v[16:19], v[224:227]\n" \
  "s_waitcnt lgkmcnt(0)\n" \
  "v_mfma_i32_16x16x64_i8 v[228:231], v[108:111], v[24:27], v[228:231]\n" \
  "ds_read_b64_tr_b8 v[16:17], v2 offset:32768\n" \
  "v_mfma_i32_16x16x64_i8 v[228:231], v[124:127], v[28:31], v[228:231]\n" \
  "ds_read_b64_tr_b8 v[18:19], v2 offset:33792\n" \
  "v_mfma_i32_16x16x64_i8 v[196:199], v[120:123], v[28:31], v[196:199]\n" \
  "ds_read_b64_tr_b8 v[20:21], v2 offset:40960\n" \
  "v_mfma_i32_16x16x64_i8 v[196:199], v[104:107], v[24:27], v[196:199]\n" \
  "ds_read_b64_tr_b8 v[22:23], v2 offset:41984\n" \
  "v_mfma_i32_16x16x64_i8 v[164:167], v[100:103], v[24:27], v[164:167]\n" \
  "global_load_dwordx4 v[72:75], v8, s[40:41] offset:2048 sc0 sc1\n" \
  "v_mfma_i32_16x16x64_i8 v[164:167], v[116:119], v[28:31], v[164:167]\n" \
  "global_load_dwordx4 v[76:79], v8, s[40:41] offset:3072 sc0 sc1\n" \
  "v_mfma_i32_16x16x64_i8 v[132:135], v[112:115], v[28:31], v[132:135]\n" \
  "v_add_u32 v8, 0x8000, v8\n" \
  "v_mfma_i32_16x16x64_i8 v[132:135], v[96:99], v[24:27], v[132:135]\n" \
  "s_waitcnt lgkmcnt(0)\n" \
  "v_mfma_i32_16x16x64_i8 v[136:139], v[96:99], v[16:19], v[136:139]\n" \
  "ds_read_b64_tr_b8 v[24:25], v3 offset:32768\n" \
  "v_mfma_i32_16x16x64_i8 v[136:139], v[112:115], v[20:23], v[136:139]\n" \
  "ds_read_b64_tr_b8 v[26:27], v3 offset:33792\n" \
  "v_mfma_i32_16x16x64_i8 v[168:171], v[116:119], v[20:23], v[168:171]\n" \
  "ds_read_b64_tr_b8 v[28:29], v3 offset:40960\n" \
  "v_mfma_i32_16x16x64_i8 v[168:171], v[100:103], v[16:19], v[168:171]\n" \
  "ds_read_b64_tr_b8 v[30:31], v3 offset:41984\n" \
  "v_mfma_i32_16x16x64_i8 v[200:203], v[104:107], v[16:19], v[200:203]\n" \
  "global_load_dwordx4 v[80:83], v8, s[40:41] offset:0 sc0 sc1\n" \
  "v_mfma_i32_16x16x64_i8 v[200:203], v[120:123], v[20:23], v[200:203]\n" \
  "global_load_dwordx4 v[84:87], v8, s[40:41] offset:1024 sc0 sc1\n" \
  "v_mfma_i32_16x16x64_i8 v[232:235], v[124:127], v[20:23], v[232:235]\n" \
  "v_mfma_i32_16x16x64_i8 v[232:235], v[108:111], v[16:19], v[232:235]\n" \
  "s_waitcnt lgkmcnt(0)\n" \
  "v_mfma_i32_16x16x64_i8 v[236:239], v[108:111], v[24:27], v[236:239]\n" \
  "ds_read_b64_tr_b8 v[16:17], v4 offset:32768\n" \
  "v_mfma_i32_16x16x64_i8 v[236:239], v[124:127], v[28:31], v[236:239]\n" \
  "ds_read_b64_tr_b8 v[18:19], v4 offset:33792\n" \
  "v_mfma_i32_16x16x64_i8 v[204:207], v[120:123], v[28:31], v[204:207]\n" \
  "ds_read_b64_tr_b8 v[20:21], v4 offset:40960\n" \
  "v_mfma_i32_16x16x64_i8 v[204:207], v[104:107], v[24:27], v[204:207]\n" \
  "ds_read_b64_tr_b8 v[22:23], v4 offset:41984\n" \
  "v_mfma_i32_16x16x64_i8 v[172:175], v[100:103], v[24:27], v[172:175]\n" \
  "global_load_dwordx4 v[88:91], v8, s[40:41] offset:2048 sc0 sc1\n" \
  "v_mfma_i32_16x16x64_i8 v[172:175], v[116:119], v[28:31], v[172:175]\n" \
  "global_load_dwordx4 v[92:95], v8, s[40:41] offset:3072 sc0 sc1\n" \
  "v_mfma_i32_16x16x64_i8 v[140:143], v[112:115], v[28:31], v[140:143]\n" \
  "v_add_u32 v8, 0x8000, v8\n" \
  "v_mfma_i32_16x16x64_i8 v[140:143], v[96:99], v[24:27], v[140:143]\n" \
  "s_waitcnt lgkmcnt(0)\n" \
  "v_mfma_i32_16x16x64_i8 v[144:147], v[96:99], v[16:19], v[144:147]\n" \
  "ds_read_b64_tr_b8 v[24:25], v5 offset:32768\n" \
  "v_mfma_i32_16x16x64_i8 v[144:147], v[112:115], v[20:23], v[144:147]\n" \
  "ds_read_b64_tr_b8 v[26:27], v5 offset:33792\n" \
  "v_mfma_i32_16x16x64_i8 v[176:179], v[116:119], v[20:23], v[176:179]\n" \
  "ds_read_b64_tr_b8 v[28:29], v5 offset:40960\n" \
  "v_mfma_i32_16x16x64_i8 v[176:179], v[100:103], v[16:19], v[176:179]\n" \
  "ds_read_b64_tr_b8 v[30:31], v5 offset:41984\n" \
  "v_mfma_i32_16x16x64_i8 v[208:211], v[104:107], v[16:19], v[208:211]\n" \
  "ds_read_b32 v10, v9\n" \
  "v_mfma_i32_16x16x64_i8 v[208:211], v[120:123], v[20:23], v[208:211]\n" \
  "v_mfma_i32_16x16x64_i8 v[240:243], v[124:127], v[20:23], v[240:243]\n" \
  "v_mfma_i32_16x16x64_i8 v[240:243], v[108:111], v[16:19], v[240:243]\n" \
  "s_waitcnt lgkmcnt(1)\n" \
  "v_mfma_i32_16x16x64_i8 v[244:247], v[108:111], v[24:27], v[244:247]\n" \
  "ds_read_b64_tr_b8 v[16:17], v6 offset:32768\n" \
  "v_mfma_i32_16x16x64_i8 v[244:247], v[124:127], v[28:31], v[244:247]\n" \
  "ds_read_b64_tr_b8 v[18:19], v6 offset:33792\n" \
  "v_mfma_i32_16x16x64_i8 v[212:215], v[120:123], v[28:31], v[212:215]\n" \
  "ds_read_b64_tr_b8 v[20:21], v6 offset:40960\n" \
  "v_mfma_i32_16x16x64_i8 v[212:215], v[104:107], v[24:27], v[212:215]\n" \
  "ds_read_b64_tr_b8 v[22:23], v6 offset:41984\n" \
  "v_mfma_i32_16x16x64_i8 v[180:183], v[100:103], v[24:27], v[180:183]\n" \
  "v_mfma_i32_16x16x64_i8 v[180:183], v[116:119], v[28:31], v[180:183]\n" \
  "v_mfma_i32_16x16x64_i8 v[148:151], v[112:115], v[28:31], v[148:151]\n" \
  "v_mfma_i32_16x16x64_i8 v[148:151], v[96:99], v[24:27], v[148:151]\n" \
  "s_waitcnt lgkmcnt(0)\n" \
  "v_mfma_i32_16x16x64_i8 v[152:155], v[96:99], v[16:19], v[152:155]\n" \
  "ds_read_b64_tr_b8 v[24:25], v7 offset:32768\n" \
  "v_mfma_i32_16x16x64_i8 v[152:155], v[112:115], v[20:23], v[152:155]\n" \
  "ds_read_b64_tr_b8 v[26:27], v7 offset:33792\n" \
  "v_mfma_i32_16x16x64_i8 v[184:187], v[116:119], v[20:23], v[184:187]\n" \
  "ds_read_b64_tr_b8 v[28:29], v7 offset:40960\n" \
  "v_mfma_i32_16x16x64_i8 v[184:187], v[100:103], v[16:19], v[184:187]\n" \
  "ds_read_b64_tr_b8 v[30:31], v7 offset:41984\n" \
  "v_mfma_i32_16x16x64_i8 v[216:219], v[104:107], v[16:19], v[216:219]\n" \
  "v_mfma_i32_16x16x64_i8 v[216:219], v[120:123], v[20:23], v[216:219]\n" \
  "v_mfma_i32_16x16x64_i8 v[248:251], v[124:127], v[20:23], v[248:251]\n" \
  "v_cmp_gt_u32 vcc, 64, v10\n" \
  "v_mfma_i32_16x16x64_i8 v[248:251], v[108:111], v[16:19], v[248:251]\n" \
  "s_waitcnt lgkmcnt(0)\n" \
  "v_mfma_i32_16x16x64_i8 v[252:255], v[108:111], v[24:27], v[252:255]\n" \
  "s_cbranch_vccnz LS29_%=\n" \
  "LB29_%=:\n" \
  "v_add_u32 v0, 0xc000, v0\n" \
  "ds_read_b64_tr_b8 v[16:17], v0 offset:0\n" \
  "v_mfma_i32_16x16x64_i8 v[252:255], v[124:127], v[28:31], v[252:255]\n" \
  "ds_read_b64_tr_b8 v[18:19], v0 offset:1024\n" \
  "v_mfma_i32_16x16x64_i8 v[220:223], v[120:123], v[28:31], v[220:223]\n" \
  "ds_read_b64_tr_b8 v[20:21], v0 offset:8192\n" \
  "v_mfma_i32_16x16x64_i8 v[220:223], v[104:107], v[24:27], v[220:223]\n" \
  "ds_read_b64_tr_b8 v[22:23], v0 offset:9216\n" \
  "v_mfma_i32_16x16x64_i8 v[188:191], v[100:103], v[24:27], v[188:191]\n" \
  "ds_add_u32 v12, v11\n" \
  "v_mfma_i32_16x16x64_i8 v[188:191], v[116:119], v[28:31], v[188:191]\n" \
  "v_mfma_i32_16x16x64_i8 v[156:159], v[112:115], v[28:31], v[156:159]\n" \
  "v_mfma_i32_16x16x64_i8 v[156:159], v[96:99], v[24:27], v[156:159]\n" \
  "s_waitcnt vmcnt(8) lgkmcnt(1)\n" \
  "v_mfma_i32_16x16x64_i8 v[128:131], v[32:35], v[16:19], v[128:131]\n" \
  "v_add_u32 v1, 0xc000, v1\n" \
  "ds_read_b64_tr_b8 v[24:25], v1 offset:0\n" \
  "ds_read_b32 v10, v9 offset:48\n" \
  "v_mbcnt_lo_u32_b32 v12, -1, 0\n" \
  "v_mbcnt_hi_u32_b32 v12, -1, v12\n" \
  "v_mfma_i32_16x16x64_i8 v[128:131], v[48:51], v[20:23], v[128:131]\n" \
  "ds_read_b64_tr_b8 v[26:27], v1 offset:1024\n" \
  "v_and_b32 v13, 31, v12\n" \
  "v_lshrrev_b32 v8, 5, v12\n" \
  "v_lshlrev_b32 v8, 19, v8\n" \
  "v_mfma_i32_16x16x64_i8 v[160:163], v[52:55], v[20:23], v[160:163]\n" \
  "ds_read_b64_tr_b8 v[28:29], v1 offset:8192\n" \
  "v_lshl_add_u32 v96, v13, 2, v8\n" \
  "v_add_u32 v97, 65536, v96\n" \
  "v_add_u32 v98, 131072, v96\n" \
  "v_mfma_i32_16x16x64_i8 v[160:163], v[36:39], v[16:19], v[160:163]\n" \
  "ds_read_b64_tr_b8 v[30:31], v1 offset:9216\n" \
  "v_add_u32 v99, 196608, v96\n" \
  "v_add_u32 v100, 262144, v96\n" \
  "v_add_u32 v101, 327680, v96\n" \
  "v_mfma_i32_16x16x64_i8 v[192:195], v[40:43], v[16:19], v[192:195]\n" \
  "v_add_u32 v102, 393216, v96\n" \
  "v_add_u32 v103, 458752, v96\n" \
  "v_mov_b32 v11, s50\n" \
  "v_mfma_i32_16x16x64_i8 v[192:195], v[56:59], v[20:23], v[192:195]\n" \
  "v_mul_f32 v11, s51, v11\n" \
  "s_waitcnt lgkmcnt(3)\n" \
  "v_readfirstlane_b32 s47, v10\n" \
  "v_mfma_i32_16x16x64_i8 v[224:227], v[60:63], v[20:23], v[224:227]\n" \
  "v_readfirstlane_b32 s50, v11\n" \
  "s_mov_b32 s51, 0\n" \
  "s_cmp_eq_u32 s47, 0\n" \
  "v_mfma_i32_16x16x64_i8 v[224:227], v[44:47], v[16:19], v[224:227]\n" \
  "s_cselect_b32 s50, s50, 0x7fc00000\n" \
  "s_cselect_b32 s51, s51, 0x7fc00000\n" \
  "s_mov_b32 s70, s50\n" \
  "s_waitcnt lgkmcnt(0)\n" \
  "v_mfma_i32_16x16x64_i8 v[228:231], v[44:47], v[24:27], v[228:231]\n" \
  "ds_read_b64_tr_b8 v[16:17], v0 offset:16384\n" \
  "s_mov_b32 s71, s50\n" \
  "s_mov_b64 s[72:73], %[sout]\n" \
  "v_mfma_i32_16x16x64_i8 v[228:231], v[60:63], v[28:31], v[228:231]\n" \
  "ds_read_b64_tr_b8 v[18:19], v0 offset:17408\n" \
  "v_mfma_i32_16x16x64_i8 v[196:199], v[56:59], v[28:31], v[196:199]\n" \
  "ds_read_b64_tr_b8 v[20:21], v0 offset:24576\n" \
  "v_mfma_i32_16x16x64_i8 v[196:199], v[40:43], v[24:27], v[196:199]\n" \
  "ds_read_b64_tr_b8 v[22:23], v0 offset:25600\n" \
  "v_mfma_i32_16x16x64_i8 v[164:167], v[36:39], v[24:27], v[164:167]\n" \
  "v_mfma_i32_16x16x64_i8 v[164:167], v[52:55], v[28:31], v[164:167]\n" \
  "v_mfma_i32_16x16x64_i8 v[132:135], v[48:51], v[28:31], v[132:135]\n" \
  "v_mfma_i32_16x16x64_i8 v[132:135], v[32:35], v[24:27], v[132:135]\n" \
  "s_waitcnt vmcnt(0) lgkmcnt(0)\n" \
  "v_mfma_i32_16x16x64_i8 v[128:131], v[64:67], v[16:19], v[128:131]\n" \
  "ds_read_b64_tr_b8 v[24:25], v1 offset:16384\n" \
  "v_mfma_i32_16x16x64_i8 v[128:131], v[80:83], v[20:23], v[128:131]\n" \
  "ds_read_b64_tr_b8 v[26:27], v1 offset:17408\n" \
  "v_mfma_i32_16x16x64_i8 v[160:163], v[84:87], v[20:23], v[160:163]\n" \
  "ds_read_b64_tr_b8 v[28:29], v1 offset:24576\n" \
  "v_mfma_i32_16x16x64_i8 v[160:163], v[68:71], v[16:19], v[160:163]\n" \
  "ds_read_b64_tr_b8 v[30:31], v1 offset:25600\n" \
  "v_mfma_i32_16x16x64_i8 v[192:195], v[72:75], v[16:19], v[192:195]\n" \
  "v_mfma_i32_16x16x64_i8 v[192:195], v[88:91], v[20:23], v[192:195]\n" \
  "v_mfma_i32_16x16x64_i8 v[224:227], v[92:95], v[20:23], v[224:227]\n" \
  "v_mfma_i32_16x16x64_i8 v[224:227], v[76:79], v[16:19], v[224:227]\n" \
  "s_waitcnt lgkmcnt(0)\n" \
  "v_mfma_i32_16x16x64_i8 v[228:231], v[76:79], v[24:27], v[228:231]\n" \
  "v_add_u32 v2, 0xc000, v2\n" \
  "ds_read_b64_tr_b8 v[16:17], v2 offset:0\n" \
  "v_mfma_i32_16x16x64_i8 v[228:231], v[92:95], v[28:31], v[228:231]\n" \
  "ds_read_b64_tr_b8 v[18:19], v2 offset:1024\n" \
  "v_mfma_i32_16x16x64_i8 v[196:199], v[88:91], v[28:31], v[196:199]\n" \
  "ds_read_b64_tr_b8 v[20:21], v2 offset:8192\n" \
  "v_mfma_i32_16x16x64_i8 v[196:199], v[72:75], v[24:27], v[196:199]\n" \
  "ds_read_b64_tr_b8 v[22:23], v2 offset:9216\n" \
  "v_mfma_i32_16x16x64_i8 v[164:167], v[68:71], v[24:27], v[164:167]\n" \
  "v_mfma_i32_16x16x64_i8 v[164:167], v[84:87], v[28:31], v[164:167]\n" \
  "v_mfma_i32_16x16x64_i8 v[132:135], v[80:83], v[28:31], v[132:135]\n" \
  "v_mfma_i32_16x16x64_i8 v[132:135], v[64:67], v[24:27], v[132:135]\n" \
  "s_waitcnt lgkmcnt(0)\n" \
  "v_mfma_i32_16x16x64_i8 v[136:139], v[32:35], v[16:19], v[136:139]\n" \
  "v_add_u32 v3, 0xc000, v3\n" \
  "ds_read_b64_tr_b8 v[24:25], v3 offset:0\n" \
  "s_nop 1\n" \
  "v_mfma_i32_16x16x64_i8 v[136:139], v[48:51], v[20:23], v[136:139]\n" \
  "ds_read_b64_tr_b8 v[26:27], v3 offset:1024\n" \
  "s_nop 1\n" \
  "v_mfma_i32_16x16x64_i8 v[168:171], v[52:55], v[20:23], v[168:171]\n" \
  "ds_read_b64_tr_b8 v[28:29], v3 offset:8192\n" \
  "s_nop 1\n" \
  "v_mfma_i32_16x16x64_i8 v[168:171], v[36:39], v[16:19], v[168:171]\n" \
  "ds_read_b64_tr_b8 v[30:31], v3 offset:9216\n" \
  "s_xor_b32 s47, %[sw7], 0\n" \
  "s_add_u32 s62, s72, s47\n" \
  "s_addc_u32 s63, s73, 0\n" \
  "v_mfma_i32_16x16x64_i8 v[200:203], v[40:43], v[16:19], v[200:203]\n" \
  "s_add_u32 s64, s62, 1048576\n" \
  "s_addc_u32 s65, s63, 0\n" \
  "s_add_u32 s66, s62, 2097152\n" \
  "v_mfma_i32_16x16x64_i8 v[200:203], v[56:59], v[20:23], v[200:203]\n" \
  "s_addc_u32 s67, s63, 0\n" \
  "s_add_u32 s68, s62, 3145728\n" \
  "s_addc_u32 s69, s63, 0\n" \
  "v_mfma_i32_16x16x64_i8 v[232:235], v[60:63], v[20:23], v[232:235]\n" \
  "v_cvt_f32_i32 v128, v128\n" \
  "v_cvt_f32_i32 v129, v129\n" \
  "v_cvt_f32_i32 v130, v130\n" \
  "v_mfma_i32_16x16x64_i8 v[232:235], v[44:47], v[16:19], v[232:235]\n" \
  "v_cvt_f32_i32 v131, v131\n" \
  "v_cvt_f32_i32 v132, v132\n" \
  "v_cvt_f32_i32 v133, v133\n" \
  "s_waitcnt lgkmcnt(0)\n" \
  "v_mfma_i32_16x16x64_i8 v[236:239], v[44:47], v[24:27], v[236:239]\n" \
  "ds_read_b64_tr_b8 v[16:17], v2 offset:16384\n" \
  "v_cvt_f32_i32 v134, v134\n" \
  "v_cvt_f32_i32 v135, v135\n" \
  "v_pk_mul_f32 v[128:129], v[128:129], s[70:71]\n" \
  "v_mfma_i32_16x16x64_i8 v[236:239], v[60:63], v[28:31], v[236:239]\n" \
  "ds_read_b64_tr_b8 v[18:19], v2 offset:17408\n" \
  "v_pk_mul_f32 v[130:131], v[130:131], s[70:71]\n" \
  "v_pk_mul_f32 v[132:133], v[132:133], s[70:71]\n" \
  "v_pk_mul_f32 v[134:135], v[134:135], s[70:71]\n" \
  "v_mfma_i32_16x16x64_i8 v[204:207], v[56:59], v[28:31], v[204:207]\n" \
  "ds_read_b64_tr_b8 v[20:21], v2 offset:24576\n" \
  "v_max_f32 v128, s51, v128\n" \
  "v_max_f32 v129, s51, v129\n" \
  "v_max_f32 v130, s51, v130\n" \
  "v_mfma_i32_16x16x64_i8 v[204:207], v[40:43], v[24:27], v[204:207]\n" \
  "ds_read_b64_tr_b8 v[22:23], v2 offset:25600\n" \
  "v_max_f32 v131, s51, v131\n" \
  "v_max_f32 v132, s51, v132\n" \
  "v_max_f32 v133, s51, v133\n" \
  "v_mfma_i32_16x16x64_i8 v[172:175], v[36:39], v[24:27], v[172:175]\n" \
  "v_max_f32 v134, s51, v134\n" \
  "v_max_f32 v135, s51, v135\n" \
  "s_nop 1\n" \
  "v_mfma_i32_16x16x64_i8 v[172:175], v[52:55], v[28:31], v[172:175]\n" \
  "v_permlane16_swap_b32 v128, v132\n" \
  "v_permlane16_swap_b32 v129, v133\n" \
  "v_permlane16_swap_b32 v130, v134\n" \
  "v_mfma_i32_16x16x64_i8 v[140:143], v[48:51], v[28:31], v[140:143]\n" \
  "v_permlane16_swap_b32 v131, v135\n" \
  "global_store_dword v96, v128, s[62:63] sc1\n" \
  "global_store_dword v100, v132, s[62:63] sc1\n" \
  "v_mfma_i32_16x16x64_i8 v[140:143], v[32:35], v[24:27], v[140:143]\n" \
  "global_store_dword v97, v129, s[62:63] sc1\n" \
  "global_store_dword v101, v133, s[62:63] sc1\n" \
  "global_store_dword v98, v130, s[62:63] sc1\n" \
  "s_waitcnt lgkmcnt(0)\n" \
  "v_mfma_i32_16x16x64_i8 v[136:139], v[64:67], v[16:19], v[136:139]\n" \
  "ds_read_b64_tr_b8 v[24:25], v3 offset:16384\n" \
  "global_store_dword v102, v134, s[62:63] sc1\n" \
  "global_store_dword v99, v131, s[62:63] sc1\n" \
  "global_store_dword v103, v135, s[62:63] sc1\n" \
  "v_mfma_i32_16x16x64_i8 v[136:139], v[80:83], v[20:23], v[136:139]\n" \
  "ds_read_b64_tr_b8 v[26:27], v3 offset:17408\n" \
  "v_cvt_f32_i32 v160, v160\n" \
  "v_cvt_f32_i32 v161, v161\n" \
  "v_cvt_f32_i32 v162, v162\n" \
  "v_mfma_i32_16x16x64_i8 v[168:171], v[84:87], v[20:23], v[168:171]\n" \
  "ds_read_b64_tr_b8 v[28:29], v3 offset:24576\n" \
  "v_cvt_f32_i32 v163, v163\n" \
  "v_cvt_f32_i32 v164, v164\n" \
  "v_cvt_f32_i32 v165, v165\n" \
  "v_mfma_i32_16x16x64_i8 v[168:171], v[68:71], v[16:19], v[168:171]\n" \
  "ds_read_b64_tr_b8 v[30:31], v3 offset:25600\n" \
  "v_cvt_f32_i32 v166, v166\n" \
  "v_cvt_f32_i32 v167, v167\n" \
  "v_pk_mul_f32 v[160:161], v[160:161], s[70:71]\n" \
  "v_mfma_i32_16x16x64_i8 v[200:203], v[72:75], v[16:19], v[200:203]\n" \
  "v_pk_mul_f32 v[162:163], v[162:163], s[70:71]\n" \
  "v_pk_mul_f32 v[164:165], v[164:165], s[70:71]\n" \
  "v_pk_mul_f32 v[166:167], v[166:167], s[70:71]\n" \
  "v_mfma_i32_16x16x64_i8 v[200:203], v[88:91], v[20:23], v[200:203]\n" \
  "v_max_f32 v160, s51, v160\n" \
  "v_max_f32 v161, s51, v161\n" \
  "v_max_f32 v162, s51, v162\n" \
  "v_mfma_i32_16x16x64_i8 v[232:235], v[92:95], v[20:23], v[232:235]\n" \
  "v_max_f32 v163, s51, v163\n" \
  "v_max_f32 v164, s51, v164\n" \
  "v_max_f32 v165, s51, v165\n" \
  "v_mfma_i32_16x16x64_i8 v[232:235], v[76:79], v[16:19], v[232:235]\n" \
  "v_max_f32 v166, s51, v166\n" \
  "v_max_f32 v167, s51, v167\n" \
  "s_nop 1\n" \
  "s_waitcnt lgkmcnt(0)\n" \
  "v_mfma_i32_16x16x64_i8 v[236:239], v[76:79], v[24:27], v[236:239]\n" \
  "v_add_u32 v4, 0xc000, v4\n" \
  "ds_read_b64_tr_b8 v[16:17], v4 offset:0\n" \
  "v_permlane16_swap_b32 v160, v164\n" \
  "v_permlane16_swap_b32 v161, v165\n" \
  "v_permlane16_swap_b32 v162, v166\n" \
  "v_mfma_i32_16x16x64_i8 v[236:239], v[92:95], v[28:31], v[236:239]\n" \
  "ds_read_b64_tr_b8 v[18:19], v4 offset:1024\n" \
  "v_permlane16_swap_b32 v163, v167\n" \
  "global_store_dword v96, v160, s[64:65] sc1\n" \
  "global_store_dword v100, v164, s[64:65] sc1\n" \
  "v_mfma_i32_16x16x64_i8 v[204:207], v[88:91], v[28:31], v[204:207]\n" \
  "ds_read_b64_tr_b8 v[20:21], v4 offset:8192\n" \
  "global_store_dword v97, v161, s[64:65] sc1\n" \
  "global_store_dword v101, v165, s[64:65] sc1\n" \
  "global_store_dword v98, v162, s[64:65] sc1\n" \
  "v_mfma_i32_16x16x64_i8 v[204:207], v[72:75], v[24:27], v[204:207]\n" \
  "ds_read_b64_tr_b8 v[22:23], v4 offset:9216\n" \
  "global_store_dword v102, v166, s[64:65] sc1\n" \
  "global_store_dword v99, v163, s[64:65] sc1\n" \
  "global_store_dword v103, v167, s[64:65] sc1\n" \
  "v_mfma_i32_16x16x64_i8 v[172:175], v[68:71], v[24:27], v[172:175]\n" \
  "v_cvt_f32_i32 v192, v192\n" \
  "v_cvt_f32_i32 v193, v193\n" \
  "v_cvt_f32_i32 v194, v194\n" \
  "v_mfma_i32_16x16x64_i8 v[172:175], v[84:87], v[28:31], v[172:175]\n" \
  "v_cvt_f32_i32 v195, v195\n" \
  "v_cvt_f32_i32 v196, v196\n" \
  "v_cvt_f32_i32 v197, v197\n" \
  "v_mfma_i32_16x16x64_i8 v[140:143], v[80:83], v[28:31], v[140:143]\n" \
  "v_cvt_f32_i32 v198, v198\n" \
  "v_cvt_f32_i32 v199, v199\n" \
  "v_pk_mul_f32 v[192:193], v[192:193], s[70:71]\n" \
  "v_mfma_i32_16x16x64_i8 v[140:143], v[64:67], v[24:27], v[140:143]\n" \
  "v_pk_mul_f32 v[194:195], v[194:195], s[70:71]\n" \
  "v_pk_mul_f32 v[196:197], v[196:197], s[70:71]\n" \
  "v_pk_mul_f32 v[198:199], v[198:199], s[70:71]\n" \
  "s_waitcnt lgkmcnt(0)\n" \
  "v_mfma_i32_16x16x64_i8 v[144:147], v[32:35], v[16:19], v[144:147]\n" \
  "v_add_u32 v5, 0xc000, v5\n" \
  "ds_read_b64_tr_b8 v[24:25], v5 offset:0\n" \
  "s_nop 1\n" \
  "v_mfma_i32_16x16x64_i8 v[144:147], v[48:51], v[20:23], v[144:147]\n" \
  "ds_read_b64_tr_b8 v[26:27], v5 offset:1024\n" \
  "s_nop 1\n" \
  "v_mfma_i32_16x16x64_i8 v[176:179], v[52:55], v[20:23], v[176:179]\n" \
  "ds_read_b64_tr_b8 v[28:29], v5 offset:8192\n" \
  "s_nop 1\n" \
  "v_mfma_i32_16x16x64_i8 v[176:179], v[36:39], v[16:19], v[176:179]\n" \
  "ds_read_b64_tr_b8 v[30:31], v5 offset:9216\n" \
  "v_max_f32 v192, s51, v192\n" \
  "v_max_f32 v193, s51, v193\n" \
  "v_max_f32 v194, s51, v194\n" \
  "v_mfma_i32_16x16x64_i8 v[208:211], v[40:43], v[16:19], v[208:211]\n" \
  "v_max_f32 v195, s51, v195\n" \
  "v_max_f32 v196, s51, v196\n" \
  "v_max_f32 v197, s51, v197\n" \
  "v_mfma_i32_16x16x64_i8 v[208:211], v[56:59], v[20:23], v[208:211]\n" \
  "v_max_f32 v198, s51, v198\n" \
  "v_max_f32 v199, s51, v199\n" \
  "s_nop 1\n" \
  "v_mfma_i32_16x16x64_i8 v[240:243], v[60:63], v[20:23], v[240:243]\n" \
  "v_permlane16_swap_b32 v192, v196\n" \
  "v_permlane16_swap_b32 v193, v197\n" \
  "v_permlane16_swap_b32 v194, v198\n" \
  "v_mfma_i32_16x16x64_i8 v[240:243], v[44:47], v[16:19], v[240:243]\n" \
  "v_permlane16_swap_b32 v195, v199\n" \
  "global_store_dword v96, v192, s[66:67] sc1\n" \
  "global_store_dword v100, v196, s[66:67] sc1\n" \
  "s_waitcnt lgkmcnt(0)\n" \
  "v_mfma_i32_16x16x64_i8 v[244:247], v[44:47], v[24:27], v[244:247]\n" \
  "ds_read_b64_tr_b8 v[16:17], v4 offset:16384\n" \
  "global_store_dword v97, v193, s[66:67] sc1\n" \
  "global_store_dword v101, v197, s[66:67] sc1\n" \
  "global_store_dword v98, v194, s[66:67] sc1\n" \
  "v_mfma_i32_16x16x64_i8 v[244:247], v[60:63], v[28:31], v[244:247]\n" \
  "ds_read_b64_tr_b8 v[18:19], v4 offset:17408\n" \
  "global_store_dword v102, v198, s[66:67] sc1\n" \
  "global_store_dword v99, v195, s[66:67] sc1\n" \
  "global_store_dword v103, v199, s[66:67] sc1\n" \
  "v_mfma_i32_16x16x64_i8 v[212:215], v[56:59], v[28:31], v[212:215]\n" \
  "ds_read_b64_tr_b8 v[20:21], v4 offset:24576\n" \
  "v_cvt_f32_i32 v224, v224\n" \
  "v_cvt_f32_i32 v225, v225\n" \
  "v_cvt_f32_i32 v226, v226\n" \
  "v_mfma_i32_16x16x64_i8 v[212:215], v[40:43], v[24:27], v[212:215]\n" \
  "ds_read_b64_tr_b8 v[22:23], v4 offset:25600\n" \
  "v_cvt_f32_i32 v227, v227\n" \
  "v_cvt_f32_i32 v228, v228\n" \
  "v_cvt_f32_i32 v229, v229\n" \
  "v_mfma_i32_16x16x64_i8 v[180:183], v[36:39], v[24:27], v[180:183]\n" \
  "v_cvt_f32_i32 v230, v230\n" \
  "v_cvt_f32_i32 v231, v231\n" \
  "v_pk_mul_f32 v[224:225], v[224:225], s[70:71]\n" \
  "v_mfma_i32_16x16x64_i8 v[180:183], v[52:55], v[28:31], v[180:183]\n" \
  "v_pk_mul_f32 v[226:227], v[226:227], s[70:71]\n" \
  "v_pk_mul_f32 v[228:229], v[228:229], s[70:71]\n" \
  "v_pk_mul_f32 v[230:231], v[230:231], s[70:71]\n" \
  "v_mfma_i32_16x16x64_i8 v[148:151], v[48:51], v[28:31], v[148:151]\n" \
  "v_max_f32 v224, s51, v224\n" \
  "v_max_f32 v225, s51, v225\n" \
  "v_max_f32 v226, s51, v226\n" \
  "v_mfma_i32_16x16x64_i8 v[148:151], v[32:35], v[24:27], v[148:151]\n" \
  "v_max_f32 v227, s51, v227\n" \
  "v_max_f32 v228, s51, v228\n" \
  "v_max_f32 v229, s51, v229\n" \
  "s_waitcnt lgkmcnt(0)\n" \
  "v_mfma_i32_16x16x64_i8 v[144:147], v[64:67], v[16:19], v[144:147]\n" \
  "ds_read_b64_tr_b8 v[24:25], v5 offset:16384\n" \
  "v_max_f32 v230, s51, v230\n" \
  "v_max_f32 v231, s51, v231\n" \
  "s_nop 1\n" \
  "v_mfma_i32_16x16x64_i8 v[144:147], v[80:83], v[20:23], v[144:147]\n" \
  "ds_read_b64_tr_b8 v[26:27], v5 offset:17408\n" \
  "v_permlane16_swap_b32 v224, v228\n" \
  "v_permlane16_swap_b32 v225, v229\n" \
  "v_permlane16_swap_b32 v226, v230\n" \
  "v_mfma_i32_16x16x64_i8 v[176:179], v[84:87], v[20:23], v[176:179]\n" \
  "ds_read_b64_tr_b8 v[28:29], v5 offset:24576\n" \
  "v_permlane16_swap_b32 v227, v231\n" \
  "global_store_dword v96, v224, s[68:69] sc1\n" \
  "global_store_dword v100, v228, s[68:69] sc1\n" \
  "v_mfma_i32_16x16x64_i8 v[176:179], v[68:71], v[16:19], v[176:179]\n" \
  "ds_read_b64_tr_b8 v[30:31], v5 offset:25600\n" \
  "global_store_dword v97, v225, s[68:69] sc1\n" \
  "global_store_dword v101, v229, s[68:69] sc1\n" \
  "global_store_dword v98, v226, s[68:69] sc1\n" \
  "v_mfma_i32_16x16x64_i8 v[208:211], v[72:75], v[16:19], v[208:211]\n" \
  "global_store_dword v102, v230, s[68:69] sc1\n" \
  "global_store_dword v99, v227, s[68:69] sc1\n" \
  "global_store_dword v103, v231, s[68:69] sc1\n" \
  "v_mfma_i32_16x16x64_i8 v[208:211], v[88:91], v[20:23], v[208:211]\n" \
  "s_xor_b32 s47, %[sw7], 128\n" \
  "s_add_u32 s62, s72, s47\n" \
  "s_addc_u32 s63, s73, 0\n" \
  "v_mfma_i32_16x16x64_i8 v[240:243], v[92:95], v[20:23], v[240:243]\n" \
  "s_add_u32 s64, s62, 1048576\n" \
  "s_addc_u32 s65, s63, 0\n" \
  "s_add_u32 s66, s62, 2097152\n" \
  "v_mfma_i32_16x16x64_i8 v[240:243], v[76:79], v[16:19], v[240:243]\n" \
  "s_addc_u32 s67, s63, 0\n" \
  "s_add_u32 s68, s62, 3145728\n" \
  "s_addc_u32 s69, s63, 0\n" \
  "s_waitcnt lgkmcnt(0)\n" \
  "v_mfma_i32_16x16x64_i8 v[244:247], v[76:79], v[24:27], v[244:247]\n" \
  "v_add_u32 v6, 0xc000, v6\n" \
  "ds_read_b64_tr_b8 v[16:17], v6 offset:0\n" \
  "v_cvt_f32_i32 v136, v136\n" \
  "v_cvt_f32_i32 v137, v137\n" \
  "v_cvt_f32_i32 v138, v138\n" \
  "v_mfma_i32_16x16x64_i8 v[244:247], v[92:95], v[28:31], v[244:247]\n" \
  "ds_read_b64_tr_b8 v[18:19], v6 offset:1024\n" \
  "v_cvt_f32_i32 v139, v139\n" \
  "v_cvt_f32_i32 v140, v140\n" \
  "v_cvt_f32_i32 v141, v141\n" \
  "v_mfma_i32_16x16x64_i8 v[212:215], v[88:91], v[28:31], v[212:215]\n" \
  "ds_read_b64_tr_b8 v[20:21], v6 offset:8192\n" \
  "v_cvt_f32_i32 v142, v142\n" \
  "v_cvt_f32_i32 v143, v143\n" \
  "v_pk_mul_f32 v[136:137], v[136:137], s[70:71]\n" \
  "v_mfma_i32_16x16x64_i8 v[212:215], v[72:75], v[24:27], v[212:215]\n" \
  "ds_read_b64_tr_b8 v[22:23], v6 offset:9216\n" \
  "v_pk_mul_f32 v[138:139], v[138:139], s[70:71]\n" \
  "v_pk_mul_f32 v[140:141], v[140:141], s[70:71]\n" \
  "v_pk_mul_f32 v[142:143], v[142:143], s[70:71]\n" \
  "v_mfma_i32_16x16x64_i8 v[180:183], v[68:71], v[24:27], v[180:183]\n" \
  "v_max_f32 v136, s51, v136\n" \
  "v_max_f32 v137, s51, v137\n" \
  "v_max_f32 v138, s51, v138\n" \
  "v_mfma_i32_16x16x64_i8 v[180:183], v[84:87], v[28:31], v[180:183]\n" \
  "v_max_f32 v139, s51, v139\n" \
  "v_max_f32 v140, s51, v140\n" \
  "v_max_f32 v141, s51, v141\n" \
  "v_mfma_i32_16x16x64_i8 v[148:151], v[80:83], v[28:31], v[148:151]\n" \
  "v_max_f32 v142, s51, v142\n" \
  "v_max_f32 v143, s51, v143\n" \
  "s_nop 1\n" \
  "v_mfma_i32_16x16x64_i8 v[148:151], v[64:67], v[24:27], v[148:151]\n" \
  "v_permlane16_swap_b32 v136, v140\n" \
  "v_permlane16_swap_b32 v137, v141\n" \
  "v_permlane16_swap_b32 v138, v142\n" \
  "s_waitcnt lgkmcnt(0)\n" \
  "v_mfma_i32_16x16x64_i8 v[152:155], v[32:35], v[16:19], v[152:155]\n" \
  "v_add_u32 v7, 0xc000, v7\n" \
  "ds_read_b64_tr_b8 v[24:25], v7 offset:0\n" \
  "s_nop 1\n" \
  "v_mfma_i32_16x16x64_i8 v[152:155], v[48:51], v[20:23], v[152:155]\n" \
  "ds_read_b64_tr_b8 v[26:27], v7 offset:1024\n" \
  "s_nop 1\n" \
  "v_mfma_i32_16x16x64_i8 v[184:187], v[52:55], v[20:23], v[184:187]\n" \
  "ds_read_b64_tr_b8 v[28:29], v7 offset:8192\n" \
  "s_nop 1\n" \
  "v_mfma_i32_16x16x64_i8 v[184:187], v[36:39], v[16:19], v[184:187]\n" \
  "ds_read_b64_tr_b8 v[30:31], v7 offset:9216\n" \
  "v_permlane16_swap_b32 v139, v143\n" \
  "global_store_dword v96, v136, s[62:63] sc1\n" \
  "global_store_dword v100, v140, s[62:63] sc1\n" \
  "v_mfma_i32_16x16x64_i8 v[216:219], v[40:43], v[16:19], v[216:219]\n" \
  "global_store_dword v97, v137, s[62:63] sc1\n" \
  "global_store_dword v101, v141, s[62:63] sc1\n" \
  "global_store_dword v98, v138, s[62:63] sc1\n" \
  "v_mfma_i32_16x16x64_i8 v[216:219], v[56:59], v[20:23], v[216:219]\n" \
  "global_store_dword v102, v142, s[62:63] sc1\n" \
  "global_store_dword v99, v139, s[62:63] sc1\n" \
  "global_store_dword v103, v143, s[62:63] sc1\n" \
  "v_mfma_i32_16x16x64_i8 v[248:251], v[60:63], v[20:23], v[248:251]\n" \
  "v_cvt_f32_i32 v168, v168\n" \
  "v_cvt_f32_i32 v169, v169\n" \
  "v_cvt_f32_i32 v170, v170\n" \
  "v_mfma_i32_16x16x64_i8 v[248:251], v[44:47], v[16:19], v[248:251]\n" \
  "v_cvt_f32_i32 v171, v171\n" \
  "v_cvt_f32_i32 v172, v172\n" \
  "v_cvt_f32_i32 v173, v173\n" \
  "s_waitcnt lgkmcnt(0)\n" \
  "v_mfma_i32_16x16x64_i8 v[252:255], v[44:47], v[24:27], v[252:255]\n" \
  "ds_read_b64_tr_b8 v[16:17], v6 offset:16384\n" \
  "v_cvt_f32_i32 v174, v174\n" \
  "v_cvt_f32_i32 v175, v175\n" \
  "v_pk_mul_f32 v[168:169], v[168:169], s[70:71]\n" \
  "v_mfma_i32_16x16x64_i8 v[252:255], v[60:63], v[28:31], v[252:255]\n" \
  "ds_read_b64_tr_b8 v[18:19], v6 offset:17408\n" \
  "v_pk_mul_f32 v[170:171], v[170:171], s[70:71]\n" \
  "v_pk_mul_f32 v[172:173], v[172:173], s[70:71]\n" \
  "v_pk_mul_f32 v[174:175], v[174:175], s[70:71]\n" \
  "v_mfma_i32_16x16x64_i8 v[220:223], v[56:59], v[28:31], v[220:223]\n" \
  "ds_read_b64_tr_b8 v[20:21], v6 offset:24576\n" \
  "v_max_f32 v168, s51, v168\n" \
  "v_max_f32 v169, s51, v169\n" \
  "v_max_f32 v170, s51, v170\n" \
  "v_mfma_i32_16x16x64_i8 v[220:223], v[40:43], v[24:27], v[220:223]\n" \
  "ds_read_b64_tr_b8 v[22:23], v6 offset:25600\n" \
  "v_max_f32 v171, s51, v171\n" \
  "v_max_f32 v172, s51, v172\n" \
  "v_max_f32 v173, s51, v173\n" \
  "v_mfma_i32_16x16x64_i8 v[188:191], v[36:39], v[24:27], v[188:191]\n" \
  "v_max_f32 v174, s51, v174\n" \
  "v_max_f32 v175, s51, v175\n" \
  "s_nop 1\n" \
  "v_mfma_i32_16x16x64_i8 v[188:191], v[52:55], v[28:31], v[188:191]\n" \
  "v_permlane16_swap_b32 v168, v172\n" \
  "v_permlane16_swap_b32 v169, v173\n" \
  "v_permlane16_swap_b32 v170, v174\n" \
  "v_mfma_i32_16x16x64_i8 v[156:159], v[48:51], v[28:31], v[156:159]\n" \
  "v_permlane16_swap_b32 v171, v175\n" \
  "global_store_dword v96, v168, s[64:65] sc1\n" \
  "global_store_dword v100, v172, s[64:65] sc1\n" \
  "v_mfma_i32_16x16x64_i8 v[156:159], v[32:35], v[24:27], v[156:159]\n" \
  "global_store_dword v97, v169, s[64:65] sc1\n" \
  "global_store_dword v101, v173, s[64:65] sc1\n" \
  "global_store_dword v98, v170, s[64:65] sc1\n" \
  "s_waitcnt lgkmcnt(0)\n" \
  "v_mfma_i32_16x16x64_i8 v[152:155], v[64:67], v[16:19], v[152:155]\n" \
  "ds_read_b64_tr_b8 v[24:25], v7 offset:16384\n" \
  "global_store_dword v102, v174, s[64:65] sc1\n" \
  "global_store_dword v99, v171, s[64:65] sc1\n" \
  "global_store_dword v103, v175, s[64:65] sc1\n" \
  "v_mfma_i32_16x16x64_i8 v[152:155], v[80:83], v[20:23], v[152:155]\n" \
  "ds_read_b64_tr_b8 v[26:27], v7 offset:17408\n" \
  "v_cvt_f32_i32 v200, v200\n" \
  "v_cvt_f32_i32 v201, v201\n" \
  "v_cvt_f32_i32 v202, v202\n" \
  "v_mfma_i32_16x16x64_i8 v[184:187], v[84:87], v[20:23], v[184:187]\n" \
  "ds_read_b64_tr_b8 v[28:29], v7 offset:24576\n" \
  "v_cvt_f32_i32 v203, v203\n" \
  "v_cvt_f32_i32 v204, v204\n" \
  "v_cvt_f32_i32 v205, v205\n" \
  "v_mfma_i32_16x16x64_i8 v[184:187], v[68:71], v[16:19], v[184:187]\n" \
  "ds_read_b64_tr_b8 v[30:31], v7 offset:25600\n" \
  "v_cvt_f32_i32 v206, v206\n" \
  "v_cvt_f32_i32 v207, v207\n" \
  "v_pk_mul_f32 v[200:201], v[200:201], s[70:71]\n" \
  "v_mfma_i32_16x16x64_i8 v[216:219], v[72:75], v[16:19], v[216:219]\n" \
  "v_pk_mul_f32 v[202:203], v[202:203], s[70:71]\n" \
  "v_pk_mul_f32 v[204:205], v[204:205], s[70:71]\n" \
  "v_pk_mul_f32 v[206:207], v[206:207], s[70:71]\n" \
  "v_mfma_i32_16x16x64_i8 v[216:219], v[88:91], v[20:23], v[216:219]\n" \
  "v_max_f32 v200, s51, v200\n" \
  "v_max_f32 v201, s51, v201\n" \
  "v_max_f32 v202, s51, v202\n" \
  "v_mfma_i32_16x16x64_i8 v[248:251], v[92:95], v[20:23], v[248:251]\n" \
  "v_max_f32 v203, s51, v203\n" \
  "v_max_f32 v204, s51, v204\n" \
  "v_max_f32 v205, s51, v205\n" \
  "v_mfma_i32_16x16x64_i8 v[248:251], v[76:79], v[16:19], v[248:251]\n" \
  "v_max_f32 v206, s51, v206\n" \
  "v_max_f32 v207, s51, v207\n" \
  "s_nop 1\n" \
  "s_waitcnt lgkmcnt(0)\n" \
  "v_mfma_i32_16x16x64_i8 v[252:255], v[76:79], v[24:27], v[252:255]\n" \
  "v_permlane16_swap_b32 v200, v204\n" \
  "v_permlane16_swap_b32 v201, v205\n" \
  "v_permlane16_swap_b32 v202, v206\n" \
  "v_mfma_i32_16x16x64_i8 v[252:255], v[92:95], v[28:31], v[252:255]\n" \
  "v_permlane16_swap_b32 v203, v207\n" \
  "global_store_dword v96, v200, s[66:67] sc1\n" \
  "global_store_dword v100, v204, s[66:67] sc1\n" \
  "v_mfma_i32_16x16x64_i8 v[220:223], v[88:91], v[28:31], v[220:223]\n" \
  "global_store_dword v97, v201, s[66:67] sc1\n" \
  "global_store_dword v101, v205, s[66:67] sc1\n" \
  "global_store_dword v98, v202, s[66:67] sc1\n" \
  "v_mfma_i32_16x16x64_i8 v[220:223], v[72:75], v[24:27], v[220:223]\n" \
  "global_store_dword v102, v206, s[66:67] sc1\n" \
  "global_store_dword v99, v203, s[66:67] sc1\n" \
  "global_store_dword v103, v207, s[66:67] sc1\n" \
  "v_mfma_i32_16x16x64_i8 v[188:191], v[68:71], v[24:27], v[188:191]\n" \
  "v_cvt_f32_i32 v232, v232\n" \
  "v_cvt_f32_i32 v233, v233\n" \
  "v_cvt_f32_i32 v234, v234\n" \
  "v_mfma_i32_16x16x64_i8 v[188:191], v[84:87], v[28:31], v[188:191]\n" \
  "v_cvt_f32_i32 v235, v235\n" \
  "v_cvt_f32_i32 v236, v236\n" \
  "v_cvt_f32_i32 v237, v237\n" \
  "v_mfma_i32_16x16x64_i8 v[156:159], v[80:83], v[28:31], v[156:159]\n" \
  "v_cvt_f32_i32 v238, v238\n" \
  "v_cvt_f32_i32 v239, v239\n" \
  "v_pk_mul_f32 v[232:233], v[232:233], s[70:71]\n" \
  "v_mfma_i32_16x16x64_i8 v[156:159], v[64:67], v[24:27], v[156:159]\n" \
  "v_pk_mul_f32 v[234:235], v[234:235], s[70:71]\n" \
  "v_pk_mul_f32 v[236:237], v[236:237], s[70:71]\n" \
  "v_pk_mul_f32 v[238:239], v[238:239], s[70:71]\n" \
  "v_max_f32 v232, s51, v232\n" \
  "v_max_f32 v233, s51, v233\n" \
  "v_max_f32 v234, s51, v234\n" \
  "v_max_f32 v235, s51, v235\n" \
  "v_max_f32 v236, s51, v236\n" \
  "v_max_f32 v237, s51, v237\n" \
  "v_max_f32 v238, s51, v238\n" \
  "v_max_f32 v239, s51, v239\n" \
  "s_nop 1\n" \
  "v_permlane16_swap_b32 v232, v236\n" \
  "v_permlane16_swap_b32 v233, v237\n" \
  "v_permlane16_swap_b32 v234, v238\n" \
  "v_permlane16_swap_b32 v235, v239\n" \
  "global_store_dword v96, v232, s[68:69] sc1\n" \
  "global_store_dword v100, v236, s[68:69] sc1\n" \
  "global_store_dword v97, v233, s[68:69] sc1\n" \
  "global_store_dword v101, v237, s[68:69] sc1\n" \
  "global_store_dword v98, v234, s[68:69] sc1\n" \
  "global_store_dword v102, v238, s[68:69] sc1\n" \
  "global_store_dword v99, v235, s[68:69] sc1\n" \
  "global_store_dword v103, v239, s[68:69] sc1\n" \
  "s_xor_b32 s47, %[sw7], 256\n" \
  "s_add_u32 s62, s72, s47\n" \
  "s_addc_u32 s63, s73, 0\n" \
  "s_add_u32 s64, s62, 1048576\n" \
  "s_addc_u32 s65, s63, 0\n" \
  "s_add_u32 s66, s62, 2097152\n" \
  "s_addc_u32 s67, s63, 0\n" \
  "s_add_u32 s68, s62, 3145728\n" \
  "s_addc_u32 s69, s63, 0\n" \
  "v_cvt_f32_i32 v144, v144\n" \
  "v_cvt_f32_i32 v145, v145\n" \
  "v_cvt_f32_i32 v146, v146\n" \
  "v_cvt_f32_i32 v147, v147\n" \
  "v_cvt_f32_i32 v148, v148\n" \
  "v_cvt_f32_i32 v149, v149\n" \
  "v_cvt_f32_i32 v150, v150\n" \
  "v_cvt_f32_i32 v151, v151\n" \
  "v_pk_mul_f32 v[144:145], v[144:145], s[70:71]\n" \
  "v_pk_mul_f32 v[146:147], v[146:147], s[70:71]\n" \
  "v_pk_mul_f32 v[148:149], v[148:149], s[70:71]\n" \
  "v_pk_mul_f32 v[150:151], v[150:151], s[70:71]\n" \
  "v_max_f32 v144, s51, v144\n" \
  "v_max_f32 v145, s51, v145\n" \
  "v_max_f32 v146, s51, v146\n" \
  "v_max_f32 v147, s51, v147\n" \
  "v_max_f32 v148, s51, v148\n" \
  "v_max_f32 v149, s51, v149\n" \
  "v_max_f32 v150, s51, v150\n" \
  "v_max_f32 v151, s51, v151\n" \
  "s_nop 1\n" \
  "v_permlane16_swap_b32 v144, v148\n" \
  "v_permlane16_swap_b32 v145, v149\n" \
  "v_permlane16_swap_b32 v146, v150\n" \
  "v_permlane16_swap_b32 v147, v151\n" \
  "global_store_dword v96, v144, s[62:63] sc1\n" \
  "global_store_dword v100, v148, s[62:63] sc1\n" \
  "global_store_dword v97, v145, s[62:63] sc1\n" \
  "global_store_dword v101, v149, s[62:63] sc1\n" \
  "global_store_dword v98, v146, s[62:63] sc1\n" \
  "global_store_dword v102, v150, s[62:63] sc1\n" \
  "global_store_dword v99, v147, s[62:63] sc1\n" \
  "global_store_dword v103, v151, s[62:63] sc1\n" \
  "v_cvt_f32_i32 v176, v176\n" \
  "v_cvt_f32_i32 v177, v177\n" \
  "v_cvt_f32_i32 v178, v178\n" \
  "v_cvt_f32_i32 v179, v179\n" \
  "v_cvt_f32_i32 v180, v180\n" \
  "v_cvt_f32_i32 v181, v181\n" \
  "v_cvt_f32_i32 v182, v182\n" \
  "v_cvt_f32_i32 v183, v183\n" \
  "v_pk_mul_f32 v[176:177], v[176:177], s[70:71]\n" \
  "v_pk_mul_f32 v[178:179], v[178:179], s[70:71]\n" \
  "v_pk_mul_f32 v[180:181], v[180:181], s[70:71]\n" \
  "v_pk_mul_f32 v[182:183], v[182:183], s[70:71]\n" \
  "v_max_f32 v176, s51, v176\n" \
  "v_max_f32 v177, s51, v177\n" \
  "v_max_f32 v178, s51, v178\n" \
  "v_max_f32 v179, s51, v179\n" \
  "v_max_f32 v180, s51, v180\n" \
  "v_max_f32 v181, s51, v181\n" \
  "v_max_f32 v182, s51, v182\n" \
  "v_max_f32 v183, s51, v183\n" \
  "s_nop 1\n" \
  "v_permlane16_swap_b32 v176, v180\n" \
  "v_permlane16_swap_b32 v177, v181\n" \
  "v_permlane16_swap_b32 v178, v182\n" \
  "v_permlane16_swap_b32 v179, v183\n" \
  "global_store_dword v96, v176, s[64:65] sc1\n" \
  "global_store_dword v100, v180, s[64:65] sc1\n" \
  "global_store_dword v97, v177, s[64:65] sc1\n" \
  "global_store_dword v101, v181, s[64:65] sc1\n" \
  "global_store_dword v98, v178, s[64:65] sc1\n" \
  "global_store_dword v102, v182, s[64:65] sc1\n" \
  "global_store_dword v99, v179, s[64:65] sc1\n" \
  "global_store_dword v103, v183, s[64:65] sc1\n" \
  "v_cvt_f32_i32 v208, v208\n" \
  "v_cvt_f32_i32 v209, v209\n" \
  "v_cvt_f32_i32 v210, v210\n" \
  "v_cvt_f32_i32 v211, v211\n" \
  "v_cvt_f32_i32 v212, v212\n" \
  "v_cvt_f32_i32 v213, v213\n" \
  "v_cvt_f32_i32 v214, v214\n" \
  "v_cvt_f32_i32 v215, v215\n" \
  "v_pk_mul_f32 v[208:209], v[208:209], s[70:71]\n" \
  "v_pk_mul_f32 v[210:211], v[210:211], s[70:71]\n" \
  "v_pk_mul_f32 v[212:213], v[212:213], s[70:71]\n" \
  "v_pk_mul_f32 v[214:215], v[214:215], s[70:71]\n" \
  "v_max_f32 v208, s51, v208\n" \
  "v_max_f32 v209, s51, v209\n" \
  "v_max_f32 v210, s51, v210\n" \
  "v_max_f32 v211, s51, v211\n" \
  "v_max_f32 v212, s51, v212\n" \
  "v_max_f32 v213, s51, v213\n" \
  "v_max_f32 v214, s51, v214\n" \
  "v_max_f32 v215, s51, v215\n" \
  "s_nop 1\n" \
  "v_permlane16_swap_b32 v208, v212\n" \
  "v_permlane16_swap_b32 v209, v213\n" \
  "v_permlane16_swap_b32 v210, v214\n" \
  "v_permlane16_swap_b32 v211, v215\n" \
  "global_store_dword v96, v208, s[66:67] sc1\n" \
  "global_store_dword v100, v212, s[66:67] sc1\n" \
  "global_store_dword v97, v209, s[66:67] sc1\n" \
  "global_store_dword v101, v213, s[66:67] sc1\n" \
  "global_store_dword v98, v210, s[66:67] sc1\n" \
  "global_store_dword v102, v214, s[66:67] sc1\n" \
  "global_store_dword v99, v211, s[66:67] sc1\n" \
  "global_store_dword v103, v215, s[66:67] sc1\n" \
  "v_cvt_f32_i32 v240, v240\n" \
  "v_cvt_f32_i32 v241, v241\n" \
  "v_cvt_f32_i32 v242, v242\n" \
  "v_cvt_f32_i32 v243, v243\n" \
  "v_cvt_f32_i32 v244, v244\n" \
  "v_cvt_f32_i32 v245, v245\n" \
  "v_cvt_f32_i32 v246, v246\n" \
  "v_cvt_f32_i32 v247, v247\n" \
  "v_pk_mul_f32 v[240:241], v[240:241], s[70:71]\n" \
  "v_pk_mul_f32 v[242:243], v[242:243], s[70:71]\n" \
  "v_pk_mul_f32 v[244:245], v[244:245], s[70:71]\n" \
  "v_pk_mul_f32 v[246:247], v[246:247], s[70:71]\n" \
  "v_max_f32 v240, s51, v240\n" \
  "v_max_f32 v241, s51, v241\n" \
  "v_max_f32 v242, s51, v242\n" \
  "v_max_f32 v243, s51, v243\n" \
  "v_max_f32 v244, s51, v244\n" \
  "v_max_f32 v245, s51, v245\n" \
  "v_max_f32 v246, s51, v246\n" \
  "v_max_f32 v247, s51, v247\n" \
  "s_nop 1\n" \
  "v_permlane16_swap_b32 v240, v244\n" \
  "v_permlane16_swap_b32 v241, v245\n" \
  "v_permlane16_swap_b32 v242, v246\n" \
  "v_permlane16_swap_b32 v243, v247\n" \
  "global_store_dword v96, v240, s[68:69] sc1\n" \
  "global_store_dword v100, v244, s[68:69] sc1\n" \
  "global_store_dword v97, v241, s[68:69] sc1\n" \
  "global_store_dword v101, v245, s[68:69] sc1\n" \
  "global_store_dword v98, v242, s[68:69] sc1\n" \
  "global_store_dword v102, v246, s[68:69] sc1\n" \
  "global_store_dword v99, v243, s[68:69] sc1\n" \
  "global_store_dword v103, v247, s[68:69] sc1\n" \
  "s_nop 7\n" \
  "s_xor_b32 s47, %[sw7], 384\n" \
  "s_add_u32 s62, s72, s47\n" \
  "s_addc_u32 s63, s73, 0\n" \
  "s_add_u32 s64, s62, 1048576\n" \
  "s_addc_u32 s65, s63, 0\n" \
  "s_add_u32 s66, s62, 2097152\n" \
  "s_addc_u32 s67, s63, 0\n" \
  "s_add_u32 s68, s62, 3145728\n" \
  "s_addc_u32 s69, s63, 0\n" \
  "v_cvt_f32_i32 v152, v152\n" \
  "v_cvt_f32_i32 v153, v153\n" \
  "v_cvt_f32_i32 v154, v154\n" \
  "v_cvt_f32_i32 v155, v155\n" \
  "v_cvt_f32_i32 v156, v156\n" \
  "v_cvt_f32_i32 v157, v157\n" \
  "v_cvt_f32_i32 v158, v158\n" \
  "v_cvt_f32_i32 v159, v159\n" \
  "v_pk_mul_f32 v[152:153], v[152:153], s[70:71]\n" \
  "v_pk_mul_f32 v[154:155], v[154:155], s[70:71]\n" \
  "v_pk_mul_f32 v[156:157], v[156:157], s[70:71]\n" \
  "v_pk_mul_f32 v[158:159], v[158:159], s[70:71]\n" \
  "v_max_f32 v152, s51, v152\n" \
  "v_max_f32 v153, s51, v153\n" \
  "v_max_f32 v154, s51, v154\n" \
  "v_max_f32 v155, s51, v155\n" \
  "v_max_f32 v156, s51, v156\n" \
  "v_max_f32 v157, s51, v157\n" \
  "v_max_f32 v158, s51, v158\n" \
  "v_max_f32 v159, s51, v159\n" \
  "s_nop 1\n" \
  "v_permlane16_swap_b32 v152, v156\n" \
  "v_permlane16_swap_b32 v153, v157\n" \
  "v_permlane16_swap_b32 v154, v158\n" \
  "v_permlane16_swap_b32 v155, v159\n" \
  "global_store_dword v96, v152, s[62:63] sc1\n" \
  "global_store_dword v100, v156, s[62:63] sc1\n" \
  "global_store_dword v97, v153, s[62:63] sc1\n" \
  "global_store_dword v101, v157, s[62:63] sc1\n" \
  "global_store_dword v98, v154, s[62:63] sc1\n" \
  "global_store_dword v102, v158, s[62:63] sc1\n" \
  "global_store_dword v99, v155, s[62:63] sc1\n" \
  "global_store_dword v103, v159, s[62:63] sc1\n" \
  "v_cvt_f32_i32 v184, v184\n" \
  "v_cvt_f32_i32 v185, v185\n" \
  "v_cvt_f32_i32 v186, v186\n" \
  "v_cvt_f32_i32 v187, v187\n" \
  "v_cvt_f32_i32 v188, v188\n" \
  "v_cvt_f32_i32 v189, v189\n" \
  "v_cvt_f32_i32 v190, v190\n" \
  "v_cvt_f32_i32 v191, v191\n" \
  "v_pk_mul_f32 v[184:185], v[184:185], s[70:71]\n" \
  "v_pk_mul_f32 v[186:187], v[186:187], s[70:71]\n" \
  "v_pk_mul_f32 v[188:189], v[188:189], s[70:71]\n" \
  "v_pk_mul_f32 v[190:191], v[190:191], s[70:71]\n" \
  "v_max_f32 v184, s51, v184\n" \
  "v_max_f32 v185, s51, v185\n" \
  "v_max_f32 v186, s51, v186\n" \
  "v_max_f32 v187, s51, v187\n" \
  "v_max_f32 v188, s51, v188\n" \
  "v_max_f32 v189, s51, v189\n" \
  "v_max_f32 v190, s51, v190\n" \
  "v_max_f32 v191, s51, v191\n" \
  "s_nop 1\n" \
  "v_permlane16_swap_b32 v184, v188\n" \
  "v_permlane16_swap_b32 v185, v189\n" \
  "v_permlane16_swap_b32 v186, v190\n" \
  "v_permlane16_swap_b32 v187, v191\n" \
  "global_store_dword v96, v184, s[64:65] sc1\n" \
  "global_store_dword v100, v188, s[64:65] sc1\n" \
  "global_store_dword v97, v185, s[64:65] sc1\n" \
  "global_store_dword v101, v189, s[64:65] sc1\n" \
  "global_store_dword v98, v186, s[64:65] sc1\n" \
  "global_store_dword v102, v190, s[64:65] sc1\n" \
  "global_store_dword v99, v187, s[64:65] sc1\n" \
  "global_store_dword v103, v191, s[64:65] sc1\n" \
  "v_cvt_f32_i32 v216, v216\n" \
  "v_cvt_f32_i32 v217, v217\n" \
  "v_cvt_f32_i32 v218, v218\n" \
  "v_cvt_f32_i32 v219, v219\n" \
  "v_cvt_f32_i32 v220, v220\n" \
  "v_cvt_f32_i32 v221, v221\n" \
  "v_cvt_f32_i32 v222, v222\n" \
  "v_cvt_f32_i32 v223, v223\n" \
  "v_pk_mul_f32 v[216:217], v[216:217], s[70:71]\n" \
  "v_pk_mul_f32 v[218:219], v[218:219], s[70:71]\n" \
  "v_pk_mul_f32 v[220:221], v[220:221], s[70:71]\n" \
  "v_pk_mul_f32 v[222:223], v[222:223], s[70:71]\n" \
  "v_max_f32 v216, s51, v216\n" \
  "v_max_f32 v217, s51, v217\n" \
  "v_max_f32 v218, s51, v218\n" \
  "v_max_f32 v219, s51, v219\n" \
  "v_max_f32 v220, s51, v220\n" \
  "v_max_f32 v221, s51, v221\n" \
  "v_max_f32 v222, s51, v222\n" \
  "v_max_f32 v223, s51, v223\n" \
  "s_nop 1\n" \
  "v_permlane16_swap_b32 v216, v220\n" \
  "v_permlane16_swap_b32 v217, v221\n" \
  "v_permlane16_swap_b32 v218, v222\n" \
  "v_permlane16_swap_b32 v219, v223\n" \
  "global_store_dword v96, v216, s[66:67] sc1\n" \
  "global_store_dword v100, v220, s[66:67] sc1\n" \
  "global_store_dword v97, v217, s[66:67] sc1\n" \
  "global_store_dword v101, v221, s[66:67] sc1\n" \
  "global_store_dword v98, v218, s[66:67] sc1\n" \
  "global_store_dword v102, v222, s[66:67] sc1\n" \
  "global_store_dword v99, v219, s[66:67] sc1\n" \
  "global_store_dword v103, v223, s[66:67] sc1\n" \
  "v_cvt_f32_i32 v248, v248\n" \
  "v_cvt_f32_i32 v249, v249\n" \
  "v_cvt_f32_i32 v250, v250\n" \
  "v_cvt_f32_i32 v251, v251\n" \
  "v_cvt_f32_i32 v252, v252\n" \
  "v_cvt_f32_i32 v253, v253\n" \
  "v_cvt_f32_i32 v254, v254\n" \
  "v_cvt_f32_i32 v255, v255\n" \
  "v_pk_mul_f32 v[248:249], v[248:249], s[70:71]\n" \
  "v_pk_mul_f32 v[250:251], v[250:251], s[70:71]\n" \
  "v_pk_mul_f32 v[252:253], v[252:253], s[70:71]\n" \
  "v_pk_mul_f32 v[254:255], v[254:255], s[70:71]\n" \
  "v_max_f32 v248, s51, v248\n" \
  "v_max_f32 v249, s51, v249\n" \
  "v_max_f32 v250, s51, v250\n" \
  "v_max_f32 v251, s51, v251\n" \
  "v_max_f32 v252, s51, v252\n" \
  "v_max_f32 v253, s51, v253\n" \
  "v_max_f32 v254, s51, v254\n" \
  "v_max_f32 v255, s51, v255\n" \
  "s_nop 1\n" \
  "v_permlane16_swap_b32 v248, v252\n" \
  "v_permlane16_swap_b32 v249, v253\n" \
  "v_permlane16_swap_b32 v250, v254\n" \
  "v_permlane16_swap_b32 v251, v255\n" \
  "global_store_dword v96, v248, s[68:69] sc1\n" \
  "global_store_dword v100, v252, s[68:69] sc1\n" \
  "global_store_dword v97, v249, s[68:69] sc1\n" \
  "global_store_dword v101, v253, s[68:69] sc1\n" \
  "global_store_dword v98, v250, s[68:69] sc1\n" \
  "global_store_dword v102, v254, s[68:69] sc1\n" \
  "global_store_dword v99, v251, s[68:69] sc1\n" \
  "global_store_dword v103, v255, s[68:69] sc1\n" \
  "s_endpgm\n" \
  "LS0_%=:\n" \
  "s_sleep 1\n" \
  "ds_read_b32 v10, v9\n" \
  "s_waitcnt lgkmcnt(0)\n" \
  "v_cmp_gt_u32 vcc, 4, v10\n" \
  "s_and_b64 vcc, vcc, 13\n" \
  "s_cbranch_vccnz LS0_%=\n" \
  "s_branch LB0_%=\n" \
  "LS1_%=:\n" \
  "s_sleep 1\n" \
  "ds_read_b32 v10, v9\n" \
  "s_waitcnt lgkmcnt(0)\n" \
  "v_cmp_gt_u32 vcc, 6, v10\n" \
  "s_and_b64 vcc, vcc, 13\n" \
  "s_cbranch_vccnz LS1_%=\n" \
  "s_branch LB1_%=\n" \
  "LSX_%=:\n" \
  "s_sleep 1\n" \
  "ds_read_b32 v10, v9\n" \
  "s_waitcnt lgkmcnt(0)\n" \
  "v_cmp_gt_u32 vcc, 6, v10\n" \
  "s_and_b64 vcc, vcc, 2\n" \
  "s_cbranch_vccnz LSX_%=\n" \
  "s_branch LBX_%=\n" \
  "LS2_%=:\n" \
  "s_sleep 1\n" \
  "ds_read_b32 v10, v9\n" \
  "s_waitcnt lgkmcnt(0)\n" \
  "v_cmp_gt_u32 vcc, 8, v10\n" \
  "s_cbranch_vccnz LS2_%=\n" \
  "s_branch LB2_%=\n" \
  "LS3_%=:\n" \
  "s_sleep 1\n" \
  "ds_read_b32 v10, v9\n" \
  "s_waitcnt lgkmcnt(0)\n" \
  "v_cmp_gt_u32 vcc, 10, v10\n" \
  "s_cbranch_vccnz LS3_%=\n" \
  "s_branch LB3_%=\n" \
  "LS4_%=:\n" \
  "s_sleep 1\n" \
  "ds_read_b32 v10, v9\n" \
  "s_waitcnt lgkmcnt(0)\n" \
  "v_cmp_gt_u32 vcc, 12, v10\n" \
  "s_cbranch_vccnz LS4_%=\n" \
  "s_branch LB4_%=\n" \
  "LS5_%=:\n" \
  "s_sleep 1\n" \
  "ds_read_b32 v10, v9\n" \
  "s_waitcnt lgkmcnt(0)\n" \
  "v_cmp_gt_u32 vcc, 14, v10\n" \
  "s_cbranch_vccnz LS5_%=\n" \
  "s_branch LB5_%=\n" \
  "LS6_%=:\n" \
  "s_sleep 1\n" \
  "ds_read_b32 v10, v9\n" \
  "s_waitcnt lgkmcnt(0)\n" \
  "v_cmp_gt_u32 vcc, 16, v10\n" \
  "s_cbranch_vccnz LS6_%=\n" \
  "s_branch LB6_%=\n" \
  "LS7_%=:\n" \
  "s_sleep 1\n" \
  "ds_read_b32 v10, v9\n" \
  "s_waitcnt lgkmcnt(0)\n" \
  "v_cmp_gt_u32 vcc, 18, v10\n" \
  "s_cbranch_vccnz LS7_%=\n" \
  "s_branch LB7_%=\n" \
  "LS8_%=:\n" \
  "s_sleep 1\n" \
  "ds_read_b32 v10, v9\n" \
  "s_waitcnt lgkmcnt(0)\n" \
  "v_cmp_gt_u32 vcc, 20, v10\n" \
  "s_cbranch_vccnz LS8_%=\n" \
  "s_branch LB8_%=\n" \
  "LS9_%=:\n" \
  "s_sleep 1\n" \
  "ds_read_b32 v10, v9\n" \
  "s_waitcnt lgkmcnt(0)\n" \
  "v_cmp_gt_u32 vcc, 22, v10\n" \
  "s_cbranch_vccnz LS9_%=\n" \
  "s_branch LB9_%=\n" \
  "LS10_%=:\n" \
  "s_sleep 1\n" \
  "ds_read_b32 v10, v9\n" \
  "s_waitcnt lgkmcnt(0)\n" \
  "v_cmp_gt_u32 vcc, 24, v10\n" \
  "s_cbranch_vccnz LS10_%=\n" \
  "s_branch LB10_%=\n" \
  "LS11_%=:\n" \
  "s_sleep 1\n" \
  "ds_read_b32 v10, v9\n" \
  "s_waitcnt lgkmcnt(0)\n" \
  "v_cmp_gt_u32 vcc, 26, v10\n" \
  "s_cbranch_vccnz LS11_%=\n" \
  "s_branch LB11_%=\n" \
  "LS12_%=:\n" \
  "s_sleep 1\n" \
  "ds_read_b32 v10, v9\n" \
  "s_waitcnt lgkmcnt(0)\n" \
  "v_cmp_gt_u32 vcc, 28, v10\n" \
  "s_cbranch_vccnz LS12_%=\n" \
  "s_branch LB12_%=\n" \
  "LS13_%=:\n" \
  "s_sleep 1\n" \
  "ds_read_b32 v10, v9\n" \
  "s_waitcnt lgkmcnt(0)\n" \
  "v_cmp_gt_u32 vcc, 30, v10\n" \
  "s_cbranch_vccnz LS13_%=\n" \
  "s_branch LB13_%=\n" \
  "LS14_%=:\n" \
  "s_sleep 1\n" \
  "ds_read_b32 v10, v9\n" \
  "s_waitcnt lgkmcnt(0)\n" \
  "v_cmp_gt_u32 vcc, 32, v10\n" \
  "s_cbranch_vccnz LS14_%=\n" \
  "s_branch LB14_%=\n" \
  "LS15_%=:\n" \
  "s_sleep 1\n" \
  "ds_read_b32 v10, v9\n" \
  "s_waitcnt lgkmcnt(0)\n" \
  "v_cmp_gt_u32 vcc, 34, v10\n" \
  "s_cbranch_vccnz LS15_%=\n" \
  "s_branch LB15_%=\n" \
  "LS16_%=:\n" \
  "s_sleep 1\n" \
  "ds_read_b32 v10, v9\n" \
  "s_waitcnt lgkmcnt(0)\n" \
  "v_cmp_gt_u32 vcc, 36, v10\n" \
  "s_cbranch_vccnz LS16_%=\n" \
  "s_branch LB16_%=\n" \
  "LS17_%=:\n" \
  "s_sleep 1\n" \
  "ds_read_b32 v10, v9\n" \
  "s_waitcnt lgkmcnt(0)\n" \
  "v_cmp_gt_u32 vcc, 38, v10\n" \
  "s_cbranch_vccnz LS17_%=\n" \
  "s_branch LB17_%=\n" \
  "LS18_%=:\n" \
  "s_sleep 1\n" \
  "ds_read_b32 v10, v9\n" \
  "s_waitcnt lgkmcnt(0)\n" \
  "v_cmp_gt_u32 vcc, 40, v10\n" \
  "s_cbranch_vccnz LS18_%=\n" \
  "s_branch LB18_%=\n" \
  "LS19_%=:\n" \
  "s_sleep 1\n" \
  "ds_read_b32 v10, v9\n" \
  "s_waitcnt lgkmcnt(0)\n" \
  "v_cmp_gt_u32 vcc, 42, v10\n" \
  "s_cbranch_vccnz LS19_%=\n" \
  "s_branch LB19_%=\n" \
  "LS20_%=:\n" \
  "s_sleep 1\n" \
  "ds_read_b32 v10, v9\n" \
  "s_waitcnt lgkmcnt(0)\n" \
  "v_cmp_gt_u32 vcc, 44, v10\n" \
  "s_cbranch_vccnz LS20_%=\n" \
  "s_branch LB20_%=\n" \
  "LS21_%=:\n" \
  "s_sleep 1\n" \
  "ds_read_b32 v10, v9\n" \
  "s_waitcnt lgkmcnt(0)\n" \
  "v_cmp_gt_u32 vcc, 46, v10\n" \
  "s_cbranch_vccnz LS21_%=\n" \
  "s_branch LB21_%=\n" \
  "LS22_%=:\n" \
  "s_sleep 1\n" \
  "ds_read_b32 v10, v9\n" \
  "s_waitcnt lgkmcnt(0)\n" \
  "v_cmp_gt_u32 vcc, 48, v10\n" \
  "s_cbranch_vccnz LS22_%=\n" \
  "s_branch LB22_%=\n" \
  "LS23_%=:\n" \
  "s_sleep 1\n" \
  "ds_read_b32 v10, v9\n" \
  "s_waitcnt lgkmcnt(0)\n" \
  "v_cmp_gt_u32 vcc, 50, v10\n" \
  "s_cbranch_vccnz LS23_%=\n" \
  "s_branch LB23_%=\n" \
  "LS24_%=:\n" \
  "s_sleep 1\n" \
  "ds_read_b32 v10, v9\n" \
  "s_waitcnt lgkmcnt(0)\n" \
  "v_cmp_gt_u32 vcc, 52, v10\n" \
  "s_cbranch_vccnz LS24_%=\n" \
  "s_branch LB24_%=\n" \
  "LS25_%=:\n" \
  "s_sleep 1\n" \
  "ds_read_b32 v10, v9\n" \
  "s_waitcnt lgkmcnt(0)\n" \
  "v_cmp_gt_u32 vcc, 54, v10\n" \
  "s_cbranch_vccnz LS25_%=\n" \
  "s_branch LB25_%=\n" \
  "LS26_%=:\n" \
  "s_sleep 1\n" \
  "ds_read_b32 v10, v9\n" \
  "s_waitcnt lgkmcnt(0)\n" \
  "v_cmp_gt_u32 vcc, 56, v10\n" \
  "s_cbranch_vccnz LS26_%=\n" \
  "s_branch LB26_%=\n" \
  "LS27_%=:\n" \
  "s_sleep 1\n" \
  "ds_read_b32 v10, v9\n" \
  "s_waitcnt lgkmcnt(0)\n" \
  "v_cmp_gt_u32 vcc, 58, v10\n" \
  "s_cbranch_vccnz LS27_%=\n" \
  "s_branch LB27_%=\n" \
  "LS28_%=:\n" \
  "s_sleep 1\n" \
  "ds_read_b32 v10, v9\n" \
  "s_waitcnt lgkmcnt(0)\n" \
  "v_cmp_gt_u32 vcc, 60, v10\n" \
  "s_cbranch_vccnz LS28_%=\n" \
  "s_branch LB28_%=\n" \
  "LS29_%=:\n" \
  "s_sleep 1\n" \
  "ds_read_b32 v10, v9\n" \
  "s_waitcnt lgkmcnt(0)\n" \
  "v_cmp_gt_u32 vcc, 64, v10\n" \
  "s_cbranch_vccnz LS29_%=\n" \
  "s_branch LB29_%=\n"
#define CONSUMER_CLOB "v0", "v1", "v2", "v3", "v4", "v5", "v6", "v7", "v8", "v9", "v10", "v11", "v12", "v13", "v14", "v15", "v16", "v17", "v18", "v19", "v20", "v21", "v22", "v23", "v24", "v25", "v26", "v27", "v28", "v29", "v30", "v31", "v32", "v33", "v34", "v35", "v36", "v37", "v38", "v39", "v40", "v41", "v42", "v43", "v44", "v45", "v46", "v47", "v48", "v49", "v50", "v51", "v52", "v53", "v54", "v55", "v56", "v57", "v58", "v59", "v60", "v61", "v62", "v63", "v64", "v65", "v66", "v67", "v68", "v69", "v70", "v71", "v72", "v73", "v74", "v75", "v76", "v77", "v78", "v79", "v80", "v81", "v82", "v83", "v84", "v85", "v86", "v87", "v88", "v89", "v90", "v91", "v92", "v93", "v94", "v95", "v96", "v97", "v98", "v99", "v100", "v101", "v102", "v103", "v104", "v105", "v106", "v107", "v108", "v109", "v110", "v111", "v112", "v113", "v114", "v115", "v116", "v117", "v118", "v119", "v120", "v121", "v122", "v123", "v124", "v125", "v126", "v127", "v128", "v129", "v130", "v131", "v132", "v133", "v134", "v135", "v136", "v137", "v138", "v139", "v140", "v141", "v142", "v143", "v144", "v145", "v146", "v147", "v148", "v149", "v150", "v151", "v152", "v153", "v154", "v155", "v156", "v157", "v158", "v159", "v160", "v161", "v162", "v163", "v164", "v165", "v166", "v167", "v168", "v169", "v170", "v171", "v172", "v173", "v174", "v175", "v176", "v177", "v178", "v179", "v180", "v181", "v182", "v183", "v184", "v185", "v186", "v187", "v188", "v189", "v190", "v191", "v192", "v193", "v194", "v195", "v196", "v197", "v198", "v199", "v200", "v201", "v202", "v203", "v204", "v205", "v206", "v207", "v208", "v209", "v210", "v211", "v212", "v213", "v214", "v215", "v216", "v217", "v218", "v219", "v220", "v221", "v222", "v223", "v224", "v225", "v226", "v227", "v228", "v229", "v230", "v231", "v232", "v233", "v234", "v235", "v236", "v237", "v238", "v239", "v240", "v241", "v242", "v243", "v244", "v245", "v246", "v247", "v248", "v249", "v250", "v251", "v252", "v253", "v254", "v255", "s40", "s41", "s42", "s43", "s44", "s45", "s46", "s47", "s48", "s49", "s50", "s51", "s52", "s53", "s54", "s55", "s56", "s57", "s58", "s59", "s60", "s61", "s62", "s63", "s64", "s65", "s66", "s67", "s68", "s69", "s70", "s71", "s72", "s73", "vcc", "scc", "memory"


#define LANE() int lane; asm volatile("v_mbcnt_lo_u32_b32 %0, -1, 0\n\tv_mbcnt_hi_u32_b32 %0, -1, %0" : "=v"(lane))

__device__ __forceinline__ int step_krow(int s, int rank) {
  if (s < 8) return rank * KC + s * 64;
  const int t = s - 8, d = t - ((t * 37) >> 8) * 7;
  return ((rank + 1 + d) & (NR - 1)) * KC + (t & 7) * 64;
}

__global__ __launch_bounds__(NTHREADS, 2)
void a2a_gemm_kernel(const signed char* __restrict__ X, const signed char* __restrict__ W,
                     const float* __restrict__ sx, const float* __restrict__ sw,
                     float* __restrict__ out, Peers peers, char* __restrict__ ws, int rank) {
  const unsigned long long ts0 = __builtin_amdgcn_s_memrealtime();
  extern __shared__ __attribute__((aligned(16))) char smem[];
  LAS char* L = (LAS char*)smem;
  volatile LAS unsigned* ctl = (volatile LAS unsigned*)(L + CTRL);
  const int wave = __builtin_amdgcn_readfirstlane(threadIdx.x >> 6);
  const int b = blockIdx.x, xcd = b & 7, idx = b >> 3;
  const int mtile = idx & 1, ntile = xcd * 16 + (idx >> 1);
  if (threadIdx.x < 32) { const unsigned t = threadIdx.x; ctl[t] = t == 1 ? (unsigned)(8 - ABIAS) : t == 3 ? BIG : 0u; }
  if (threadIdx.x >= 32 && threadIdx.x < 36) ctl[64 * (threadIdx.x - 31)] = 0u;
  asm volatile("s_waitcnt lgkmcnt(0)\n\ts_barrier" ::: "memory");
  auto min_free = [&]() -> unsigned {
    const unsigned f0 = ctl[64], f1 = ctl[128], f2 = ctl[192], f3 = ctl[256];
    return __builtin_amdgcn_readfirstlane(min(min(f0, f1), min(f2, f3)));
  };
  if (wave == 0) {
    __builtin_amdgcn_s_setprio(3);
    LANE();
    unsigned* ctr = (unsigned*)(ws + WS_CTR + (size_t)b * 128);
    const int slot = idx >> 2, rb = (idx & 3) * 8 + xcd, rot = rb % 7;
    v4i v[7];
    const GAS char* src = (const GAS char*)X + (size_t)(rb * 16 + (lane & 15)) * KC + (lane >> 4) * 16;
#pragma unroll
    for (int u = 0; u < 7; ++u) {
      const int dd = u + rot >= 7 ? u + rot - 7 : u + rot;
      v[u] = *(const GAS v4i*)(src + (size_t)((rank + 7 - dd) & (NR - 1)) * (MLOC * KC) + ((slot * 7 + dd) & 7) * 64);
    }
    unsigned call = __hip_atomic_load(ctr, __ATOMIC_RELAXED, __HIP_MEMORY_SCOPE_AGENT) + 1u;
    ctl[20] = 1u;
    char* fp = peers.ws[(rank + 7) & (NR - 1)];
#pragma unroll
    for (int dd = 1; dd < 7; ++dd) fp = lane == dd ? peers.ws[(rank + 7 - dd) & (NR - 1)] : fp;
    { const int dd = lane < 7 ? lane : 0;
      fp += slot == 0 ? WS_FLAG + (size_t)(rb >> 4) * FLAG_MT + dd * 2048 + (rb & 15) * 128 : WS_FLAG2 + (size_t)(slot * 7 + dd) * 128; }
    unsigned* cnt = (unsigned*)(ws + WS_CNT + slot * 128);
    call = __builtin_amdgcn_readfirstlane(call);
    const unsigned doff = (unsigned)(WS_A + (size_t)(call & 1u) * A_PAR) + (unsigned)(slot * 7 * 32768 + rb * 1024 + lane * 16);
    asm volatile("" : "+v"(v[0]), "+v"(v[1]), "+v"(v[2]), "+v"(v[3]), "+v"(v[4]), "+v"(v[5]), "+v"(v[6]));
    if (slot > 0) { const unsigned long long tw = (unsigned long long)(PBASE + (((b - 32) * PG) >> 5));
      while (__builtin_amdgcn_s_memrealtime() - ts0 < tw) __builtin_amdgcn_s_sleep(1); }
#pragma unroll
    for (int u = 0; u < 7; ++u) {
      const int dd = u + rot >= 7 ? u + rot - 7 : u + rot;
      asm volatile("global_store_dwordx4 %0, %1, %2 sc0 sc1\n\ts_nop 1"
                   :: "v"(doff + (unsigned)(dd * 32768)), "v"(v[u]), "s"(peers.ws[(rank + 7 - dd) & (NR - 1)]) : "memory");
    }
    asm volatile("s_waitcnt vmcnt(0)" ::: "memory");
    bool pub = true;
    if (slot > 0) {
      unsigned old = 0;
      if (lane == 0) old = __hip_atomic_fetch_add(cnt, 1u, __ATOMIC_RELAXED, __HIP_MEMORY_SCOPE_AGENT);
      pub = ((__builtin_amdgcn_readfirstlane(old) + 1u) & 31u) == 0u;
    }
    if (pub) asm volatile("s_mov_b64 exec, 0x7f\n\tglobal_store_dword %0, %1, off sc0 sc1\n\ts_mov_b64 exec, -1" :: "v"(fp), "v"(call) : "memory");
    if (lane == 0) __hip_atomic_store(ctr, call, __ATOMIC_RELAXED, __HIP_MEMORY_SCOPE_AGENT);
    return;
  }
  if (wave == 1) {
    __builtin_amdgcn_s_setprio(3);
    LANE();
    unsigned* ctr = (unsigned*)(ws + WS_CTR + (size_t)b * 128 + 64);
    auto srx = __builtin_amdgcn_make_buffer_rsrc((void*)(X + (size_t)rank * MLOC * KC), 0, MLOC * KC, 0x00020000);
    unsigned dummy = 0;
    { const unsigned so = (unsigned)(idx * 8192), va = (unsigned)(lane * 128);
      asm volatile("buffer_load_dword %0, %1, %2, %3 offen" : "+v"(dummy) : "v"(va), "s"(srx), "s"(so) : "memory"); }
    unsigned call = __hip_atomic_load(ctr, __ATOMIC_RELAXED, __HIP_MEMORY_SCOPE_AGENT) + 1u;
    call = __builtin_amdgcn_readfirstlane(call);
    ctl[16] = (call & 1u) * (unsigned)A_PAR;
    if (lane == 0) __hip_atomic_store(ctr, call, __ATOMIC_RELAXED, __HIP_MEMORY_SCOPE_AGENT);
    { const char* wp = ws + WS_A + (size_t)(call & 1u) * A_PAR + PAD;
      asm volatile("global_load_dword %0, %1, off" : "+v"(dummy) : "v"(wp) : "memory"); }
    unsigned* tmo = (unsigned*)(ws + WS_TMO);
    const char* fl = ws + WS_FLAG + (size_t)mtile * FLAG_MT + lane * 128;
    const char* fl2 = ws + WS_FLAG2 + (7 + (lane < 49 ? lane : 48)) * 128;
    const unsigned long long t0 = __builtin_amdgcn_s_memrealtime();
    while (__builtin_amdgcn_s_memrealtime() - ts0 < (unsigned long long)POLL0) __builtin_amdgcn_s_sleep(2);
    int done = 0;
    for (;;) {
      int cnt;
      if (done < 7) {
        unsigned f0, f1;
        asm volatile("global_load_dword %0, %2, off sc0 sc1\n\tglobal_load_dword %1, %3, off sc0 sc1\n\ts_waitcnt vmcnt(0)"
                     : "=&v"(f0), "=&v"(f1) : "v"(fl), "v"(fl + (lane < 48 ? 8192 : 0)) : "memory");
        const unsigned long long m0 = __ballot(f0 < call), m1 = __ballot(f1 < call);
        cnt = m0 ? (__builtin_ctzll(m0) >> 4) : m1 ? 4 + (__builtin_ctzll(m1) >> 4) : 7;
      } else {
        unsigned f0;
        asm volatile("global_load_dword %0, %1, off sc0 sc1\n\ts_waitcnt vmcnt(0)" : "=&v"(f0) : "v"(fl2) : "memory");
        const unsigned long long m0 = __ballot(f0 < call);
        cnt = m0 ? 7 + __builtin_ctzll(m0) : 56;
      }
      const bool again = cnt == 7 && done < 7;
      if (cnt > done) {
        done = cnt;

        ctl[1] = done == 56 ? BIG : (unsigned)(8 - ABIAS + done);
      }
      if (done == 56) break;
      if (__builtin_amdgcn_s_memrealtime() - t0 > 3000ull) {
        if (__hip_atomic_load(tmo, __ATOMIC_RELAXED, __HIP_MEMORY_SCOPE_AGENT) == call ||
            __builtin_amdgcn_s_memrealtime() - t0 > DEADLINE) {
          __hip_atomic_store(tmo, call, __ATOMIC_RELAXED, __HIP_MEMORY_SCOPE_AGENT);
          ctl[12] = 1u;
          asm volatile("s_waitcnt lgkmcnt(0)" ::: "memory");
          ctl[1] = BIG;
          break;
        }
        __builtin_amdgcn_s_sleep(16);
      }
      if (!again) { if ((int)min_free() + 14 < done + 8) __builtin_amdgcn_s_sleep(16); else __builtin_amdgcn_s_sleep(2); }
    }
    asm volatile("s_waitcnt vmcnt(0)" : "+v"(dummy) :: "memory");
    return;
  }
  if (wave < 4) {
    const int role = wave - 2, lw = role * 2;
    LANE();
    __builtin_amdgcn_s_setprio(1);
    auto srd = __builtin_amdgcn_make_buffer_rsrc((void*)W, 0, KK * NN, 0x00020000);
    const int sl = ((lane >> 4) & 3) << 1;
    const unsigned voff0 = (unsigned)((lane >> 3) * NN + (((lane & 7) ^ sl) << 4) + ntile * 128);
    unsigned vo[8];
#pragma unroll
    for (int e = 0; e < 8; ++e) vo[e] = (voff0 ^ (((e >> 1) & 1) << 4)) + (unsigned)(e * 8 * NN - (e & 3) * 1024);
    auto issue_step = [&](int s) {
      const unsigned m0a = (unsigned)((s % NS) * SLOT), m0b = m0a + 4096u, so = (unsigned)(step_krow(s, rank) * NN);
      asm volatile("s_mov_b32 m0, %0\n\ts_nop 0\n\t"
                   "buffer_load_dwordx4 %4, %2, %3 offen lds\n\tbuffer_load_dwordx4 %5, %2, %3 offen offset:1024 lds\n\t"
                   "buffer_load_dwordx4 %6, %2, %3 offen offset:2048 lds\n\tbuffer_load_dwordx4 %7, %2, %3 offen offset:3072 lds\n\t"
                   "s_mov_b32 m0, %1\n\ts_nop 0\n\t"
                   "buffer_load_dwordx4 %8, %2, %3 offen lds\n\tbuffer_load_dwordx4 %9, %2, %3 offen offset:1024 lds\n\t"
                   "buffer_load_dwordx4 %10, %2, %3 offen offset:2048 lds\n\tbuffer_load_dwordx4 %11, %2, %3 offen offset:3072 lds"
                   :: "s"(m0a), "s"(m0b), "s"(srd), "s"(so), "v"(vo[0]), "v"(vo[1]), "v"(vo[2]), "v"(vo[3]), "v"(vo[4]), "v"(vo[5]), "v"(vo[6]), "v"(vo[7]) : "memory");
    };
    const int par = role == 0 ? mtile : (mtile ^ 1);
    while (ctl[20] == 0u) __builtin_amdgcn_s_sleep(1);
    if (par) ctl[lw] = (unsigned)CL;
    else {
#pragma unroll
      for (int s = 0; s < CL; ++s) issue_step(s);
#pragma unroll
      for (int r = CL - 1; r >= 0; --r) { asm volatile("s_waitcnt vmcnt(%0)" :: "i"(8 * r) : "memory"); ctl[lw] = (unsigned)(r == 0 ? 2 * CL : CL - r); }
    }
    unsigned dummy = 0;
    unsigned slowmask;
    { const unsigned a0 = (unsigned)(((unsigned long long)W >> 7) + xcd * 16);
      slowmask = (0x0101u << ((3u - a0) & 7u)) & 0xffffu;
      const unsigned d63 = (63u - a0) & 63u; if (d63 < 16u) slowmask |= 1u << d63; }
    const int lp = idx >> 1;
    const bool slow = (slowmask >> lp) & 1;
    const int nsl = __builtin_popcount(slowmask), nn = 32 - 2 * nsl;
    const int hr = 2 * (lp - __builtin_popcount(slowmask & ((1u << lp) - 1))) + mtile;
    const int item = hr + nn * lane, hq = item >> 9, hrow = item & 511;
    unsigned hcol;
    { unsigned m = slowmask; for (int t = 0; t < 3; ++t) if (t < hq) m &= m - 1;
      hcol = (hq < nsl) ? (unsigned)((xcd * 16 + (m ? __builtin_ctz(m) : 0)) * 128 + (hrow & 63) * NN) : 0x80000000u; }
    int hnext = HSTART;
    auto sra = __builtin_amdgcn_make_buffer_rsrc((void*)(ws + WS_A), 0, (int)(2 * A_PAR), 0x00020000);
    const unsigned vt = (unsigned)((lane >> 3) * 32768 + mtile * 16384 + lp * 1024 + (lane & 7) * 128);
    int anext = 16;
    bool pend = false;
    for (int k = par ? 1 : 2; k * CL < NSTEP; k += 2) {
      const int st = k * CL, en = st + CL;
      const bool lead = role == 0 && !(slow && st >= HSTART);
      int need = lead ? st - LLEAD : st - LFOL;
      if (need > LASTN) need = LASTN;
      if (need < en - NS) need = en - NS;
      if (en > 11 && need < 4) need = 4;
      if (st < 16 && need < (st - 4) / 2) need = (st - 4) / 2;
      if (need > 0 && min_free() < (unsigned)need) {
        if (pend) { asm volatile("s_waitcnt vmcnt(0)" ::: "memory"); ctl[lw] = (unsigned)st; pend = false; }
        while (min_free() < (unsigned)need) __builtin_amdgcn_s_sleep(3);
      }
      if (role == 0 && anext < NSTEP && anext < en + (slow ? 6 : 0) &&
          anext + 8 <= (int)__builtin_amdgcn_readfirstlane(ctl[1]) + ABIAS) {
        const unsigned so = __builtin_amdgcn_readfirstlane(ctl[16]) + (unsigned)((anext - 8) * 32768);
        asm volatile("s_nop 4\n\tbuffer_load_dword %0, %1, %2, %3 offen sc0 sc1" : "+v"(dummy) : "v"(vt), "s"(sra), "s"(so) : "memory");
        anext += 8;
      }
      if (role == 0 && !slow) {
        while (hnext < NSTEP && hnext < en + HAHEAD) {
          const unsigned vh = hcol + (unsigned)(step_krow(hnext + (hrow >> 6), rank) * NN);
          asm volatile("buffer_load_dword %0, %1, %2, 0 offen" : "+v"(dummy) : "v"(vh), "s"(srd) : "memory");
          hnext += 8;
        }
      }
#pragma unroll
      for (int s = 0; s < CL; ++s) issue_step(st + s);
      if (lead && st >= 4) {
        if (pend) { asm volatile("s_waitcnt vmcnt(%0)" :: "i"(8 * CL) : "memory"); ctl[lw] = (unsigned)st; }
        pend = true;
      } else {
#pragma unroll
        for (int r = CL - 1; r >= 0; --r) {
          asm volatile("s_waitcnt vmcnt(%0)" :: "i"(8 * r) : "memory");
          ctl[lw] = (unsigned)(r == 0 ? en + CL : en - r);
        }
        pend = false;
      }
    }
    asm volatile("s_waitcnt vmcnt(0)" : "+v"(dummy) :: "memory");
    ctl[lw] = BIG;
    return;
  }
  {
    const int w = wave - 4;
    const unsigned sfree = (unsigned)(CTRL + 256 + 256 * w);
    const unsigned sxb = (unsigned)(11 * SLOT + w * 8192);
    const signed char* sa0 = X + (size_t)(rank * MLOC + mtile * 256 + w * 64) * KC;
    const char* sa1 = ws + WS_A + (size_t)(mtile * 4 + w) * 4096;
    float* sout = out + (size_t)(mtile * 256 + w * 64) * NN + ntile * 128;
    asm volatile(CONSUMER_ASM
                 :
                 : [sfree] "s"(sfree), [sw7] "s"((unsigned)(w << 7)), [shold] "s"(1u), [sxb] "s"(sxb), [sa0] "s"(sa0), [sa1] "s"(sa1),
                   [sout] "s"(sout), [psx] "s"(sx), [psw] "s"(sw)
                 : CONSUMER_CLOB);
  }
}

extern "C" void kernel_launch(int rank, int nranks, ncclComm_t comm,
                              hipStream_t stream,
                              void* const* d_in, const int* in_sizes, int n_in,
                              void* d_out, int out_size,
                              const int* in_dtypes, int out_dtype,
                              void* const* d_out_all,
                              void* const* d_ws_all, size_t ws_bytes) {
  if (nranks != NR || (unsigned)rank >= (unsigned)NR || ws_bytes < WS_NEED || n_in != 4 ||
      in_sizes[0] != 4096 * KC || in_sizes[1] != KK * NN || out_size != MLOC * NN) {
    fprintf(stderr, "kernel_launch: unexpected problem shape\n");
    return;
  }
  static bool attr_set = false;
  if (!attr_set) {
    HIP_CHECK(hipFuncSetAttribute((const void*)a2a_gemm_kernel, hipFuncAttributeMaxDynamicSharedMemorySize, LDS_BYTES));
    attr_set = true;
  }
  Peers peers;
  for (int p = 0; p < NR; ++p) peers.ws[p] = (char*)d_ws_all[p];
  a2a_gemm_kernel<<<NWG, NTHREADS, LDS_BYTES, stream>>>(
      (const signed char*)d_in[0], (const signed char*)d_in[1], (const float*)d_in[2], (const float*)d_in[3],
      (float*)d_out, peers, peers.ws[rank], rank);
  HIP_CHECK(hipGetLastError());
}
```
